```python
import numpy as np

kernel_name = "gpu_a2a_gemm_m4096_k4096_n32768_bf16_gelu"


ABSMAX_RTOL = 0.008

COMPUTE_DTYPE = "bfloat16"
EPILOGUE = "gelu"
IN_QUANT = None
OUT_QUANT = None
IN_DTYPES = {"x": "bfloat16", "w_mat": "bfloat16"}
WORLD_SIZE = 8

def _to_bf16(x):
    u = np.ascontiguousarray(x, dtype=np.float32).view(np.uint32)
    return ((u + 0x7FFF + ((u >> 16) & 1)) & 0xFFFF0000).view(np.float32)

def setup_inputs():

    x = _to_bf16(np.random.randn(4096, 4096).astype(np.float32))
    w_mat = _to_bf16((np.random.randn(4096, 32768) / np.sqrt(4096)).astype(np.float32))
    return {"x": x, "w_mat": w_mat}

def _epilogue(y):
    c = 0.7978845608028654
    return (0.5 * y * (1.0 + np.tanh(c * (y + 0.044715 * y ** 3)))).astype(np.float32)

def reference(x, w_mat):
    y = (x @ w_mat).astype(np.float32)
    return _epilogue(y)
```

```cpp
#include <hip/hip_runtime.h>
#include <rccl/rccl.h>
#include <cstdio>

#define HIP_CHECK(expr) do { hipError_t _e = (expr); if (_e != hipSuccess) { \
    fprintf(stderr, "HIP error %s at %s:%d\n", hipGetErrorString(_e), __FILE__, __LINE__); \
    return; } } while (0)

#define LAS __attribute__((address_space(3)))
constexpr int NRANKS = 8;
constexpr unsigned LDS_BYTES = 163840;
constexpr unsigned CNT_OFF = 0;
constexpr unsigned FLAG_OFF = 65536;
constexpr unsigned TMO_OFF = FLAG_OFF + 1152;
constexpr unsigned DEC_OFF = 131072;
constexpr size_t DATA_OFF = 1u << 20;
constexpr size_t WS_NEED = DATA_OFF + 2 * (4u << 20);

struct Args {
    const char* x; const char* w; char* out; char* wsme;
    char* ws[NRANKS];
    unsigned long long* stamps;
    int rank, pad;
};
static_assert(sizeof(Args) == 112, "kernel argument layout");

__global__ __launch_bounds__(256, 1) void a2a_gemm_gelu(const Args a) {
    extern __shared__ __attribute__((aligned(1024))) unsigned char shm[];
    const unsigned ldsbase = (unsigned)(size_t)(LAS unsigned char*)shm;
    asm volatile(
    "s_load_dwordx8 s[8:15], %[ka], 0x0\n"
    "s_load_dword s22, %[ka], 0x68\n"
    "s_mov_b32 s23, %[bx]\n"
    "s_lshr_b32 s27, s23, 5\n"
    "s_lshl_b32 s67, s27, 3\n"
    "s_add_u32 s67, s67, 32\n"
    "s_load_dwordx2 s[20:21], %[ka], s67\n"
    "v_and_b32 v63, 63, v0\n"
    "v_lshrrev_b32 v1, 6, v0\n"
    "v_lshrrev_b32 v2, 4, v63\n"
    "v_readfirstlane_b32 s24, v1\n"
    "v_and_b32 v3, 15, v63\n"
    "v_and_b32 v4, 7, v63\n"
    "s_and_b32 s67, s23, 7\n"
    "s_lshr_b32 s77, s23, 3\n"
    "s_and_b32 s25, s77, 1\n"
    "s_lshr_b32 s77, s77, 1\n"
    "s_lshl_b32 s67, s67, 4\n"
    "s_add_u32 s26, s67, s77\n"
    "s_and_b32 s67, s24, 1\n"
    "s_lshl_b32 s67, s67, 20\n"
    "s_lshr_b32 s77, s24, 1\n"
    "s_lshl_b32 s77, s77, 8\n"
    "s_add_u32 s67, s67, s77\n"
    "v_lshlrev_b32 v5, 2, v2\n"
    "v_xor_b32 v5, v5, v3\n"
    "v_lshlrev_b32 v5, 4, v5\n"
    "v_lshl_add_u32 v6, v2, 16, s67\n"
    "v_add_u32 v56, v6, v5\n"
    "v_add_u32 v57, 261120, v56\n"
    "v_xor_b32 v5, 16, v5\n"
    "v_add_u32 v5, v6, v5\n"
    "v_add_u32 v58, 522240, v5\n"
    "v_add_u32 v59, 783360, v5\n"
    "v_lshrrev_b32 v5, 3, v63\n"
    "s_lshl_b32 s67, s24, 16\n"
    "v_lshl_add_u32 v5, v5, 10, s67\n"
    "v_xor_b32 v6, v4, v2\n"
    "v_lshlrev_b32 v6, 4, v6\n"
    "v_add_u32 v7, v5, v6\n"
    "v_xor_b32 v6, 64, v6\n"
    "v_add_u32 v6, v5, v6\n"
    "v_add_u32 v48, 0, v7\n"
    "v_add_u32 v49, 7168, v6\n"
    "v_add_u32 v50, 14336, v7\n"
    "v_add_u32 v51, 21504, v6\n"
    "v_add_u32 v52, 32768, v7\n"
    "v_add_u32 v53, 39936, v6\n"
    "v_add_u32 v54, 47104, v7\n"
    "v_add_u32 v55, 54272, v6\n"
    "s_lshl_b32 s70, s24, 13\n"
    "s_add_u32 s70, s70, %[lb]\n"
    "s_lshr_b32 s67, s24, 1\n"
    "s_lshl_b32 s67, s67, 13\n"
    "s_and_b32 s77, s24, 1\n"
    "s_lshl_b32 s77, s77, 12\n"
    "s_add_u32 s71, s67, s77\n"
    "s_add_u32 s71, s71, %[lb]\n"
    "s_add_u32 s71, s71, 65536\n"
    "s_lshl_b32 s34, s25, 18\n"
    "s_lshl_b32 s35, s26, 9\n"
    "s_lshl_b32 s40, s23, 7\n"
    "s_lshl_b32 s67, s24, 2\n"
    "s_add_u32 s40, s40, s67\n"
    "v_mov_b32 v46, s40\n"
    "v_bfe_u32 v5, v63, 3, 2\n"
    "s_lshl_b32 s67, s23, 4\n"
    "s_lshl_b32 s77, s24, 2\n"
    "s_add_u32 s67, s67, s77\n"
    "v_add_u32 v5, s67, v5\n"
    "v_lshlrev_b32 v5, 10, v5\n"
    "v_lshrrev_b32 v6, 5, v63\n"
    "v_lshl_add_u32 v5, v6, 7, v5\n"
    "v_lshl_add_u32 v42, v4, 4, v5\n"
    "v_and_b32 v43, 0x7ffff, v42\n"
    "s_waitcnt lgkmcnt(0)\n"
    "s_mov_b64 s[16:17], s[12:13]\n"
    "s_mov_b64 s[18:19], s[14:15]\n"
    "global_load_dword v46, v46, s[18:19] sc1\n"
    "s_mov_b32 s12, s10\n"
    "s_and_b32 s13, s11, 0xffff\n"
    "s_mov_b32 s14, 0x10000000\n"
    "s_mov_b32 s15, 0x20000\n"
    "s_mov_b32 s84, s8\n"
    "s_and_b32 s85, s9, 0xffff\n"
    "s_mov_b32 s86, 0x400000\n"
    "s_mov_b32 s87, 0x20000\n"
    "s_and_b32 s22, s22, 7\n"
    "s_lshl_b32 s73, s22, 25\n"
    "s_add_u32 s73, s73, s35\n"
    "s_lshl_b32 s72, s22, 19\n"
    "s_add_u32 s72, s72, s34\n"
    "s_add_u32 m0, s71, 0\n"
    "s_nop 0\n"
    "buffer_load_dwordx4 v56, s[12:15], s73 offen offset:0  lds\n"
    "v_lshlrev_b32 v5, 7, v3\n"
    "v_lshrrev_b32 v6, 1, v3\n"
    "v_xor_b32 v6, v6, v2\n"
    "v_lshl_add_u32 v5, v6, 4, v5\n"
    "s_lshr_b32 s67, s24, 1\n"
    "s_lshl_b32 s67, s67, 14\n"
    "buffer_load_dwordx4 v57, s[12:15], s73 offen offset:1024  lds\n"
    "s_add_u32 s67, s67, %[lb]\n"
    "v_add_u32 v32, s67, v5\n"
    "v_xor_b32 v33, 64, v32\n"
    "v_bfe_u32 v5, v63, 2, 2\n"
    "v_and_b32 v6, 3, v63\n"
    "v_and_b32 v7, 1, v2\n"
    "buffer_load_dwordx4 v58, s[12:15], s73 offen offset:2048  lds\n"
    "v_xor_b32 v7, v7, v6\n"
    "v_lshlrev_b32 v7, 4, v7\n"
    "v_lshl_add_u32 v7, v2, 11, v7\n"
    "v_lshl_add_u32 v7, v5, 8, v7\n"
    "v_lshl_add_u32 v7, v5, 6, v7\n"
    "v_and_b32 v6, 1, v6\n"
    "buffer_load_dwordx4 v59, s[12:15], s73 offen offset:3072  lds\n"
    "v_lshl_add_u32 v7, v6, 3, v7\n"
    "s_and_b32 s67, s24, 1\n"
    "s_lshl_b32 s67, s67, 13\n"
    "s_add_u32 s67, s67, %[lb]\n"
    "s_add_u32 s67, s67, 65536\n"
    "v_add_u32 v24, s67, v7\n"
    "s_add_u32 m0, s70, 0\n"
    "s_nop 0\n"
    "buffer_load_dwordx4 v48, s[84:87], s72 offen offset:0 sc0 sc1 lds\n"
    "v_xor_b32 v25, 8, v24\n"
    "v_xor_b32 v26, 64, v24\n"
    "v_xor_b32 v27, 72, v24\n"
    "v_xor_b32 v28, 128, v24\n"
    "v_xor_b32 v29, 136, v24\n"
    "v_xor_b32 v30, 192, v24\n"
    "buffer_load_dwordx4 v49, s[84:87], s72 offen offset:1024 sc0 sc1 lds\n"
    "v_xor_b32 v31, 200, v24\n"
    "v_add_u32 v16, 0x10000, v24\n"
    "v_add_u32 v17, 0x10000, v25\n"
    "v_add_u32 v18, 0x10000, v26\n"
    "v_add_u32 v19, 0x10000, v27\n"
    "v_add_u32 v20, 0x10000, v28\n"
    "buffer_load_dwordx4 v50, s[84:87], s72 offen offset:2048 sc0 sc1 lds\n"
    "v_add_u32 v21, 0x10000, v29\n"
    "v_add_u32 v22, 0x10000, v30\n"
    "v_add_u32 v23, 0x10000, v31\n"
    "v_subrev_u32 v5, 8, v63\n"
    "v_mul_u32_u24 v6, 9363, v5\n"
    "v_lshrrev_b32 v6, 16, v6\n"
    "buffer_load_dwordx4 v51, s[84:87], s72 offen offset:3072 sc0 sc1 lds\n"
    "v_mul_u32_u24 v7, 7, v6\n"
    "v_sub_u32 v7, v5, v7\n"
    "s_add_u32 s67, s22, 1\n"
    "v_add_u32 v7, s67, v7\n"
    "v_and_b32 v7, 7, v7\n"
    "v_cmp_gt_u32 vcc, 8, v63\n"
    "s_add_u32 m0, s70, 4096\n"
    "s_nop 0\n"
    "buffer_load_dwordx4 v52, s[84:87], s72 offen offset:0 sc0 sc1 lds\n"
    "v_mov_b32 v5, s22\n"
    "s_nop 0\n"
    "v_cndmask_b32 v7, v7, v5, vcc\n"
    "v_cndmask_b32 v6, v6, v63, vcc\n"
    "v_lshlrev_b32 v5, 19, v7\n"
    "v_lshl_add_u32 v5, v6, 7, v5\n"
    "buffer_load_dwordx4 v53, s[84:87], s72 offen offset:1024 sc0 sc1 lds\n"
    "v_add_u32 v40, s34, v5\n"
    "v_lshlrev_b32 v5, 25, v7\n"
    "v_lshl_add_u32 v5, v6, 22, v5\n"
    "v_add_u32 v41, s35, v5\n"
    "v_lshrrev_b32 v5, 3, v63\n"
    "v_lshlrev_b32 v5, 7, v5\n"
    "buffer_load_dwordx4 v54, s[84:87], s72 offen offset:2048 sc0 sc1 lds\n"
    "v_lshl_add_u32 v44, v4, 2, v5\n"
    "v_add_u32 v44, 65536, v44\n"
    "v_lshlrev_b32 v9, 2, v4\n"
    "v_add_u32 v9, 66560, v9\n"
    "s_lshl_b32 s67, s22, 7\n"
    "s_add_u32 s67, s67, 65536\n"
    "buffer_load_dwordx4 v55, s[84:87], s72 offen offset:3072 sc0 sc1 lds\n"
    "v_mov_b32 v60, s67\n"
    "v_mov_b32 v11, v4\n"
    "s_lshr_b32 s67, s24, 1\n"
    "s_lshl_b32 s67, s67, 23\n"
    "s_and_b32 s77, s24, 1\n"
    "s_lshl_b32 s77, s77, 8\n"
    "s_add_u32 s73, s73, 0x200000\n"
    "s_add_u32 m0, s71, 16384\n"
    "s_nop 0\n"
    "buffer_load_dwordx4 v56, s[12:15], s73 offen offset:0  lds\n"
    "s_add_u32 s67, s67, s77\n"
    "v_lshl_add_u32 v39, v3, 16, s67\n"
    "v_lshl_add_u32 v39, v2, 4, v39\n"
    "s_xor_b32 s33, s27, s22\n"
    "s_lshl_b32 s67, s23, 14\n"
    "v_lshl_add_u32 v5, v63, 7, s67\n"
    "buffer_load_dwordx4 v57, s[12:15], s73 offen offset:1024  lds\n"
    "s_and_b32 s67, s23, 7\n"
    "s_lshr_b32 s77, s23, 4\n"
    "s_lshl_b32 s77, s77, 3\n"
    "s_or_b32 s67, s67, s77\n"
    "s_lshl_b32 s67, s67, 7\n"
    "s_add_u32 s36, s67, 131072\n"
    "buffer_load_dwordx4 v58, s[12:15], s73 offen offset:2048  lds\n"
    "s_cmp_eq_u32 s24, 0\n"
    "s_cselect_b32 s38, 1, 0\n"
    "buffer_load_dwordx4 v59, s[12:15], s73 offen offset:3072  lds\n"
    "s_waitcnt vmcnt(16)\n"
    "global_load_dwordx4 a[224:227], v42, s[8:9] offset:0\n"
    "global_load_dwordx4 a[228:231], v42, s[8:9] offset:256\n"
    "global_load_dwordx4 a[232:235], v42, s[8:9] offset:512\n"
    "global_load_dwordx4 a[236:239], v42, s[8:9] offset:768\n"
    "v_readfirstlane_b32 s66, v46\n"
    "s_and_b32 s67, s23, 31\n"
    "s_or_b32 s67, s67, s24\n"
    "s_cmp_eq_u32 s67, 0\n"
    "s_cselect_b32 s67, s33, 0\n"
    "s_lshl_b32 s77, s22, 2\n"
    "s_add_u32 s77, s77, 66560\n"
    "s_cmp_lg_u32 s67, 0\n"
    "s_cselect_b32 s37, s77, 0\n"
    "s_add_u32 s66, s66, 1\n"
    "s_cmp_eq_u32 s37, 0\n"
    "s_cbranch_scc1 Lnostart%=\n"
    "v_mov_b32 v35, s37\n"
    "v_mov_b32 v47, s66\n"
    "s_mov_b64 exec, 1\n"
    "global_store_dword v35, v47, s[20:21] sc0 sc1\n"
    "s_mov_b64 exec, -1\n"
    "Lnostart%=:\n"
    "v_mov_b32 v35, s40\n"
    "v_mov_b32 v47, s66\n"
    "s_mov_b64 exec, 1\n"
    "global_store_dword v35, v47, s[18:19] sc1\n"
    "s_mov_b64 exec, -1\n"
    "s_add_u32 s77, s72, 128\n"
    "s_add_u32 s78, s73, 0x200000\n"
    "s_add_u32 m0, s70, 32768\n"
    "s_nop 0\n"
    "buffer_load_dwordx4 v48, s[84:87], s77 offen offset:0 sc0 sc1 lds\n"
    "s_lshr_b32 s90, s23, 3\n"
    "s_and_b32 s91, s90, 1\n"
    "s_mul_i32 s92, s90, 43\n"
    "s_lshr_b32 s92, s92, 7\n"
    "s_mul_i32 s92, s92, 3\n"
    "s_sub_u32 s92, s90, s92\n"
    "buffer_load_dwordx4 v49, s[84:87], s77 offen offset:1024 sc0 sc1 lds\n"
    "s_mov_b32 s42, 2\n"
    "s_mov_b32 s43, 0\n"
    "s_add_u32 s90, s91, 2\n"
    "s_bitset1_b32 s42, s90\n"
    "s_add_u32 s90, s92, 4\n"
    "s_bitset1_b32 s42, s90\n"
    "buffer_load_dwordx4 v50, s[84:87], s77 offen offset:2048 sc0 sc1 lds\n"
    "s_add_u32 s90, s91, 7\n"
    "s_bitset1_b32 s42, s90\n"
    "s_cmp_lg_u32 s33, 0\n"
    "s_cselect_b32 s42, s42, 0\n"
    "s_lshl_b32 s44, s42, 2\n"
    "s_or_b32 s44, s44, 2\n"
    "buffer_load_dwordx4 v51, s[84:87], s77 offen offset:3072 sc0 sc1 lds\n"
    "s_mov_b32 s45, 0\n"
    "s_and_b32 s90, s38, 1\n"
    "s_cmp_lg_u32 s33, 0\n"
    "s_cselect_b32 s90, s90, 0\n"
    "s_cmp_lg_u32 s90, 0\n"
    "s_cselect_b32 s44, s44, 0\n"
    "s_add_u32 m0, s70, 36864\n"
    "s_nop 0\n"
    "buffer_load_dwordx4 v52, s[84:87], s77 offen offset:0 sc0 sc1 lds\n"
    "s_and_b32 s67, s66, 1\n"
    "s_lshl_b32 s67, s67, 22\n"
    "s_lshl_b32 s90, s22, 19\n"
    "s_add_u32 s90, s90, 1048576\n"
    "s_add_u32 s90, s90, s67\n"
    "s_add_u32 s82, s20, s90\n"
    "buffer_load_dwordx4 v53, s[84:87], s77 offen offset:1024 sc0 sc1 lds\n"
    "s_addc_u32 s83, s21, 0\n"
    "s_add_u32 s90, s67, 1048576\n"
    "s_add_u32 s88, s18, s90\n"
    "s_addc_u32 s89, s19, 0\n"
    "s_and_b32 s89, s89, 0xffff\n"
    "s_lshl_b32 s48, 1, s22\n"
    "buffer_load_dwordx4 v54, s[84:87], s77 offen offset:2048 sc0 sc1 lds\n"
    "s_mov_b32 s49, 0\n"
    "s_mov_b32 s50, 0\n"
    "s_mov_b32 s51, 8\n"
    "s_mov_b32 s52, 0\n"
    "s_mov_b32 s39, 0\n"
    "s_or_b64 s[28:29], s[44:45], s[42:43]\n"
    "buffer_load_dwordx4 v55, s[84:87], s77 offen offset:3072 sc0 sc1 lds\n"
    "s_or_b64 s[64:65], s[28:29], 2\n"
    "s_mov_b64 s[96:97], 0\n"
    "s_mov_b64 s[98:99], 4\n"
    "s_mov_b64 s[62:63], 12\n"
    "s_mov_b64 s[60:61], 12\n"
    "s_mov_b32 s79, 0\n"
    "s_add_u32 m0, s71, 32768\n"
    "s_nop 0\n"
    "buffer_load_dwordx4 v56, s[12:15], s78 offen offset:0  lds\n"
    "v_mov_b32 v62, 1\n"
    "buffer_load_dwordx4 v57, s[12:15], s78 offen offset:1024  lds\n"
    "buffer_load_dwordx4 v58, s[12:15], s78 offen offset:2048  lds\n"
    "buffer_load_dwordx4 v59, s[12:15], s78 offen offset:3072  lds\n"
    "s_add_u32 s78, s78, 0x200000\n"
    "s_add_u32 m0, s71, 49152\n"
    "s_nop 0\n"
    "buffer_load_dwordx4 v56, s[12:15], s78 offen offset:0  lds\n"
    "buffer_load_dwordx4 v57, s[12:15], s78 offen offset:1024  lds\n"
    "buffer_load_dwordx4 v58, s[12:15], s78 offen offset:2048  lds\n"
    "buffer_load_dwordx4 v59, s[12:15], s78 offen offset:3072  lds\n"
    "s_mov_b32 s41, 0\n"
    "s_mov_b32 s30, 1600\n"
    "s_mov_b32 s4, 0\n"
    "s_mov_b32 s56, 0\n"
    "s_mov_b32 s80, 0\n"
    "v_lshlrev_b32 v45, 2, v63\n"
    "v_and_b32 v45, 28, v45\n"
    "v_add_u32 v45, s36, v45\n"
    "global_load_dword v45, v45, s[18:19] sc1\n"
    "s_add_u32 s77, s78, 0x200000\n"
    "s_add_u32 s72, s72, 256\n"
    "s_add_u32 s73, s77, 0x400000\n"
    "s_mov_b32 s74, 0\n"
    "s_mov_b32 s75, 2\n"
    "s_mov_b32 s76, 3\n"
    "s_cmp_eq_u32 s37, 0\n"
    "s_cbranch_scc1 Lw18%=\n"
    "s_waitcnt vmcnt(19)\n"
    "s_branch Lw22%=\n"
    "Lw18%=:\n"
    "s_waitcnt vmcnt(18)\n"
    "Lw22%=:\n"
    "s_cmp_eq_u32 s33, 0\n"
    "s_cbranch_scc1 Lnopush0%=\n"
    "s_mov_b64 exec, 0xffffffff\n"
    "global_store_dwordx4 v43, a[224:227], s[82:83] sc0 sc1\n"
    "s_mov_b64 exec, -1\n"
    "Lnopush0%=:\n"
    "s_barrier\n"
    "ds_read_b64_tr_b16 a[160:161], v24 offset:0\n"
    "ds_read_b64_tr_b16 a[162:163], v24 offset:1024\n"
    "ds_read_b64_tr_b16 a[164:165], v25 offset:0\n"
    "ds_read_b64_tr_b16 a[166:167], v25 offset:1024\n"
    "ds_read_b64_tr_b16 a[168:169], v26 offset:0\n"
    "ds_read_b64_tr_b16 a[170:171], v26 offset:1024\n"
    "ds_read_b64_tr_b16 a[172:173], v27 offset:0\n"
    "ds_read_b64_tr_b16 a[174:175], v27 offset:1024\n"
    "ds_read_b64_tr_b16 a[176:177], v28 offset:0\n"
    "ds_read_b64_tr_b16 a[178:179], v28 offset:1024\n"
    "ds_read_b64_tr_b16 a[180:181], v29 offset:0\n"
    "ds_read_b64_tr_b16 a[182:183], v29 offset:1024\n"
    "ds_read_b64_tr_b16 a[184:185], v30 offset:0\n"
    "ds_read_b64_tr_b16 a[186:187], v30 offset:1024\n"
    "ds_read_b64_tr_b16 a[188:189], v31 offset:0\n"
    "ds_read_b64_tr_b16 a[190:191], v31 offset:1024\n"
    "ds_read_b128 a[64:67], v32 offset:0\n"
    "ds_read_b128 a[68:71], v32 offset:2048\n"
    "ds_read_b128 a[72:75], v32 offset:4096\n"
    "ds_read_b128 a[76:79], v32 offset:6144\n"
    "ds_read_b128 a[80:83], v32 offset:8192\n"
    "ds_read_b128 a[84:87], v32 offset:10240\n"
    "ds_read_b128 a[88:91], v32 offset:12288\n"
    "ds_read_b128 a[92:95], v32 offset:14336\n"
    "ds_read_b128 a[96:99], v33 offset:0\n"
    "ds_read_b128 a[100:103], v33 offset:2048\n"
    "ds_read_b128 a[104:107], v33 offset:4096\n"
    "ds_read_b128 a[108:111], v33 offset:6144\n"
    "ds_read_b128 a[112:115], v33 offset:8192\n"
    "ds_read_b128 a[116:119], v33 offset:10240\n"
    "ds_read_b128 a[120:123], v33 offset:12288\n"
    "ds_read_b128 a[124:127], v33 offset:14336\n"
    "s_waitcnt lgkmcnt(0)\n"
    "s_barrier\n"
    "v_mfma_f32_16x16x32_bf16 v[64:67], a[160:163], a[64:67], 0\n"
    "ds_read_b64_tr_b16 a[192:193], v24 offset:16384\n"
    "s_add_u32 m0, s71, 65536\n"
    "v_mfma_f32_16x16x32_bf16 v[96:99], a[160:163], a[68:71], 0\n"
    "buffer_load_dwordx4 v56, s[12:15], s77 offen offset:0  lds\n"
    "v_mfma_f32_16x16x32_bf16 v[100:103], a[164:167], a[68:71], 0\n"
    "v_mfma_f32_16x16x32_bf16 v[68:71], a[164:167], a[64:67], 0\n"
    "ds_read_b64_tr_b16 a[194:195], v24 offset:17408\n"
    "v_mfma_f32_16x16x32_bf16 v[128:131], a[160:163], a[72:75], 0\n"
    "buffer_load_dwordx4 v57, s[12:15], s77 offen offset:1024  lds\n"
    "v_mfma_f32_16x16x32_bf16 v[160:163], a[160:163], a[76:79], 0\n"
    "v_mfma_f32_16x16x32_bf16 v[164:167], a[164:167], a[76:79], 0\n"
    "ds_read_b64_tr_b16 a[196:197], v25 offset:16384\n"
    "v_mfma_f32_16x16x32_bf16 v[132:135], a[164:167], a[72:75], 0\n"
    "buffer_load_dwordx4 v58, s[12:15], s77 offen offset:2048  lds\n"
    "v_mfma_f32_16x16x32_bf16 v[192:195], a[160:163], a[80:83], 0\n"
    "v_mfma_f32_16x16x32_bf16 v[224:227], a[160:163], a[84:87], 0\n"
    "ds_read_b64_tr_b16 a[198:199], v25 offset:17408\n"
    "v_mfma_f32_16x16x32_bf16 v[228:231], a[164:167], a[84:87], 0\n"
    "buffer_load_dwordx4 v59, s[12:15], s77 offen offset:3072  lds\n"
    "v_mfma_f32_16x16x32_bf16 v[196:199], a[164:167], a[80:83], 0\n"
    "s_add_u32 s77, s77, 0x200000\n"
    "v_mfma_f32_16x16x32_bf16 a[0:3], a[160:163], a[88:91], 0\n"
    "ds_read_b64_tr_b16 a[200:201], v26 offset:16384\n"
    "s_add_u32 m0, s71, 81920\n"
    "v_mfma_f32_16x16x32_bf16 a[32:35], a[160:163], a[92:95], 0\n"
    "buffer_load_dwordx4 v56, s[12:15], s77 offen offset:0  lds\n"
    "v_mfma_f32_16x16x32_bf16 a[36:39], a[164:167], a[92:95], 0\n"
    "v_mfma_f32_16x16x32_bf16 a[4:7], a[164:167], a[88:91], 0\n"
    "ds_read_b64_tr_b16 a[202:203], v26 offset:17408\n"
    "v_mfma_f32_16x16x32_bf16 a[8:11], a[168:171], a[88:91], 0\n"
    "buffer_load_dwordx4 v57, s[12:15], s77 offen offset:1024  lds\n"
    "v_mfma_f32_16x16x32_bf16 a[40:43], a[168:171], a[92:95], 0\n"
    "v_mfma_f32_16x16x32_bf16 a[44:47], a[172:175], a[92:95], 0\n"
    "ds_read_b64_tr_b16 a[204:205], v27 offset:16384\n"
    "v_mfma_f32_16x16x32_bf16 a[12:15], a[172:175], a[88:91], 0\n"
    "buffer_load_dwordx4 v58, s[12:15], s77 offen offset:2048  lds\n"
    "v_mfma_f32_16x16x32_bf16 v[200:203], a[168:171], a[80:83], 0\n"
    "v_mfma_f32_16x16x32_bf16 v[232:235], a[168:171], a[84:87], 0\n"
    "ds_read_b64_tr_b16 a[206:207], v27 offset:17408\n"
    "v_mfma_f32_16x16x32_bf16 v[236:239], a[172:175], a[84:87], 0\n"
    "buffer_load_dwordx4 v59, s[12:15], s77 offen offset:3072  lds\n"
    "v_mfma_f32_16x16x32_bf16 v[204:207], a[172:175], a[80:83], 0\n"
    "v_mfma_f32_16x16x32_bf16 v[136:139], a[168:171], a[72:75], 0\n"
    "ds_read_b64_tr_b16 a[208:209], v28 offset:16384\n"
    "s_add_u32 m0, s70, 0\n"
    "v_mfma_f32_16x16x32_bf16 v[168:171], a[168:171], a[76:79], 0\n"
    "buffer_load_dwordx4 v48, s[84:87], s72 offen offset:0 sc0 sc1 lds\n"
    "v_mfma_f32_16x16x32_bf16 v[172:175], a[172:175], a[76:79], 0\n"
    "v_mfma_f32_16x16x32_bf16 v[140:143], a[172:175], a[72:75], 0\n"
    "ds_read_b64_tr_b16 a[210:211], v28 offset:17408\n"
    "v_mfma_f32_16x16x32_bf16 v[72:75], a[168:171], a[64:67], 0\n"
    "buffer_load_dwordx4 v49, s[84:87], s72 offen offset:1024 sc0 sc1 lds\n"
    "v_mfma_f32_16x16x32_bf16 v[104:107], a[168:171], a[68:71], 0\n"
    "v_mfma_f32_16x16x32_bf16 v[108:111], a[172:175], a[68:71], 0\n"
    "ds_read_b64_tr_b16 a[212:213], v29 offset:16384\n"
    "v_mfma_f32_16x16x32_bf16 v[76:79], a[172:175], a[64:67], 0\n"
    "buffer_load_dwordx4 v50, s[84:87], s72 offen offset:2048 sc0 sc1 lds\n"
    "v_mfma_f32_16x16x32_bf16 v[80:83], a[176:179], a[64:67], 0\n"
    "v_mfma_f32_16x16x32_bf16 v[112:115], a[176:179], a[68:71], 0\n"
    "ds_read_b64_tr_b16 a[214:215], v29 offset:17408\n"
    "v_mfma_f32_16x16x32_bf16 v[116:119], a[180:183], a[68:71], 0\n"
    "buffer_load_dwordx4 v51, s[84:87], s72 offen offset:3072 sc0 sc1 lds\n"
    "v_mfma_f32_16x16x32_bf16 v[84:87], a[180:183], a[64:67], 0\n"
    "v_mfma_f32_16x16x32_bf16 v[144:147], a[176:179], a[72:75], 0\n"
    "ds_read_b64_tr_b16 a[216:217], v30 offset:16384\n"
    "s_add_u32 m0, s70, 4096\n"
    "v_mfma_f32_16x16x32_bf16 v[176:179], a[176:179], a[76:79], 0\n"
    "buffer_load_dwordx4 v52, s[84:87], s72 offen offset:0 sc0 sc1 lds\n"
    "v_mfma_f32_16x16x32_bf16 v[180:183], a[180:183], a[76:79], 0\n"
    "v_mfma_f32_16x16x32_bf16 v[148:151], a[180:183], a[72:75], 0\n"
    "ds_read_b64_tr_b16 a[218:219], v30 offset:17408\n"
    "v_mfma_f32_16x16x32_bf16 v[208:211], a[176:179], a[80:83], 0\n"
    "buffer_load_dwordx4 v53, s[84:87], s72 offen offset:1024 sc0 sc1 lds\n"
    "v_mfma_f32_16x16x32_bf16 v[240:243], a[176:179], a[84:87], 0\n"
    "v_mfma_f32_16x16x32_bf16 v[244:247], a[180:183], a[84:87], 0\n"
    "ds_read_b64_tr_b16 a[220:221], v31 offset:16384\n"
    "v_mfma_f32_16x16x32_bf16 v[212:215], a[180:183], a[80:83], 0\n"
    "buffer_load_dwordx4 v54, s[84:87], s72 offen offset:2048 sc0 sc1 lds\n"
    "v_mfma_f32_16x16x32_bf16 a[16:19], a[176:179], a[88:91], 0\n"
    "v_mfma_f32_16x16x32_bf16 a[48:51], a[176:179], a[92:95], 0\n"
    "ds_read_b64_tr_b16 a[222:223], v31 offset:17408\n"
    "v_mfma_f32_16x16x32_bf16 a[52:55], a[180:183], a[92:95], 0\n"
    "buffer_load_dwordx4 v55, s[84:87], s72 offen offset:3072 sc0 sc1 lds\n"
    "v_mfma_f32_16x16x32_bf16 a[20:23], a[180:183], a[88:91], 0\n"
    "v_mfma_f32_16x16x32_bf16 a[24:27], a[184:187], a[88:91], 0\n"
    "v_mfma_f32_16x16x32_bf16 a[56:59], a[184:187], a[92:95], 0\n"
    "v_mfma_f32_16x16x32_bf16 a[60:63], a[188:191], a[92:95], 0\n"
    "s_add_u32 s75, s75, 1\n"
    "v_mfma_f32_16x16x32_bf16 a[28:31], a[188:191], a[88:91], 0\n"
    "v_mfma_f32_16x16x32_bf16 v[216:219], a[184:187], a[80:83], 0\n"
    "v_mfma_f32_16x16x32_bf16 v[248:251], a[184:187], a[84:87], 0\n"
    "v_mfma_f32_16x16x32_bf16 v[252:255], a[188:191], a[84:87], 0\n"
    "v_mfma_f32_16x16x32_bf16 v[220:223], a[188:191], a[80:83], 0\n"
    "v_mfma_f32_16x16x32_bf16 v[152:155], a[184:187], a[72:75], 0\n"
    "v_mfma_f32_16x16x32_bf16 v[184:187], a[184:187], a[76:79], 0\n"
    "v_mfma_f32_16x16x32_bf16 v[188:191], a[188:191], a[76:79], 0\n"
    "v_mfma_f32_16x16x32_bf16 v[156:159], a[188:191], a[72:75], 0\n"
    "v_mfma_f32_16x16x32_bf16 v[88:91], a[184:187], a[64:67], 0\n"
    "v_mfma_f32_16x16x32_bf16 v[120:123], a[184:187], a[68:71], 0\n"
    "v_mfma_f32_16x16x32_bf16 v[124:127], a[188:191], a[68:71], 0\n"
    "v_mfma_f32_16x16x32_bf16 v[92:95], a[188:191], a[64:67], 0\n"
    "s_waitcnt vmcnt(17) lgkmcnt(0)\n"
    "s_barrier\n"
    "v_mfma_f32_16x16x32_bf16 v[64:67], a[192:195], a[96:99], v[64:67]\n"
    "ds_read_b64_tr_b16 a[160:161], v24 offset:32768\n"
    "v_mfma_f32_16x16x32_bf16 v[96:99], a[192:195], a[100:103], v[96:99]\n"
    "ds_read_b128 a[64:67], v32 offset:32768\n"
    "v_mfma_f32_16x16x32_bf16 v[100:103], a[196:199], a[100:103], v[100:103]\n"
    "v_mfma_f32_16x16x32_bf16 v[68:71], a[196:199], a[96:99], v[68:71]\n"
    "ds_read_b64_tr_b16 a[162:163], v24 offset:33792\n"
    "v_mfma_f32_16x16x32_bf16 v[128:131], a[192:195], a[104:107], v[128:131]\n"
    "ds_read_b128 a[68:71], v32 offset:34816\n"
    "s_add_u32 m0, s71, 0\n"
    "v_mfma_f32_16x16x32_bf16 v[160:163], a[192:195], a[108:111], v[160:163]\n"
    "buffer_load_dwordx4 v56, s[12:15], s73 offen offset:0  lds\n"
    "v_mfma_f32_16x16x32_bf16 v[164:167], a[196:199], a[108:111], v[164:167]\n"
    "ds_read_b64_tr_b16 a[164:165], v25 offset:32768\n"
    "v_mfma_f32_16x16x32_bf16 v[132:135], a[196:199], a[104:107], v[132:135]\n"
    "ds_read_b128 a[72:75], v32 offset:36864\n"
    "v_mfma_f32_16x16x32_bf16 v[192:195], a[192:195], a[112:115], v[192:195]\n"
    "v_mfma_f32_16x16x32_bf16 v[224:227], a[192:195], a[116:119], v[224:227]\n"
    "ds_read_b64_tr_b16 a[166:167], v25 offset:33792\n"
    "v_mfma_f32_16x16x32_bf16 v[228:231], a[196:199], a[116:119], v[228:231]\n"
    "ds_read_b128 a[76:79], v32 offset:38912\n"
    "v_mfma_f32_16x16x32_bf16 v[196:199], a[196:199], a[112:115], v[196:199]\n"
    "buffer_load_dwordx4 v57, s[12:15], s73 offen offset:1024  lds\n"
    "v_mfma_f32_16x16x32_bf16 a[0:3], a[192:195], a[120:123], a[0:3]\n"
    "ds_read_b64_tr_b16 a[168:169], v26 offset:32768\n"
    "v_mfma_f32_16x16x32_bf16 a[32:35], a[192:195], a[124:127], a[32:35]\n"
    "ds_read_b128 a[80:83], v32 offset:40960\n"
    "v_mfma_f32_16x16x32_bf16 a[36:39], a[196:199], a[124:127], a[36:39]\n"
    "v_mfma_f32_16x16x32_bf16 a[4:7], a[196:199], a[120:123], a[4:7]\n"
    "ds_read_b64_tr_b16 a[170:171], v26 offset:33792\n"
    "v_mfma_f32_16x16x32_bf16 a[8:11], a[200:203], a[120:123], a[8:11]\n"
    "ds_read_b128 a[84:87], v32 offset:43008\n"
    "v_mfma_f32_16x16x32_bf16 a[40:43], a[200:203], a[124:127], a[40:43]\n"
    "buffer_load_dwordx4 v58, s[12:15], s73 offen offset:2048  lds\n"
    "v_mfma_f32_16x16x32_bf16 a[44:47], a[204:207], a[124:127], a[44:47]\n"
    "ds_read_b64_tr_b16 a[172:173], v27 offset:32768\n"
    "v_mfma_f32_16x16x32_bf16 a[12:15], a[204:207], a[120:123], a[12:15]\n"
    "ds_read_b128 a[88:91], v32 offset:45056\n"
    "v_mfma_f32_16x16x32_bf16 v[200:203], a[200:203], a[112:115], v[200:203]\n"
    "v_mfma_f32_16x16x32_bf16 v[232:235], a[200:203], a[116:119], v[232:235]\n"
    "ds_read_b64_tr_b16 a[174:175], v27 offset:33792\n"
    "v_mfma_f32_16x16x32_bf16 v[236:239], a[204:207], a[116:119], v[236:239]\n"
    "ds_read_b128 a[92:95], v32 offset:47104\n"
    "v_mfma_f32_16x16x32_bf16 v[204:207], a[204:207], a[112:115], v[204:207]\n"
    "buffer_load_dwordx4 v59, s[12:15], s73 offen offset:3072  lds\n"
    "v_mfma_f32_16x16x32_bf16 v[136:139], a[200:203], a[104:107], v[136:139]\n"
    "ds_read_b64_tr_b16 a[176:177], v28 offset:32768\n"
    "s_add_u32 s73, s73, 0x200000\n"
    "v_mfma_f32_16x16x32_bf16 v[168:171], a[200:203], a[108:111], v[168:171]\n"
    "ds_read_b128 a[128:131], v33 offset:32768\n"
    "v_mfma_f32_16x16x32_bf16 v[172:175], a[204:207], a[108:111], v[172:175]\n"
    "v_mfma_f32_16x16x32_bf16 v[140:143], a[204:207], a[104:107], v[140:143]\n"
    "ds_read_b64_tr_b16 a[178:179], v28 offset:33792\n"
    "v_mfma_f32_16x16x32_bf16 v[72:75], a[200:203], a[96:99], v[72:75]\n"
    "ds_read_b128 a[132:135], v33 offset:34816\n"
    "s_add_u32 m0, s71, 16384\n"
    "v_mfma_f32_16x16x32_bf16 v[104:107], a[200:203], a[100:103], v[104:107]\n"
    "buffer_load_dwordx4 v56, s[12:15], s73 offen offset:0  lds\n"
    "v_mfma_f32_16x16x32_bf16 v[108:111], a[204:207], a[100:103], v[108:111]\n"
    "ds_read_b64_tr_b16 a[180:181], v29 offset:32768\n"
    "v_mfma_f32_16x16x32_bf16 v[76:79], a[204:207], a[96:99], v[76:79]\n"
    "ds_read_b128 a[136:139], v33 offset:36864\n"
    "v_mfma_f32_16x16x32_bf16 v[80:83], a[208:211], a[96:99], v[80:83]\n"
    "v_mfma_f32_16x16x32_bf16 v[112:115], a[208:211], a[100:103], v[112:115]\n"
    "ds_read_b64_tr_b16 a[182:183], v29 offset:33792\n"
    "v_mfma_f32_16x16x32_bf16 v[116:119], a[212:215], a[100:103], v[116:119]\n"
    "ds_read_b128 a[140:143], v33 offset:38912\n"
    "v_mfma_f32_16x16x32_bf16 v[84:87], a[212:215], a[96:99], v[84:87]\n"
    "buffer_load_dwordx4 v57, s[12:15], s73 offen offset:1024  lds\n"
    "v_mfma_f32_16x16x32_bf16 v[144:147], a[208:211], a[104:107], v[144:147]\n"
    "ds_read_b64_tr_b16 a[184:185], v30 offset:32768\n"
    "v_mfma_f32_16x16x32_bf16 v[176:179], a[208:211], a[108:111], v[176:179]\n"
    "ds_read_b128 a[144:147], v33 offset:40960\n"
    "v_mfma_f32_16x16x32_bf16 v[180:183], a[212:215], a[108:111], v[180:183]\n"
    "v_mfma_f32_16x16x32_bf16 v[148:151], a[212:215], a[104:107], v[148:151]\n"
    "ds_read_b64_tr_b16 a[186:187], v30 offset:33792\n"
    "v_mfma_f32_16x16x32_bf16 v[208:211], a[208:211], a[112:115], v[208:211]\n"
    "ds_read_b128 a[148:151], v33 offset:43008\n"
    "v_mfma_f32_16x16x32_bf16 v[240:243], a[208:211], a[116:119], v[240:243]\n"
    "buffer_load_dwordx4 v58, s[12:15], s73 offen offset:2048  lds\n"
    "v_mfma_f32_16x16x32_bf16 v[244:247], a[212:215], a[116:119], v[244:247]\n"
    "ds_read_b64_tr_b16 a[188:189], v31 offset:32768\n"
    "v_mfma_f32_16x16x32_bf16 v[212:215], a[212:215], a[112:115], v[212:215]\n"
    "ds_read_b128 a[152:155], v33 offset:45056\n"
    "v_mfma_f32_16x16x32_bf16 a[16:19], a[208:211], a[120:123], a[16:19]\n"
    "v_mfma_f32_16x16x32_bf16 a[48:51], a[208:211], a[124:127], a[48:51]\n"
    "ds_read_b64_tr_b16 a[190:191], v31 offset:33792\n"
    "v_mfma_f32_16x16x32_bf16 a[52:55], a[212:215], a[124:127], a[52:55]\n"
    "ds_read_b128 a[156:159], v33 offset:47104\n"
    "v_mfma_f32_16x16x32_bf16 a[20:23], a[212:215], a[120:123], a[20:23]\n"
    "buffer_load_dwordx4 v59, s[12:15], s73 offen offset:3072  lds\n"
    "v_mfma_f32_16x16x32_bf16 a[24:27], a[216:219], a[120:123], a[24:27]\n"
    "s_add_u32 s76, s76, 1\n"
    "v_mfma_f32_16x16x32_bf16 a[56:59], a[216:219], a[124:127], a[56:59]\n"
    "v_readlane_b32 s72, v40, s75\n"
    "v_mfma_f32_16x16x32_bf16 a[60:63], a[220:223], a[124:127], a[60:63]\n"
    "v_mfma_f32_16x16x32_bf16 a[28:31], a[220:223], a[120:123], a[28:31]\n"
    "v_readlane_b32 s73, v41, s76\n"
    "v_mfma_f32_16x16x32_bf16 v[216:219], a[216:219], a[112:115], v[216:219]\n"
    "s_cmp_eq_u32 s75, 8\n"
    "v_mfma_f32_16x16x32_bf16 v[248:251], a[216:219], a[116:119], v[248:251]\n"
    "s_cselect_b32 s84, s88, s84\n"
    "s_cselect_b32 s85, s89, s85\n"
    "v_mfma_f32_16x16x32_bf16 v[252:255], a[220:223], a[116:119], v[252:255]\n"
    "v_mfma_f32_16x16x32_bf16 v[220:223], a[220:223], a[112:115], v[220:223]\n"
    "s_cmp_ge_u32 s75, 64\n"
    "s_cselect_b32 s72, 1073741824, s72\n"
    "v_mfma_f32_16x16x32_bf16 v[152:155], a[216:219], a[104:107], v[152:155]\n"
    "v_mfma_f32_16x16x32_bf16 v[184:187], a[216:219], a[108:111], v[184:187]\n"
    "s_cmp_ge_u32 s76, 64\n"
    "s_cselect_b32 s73, 1073741824, s73\n"
    "v_mfma_f32_16x16x32_bf16 v[188:191], a[220:223], a[108:111], v[188:191]\n"
    "v_mfma_f32_16x16x32_bf16 v[156:159], a[220:223], a[104:107], v[156:159]\n"
    "v_mfma_f32_16x16x32_bf16 v[88:91], a[216:219], a[96:99], v[88:91]\n"
    "v_mfma_f32_16x16x32_bf16 v[120:123], a[216:219], a[100:103], v[120:123]\n"
    "s_add_u32 s79, s79, 1\n"
    "v_mfma_f32_16x16x32_bf16 v[124:127], a[220:223], a[100:103], v[124:127]\n"
    "v_mfma_f32_16x16x32_bf16 v[92:95], a[220:223], a[96:99], v[92:95]\n"
    "s_waitcnt lgkmcnt(0)\n"
    "s_barrier\n"
    "v_mfma_f32_16x16x32_bf16 v[64:67], a[160:163], a[64:67], v[64:67]\n"
    "ds_read_b64_tr_b16 a[192:193], v24 offset:49152\n"
    "v_mfma_f32_16x16x32_bf16 v[96:99], a[160:163], a[68:71], v[96:99]\n"
    "s_add_u32 m0, s70, 32768\n"
    "v_mfma_f32_16x16x32_bf16 v[100:103], a[164:167], a[68:71], v[100:103]\n"
    "buffer_load_dwordx4 v48, s[84:87], s72 offen offset:0 sc0 sc1 lds\n"
    "v_mfma_f32_16x16x32_bf16 v[68:71], a[164:167], a[64:67], v[68:71]\n"
    "ds_read_b64_tr_b16 a[194:195], v24 offset:50176\n"
    "v_mfma_f32_16x16x32_bf16 v[128:131], a[160:163], a[72:75], v[128:131]\n"
    "v_mfma_f32_16x16x32_bf16 v[160:163], a[160:163], a[76:79], v[160:163]\n"
    "buffer_load_dwordx4 v49, s[84:87], s72 offen offset:1024 sc0 sc1 lds\n"
    "v_mfma_f32_16x16x32_bf16 v[164:167], a[164:167], a[76:79], v[164:167]\n"
    "ds_read_b64_tr_b16 a[196:197], v25 offset:49152\n"
    "v_mfma_f32_16x16x32_bf16 v[132:135], a[164:167], a[72:75], v[132:135]\n"
    "v_mfma_f32_16x16x32_bf16 v[192:195], a[160:163], a[80:83], v[192:195]\n"
    "buffer_load_dwordx4 v50, s[84:87], s72 offen offset:2048 sc0 sc1 lds\n"
    "v_mfma_f32_16x16x32_bf16 v[224:227], a[160:163], a[84:87], v[224:227]\n"
    "ds_read_b64_tr_b16 a[198:199], v25 offset:50176\n"
    "v_mfma_f32_16x16x32_bf16 v[228:231], a[164:167], a[84:87], v[228:231]\n"
    "v_mfma_f32_16x16x32_bf16 v[196:199], a[164:167], a[80:83], v[196:199]\n"
    "buffer_load_dwordx4 v51, s[84:87], s72 offen offset:3072 sc0 sc1 lds\n"
    "v_mfma_f32_16x16x32_bf16 a[0:3], a[160:163], a[88:91], a[0:3]\n"
    "ds_read_b64_tr_b16 a[200:201], v26 offset:49152\n"
    "v_mfma_f32_16x16x32_bf16 a[32:35], a[160:163], a[92:95], a[32:35]\n"
    "s_add_u32 m0, s70, 36864\n"
    "v_mfma_f32_16x16x32_bf16 a[36:39], a[164:167], a[92:95], a[36:39]\n"
    "buffer_load_dwordx4 v52, s[84:87], s72 offen offset:0 sc0 sc1 lds\n"
    "v_mfma_f32_16x16x32_bf16 a[4:7], a[164:167], a[88:91], a[4:7]\n"
    "ds_read_b64_tr_b16 a[202:203], v26 offset:50176\n"
    "v_mfma_f32_16x16x32_bf16 a[8:11], a[168:171], a[88:91], a[8:11]\n"
    "v_mfma_f32_16x16x32_bf16 a[40:43], a[168:171], a[92:95], a[40:43]\n"
    "buffer_load_dwordx4 v53, s[84:87], s72 offen offset:1024 sc0 sc1 lds\n"
    "v_mfma_f32_16x16x32_bf16 a[44:47], a[172:175], a[92:95], a[44:47]\n"
    "ds_read_b64_tr_b16 a[204:205], v27 offset:49152\n"
    "v_mfma_f32_16x16x32_bf16 a[12:15], a[172:175], a[88:91], a[12:15]\n"
    "v_mfma_f32_16x16x32_bf16 v[200:203], a[168:171], a[80:83], v[200:203]\n"
    "buffer_load_dwordx4 v54, s[84:87], s72 offen offset:2048 sc0 sc1 lds\n"
    "v_mfma_f32_16x16x32_bf16 v[232:235], a[168:171], a[84:87], v[232:235]\n"
    "ds_read_b64_tr_b16 a[206:207], v27 offset:50176\n"
    "v_mfma_f32_16x16x32_bf16 v[236:239], a[172:175], a[84:87], v[236:239]\n"
    "v_mfma_f32_16x16x32_bf16 v[204:207], a[172:175], a[80:83], v[204:207]\n"
    "buffer_load_dwordx4 v55, s[84:87], s72 offen offset:3072 sc0 sc1 lds\n"
    "v_mfma_f32_16x16x32_bf16 v[136:139], a[168:171], a[72:75], v[136:139]\n"
    "ds_read_b64_tr_b16 a[208:209], v28 offset:49152\n"
    "v_mfma_f32_16x16x32_bf16 v[168:171], a[168:171], a[76:79], v[168:171]\n"
    "v_mfma_f32_16x16x32_bf16 v[172:175], a[172:175], a[76:79], v[172:175]\n"
    "v_mfma_f32_16x16x32_bf16 v[140:143], a[172:175], a[72:75], v[140:143]\n"
    "ds_read_b64_tr_b16 a[210:211], v28 offset:50176\n"
    "v_mfma_f32_16x16x32_bf16 v[72:75], a[168:171], a[64:67], v[72:75]\n"
    "s_add_u32 m0, s71, 32768\n"
    "v_mfma_f32_16x16x32_bf16 v[104:107], a[168:171], a[68:71], v[104:107]\n"
    "buffer_load_dwordx4 v56, s[12:15], s73 offen offset:0  lds\n"
    "v_mfma_f32_16x16x32_bf16 v[108:111], a[172:175], a[68:71], v[108:111]\n"
    "ds_read_b64_tr_b16 a[212:213], v29 offset:49152\n"
    "v_mfma_f32_16x16x32_bf16 v[76:79], a[172:175], a[64:67], v[76:79]\n"
    "v_mfma_f32_16x16x32_bf16 v[80:83], a[176:179], a[64:67], v[80:83]\n"
    "v_mfma_f32_16x16x32_bf16 v[112:115], a[176:179], a[68:71], v[112:115]\n"
    "ds_read_b64_tr_b16 a[214:215], v29 offset:50176\n"
    "v_mfma_f32_16x16x32_bf16 v[116:119], a[180:183], a[68:71], v[116:119]\n"
    "v_mfma_f32_16x16x32_bf16 v[84:87], a[180:183], a[64:67], v[84:87]\n"
    "buffer_load_dwordx4 v57, s[12:15], s73 offen offset:1024  lds\n"
    "v_mfma_f32_16x16x32_bf16 v[144:147], a[176:179], a[72:75], v[144:147]\n"
    "ds_read_b64_tr_b16 a[216:217], v30 offset:49152\n"
    "v_mfma_f32_16x16x32_bf16 v[176:179], a[176:179], a[76:79], v[176:179]\n"
    "v_mfma_f32_16x16x32_bf16 v[180:183], a[180:183], a[76:79], v[180:183]\n"
    "v_mfma_f32_16x16x32_bf16 v[148:151], a[180:183], a[72:75], v[148:151]\n"
    "ds_read_b64_tr_b16 a[218:219], v30 offset:50176\n"
    "v_mfma_f32_16x16x32_bf16 v[208:211], a[176:179], a[80:83], v[208:211]\n"
    "v_mfma_f32_16x16x32_bf16 v[240:243], a[176:179], a[84:87], v[240:243]\n"
    "buffer_load_dwordx4 v58, s[12:15], s73 offen offset:2048  lds\n"
    "v_mfma_f32_16x16x32_bf16 v[244:247], a[180:183], a[84:87], v[244:247]\n"
    "ds_read_b64_tr_b16 a[220:221], v31 offset:49152\n"
    "v_mfma_f32_16x16x32_bf16 v[212:215], a[180:183], a[80:83], v[212:215]\n"
    "v_mfma_f32_16x16x32_bf16 a[16:19], a[176:179], a[88:91], a[16:19]\n"
    "v_mfma_f32_16x16x32_bf16 a[48:51], a[176:179], a[92:95], a[48:51]\n"
    "ds_read_b64_tr_b16 a[222:223], v31 offset:50176\n"
    "v_mfma_f32_16x16x32_bf16 a[52:55], a[180:183], a[92:95], a[52:55]\n"
    "v_mfma_f32_16x16x32_bf16 a[20:23], a[180:183], a[88:91], a[20:23]\n"
    "buffer_load_dwordx4 v59, s[12:15], s73 offen offset:3072  lds\n"
    "v_mfma_f32_16x16x32_bf16 a[24:27], a[184:187], a[88:91], a[24:27]\n"
    "v_mfma_f32_16x16x32_bf16 a[56:59], a[184:187], a[92:95], a[56:59]\n"
    "s_add_u32 s73, s73, 0x200000\n"
    "v_mfma_f32_16x16x32_bf16 a[60:63], a[188:191], a[92:95], a[60:63]\n"
    "s_add_u32 s75, s75, 1\n"
    "v_mfma_f32_16x16x32_bf16 a[28:31], a[188:191], a[88:91], a[28:31]\n"
    "v_mfma_f32_16x16x32_bf16 v[216:219], a[184:187], a[80:83], v[216:219]\n"
    "v_mfma_f32_16x16x32_bf16 v[248:251], a[184:187], a[84:87], v[248:251]\n"
    "v_mfma_f32_16x16x32_bf16 v[252:255], a[188:191], a[84:87], v[252:255]\n"
    "v_mfma_f32_16x16x32_bf16 v[220:223], a[188:191], a[80:83], v[220:223]\n"
    "v_mfma_f32_16x16x32_bf16 v[152:155], a[184:187], a[72:75], v[152:155]\n"
    "v_mfma_f32_16x16x32_bf16 v[184:187], a[184:187], a[76:79], v[184:187]\n"
    "v_mfma_f32_16x16x32_bf16 v[188:191], a[188:191], a[76:79], v[188:191]\n"
    "v_mfma_f32_16x16x32_bf16 v[156:159], a[188:191], a[72:75], v[156:159]\n"
    "v_mfma_f32_16x16x32_bf16 v[88:91], a[184:187], a[64:67], v[88:91]\n"
    "v_mfma_f32_16x16x32_bf16 v[120:123], a[184:187], a[68:71], v[120:123]\n"
    "v_mfma_f32_16x16x32_bf16 v[124:127], a[188:191], a[68:71], v[124:127]\n"
    "v_mfma_f32_16x16x32_bf16 v[92:95], a[188:191], a[64:67], v[92:95]\n"
    "s_waitcnt vmcnt(20) lgkmcnt(0)\n"
    "s_barrier\n"
    "v_mfma_f32_16x16x32_bf16 v[64:67], a[192:195], a[128:131], v[64:67]\n"
    "ds_read_b64_tr_b16 a[160:161], v16 offset:0\n"
    "v_mfma_f32_16x16x32_bf16 v[96:99], a[192:195], a[132:135], v[96:99]\n"
    "ds_read_b128 a[64:67], v32 offset:0\n"
    "v_mfma_f32_16x16x32_bf16 v[100:103], a[196:199], a[132:135], v[100:103]\n"
    "s_bitcmp1_b64 s[64:65], s79\n"
    "s_cbranch_scc1 Lfl1%=\n"
    "Lfl1r%=:\n"
    "v_mfma_f32_16x16x32_bf16 v[68:71], a[196:199], a[128:131], v[68:71]\n"
    "ds_read_b64_tr_b16 a[162:163], v16 offset:1024\n"
    "v_mfma_f32_16x16x32_bf16 v[128:131], a[192:195], a[136:139], v[128:131]\n"
    "ds_read_b128 a[68:71], v32 offset:2048\n"
    "s_add_u32 m0, s71, 49152\n"
    "v_mfma_f32_16x16x32_bf16 v[160:163], a[192:195], a[140:143], v[160:163]\n"
    "buffer_load_dwordx4 v56, s[12:15], s73 offen offset:0  lds\n"
    "v_mfma_f32_16x16x32_bf16 v[164:167], a[196:199], a[140:143], v[164:167]\n"
    "ds_read_b64_tr_b16 a[164:165], v17 offset:0\n"
    "v_mfma_f32_16x16x32_bf16 v[132:135], a[196:199], a[136:139], v[132:135]\n"
    "ds_read_b128 a[72:75], v32 offset:4096\n"
    "v_mfma_f32_16x16x32_bf16 v[192:195], a[192:195], a[144:147], v[192:195]\n"
    "v_mfma_f32_16x16x32_bf16 v[224:227], a[192:195], a[148:151], v[224:227]\n"
    "ds_read_b64_tr_b16 a[166:167], v17 offset:1024\n"
    "v_mfma_f32_16x16x32_bf16 v[228:231], a[196:199], a[148:151], v[228:231]\n"
    "ds_read_b128 a[76:79], v32 offset:6144\n"
    "v_mfma_f32_16x16x32_bf16 v[196:199], a[196:199], a[144:147], v[196:199]\n"
    "v_mfma_f32_16x16x32_bf16 a[0:3], a[192:195], a[152:155], a[0:3]\n"
    "ds_read_b64_tr_b16 a[168:169], v18 offset:0\n"
    "v_mfma_f32_16x16x32_bf16 a[32:35], a[192:195], a[156:159], a[32:35]\n"
    "ds_read_b128 a[80:83], v32 offset:8192\n"
    "v_mfma_f32_16x16x32_bf16 a[36:39], a[196:199], a[156:159], a[36:39]\n"
    "v_mfma_f32_16x16x32_bf16 a[4:7], a[196:199], a[152:155], a[4:7]\n"
    "ds_read_b64_tr_b16 a[170:171], v18 offset:1024\n"
    "v_mfma_f32_16x16x32_bf16 a[8:11], a[200:203], a[152:155], a[8:11]\n"
    "ds_read_b128 a[84:87], v32 offset:10240\n"
    "v_mfma_f32_16x16x32_bf16 a[40:43], a[200:203], a[156:159], a[40:43]\n"
    "buffer_load_dwordx4 v57, s[12:15], s73 offen offset:1024  lds\n"
    "v_mfma_f32_16x16x32_bf16 a[44:47], a[204:207], a[156:159], a[44:47]\n"
    "ds_read_b64_tr_b16 a[172:173], v19 offset:0\n"
    "v_mfma_f32_16x16x32_bf16 a[12:15], a[204:207], a[152:155], a[12:15]\n"
    "ds_read_b128 a[88:91], v32 offset:12288\n"
    "v_mfma_f32_16x16x32_bf16 v[200:203], a[200:203], a[144:147], v[200:203]\n"
    "v_mfma_f32_16x16x32_bf16 v[232:235], a[200:203], a[148:151], v[232:235]\n"
    "ds_read_b64_tr_b16 a[174:175], v19 offset:1024\n"
    "v_mfma_f32_16x16x32_bf16 v[236:239], a[204:207], a[148:151], v[236:239]\n"
    "ds_read_b128 a[92:95], v32 offset:14336\n"
    "v_mfma_f32_16x16x32_bf16 v[204:207], a[204:207], a[144:147], v[204:207]\n"
    "v_mfma_f32_16x16x32_bf16 v[136:139], a[200:203], a[136:139], v[136:139]\n"
    "ds_read_b64_tr_b16 a[176:177], v20 offset:0\n"
    "v_mfma_f32_16x16x32_bf16 v[168:171], a[200:203], a[140:143], v[168:171]\n"
    "ds_read_b128 a[96:99], v33 offset:0\n"
    "v_mfma_f32_16x16x32_bf16 v[172:175], a[204:207], a[140:143], v[172:175]\n"
    "v_mfma_f32_16x16x32_bf16 v[140:143], a[204:207], a[136:139], v[140:143]\n"
    "ds_read_b64_tr_b16 a[178:179], v20 offset:1024\n"
    "v_mfma_f32_16x16x32_bf16 v[72:75], a[200:203], a[128:131], v[72:75]\n"
    "ds_read_b128 a[100:103], v33 offset:2048\n"
    "v_mfma_f32_16x16x32_bf16 v[104:107], a[200:203], a[132:135], v[104:107]\n"
    "buffer_load_dwordx4 v58, s[12:15], s73 offen offset:2048  lds\n"
    "v_mfma_f32_16x16x32_bf16 v[108:111], a[204:207], a[132:135], v[108:111]\n"
    "ds_read_b64_tr_b16 a[180:181], v21 offset:0\n"
    "v_mfma_f32_16x16x32_bf16 v[76:79], a[204:207], a[128:131], v[76:79]\n"
    "ds_read_b128 a[104:107], v33 offset:4096\n"
    "v_mfma_f32_16x16x32_bf16 v[80:83], a[208:211], a[128:131], v[80:83]\n"
    "v_mfma_f32_16x16x32_bf16 v[112:115], a[208:211], a[132:135], v[112:115]\n"
    "ds_read_b64_tr_b16 a[182:183], v21 offset:1024\n"
    "v_mfma_f32_16x16x32_bf16 v[116:119], a[212:215], a[132:135], v[116:119]\n"
    "ds_read_b128 a[108:111], v33 offset:6144\n"
    "v_mfma_f32_16x16x32_bf16 v[84:87], a[212:215], a[128:131], v[84:87]\n"
    "v_mfma_f32_16x16x32_bf16 v[144:147], a[208:211], a[136:139], v[144:147]\n"
    "ds_read_b64_tr_b16 a[184:185], v22 offset:0\n"
    "v_mfma_f32_16x16x32_bf16 v[176:179], a[208:211], a[140:143], v[176:179]\n"
    "ds_read_b128 a[112:115], v33 offset:8192\n"
    "v_mfma_f32_16x16x32_bf16 v[180:183], a[212:215], a[140:143], v[180:183]\n"
    "v_mfma_f32_16x16x32_bf16 v[148:151], a[212:215], a[136:139], v[148:151]\n"
    "ds_read_b64_tr_b16 a[186:187], v22 offset:1024\n"
    "v_mfma_f32_16x16x32_bf16 v[208:211], a[208:211], a[144:147], v[208:211]\n"
    "ds_read_b128 a[116:119], v33 offset:10240\n"
    "v_mfma_f32_16x16x32_bf16 v[240:243], a[208:211], a[148:151], v[240:243]\n"
    "buffer_load_dwordx4 v59, s[12:15], s73 offen offset:3072  lds\n"
    "v_mfma_f32_16x16x32_bf16 v[244:247], a[212:215], a[148:151], v[244:247]\n"
    "ds_read_b64_tr_b16 a[188:189], v23 offset:0\n"
    "v_mfma_f32_16x16x32_bf16 v[212:215], a[212:215], a[144:147], v[212:215]\n"
    "ds_read_b128 a[120:123], v33 offset:12288\n"
    "v_mfma_f32_16x16x32_bf16 a[16:19], a[208:211], a[152:155], a[16:19]\n"
    "v_mfma_f32_16x16x32_bf16 a[48:51], a[208:211], a[156:159], a[48:51]\n"
    "ds_read_b64_tr_b16 a[190:191], v23 offset:1024\n"
    "v_mfma_f32_16x16x32_bf16 a[52:55], a[212:215], a[156:159], a[52:55]\n"
    "ds_read_b128 a[124:127], v33 offset:14336\n"
    "v_mfma_f32_16x16x32_bf16 a[20:23], a[212:215], a[152:155], a[20:23]\n"
    "v_mfma_f32_16x16x32_bf16 a[24:27], a[216:219], a[152:155], a[24:27]\n"
    "s_add_u32 s76, s76, 1\n"
    "v_mfma_f32_16x16x32_bf16 a[56:59], a[216:219], a[156:159], a[56:59]\n"
    "v_readlane_b32 s72, v40, s75\n"
    "v_mfma_f32_16x16x32_bf16 a[60:63], a[220:223], a[156:159], a[60:63]\n"
    "v_mfma_f32_16x16x32_bf16 a[28:31], a[220:223], a[152:155], a[28:31]\n"
    "v_readlane_b32 s73, v41, s76\n"
    "v_mfma_f32_16x16x32_bf16 v[216:219], a[216:219], a[144:147], v[216:219]\n"
    "s_cmp_eq_u32 s75, 8\n"
    "v_mfma_f32_16x16x32_bf16 v[248:251], a[216:219], a[148:151], v[248:251]\n"
    "s_cselect_b32 s84, s88, s84\n"
    "s_cselect_b32 s85, s89, s85\n"
    "v_mfma_f32_16x16x32_bf16 v[252:255], a[220:223], a[148:151], v[252:255]\n"
    "v_mfma_f32_16x16x32_bf16 v[220:223], a[220:223], a[144:147], v[220:223]\n"
    "s_cmp_ge_u32 s75, 64\n"
    "s_cselect_b32 s72, 1073741824, s72\n"
    "v_mfma_f32_16x16x32_bf16 v[152:155], a[216:219], a[136:139], v[152:155]\n"
    "v_mfma_f32_16x16x32_bf16 v[184:187], a[216:219], a[140:143], v[184:187]\n"
    "s_cmp_ge_u32 s76, 64\n"
    "s_cselect_b32 s73, 1073741824, s73\n"
    "v_mfma_f32_16x16x32_bf16 v[188:191], a[220:223], a[140:143], v[188:191]\n"
    "v_mfma_f32_16x16x32_bf16 v[156:159], a[220:223], a[136:139], v[156:159]\n"
    "v_mfma_f32_16x16x32_bf16 v[88:91], a[216:219], a[128:131], v[88:91]\n"
    "v_mfma_f32_16x16x32_bf16 v[120:123], a[216:219], a[132:135], v[120:123]\n"
    "s_add_u32 s79, s79, 1\n"
    "v_mfma_f32_16x16x32_bf16 v[124:127], a[220:223], a[132:135], v[124:127]\n"
    "v_mfma_f32_16x16x32_bf16 v[92:95], a[220:223], a[128:131], v[92:95]\n"
    "s_waitcnt lgkmcnt(0)\n"
    "s_barrier\n"
    ".p2align 6\n"
    "L1%=:\n"
    "LT2%=:\n"
    "v_mfma_f32_16x16x32_bf16 v[64:67], a[160:163], a[64:67], v[64:67]\n"
    "s_sub_u32 s78, s4, s56\n"
    "ds_read_b64_tr_b16 a[192:193], v16 offset:16384\n"
    "v_mfma_f32_16x16x32_bf16 v[96:99], a[160:163], a[68:71], v[96:99]\n"
    "s_memtime s[56:57]\n"
    "s_add_u32 m0, s70, 0\n"
    "v_mfma_f32_16x16x32_bf16 v[100:103], a[164:167], a[68:71], v[100:103]\n"
    "buffer_load_dwordx4 v48, s[84:87], s72 offen offset:0 sc0 sc1 lds\n"
    "v_mfma_f32_16x16x32_bf16 v[68:71], a[164:167], a[64:67], v[68:71]\n"
    "ds_read_b64_tr_b16 a[194:195], v16 offset:17408\n"
    "v_mfma_f32_16x16x32_bf16 v[128:131], a[160:163], a[72:75], v[128:131]\n"
    "v_mfma_f32_16x16x32_bf16 v[160:163], a[160:163], a[76:79], v[160:163]\n"
    "buffer_load_dwordx4 v49, s[84:87], s72 offen offset:1024 sc0 sc1 lds\n"
    "v_mfma_f32_16x16x32_bf16 v[164:167], a[164:167], a[76:79], v[164:167]\n"
    "ds_read_b64_tr_b16 a[196:197], v17 offset:16384\n"
    "v_mfma_f32_16x16x32_bf16 v[132:135], a[164:167], a[72:75], v[132:135]\n"
    "v_mfma_f32_16x16x32_bf16 v[192:195], a[160:163], a[80:83], v[192:195]\n"
    "buffer_load_dwordx4 v50, s[84:87], s72 offen offset:2048 sc0 sc1 lds\n"
    "v_mfma_f32_16x16x32_bf16 v[224:227], a[160:163], a[84:87], v[224:227]\n"
    "ds_read_b64_tr_b16 a[198:199], v17 offset:17408\n"
    "v_mfma_f32_16x16x32_bf16 v[228:231], a[164:167], a[84:87], v[228:231]\n"
    "v_mfma_f32_16x16x32_bf16 v[196:199], a[164:167], a[80:83], v[196:199]\n"
    "buffer_load_dwordx4 v51, s[84:87], s72 offen offset:3072 sc0 sc1 lds\n"
    "v_mfma_f32_16x16x32_bf16 a[0:3], a[160:163], a[88:91], a[0:3]\n"
    "ds_read_b64_tr_b16 a[200:201], v18 offset:16384\n"
    "v_mfma_f32_16x16x32_bf16 a[32:35], a[160:163], a[92:95], a[32:35]\n"
    "s_add_u32 m0, s70, 4096\n"
    "v_mfma_f32_16x16x32_bf16 a[36:39], a[164:167], a[92:95], a[36:39]\n"
    "buffer_load_dwordx4 v52, s[84:87], s72 offen offset:0 sc0 sc1 lds\n"
    "v_mfma_f32_16x16x32_bf16 a[4:7], a[164:167], a[88:91], a[4:7]\n"
    "ds_read_b64_tr_b16 a[202:203], v18 offset:17408\n"
    "v_mfma_f32_16x16x32_bf16 a[8:11], a[168:171], a[88:91], a[8:11]\n"
    "v_mfma_f32_16x16x32_bf16 a[40:43], a[168:171], a[92:95], a[40:43]\n"
    "buffer_load_dwordx4 v53, s[84:87], s72 offen offset:1024 sc0 sc1 lds\n"
    "v_mfma_f32_16x16x32_bf16 a[44:47], a[172:175], a[92:95], a[44:47]\n"
    "ds_read_b64_tr_b16 a[204:205], v19 offset:16384\n"
    "v_mfma_f32_16x16x32_bf16 a[12:15], a[172:175], a[88:91], a[12:15]\n"
    "v_mfma_f32_16x16x32_bf16 v[200:203], a[168:171], a[80:83], v[200:203]\n"
    "buffer_load_dwordx4 v54, s[84:87], s72 offen offset:2048 sc0 sc1 lds\n"
    "v_mfma_f32_16x16x32_bf16 v[232:235], a[168:171], a[84:87], v[232:235]\n"
    "ds_read_b64_tr_b16 a[206:207], v19 offset:17408\n"
    "v_mfma_f32_16x16x32_bf16 v[236:239], a[172:175], a[84:87], v[236:239]\n"
    "v_mfma_f32_16x16x32_bf16 v[204:207], a[172:175], a[80:83], v[204:207]\n"
    "buffer_load_dwordx4 v55, s[84:87], s72 offen offset:3072 sc0 sc1 lds\n"
    "v_mfma_f32_16x16x32_bf16 v[136:139], a[168:171], a[72:75], v[136:139]\n"
    "ds_read_b64_tr_b16 a[208:209], v20 offset:16384\n"
    "v_mfma_f32_16x16x32_bf16 v[168:171], a[168:171], a[76:79], v[168:171]\n"
    "v_mfma_f32_16x16x32_bf16 v[172:175], a[172:175], a[76:79], v[172:175]\n"
    "v_mfma_f32_16x16x32_bf16 v[140:143], a[172:175], a[72:75], v[140:143]\n"
    "ds_read_b64_tr_b16 a[210:211], v20 offset:17408\n"
    "v_mfma_f32_16x16x32_bf16 v[72:75], a[168:171], a[64:67], v[72:75]\n"
    "s_add_u32 m0, s71, 65536\n"
    "v_mfma_f32_16x16x32_bf16 v[104:107], a[168:171], a[68:71], v[104:107]\n"
    "buffer_load_dwordx4 v56, s[12:15], s73 offen offset:0  lds\n"
    "v_mfma_f32_16x16x32_bf16 v[108:111], a[172:175], a[68:71], v[108:111]\n"
    "ds_read_b64_tr_b16 a[212:213], v21 offset:16384\n"
    "v_mfma_f32_16x16x32_bf16 v[76:79], a[172:175], a[64:67], v[76:79]\n"
    "v_mfma_f32_16x16x32_bf16 v[80:83], a[176:179], a[64:67], v[80:83]\n"
    "v_mfma_f32_16x16x32_bf16 v[112:115], a[176:179], a[68:71], v[112:115]\n"
    "ds_read_b64_tr_b16 a[214:215], v21 offset:17408\n"
    "v_mfma_f32_16x16x32_bf16 v[116:119], a[180:183], a[68:71], v[116:119]\n"
    "v_mfma_f32_16x16x32_bf16 v[84:87], a[180:183], a[64:67], v[84:87]\n"
    "buffer_load_dwordx4 v57, s[12:15], s73 offen offset:1024  lds\n"
    "v_mfma_f32_16x16x32_bf16 v[144:147], a[176:179], a[72:75], v[144:147]\n"
    "ds_read_b64_tr_b16 a[216:217], v22 offset:16384\n"
    "v_mfma_f32_16x16x32_bf16 v[176:179], a[176:179], a[76:79], v[176:179]\n"
    "v_mfma_f32_16x16x32_bf16 v[180:183], a[180:183], a[76:79], v[180:183]\n"
    "v_mfma_f32_16x16x32_bf16 v[148:151], a[180:183], a[72:75], v[148:151]\n"
    "ds_read_b64_tr_b16 a[218:219], v22 offset:17408\n"
    "v_mfma_f32_16x16x32_bf16 v[208:211], a[176:179], a[80:83], v[208:211]\n"
    "v_mfma_f32_16x16x32_bf16 v[240:243], a[176:179], a[84:87], v[240:243]\n"
    "buffer_load_dwordx4 v58, s[12:15], s73 offen offset:2048  lds\n"
    "v_mfma_f32_16x16x32_bf16 v[244:247], a[180:183], a[84:87], v[244:247]\n"
    "ds_read_b64_tr_b16 a[220:221], v23 offset:16384\n"
    "v_mfma_f32_16x16x32_bf16 v[212:215], a[180:183], a[80:83], v[212:215]\n"
    "v_mfma_f32_16x16x32_bf16 a[16:19], a[176:179], a[88:91], a[16:19]\n"
    "v_mfma_f32_16x16x32_bf16 a[48:51], a[176:179], a[92:95], a[48:51]\n"
    "ds_read_b64_tr_b16 a[222:223], v23 offset:17408\n"
    "v_mfma_f32_16x16x32_bf16 a[52:55], a[180:183], a[92:95], a[52:55]\n"
    "v_mfma_f32_16x16x32_bf16 a[20:23], a[180:183], a[88:91], a[20:23]\n"
    "buffer_load_dwordx4 v59, s[12:15], s73 offen offset:3072  lds\n"
    "v_mfma_f32_16x16x32_bf16 a[24:27], a[184:187], a[88:91], a[24:27]\n"
    "v_mfma_f32_16x16x32_bf16 a[56:59], a[184:187], a[92:95], a[56:59]\n"
    "s_add_u32 s73, s73, 0x200000\n"
    "v_mfma_f32_16x16x32_bf16 a[60:63], a[188:191], a[92:95], a[60:63]\n"
    "s_add_u32 s75, s75, 1\n"
    "v_mfma_f32_16x16x32_bf16 a[28:31], a[188:191], a[88:91], a[28:31]\n"
    "v_mfma_f32_16x16x32_bf16 v[216:219], a[184:187], a[80:83], v[216:219]\n"
    "s_cmp_gt_u32 s78, s30\n"
    "s_cbranch_scc1 Lad2%=\n"
    "Lad2r%=:\n"
    "v_mfma_f32_16x16x32_bf16 v[248:251], a[184:187], a[84:87], v[248:251]\n"
    "v_mfma_f32_16x16x32_bf16 v[252:255], a[188:191], a[84:87], v[252:255]\n"
    "v_mfma_f32_16x16x32_bf16 v[220:223], a[188:191], a[80:83], v[220:223]\n"
    "v_mfma_f32_16x16x32_bf16 v[152:155], a[184:187], a[72:75], v[152:155]\n"
    "v_mfma_f32_16x16x32_bf16 v[184:187], a[184:187], a[76:79], v[184:187]\n"
    "v_mfma_f32_16x16x32_bf16 v[188:191], a[188:191], a[76:79], v[188:191]\n"
    "v_mfma_f32_16x16x32_bf16 v[156:159], a[188:191], a[72:75], v[156:159]\n"
    "v_mfma_f32_16x16x32_bf16 v[88:91], a[184:187], a[64:67], v[88:91]\n"
    "v_mfma_f32_16x16x32_bf16 v[120:123], a[184:187], a[68:71], v[120:123]\n"
    "v_mfma_f32_16x16x32_bf16 v[124:127], a[188:191], a[68:71], v[124:127]\n"
    "v_mfma_f32_16x16x32_bf16 v[92:95], a[188:191], a[64:67], v[92:95]\n"
    "s_waitcnt vmcnt(20) lgkmcnt(0)\n"
    "s_barrier\n"
    "v_mfma_f32_16x16x32_bf16 v[64:67], a[192:195], a[96:99], v[64:67]\n"
    "ds_read_b64_tr_b16 a[160:161], v24 offset:0\n"
    "s_memtime s[4:5]\n"
    "v_mfma_f32_16x16x32_bf16 v[96:99], a[192:195], a[100:103], v[96:99]\n"
    "ds_read_b128 a[64:67], v32 offset:32768\n"
    "v_mfma_f32_16x16x32_bf16 v[100:103], a[196:199], a[100:103], v[100:103]\n"
    "s_bitcmp1_b64 s[64:65], s79\n"
    "s_cbranch_scc1 Lfl2%=\n"
    "Lfl2r%=:\n"
    "v_mfma_f32_16x16x32_bf16 v[68:71], a[196:199], a[96:99], v[68:71]\n"
    "ds_read_b64_tr_b16 a[162:163], v24 offset:1024\n"
    "v_mfma_f32_16x16x32_bf16 v[128:131], a[192:195], a[104:107], v[128:131]\n"
    "ds_read_b128 a[68:71], v32 offset:34816\n"
    "s_add_u32 m0, s71, 81920\n"
    "v_mfma_f32_16x16x32_bf16 v[160:163], a[192:195], a[108:111], v[160:163]\n"
    "buffer_load_dwordx4 v56, s[12:15], s73 offen offset:0  lds\n"
    "v_mfma_f32_16x16x32_bf16 v[164:167], a[196:199], a[108:111], v[164:167]\n"
    "ds_read_b64_tr_b16 a[164:165], v25 offset:0\n"
    "v_mfma_f32_16x16x32_bf16 v[132:135], a[196:199], a[104:107], v[132:135]\n"
    "ds_read_b128 a[72:75], v32 offset:36864\n"
    "v_mfma_f32_16x16x32_bf16 v[192:195], a[192:195], a[112:115], v[192:195]\n"
    "v_mfma_f32_16x16x32_bf16 v[224:227], a[192:195], a[116:119], v[224:227]\n"
    "ds_read_b64_tr_b16 a[166:167], v25 offset:1024\n"
    "v_mfma_f32_16x16x32_bf16 v[228:231], a[196:199], a[116:119], v[228:231]\n"
    "ds_read_b128 a[76:79], v32 offset:38912\n"
    "v_mfma_f32_16x16x32_bf16 v[196:199], a[196:199], a[112:115], v[196:199]\n"
    "v_mfma_f32_16x16x32_bf16 a[0:3], a[192:195], a[120:123], a[0:3]\n"
    "ds_read_b64_tr_b16 a[168:169], v26 offset:0\n"
    "v_mfma_f32_16x16x32_bf16 a[32:35], a[192:195], a[124:127], a[32:35]\n"
    "ds_read_b128 a[80:83], v32 offset:40960\n"
    "v_mfma_f32_16x16x32_bf16 a[36:39], a[196:199], a[124:127], a[36:39]\n"
    "v_mfma_f32_16x16x32_bf16 a[4:7], a[196:199], a[120:123], a[4:7]\n"
    "ds_read_b64_tr_b16 a[170:171], v26 offset:1024\n"
    "v_mfma_f32_16x16x32_bf16 a[8:11], a[200:203], a[120:123], a[8:11]\n"
    "ds_read_b128 a[84:87], v32 offset:43008\n"
    "v_mfma_f32_16x16x32_bf16 a[40:43], a[200:203], a[124:127], a[40:43]\n"
    "buffer_load_dwordx4 v57, s[12:15], s73 offen offset:1024  lds\n"
    "v_mfma_f32_16x16x32_bf16 a[44:47], a[204:207], a[124:127], a[44:47]\n"
    "ds_read_b64_tr_b16 a[172:173], v27 offset:0\n"
    "v_mfma_f32_16x16x32_bf16 a[12:15], a[204:207], a[120:123], a[12:15]\n"
    "ds_read_b128 a[88:91], v32 offset:45056\n"
    "v_mfma_f32_16x16x32_bf16 v[200:203], a[200:203], a[112:115], v[200:203]\n"
    "v_mfma_f32_16x16x32_bf16 v[232:235], a[200:203], a[116:119], v[232:235]\n"
    "ds_read_b64_tr_b16 a[174:175], v27 offset:1024\n"
    "v_mfma_f32_16x16x32_bf16 v[236:239], a[204:207], a[116:119], v[236:239]\n"
    "ds_read_b128 a[92:95], v32 offset:47104\n"
    "v_mfma_f32_16x16x32_bf16 v[204:207], a[204:207], a[112:115], v[204:207]\n"
    "v_mfma_f32_16x16x32_bf16 v[136:139], a[200:203], a[104:107], v[136:139]\n"
    "ds_read_b64_tr_b16 a[176:177], v28 offset:0\n"
    "v_mfma_f32_16x16x32_bf16 v[168:171], a[200:203], a[108:111], v[168:171]\n"
    "ds_read_b128 a[128:131], v33 offset:32768\n"
    "v_mfma_f32_16x16x32_bf16 v[172:175], a[204:207], a[108:111], v[172:175]\n"
    "v_mfma_f32_16x16x32_bf16 v[140:143], a[204:207], a[104:107], v[140:143]\n"
    "ds_read_b64_tr_b16 a[178:179], v28 offset:1024\n"
    "v_mfma_f32_16x16x32_bf16 v[72:75], a[200:203], a[96:99], v[72:75]\n"
    "ds_read_b128 a[132:135], v33 offset:34816\n"
    "v_mfma_f32_16x16x32_bf16 v[104:107], a[200:203], a[100:103], v[104:107]\n"
    "buffer_load_dwordx4 v58, s[12:15], s73 offen offset:2048  lds\n"
    "v_mfma_f32_16x16x32_bf16 v[108:111], a[204:207], a[100:103], v[108:111]\n"
    "ds_read_b64_tr_b16 a[180:181], v29 offset:0\n"
    "v_mfma_f32_16x16x32_bf16 v[76:79], a[204:207], a[96:99], v[76:79]\n"
    "ds_read_b128 a[136:139], v33 offset:36864\n"
    "v_mfma_f32_16x16x32_bf16 v[80:83], a[208:211], a[96:99], v[80:83]\n"
    "v_mfma_f32_16x16x32_bf16 v[112:115], a[208:211], a[100:103], v[112:115]\n"
    "ds_read_b64_tr_b16 a[182:183], v29 offset:1024\n"
    "v_mfma_f32_16x16x32_bf16 v[116:119], a[212:215], a[100:103], v[116:119]\n"
    "ds_read_b128 a[140:143], v33 offset:38912\n"
    "v_mfma_f32_16x16x32_bf16 v[84:87], a[212:215], a[96:99], v[84:87]\n"
    "v_mfma_f32_16x16x32_bf16 v[144:147], a[208:211], a[104:107], v[144:147]\n"
    "ds_read_b64_tr_b16 a[184:185], v30 offset:0\n"
    "v_mfma_f32_16x16x32_bf16 v[176:179], a[208:211], a[108:111], v[176:179]\n"
    "ds_read_b128 a[144:147], v33 offset:40960\n"
    "v_mfma_f32_16x16x32_bf16 v[180:183], a[212:215], a[108:111], v[180:183]\n"
    "v_mfma_f32_16x16x32_bf16 v[148:151], a[212:215], a[104:107], v[148:151]\n"
    "ds_read_b64_tr_b16 a[186:187], v30 offset:1024\n"
    "v_mfma_f32_16x16x32_bf16 v[208:211], a[208:211], a[112:115], v[208:211]\n"
    "ds_read_b128 a[148:151], v33 offset:43008\n"
    "v_mfma_f32_16x16x32_bf16 v[240:243], a[208:211], a[116:119], v[240:243]\n"
    "buffer_load_dwordx4 v59, s[12:15], s73 offen offset:3072  lds\n"
    "v_mfma_f32_16x16x32_bf16 v[244:247], a[212:215], a[116:119], v[244:247]\n"
    "ds_read_b64_tr_b16 a[188:189], v31 offset:0\n"
    "v_mfma_f32_16x16x32_bf16 v[212:215], a[212:215], a[112:115], v[212:215]\n"
    "ds_read_b128 a[152:155], v33 offset:45056\n"
    "v_mfma_f32_16x16x32_bf16 a[16:19], a[208:211], a[120:123], a[16:19]\n"
    "v_mfma_f32_16x16x32_bf16 a[48:51], a[208:211], a[124:127], a[48:51]\n"
    "ds_read_b64_tr_b16 a[190:191], v31 offset:1024\n"
    "v_mfma_f32_16x16x32_bf16 a[52:55], a[212:215], a[124:127], a[52:55]\n"
    "ds_read_b128 a[156:159], v33 offset:47104\n"
    "v_mfma_f32_16x16x32_bf16 a[20:23], a[212:215], a[120:123], a[20:23]\n"
    "v_mfma_f32_16x16x32_bf16 a[24:27], a[216:219], a[120:123], a[24:27]\n"
    "s_add_u32 s76, s76, 1\n"
    "v_mfma_f32_16x16x32_bf16 a[56:59], a[216:219], a[124:127], a[56:59]\n"
    "v_readlane_b32 s72, v40, s75\n"
    "v_mfma_f32_16x16x32_bf16 a[60:63], a[220:223], a[124:127], a[60:63]\n"
    "v_mfma_f32_16x16x32_bf16 a[28:31], a[220:223], a[120:123], a[28:31]\n"
    "v_readlane_b32 s73, v41, s76\n"
    "v_mfma_f32_16x16x32_bf16 v[216:219], a[216:219], a[112:115], v[216:219]\n"
    "s_cmp_eq_u32 s75, 8\n"
    "v_mfma_f32_16x16x32_bf16 v[248:251], a[216:219], a[116:119], v[248:251]\n"
    "s_cselect_b32 s84, s88, s84\n"
    "s_cselect_b32 s85, s89, s85\n"
    "v_mfma_f32_16x16x32_bf16 v[252:255], a[220:223], a[116:119], v[252:255]\n"
    "v_mfma_f32_16x16x32_bf16 v[220:223], a[220:223], a[112:115], v[220:223]\n"
    "s_cmp_ge_u32 s75, 64\n"
    "s_cselect_b32 s72, 1073741824, s72\n"
    "v_mfma_f32_16x16x32_bf16 v[152:155], a[216:219], a[104:107], v[152:155]\n"
    "v_mfma_f32_16x16x32_bf16 v[184:187], a[216:219], a[108:111], v[184:187]\n"
    "s_cmp_ge_u32 s76, 64\n"
    "s_cselect_b32 s73, 1073741824, s73\n"
    "v_mfma_f32_16x16x32_bf16 v[188:191], a[220:223], a[108:111], v[188:191]\n"
    "v_mfma_f32_16x16x32_bf16 v[156:159], a[220:223], a[104:107], v[156:159]\n"
    "s_bitcmp1_b64 s[60:61], s79\n"
    "s_cbranch_scc1 Loe2%=\n"
    "Loe2r%=:\n"
    "v_mfma_f32_16x16x32_bf16 v[88:91], a[216:219], a[96:99], v[88:91]\n"
    "v_mfma_f32_16x16x32_bf16 v[120:123], a[216:219], a[100:103], v[120:123]\n"
    "s_add_u32 s79, s79, 1\n"
    "v_mfma_f32_16x16x32_bf16 v[124:127], a[220:223], a[100:103], v[124:127]\n"
    "v_mfma_f32_16x16x32_bf16 v[92:95], a[220:223], a[96:99], v[92:95]\n"
    "s_cmp_lg_u32 s41, 0\n"
    "s_cbranch_scc1 LW102%=\n"
    "LW2%=:\n"
    "s_waitcnt lgkmcnt(0)\n"
    "s_barrier\n"
    "LT3%=:\n"
    "v_mfma_f32_16x16x32_bf16 v[64:67], a[160:163], a[64:67], v[64:67]\n"
    "s_sub_u32 s78, s4, s56\n"
    "ds_read_b64_tr_b16 a[192:193], v24 offset:16384\n"
    "v_mfma_f32_16x16x32_bf16 v[96:99], a[160:163], a[68:71], v[96:99]\n"
    "s_memtime s[56:57]\n"
    "s_add_u32 m0, s70, 32768\n"
    "v_mfma_f32_16x16x32_bf16 v[100:103], a[164:167], a[68:71], v[100:103]\n"
    "buffer_load_dwordx4 v48, s[84:87], s72 offen offset:0 sc0 sc1 lds\n"
    "v_mfma_f32_16x16x32_bf16 v[68:71], a[164:167], a[64:67], v[68:71]\n"
    "ds_read_b64_tr_b16 a[194:195], v24 offset:17408\n"
    "v_mfma_f32_16x16x32_bf16 v[128:131], a[160:163], a[72:75], v[128:131]\n"
    "v_mfma_f32_16x16x32_bf16 v[160:163], a[160:163], a[76:79], v[160:163]\n"
    "buffer_load_dwordx4 v49, s[84:87], s72 offen offset:1024 sc0 sc1 lds\n"
    "v_mfma_f32_16x16x32_bf16 v[164:167], a[164:167], a[76:79], v[164:167]\n"
    "ds_read_b64_tr_b16 a[196:197], v25 offset:16384\n"
    "v_mfma_f32_16x16x32_bf16 v[132:135], a[164:167], a[72:75], v[132:135]\n"
    "v_mfma_f32_16x16x32_bf16 v[192:195], a[160:163], a[80:83], v[192:195]\n"
    "buffer_load_dwordx4 v50, s[84:87], s72 offen offset:2048 sc0 sc1 lds\n"
    "v_mfma_f32_16x16x32_bf16 v[224:227], a[160:163], a[84:87], v[224:227]\n"
    "ds_read_b64_tr_b16 a[198:199], v25 offset:17408\n"
    "v_mfma_f32_16x16x32_bf16 v[228:231], a[164:167], a[84:87], v[228:231]\n"
    "v_mfma_f32_16x16x32_bf16 v[196:199], a[164:167], a[80:83], v[196:199]\n"
    "buffer_load_dwordx4 v51, s[84:87], s72 offen offset:3072 sc0 sc1 lds\n"
    "v_mfma_f32_16x16x32_bf16 a[0:3], a[160:163], a[88:91], a[0:3]\n"
    "ds_read_b64_tr_b16 a[200:201], v26 offset:16384\n"
    "v_mfma_f32_16x16x32_bf16 a[32:35], a[160:163], a[92:95], a[32:35]\n"
    "s_add_u32 m0, s70, 36864\n"
    "v_mfma_f32_16x16x32_bf16 a[36:39], a[164:167], a[92:95], a[36:39]\n"
    "buffer_load_dwordx4 v52, s[84:87], s72 offen offset:0 sc0 sc1 lds\n"
    "v_mfma_f32_16x16x32_bf16 a[4:7], a[164:167], a[88:91], a[4:7]\n"
    "ds_read_b64_tr_b16 a[202:203], v26 offset:17408\n"
    "v_mfma_f32_16x16x32_bf16 a[8:11], a[168:171], a[88:91], a[8:11]\n"
    "v_mfma_f32_16x16x32_bf16 a[40:43], a[168:171], a[92:95], a[40:43]\n"
    "buffer_load_dwordx4 v53, s[84:87], s72 offen offset:1024 sc0 sc1 lds\n"
    "v_mfma_f32_16x16x32_bf16 a[44:47], a[172:175], a[92:95], a[44:47]\n"
    "ds_read_b64_tr_b16 a[204:205], v27 offset:16384\n"
    "v_mfma_f32_16x16x32_bf16 a[12:15], a[172:175], a[88:91], a[12:15]\n"
    "v_mfma_f32_16x16x32_bf16 v[200:203], a[168:171], a[80:83], v[200:203]\n"
    "buffer_load_dwordx4 v54, s[84:87], s72 offen offset:2048 sc0 sc1 lds\n"
    "v_mfma_f32_16x16x32_bf16 v[232:235], a[168:171], a[84:87], v[232:235]\n"
    "ds_read_b64_tr_b16 a[206:207], v27 offset:17408\n"
    "v_mfma_f32_16x16x32_bf16 v[236:239], a[172:175], a[84:87], v[236:239]\n"
    "v_mfma_f32_16x16x32_bf16 v[204:207], a[172:175], a[80:83], v[204:207]\n"
    "buffer_load_dwordx4 v55, s[84:87], s72 offen offset:3072 sc0 sc1 lds\n"
    "v_mfma_f32_16x16x32_bf16 v[136:139], a[168:171], a[72:75], v[136:139]\n"
    "ds_read_b64_tr_b16 a[208:209], v28 offset:16384\n"
    "v_mfma_f32_16x16x32_bf16 v[168:171], a[168:171], a[76:79], v[168:171]\n"
    "v_mfma_f32_16x16x32_bf16 v[172:175], a[172:175], a[76:79], v[172:175]\n"
    "v_mfma_f32_16x16x32_bf16 v[140:143], a[172:175], a[72:75], v[140:143]\n"
    "ds_read_b64_tr_b16 a[210:211], v28 offset:17408\n"
    "v_mfma_f32_16x16x32_bf16 v[72:75], a[168:171], a[64:67], v[72:75]\n"
    "s_add_u32 m0, s71, 0\n"
    "v_mfma_f32_16x16x32_bf16 v[104:107], a[168:171], a[68:71], v[104:107]\n"
    "buffer_load_dwordx4 v56, s[12:15], s73 offen offset:0  lds\n"
    "v_mfma_f32_16x16x32_bf16 v[108:111], a[172:175], a[68:71], v[108:111]\n"
    "ds_read_b64_tr_b16 a[212:213], v29 offset:16384\n"
    "v_mfma_f32_16x16x32_bf16 v[76:79], a[172:175], a[64:67], v[76:79]\n"
    "v_mfma_f32_16x16x32_bf16 v[80:83], a[176:179], a[64:67], v[80:83]\n"
    "v_mfma_f32_16x16x32_bf16 v[112:115], a[176:179], a[68:71], v[112:115]\n"
    "ds_read_b64_tr_b16 a[214:215], v29 offset:17408\n"
    "v_mfma_f32_16x16x32_bf16 v[116:119], a[180:183], a[68:71], v[116:119]\n"
    "v_mfma_f32_16x16x32_bf16 v[84:87], a[180:183], a[64:67], v[84:87]\n"
    "buffer_load_dwordx4 v57, s[12:15], s73 offen offset:1024  lds\n"
    "v_mfma_f32_16x16x32_bf16 v[144:147], a[176:179], a[72:75], v[144:147]\n"
    "ds_read_b64_tr_b16 a[216:217], v30 offset:16384\n"
    "v_mfma_f32_16x16x32_bf16 v[176:179], a[176:179], a[76:79], v[176:179]\n"
    "v_mfma_f32_16x16x32_bf16 v[180:183], a[180:183], a[76:79], v[180:183]\n"
    "v_mfma_f32_16x16x32_bf16 v[148:151], a[180:183], a[72:75], v[148:151]\n"
    "ds_read_b64_tr_b16 a[218:219], v30 offset:17408\n"
    "v_mfma_f32_16x16x32_bf16 v[208:211], a[176:179], a[80:83], v[208:211]\n"
    "v_mfma_f32_16x16x32_bf16 v[240:243], a[176:179], a[84:87], v[240:243]\n"
    "buffer_load_dwordx4 v58, s[12:15], s73 offen offset:2048  lds\n"
    "v_mfma_f32_16x16x32_bf16 v[244:247], a[180:183], a[84:87], v[244:247]\n"
    "ds_read_b64_tr_b16 a[220:221], v31 offset:16384\n"
    "v_mfma_f32_16x16x32_bf16 v[212:215], a[180:183], a[80:83], v[212:215]\n"
    "v_mfma_f32_16x16x32_bf16 a[16:19], a[176:179], a[88:91], a[16:19]\n"
    "v_mfma_f32_16x16x32_bf16 a[48:51], a[176:179], a[92:95], a[48:51]\n"
    "ds_read_b64_tr_b16 a[222:223], v31 offset:17408\n"
    "v_mfma_f32_16x16x32_bf16 a[52:55], a[180:183], a[92:95], a[52:55]\n"
    "v_mfma_f32_16x16x32_bf16 a[20:23], a[180:183], a[88:91], a[20:23]\n"
    "buffer_load_dwordx4 v59, s[12:15], s73 offen offset:3072  lds\n"
    "v_mfma_f32_16x16x32_bf16 a[24:27], a[184:187], a[88:91], a[24:27]\n"
    "v_mfma_f32_16x16x32_bf16 a[56:59], a[184:187], a[92:95], a[56:59]\n"
    "s_add_u32 s73, s73, 0x200000\n"
    "v_mfma_f32_16x16x32_bf16 a[60:63], a[188:191], a[92:95], a[60:63]\n"
    "s_add_u32 s75, s75, 1\n"
    "v_mfma_f32_16x16x32_bf16 a[28:31], a[188:191], a[88:91], a[28:31]\n"
    "v_mfma_f32_16x16x32_bf16 v[216:219], a[184:187], a[80:83], v[216:219]\n"
    "s_cmp_gt_u32 s78, s30\n"
    "s_cbranch_scc1 Lad3%=\n"
    "Lad3r%=:\n"
    "v_mfma_f32_16x16x32_bf16 v[248:251], a[184:187], a[84:87], v[248:251]\n"
    "v_mfma_f32_16x16x32_bf16 v[252:255], a[188:191], a[84:87], v[252:255]\n"
    "v_mfma_f32_16x16x32_bf16 v[220:223], a[188:191], a[80:83], v[220:223]\n"
    "v_mfma_f32_16x16x32_bf16 v[152:155], a[184:187], a[72:75], v[152:155]\n"
    "v_mfma_f32_16x16x32_bf16 v[184:187], a[184:187], a[76:79], v[184:187]\n"
    "v_mfma_f32_16x16x32_bf16 v[188:191], a[188:191], a[76:79], v[188:191]\n"
    "v_mfma_f32_16x16x32_bf16 v[156:159], a[188:191], a[72:75], v[156:159]\n"
    "v_mfma_f32_16x16x32_bf16 v[88:91], a[184:187], a[64:67], v[88:91]\n"
    "v_mfma_f32_16x16x32_bf16 v[120:123], a[184:187], a[68:71], v[120:123]\n"
    "v_mfma_f32_16x16x32_bf16 v[124:127], a[188:191], a[68:71], v[124:127]\n"
    "v_mfma_f32_16x16x32_bf16 v[92:95], a[188:191], a[64:67], v[92:95]\n"
    "s_waitcnt vmcnt(20) lgkmcnt(0)\n"
    "s_barrier\n"
    "v_mfma_f32_16x16x32_bf16 v[64:67], a[192:195], a[128:131], v[64:67]\n"
    "ds_read_b64_tr_b16 a[160:161], v24 offset:32768\n"
    "s_memtime s[4:5]\n"
    "v_mfma_f32_16x16x32_bf16 v[96:99], a[192:195], a[132:135], v[96:99]\n"
    "ds_read_b128 a[64:67], v32 offset:0\n"
    "v_mfma_f32_16x16x32_bf16 v[100:103], a[196:199], a[132:135], v[100:103]\n"
    "s_bitcmp1_b64 s[64:65], s79\n"
    "s_cbranch_scc1 Lfl3%=\n"
    "Lfl3r%=:\n"
    "v_mfma_f32_16x16x32_bf16 v[68:71], a[196:199], a[128:131], v[68:71]\n"
    "ds_read_b64_tr_b16 a[162:163], v24 offset:33792\n"
    "v_mfma_f32_16x16x32_bf16 v[128:131], a[192:195], a[136:139], v[128:131]\n"
    "ds_read_b128 a[68:71], v32 offset:2048\n"
    "s_add_u32 m0, s71, 16384\n"
    "v_mfma_f32_16x16x32_bf16 v[160:163], a[192:195], a[140:143], v[160:163]\n"
    "buffer_load_dwordx4 v56, s[12:15], s73 offen offset:0  lds\n"
    "v_mfma_f32_16x16x32_bf16 v[164:167], a[196:199], a[140:143], v[164:167]\n"
    "ds_read_b64_tr_b16 a[164:165], v25 offset:32768\n"
    "v_mfma_f32_16x16x32_bf16 v[132:135], a[196:199], a[136:139], v[132:135]\n"
    "ds_read_b128 a[72:75], v32 offset:4096\n"
    "v_mfma_f32_16x16x32_bf16 v[192:195], a[192:195], a[144:147], v[192:195]\n"
    "v_mfma_f32_16x16x32_bf16 v[224:227], a[192:195], a[148:151], v[224:227]\n"
    "ds_read_b64_tr_b16 a[166:167], v25 offset:33792\n"
    "v_mfma_f32_16x16x32_bf16 v[228:231], a[196:199], a[148:151], v[228:231]\n"
    "ds_read_b128 a[76:79], v32 offset:6144\n"
    "v_mfma_f32_16x16x32_bf16 v[196:199], a[196:199], a[144:147], v[196:199]\n"
    "v_mfma_f32_16x16x32_bf16 a[0:3], a[192:195], a[152:155], a[0:3]\n"
    "ds_read_b64_tr_b16 a[168:169], v26 offset:32768\n"
    "v_mfma_f32_16x16x32_bf16 a[32:35], a[192:195], a[156:159], a[32:35]\n"
    "ds_read_b128 a[80:83], v32 offset:8192\n"
    "v_mfma_f32_16x16x32_bf16 a[36:39], a[196:199], a[156:159], a[36:39]\n"
    "v_mfma_f32_16x16x32_bf16 a[4:7], a[196:199], a[152:155], a[4:7]\n"
    "ds_read_b64_tr_b16 a[170:171], v26 offset:33792\n"
    "v_mfma_f32_16x16x32_bf16 a[8:11], a[200:203], a[152:155], a[8:11]\n"
    "ds_read_b128 a[84:87], v32 offset:10240\n"
    "v_mfma_f32_16x16x32_bf16 a[40:43], a[200:203], a[156:159], a[40:43]\n"
    "buffer_load_dwordx4 v57, s[12:15], s73 offen offset:1024  lds\n"
    "v_mfma_f32_16x16x32_bf16 a[44:47], a[204:207], a[156:159], a[44:47]\n"
    "ds_read_b64_tr_b16 a[172:173], v27 offset:32768\n"
    "v_mfma_f32_16x16x32_bf16 a[12:15], a[204:207], a[152:155], a[12:15]\n"
    "ds_read_b128 a[88:91], v32 offset:12288\n"
    "v_mfma_f32_16x16x32_bf16 v[200:203], a[200:203], a[144:147], v[200:203]\n"
    "v_mfma_f32_16x16x32_bf16 v[232:235], a[200:203], a[148:151], v[232:235]\n"
    "ds_read_b64_tr_b16 a[174:175], v27 offset:33792\n"
    "v_mfma_f32_16x16x32_bf16 v[236:239], a[204:207], a[148:151], v[236:239]\n"
    "ds_read_b128 a[92:95], v32 offset:14336\n"
    "v_mfma_f32_16x16x32_bf16 v[204:207], a[204:207], a[144:147], v[204:207]\n"
    "v_mfma_f32_16x16x32_bf16 v[136:139], a[200:203], a[136:139], v[136:139]\n"
    "ds_read_b64_tr_b16 a[176:177], v28 offset:32768\n"
    "v_mfma_f32_16x16x32_bf16 v[168:171], a[200:203], a[140:143], v[168:171]\n"
    "ds_read_b128 a[96:99], v33 offset:0\n"
    "v_mfma_f32_16x16x32_bf16 v[172:175], a[204:207], a[140:143], v[172:175]\n"
    "v_mfma_f32_16x16x32_bf16 v[140:143], a[204:207], a[136:139], v[140:143]\n"
    "ds_read_b64_tr_b16 a[178:179], v28 offset:33792\n"
    "v_mfma_f32_16x16x32_bf16 v[72:75], a[200:203], a[128:131], v[72:75]\n"
    "ds_read_b128 a[100:103], v33 offset:2048\n"
    "v_mfma_f32_16x16x32_bf16 v[104:107], a[200:203], a[132:135], v[104:107]\n"
    "buffer_load_dwordx4 v58, s[12:15], s73 offen offset:2048  lds\n"
    "v_mfma_f32_16x16x32_bf16 v[108:111], a[204:207], a[132:135], v[108:111]\n"
    "ds_read_b64_tr_b16 a[180:181], v29 offset:32768\n"
    "v_mfma_f32_16x16x32_bf16 v[76:79], a[204:207], a[128:131], v[76:79]\n"
    "ds_read_b128 a[104:107], v33 offset:4096\n"
    "v_mfma_f32_16x16x32_bf16 v[80:83], a[208:211], a[128:131], v[80:83]\n"
    "v_mfma_f32_16x16x32_bf16 v[112:115], a[208:211], a[132:135], v[112:115]\n"
    "ds_read_b64_tr_b16 a[182:183], v29 offset:33792\n"
    "v_mfma_f32_16x16x32_bf16 v[116:119], a[212:215], a[132:135], v[116:119]\n"
    "ds_read_b128 a[108:111], v33 offset:6144\n"
    "v_mfma_f32_16x16x32_bf16 v[84:87], a[212:215], a[128:131], v[84:87]\n"
    "v_mfma_f32_16x16x32_bf16 v[144:147], a[208:211], a[136:139], v[144:147]\n"
    "ds_read_b64_tr_b16 a[184:185], v30 offset:32768\n"
    "v_mfma_f32_16x16x32_bf16 v[176:179], a[208:211], a[140:143], v[176:179]\n"
    "ds_read_b128 a[112:115], v33 offset:8192\n"
    "v_mfma_f32_16x16x32_bf16 v[180:183], a[212:215], a[140:143], v[180:183]\n"
    "v_mfma_f32_16x16x32_bf16 v[148:151], a[212:215], a[136:139], v[148:151]\n"
    "ds_read_b64_tr_b16 a[186:187], v30 offset:33792\n"
    "v_mfma_f32_16x16x32_bf16 v[208:211], a[208:211], a[144:147], v[208:211]\n"
    "ds_read_b128 a[116:119], v33 offset:10240\n"
    "v_mfma_f32_16x16x32_bf16 v[240:243], a[208:211], a[148:151], v[240:243]\n"
    "buffer_load_dwordx4 v59, s[12:15], s73 offen offset:3072  lds\n"
    "v_mfma_f32_16x16x32_bf16 v[244:247], a[212:215], a[148:151], v[244:247]\n"
    "ds_read_b64_tr_b16 a[188:189], v31 offset:32768\n"
    "v_mfma_f32_16x16x32_bf16 v[212:215], a[212:215], a[144:147], v[212:215]\n"
    "ds_read_b128 a[120:123], v33 offset:12288\n"
    "v_mfma_f32_16x16x32_bf16 a[16:19], a[208:211], a[152:155], a[16:19]\n"
    "v_mfma_f32_16x16x32_bf16 a[48:51], a[208:211], a[156:159], a[48:51]\n"
    "ds_read_b64_tr_b16 a[190:191], v31 offset:33792\n"
    "v_mfma_f32_16x16x32_bf16 a[52:55], a[212:215], a[156:159], a[52:55]\n"
    "ds_read_b128 a[124:127], v33 offset:14336\n"
    "v_mfma_f32_16x16x32_bf16 a[20:23], a[212:215], a[152:155], a[20:23]\n"
    "v_mfma_f32_16x16x32_bf16 a[24:27], a[216:219], a[152:155], a[24:27]\n"
    "s_add_u32 s76, s76, 1\n"
    "v_mfma_f32_16x16x32_bf16 a[56:59], a[216:219], a[156:159], a[56:59]\n"
    "v_readlane_b32 s72, v40, s75\n"
    "v_mfma_f32_16x16x32_bf16 a[60:63], a[220:223], a[156:159], a[60:63]\n"
    "v_mfma_f32_16x16x32_bf16 a[28:31], a[220:223], a[152:155], a[28:31]\n"
    "v_readlane_b32 s73, v41, s76\n"
    "v_mfma_f32_16x16x32_bf16 v[216:219], a[216:219], a[144:147], v[216:219]\n"
    "s_cmp_eq_u32 s75, 8\n"
    "v_mfma_f32_16x16x32_bf16 v[248:251], a[216:219], a[148:151], v[248:251]\n"
    "s_cselect_b32 s84, s88, s84\n"
    "s_cselect_b32 s85, s89, s85\n"
    "v_mfma_f32_16x16x32_bf16 v[252:255], a[220:223], a[148:151], v[252:255]\n"
    "v_mfma_f32_16x16x32_bf16 v[220:223], a[220:223], a[144:147], v[220:223]\n"
    "s_cmp_ge_u32 s75, 64\n"
    "s_cselect_b32 s72, 1073741824, s72\n"
    "v_mfma_f32_16x16x32_bf16 v[152:155], a[216:219], a[136:139], v[152:155]\n"
    "v_mfma_f32_16x16x32_bf16 v[184:187], a[216:219], a[140:143], v[184:187]\n"
    "s_cmp_ge_u32 s76, 64\n"
    "s_cselect_b32 s73, 1073741824, s73\n"
    "v_mfma_f32_16x16x32_bf16 v[188:191], a[220:223], a[140:143], v[188:191]\n"
    "v_mfma_f32_16x16x32_bf16 v[156:159], a[220:223], a[136:139], v[156:159]\n"
    "s_bitcmp1_b64 s[60:61], s79\n"
    "s_cbranch_scc1 Loe3%=\n"
    "Loe3r%=:\n"
    "v_mfma_f32_16x16x32_bf16 v[88:91], a[216:219], a[128:131], v[88:91]\n"
    "v_mfma_f32_16x16x32_bf16 v[120:123], a[216:219], a[132:135], v[120:123]\n"
    "s_add_u32 s79, s79, 1\n"
    "v_mfma_f32_16x16x32_bf16 v[124:127], a[220:223], a[132:135], v[124:127]\n"
    "v_mfma_f32_16x16x32_bf16 v[92:95], a[220:223], a[128:131], v[92:95]\n"
    "s_cmp_lg_u32 s41, 0\n"
    "s_cbranch_scc1 LW103%=\n"
    "LW3%=:\n"
    "s_waitcnt lgkmcnt(0)\n"
    "s_barrier\n"
    "LT4%=:\n"
    "v_mfma_f32_16x16x32_bf16 v[64:67], a[160:163], a[64:67], v[64:67]\n"
    "s_sub_u32 s78, s4, s56\n"
    "ds_read_b64_tr_b16 a[192:193], v24 offset:49152\n"
    "v_mfma_f32_16x16x32_bf16 v[96:99], a[160:163], a[68:71], v[96:99]\n"
    "s_memtime s[56:57]\n"
    "s_add_u32 m0, s70, 0\n"
    "v_mfma_f32_16x16x32_bf16 v[100:103], a[164:167], a[68:71], v[100:103]\n"
    "buffer_load_dwordx4 v48, s[84:87], s72 offen offset:0 sc0 sc1 lds\n"
    "v_mfma_f32_16x16x32_bf16 v[68:71], a[164:167], a[64:67], v[68:71]\n"
    "ds_read_b64_tr_b16 a[194:195], v24 offset:50176\n"
    "v_mfma_f32_16x16x32_bf16 v[128:131], a[160:163], a[72:75], v[128:131]\n"
    "v_mfma_f32_16x16x32_bf16 v[160:163], a[160:163], a[76:79], v[160:163]\n"
    "buffer_load_dwordx4 v49, s[84:87], s72 offen offset:1024 sc0 sc1 lds\n"
    "v_mfma_f32_16x16x32_bf16 v[164:167], a[164:167], a[76:79], v[164:167]\n"
    "ds_read_b64_tr_b16 a[196:197], v25 offset:49152\n"
    "v_mfma_f32_16x16x32_bf16 v[132:135], a[164:167], a[72:75], v[132:135]\n"
    "v_mfma_f32_16x16x32_bf16 v[192:195], a[160:163], a[80:83], v[192:195]\n"
    "buffer_load_dwordx4 v50, s[84:87], s72 offen offset:2048 sc0 sc1 lds\n"
    "v_mfma_f32_16x16x32_bf16 v[224:227], a[160:163], a[84:87], v[224:227]\n"
    "ds_read_b64_tr_b16 a[198:199], v25 offset:50176\n"
    "v_mfma_f32_16x16x32_bf16 v[228:231], a[164:167], a[84:87], v[228:231]\n"
    "v_mfma_f32_16x16x32_bf16 v[196:199], a[164:167], a[80:83], v[196:199]\n"
    "buffer_load_dwordx4 v51, s[84:87], s72 offen offset:3072 sc0 sc1 lds\n"
    "v_mfma_f32_16x16x32_bf16 a[0:3], a[160:163], a[88:91], a[0:3]\n"
    "ds_read_b64_tr_b16 a[200:201], v26 offset:49152\n"
    "v_mfma_f32_16x16x32_bf16 a[32:35], a[160:163], a[92:95], a[32:35]\n"
    "s_add_u32 m0, s70, 4096\n"
    "v_mfma_f32_16x16x32_bf16 a[36:39], a[164:167], a[92:95], a[36:39]\n"
    "buffer_load_dwordx4 v52, s[84:87], s72 offen offset:0 sc0 sc1 lds\n"
    "v_mfma_f32_16x16x32_bf16 a[4:7], a[164:167], a[88:91], a[4:7]\n"
    "ds_read_b64_tr_b16 a[202:203], v26 offset:50176\n"
    "v_mfma_f32_16x16x32_bf16 a[8:11], a[168:171], a[88:91], a[8:11]\n"
    "v_mfma_f32_16x16x32_bf16 a[40:43], a[168:171], a[92:95], a[40:43]\n"
    "buffer_load_dwordx4 v53, s[84:87], s72 offen offset:1024 sc0 sc1 lds\n"
    "v_mfma_f32_16x16x32_bf16 a[44:47], a[172:175], a[92:95], a[44:47]\n"
    "ds_read_b64_tr_b16 a[204:205], v27 offset:49152\n"
    "v_mfma_f32_16x16x32_bf16 a[12:15], a[172:175], a[88:91], a[12:15]\n"
    "v_mfma_f32_16x16x32_bf16 v[200:203], a[168:171], a[80:83], v[200:203]\n"
    "buffer_load_dwordx4 v54, s[84:87], s72 offen offset:2048 sc0 sc1 lds\n"
    "v_mfma_f32_16x16x32_bf16 v[232:235], a[168:171], a[84:87], v[232:235]\n"
    "ds_read_b64_tr_b16 a[206:207], v27 offset:50176\n"
    "v_mfma_f32_16x16x32_bf16 v[236:239], a[172:175], a[84:87], v[236:239]\n"
    "v_mfma_f32_16x16x32_bf16 v[204:207], a[172:175], a[80:83], v[204:207]\n"
    "buffer_load_dwordx4 v55, s[84:87], s72 offen offset:3072 sc0 sc1 lds\n"
    "v_mfma_f32_16x16x32_bf16 v[136:139], a[168:171], a[72:75], v[136:139]\n"
    "ds_read_b64_tr_b16 a[208:209], v28 offset:49152\n"
    "v_mfma_f32_16x16x32_bf16 v[168:171], a[168:171], a[76:79], v[168:171]\n"
    "v_mfma_f32_16x16x32_bf16 v[172:175], a[172:175], a[76:79], v[172:175]\n"
    "v_mfma_f32_16x16x32_bf16 v[140:143], a[172:175], a[72:75], v[140:143]\n"
    "ds_read_b64_tr_b16 a[210:211], v28 offset:50176\n"
    "v_mfma_f32_16x16x32_bf16 v[72:75], a[168:171], a[64:67], v[72:75]\n"
    "s_add_u32 m0, s71, 32768\n"
    "v_mfma_f32_16x16x32_bf16 v[104:107], a[168:171], a[68:71], v[104:107]\n"
    "buffer_load_dwordx4 v56, s[12:15], s73 offen offset:0  lds\n"
    "v_mfma_f32_16x16x32_bf16 v[108:111], a[172:175], a[68:71], v[108:111]\n"
    "ds_read_b64_tr_b16 a[212:213], v29 offset:49152\n"
    "v_mfma_f32_16x16x32_bf16 v[76:79], a[172:175], a[64:67], v[76:79]\n"
    "v_mfma_f32_16x16x32_bf16 v[80:83], a[176:179], a[64:67], v[80:83]\n"
    "v_mfma_f32_16x16x32_bf16 v[112:115], a[176:179], a[68:71], v[112:115]\n"
    "ds_read_b64_tr_b16 a[214:215], v29 offset:50176\n"
    "v_mfma_f32_16x16x32_bf16 v[116:119], a[180:183], a[68:71], v[116:119]\n"
    "v_mfma_f32_16x16x32_bf16 v[84:87], a[180:183], a[64:67], v[84:87]\n"
    "buffer_load_dwordx4 v57, s[12:15], s73 offen offset:1024  lds\n"
    "v_mfma_f32_16x16x32_bf16 v[144:147], a[176:179], a[72:75], v[144:147]\n"
    "ds_read_b64_tr_b16 a[216:217], v30 offset:49152\n"
    "v_mfma_f32_16x16x32_bf16 v[176:179], a[176:179], a[76:79], v[176:179]\n"
    "v_mfma_f32_16x16x32_bf16 v[180:183], a[180:183], a[76:79], v[180:183]\n"
    "v_mfma_f32_16x16x32_bf16 v[148:151], a[180:183], a[72:75], v[148:151]\n"
    "ds_read_b64_tr_b16 a[218:219], v30 offset:50176\n"
    "v_mfma_f32_16x16x32_bf16 v[208:211], a[176:179], a[80:83], v[208:211]\n"
    "v_mfma_f32_16x16x32_bf16 v[240:243], a[176:179], a[84:87], v[240:243]\n"
    "buffer_load_dwordx4 v58, s[12:15], s73 offen offset:2048  lds\n"
    "v_mfma_f32_16x16x32_bf16 v[244:247], a[180:183], a[84:87], v[244:247]\n"
    "ds_read_b64_tr_b16 a[220:221], v31 offset:49152\n"
    "v_mfma_f32_16x16x32_bf16 v[212:215], a[180:183], a[80:83], v[212:215]\n"
    "v_mfma_f32_16x16x32_bf16 a[16:19], a[176:179], a[88:91], a[16:19]\n"
    "v_mfma_f32_16x16x32_bf16 a[48:51], a[176:179], a[92:95], a[48:51]\n"
    "ds_read_b64_tr_b16 a[222:223], v31 offset:50176\n"
    "v_mfma_f32_16x16x32_bf16 a[52:55], a[180:183], a[92:95], a[52:55]\n"
    "v_mfma_f32_16x16x32_bf16 a[20:23], a[180:183], a[88:91], a[20:23]\n"
    "buffer_load_dwordx4 v59, s[12:15], s73 offen offset:3072  lds\n"
    "v_mfma_f32_16x16x32_bf16 a[24:27], a[184:187], a[88:91], a[24:27]\n"
    "v_mfma_f32_16x16x32_bf16 a[56:59], a[184:187], a[92:95], a[56:59]\n"
    "s_add_u32 s73, s73, 0x200000\n"
    "v_mfma_f32_16x16x32_bf16 a[60:63], a[188:191], a[92:95], a[60:63]\n"
    "s_add_u32 s75, s75, 1\n"
    "v_mfma_f32_16x16x32_bf16 a[28:31], a[188:191], a[88:91], a[28:31]\n"
    "v_mfma_f32_16x16x32_bf16 v[216:219], a[184:187], a[80:83], v[216:219]\n"
    "s_cmp_gt_u32 s78, s30\n"
    "s_cbranch_scc1 Lad4%=\n"
    "Lad4r%=:\n"
    "v_mfma_f32_16x16x32_bf16 v[248:251], a[184:187], a[84:87], v[248:251]\n"
    "v_mfma_f32_16x16x32_bf16 v[252:255], a[188:191], a[84:87], v[252:255]\n"
    "v_mfma_f32_16x16x32_bf16 v[220:223], a[188:191], a[80:83], v[220:223]\n"
    "v_mfma_f32_16x16x32_bf16 v[152:155], a[184:187], a[72:75], v[152:155]\n"
    "v_mfma_f32_16x16x32_bf16 v[184:187], a[184:187], a[76:79], v[184:187]\n"
    "v_mfma_f32_16x16x32_bf16 v[188:191], a[188:191], a[76:79], v[188:191]\n"
    "v_mfma_f32_16x16x32_bf16 v[156:159], a[188:191], a[72:75], v[156:159]\n"
    "v_mfma_f32_16x16x32_bf16 v[88:91], a[184:187], a[64:67], v[88:91]\n"
    "v_mfma_f32_16x16x32_bf16 v[120:123], a[184:187], a[68:71], v[120:123]\n"
    "v_mfma_f32_16x16x32_bf16 v[124:127], a[188:191], a[68:71], v[124:127]\n"
    "v_mfma_f32_16x16x32_bf16 v[92:95], a[188:191], a[64:67], v[92:95]\n"
    "s_waitcnt vmcnt(20) lgkmcnt(0)\n"
    "s_barrier\n"
    "v_mfma_f32_16x16x32_bf16 v[64:67], a[192:195], a[96:99], v[64:67]\n"
    "ds_read_b64_tr_b16 a[160:161], v16 offset:0\n"
    "s_memtime s[4:5]\n"
    "v_mfma_f32_16x16x32_bf16 v[96:99], a[192:195], a[100:103], v[96:99]\n"
    "ds_read_b128 a[64:67], v32 offset:32768\n"
    "v_mfma_f32_16x16x32_bf16 v[100:103], a[196:199], a[100:103], v[100:103]\n"
    "s_bitcmp1_b64 s[64:65], s79\n"
    "s_cbranch_scc1 Lfl4%=\n"
    "Lfl4r%=:\n"
    "v_mfma_f32_16x16x32_bf16 v[68:71], a[196:199], a[96:99], v[68:71]\n"
    "ds_read_b64_tr_b16 a[162:163], v16 offset:1024\n"
    "v_mfma_f32_16x16x32_bf16 v[128:131], a[192:195], a[104:107], v[128:131]\n"
    "ds_read_b128 a[68:71], v32 offset:34816\n"
    "s_add_u32 m0, s71, 49152\n"
    "v_mfma_f32_16x16x32_bf16 v[160:163], a[192:195], a[108:111], v[160:163]\n"
    "buffer_load_dwordx4 v56, s[12:15], s73 offen offset:0  lds\n"
    "v_mfma_f32_16x16x32_bf16 v[164:167], a[196:199], a[108:111], v[164:167]\n"
    "ds_read_b64_tr_b16 a[164:165], v17 offset:0\n"
    "v_mfma_f32_16x16x32_bf16 v[132:135], a[196:199], a[104:107], v[132:135]\n"
    "ds_read_b128 a[72:75], v32 offset:36864\n"
    "v_mfma_f32_16x16x32_bf16 v[192:195], a[192:195], a[112:115], v[192:195]\n"
    "v_mfma_f32_16x16x32_bf16 v[224:227], a[192:195], a[116:119], v[224:227]\n"
    "ds_read_b64_tr_b16 a[166:167], v17 offset:1024\n"
    "v_mfma_f32_16x16x32_bf16 v[228:231], a[196:199], a[116:119], v[228:231]\n"
    "ds_read_b128 a[76:79], v32 offset:38912\n"
    "v_mfma_f32_16x16x32_bf16 v[196:199], a[196:199], a[112:115], v[196:199]\n"
    "v_mfma_f32_16x16x32_bf16 a[0:3], a[192:195], a[120:123], a[0:3]\n"
    "ds_read_b64_tr_b16 a[168:169], v18 offset:0\n"
    "v_mfma_f32_16x16x32_bf16 a[32:35], a[192:195], a[124:127], a[32:35]\n"
    "ds_read_b128 a[80:83], v32 offset:40960\n"
    "v_mfma_f32_16x16x32_bf16 a[36:39], a[196:199], a[124:127], a[36:39]\n"
    "v_mfma_f32_16x16x32_bf16 a[4:7], a[196:199], a[120:123], a[4:7]\n"
    "ds_read_b64_tr_b16 a[170:171], v18 offset:1024\n"
    "v_mfma_f32_16x16x32_bf16 a[8:11], a[200:203], a[120:123], a[8:11]\n"
    "ds_read_b128 a[84:87], v32 offset:43008\n"
    "v_mfma_f32_16x16x32_bf16 a[40:43], a[200:203], a[124:127], a[40:43]\n"
    "buffer_load_dwordx4 v57, s[12:15], s73 offen offset:1024  lds\n"
    "v_mfma_f32_16x16x32_bf16 a[44:47], a[204:207], a[124:127], a[44:47]\n"
    "ds_read_b64_tr_b16 a[172:173], v19 offset:0\n"
    "v_mfma_f32_16x16x32_bf16 a[12:15], a[204:207], a[120:123], a[12:15]\n"
    "ds_read_b128 a[88:91], v32 offset:45056\n"
    "v_mfma_f32_16x16x32_bf16 v[200:203], a[200:203], a[112:115], v[200:203]\n"
    "v_mfma_f32_16x16x32_bf16 v[232:235], a[200:203], a[116:119], v[232:235]\n"
    "ds_read_b64_tr_b16 a[174:175], v19 offset:1024\n"
    "v_mfma_f32_16x16x32_bf16 v[236:239], a[204:207], a[116:119], v[236:239]\n"
    "ds_read_b128 a[92:95], v32 offset:47104\n"
    "v_mfma_f32_16x16x32_bf16 v[204:207], a[204:207], a[112:115], v[204:207]\n"
    "v_mfma_f32_16x16x32_bf16 v[136:139], a[200:203], a[104:107], v[136:139]\n"
    "ds_read_b64_tr_b16 a[176:177], v20 offset:0\n"
    "v_mfma_f32_16x16x32_bf16 v[168:171], a[200:203], a[108:111], v[168:171]\n"
    "ds_read_b128 a[128:131], v33 offset:32768\n"
    "v_mfma_f32_16x16x32_bf16 v[172:175], a[204:207], a[108:111], v[172:175]\n"
    "v_mfma_f32_16x16x32_bf16 v[140:143], a[204:207], a[104:107], v[140:143]\n"
    "ds_read_b64_tr_b16 a[178:179], v20 offset:1024\n"
    "v_mfma_f32_16x16x32_bf16 v[72:75], a[200:203], a[96:99], v[72:75]\n"
    "ds_read_b128 a[132:135], v33 offset:34816\n"
    "v_mfma_f32_16x16x32_bf16 v[104:107], a[200:203], a[100:103], v[104:107]\n"
    "buffer_load_dwordx4 v58, s[12:15], s73 offen offset:2048  lds\n"
    "v_mfma_f32_16x16x32_bf16 v[108:111], a[204:207], a[100:103], v[108:111]\n"
    "ds_read_b64_tr_b16 a[180:181], v21 offset:0\n"
    "v_mfma_f32_16x16x32_bf16 v[76:79], a[204:207], a[96:99], v[76:79]\n"
    "ds_read_b128 a[136:139], v33 offset:36864\n"
    "v_mfma_f32_16x16x32_bf16 v[80:83], a[208:211], a[96:99], v[80:83]\n"
    "v_mfma_f32_16x16x32_bf16 v[112:115], a[208:211], a[100:103], v[112:115]\n"
    "ds_read_b64_tr_b16 a[182:183], v21 offset:1024\n"
    "v_mfma_f32_16x16x32_bf16 v[116:119], a[212:215], a[100:103], v[116:119]\n"
    "ds_read_b128 a[140:143], v33 offset:38912\n"
    "v_mfma_f32_16x16x32_bf16 v[84:87], a[212:215], a[96:99], v[84:87]\n"
    "v_mfma_f32_16x16x32_bf16 v[144:147], a[208:211], a[104:107], v[144:147]\n"
    "ds_read_b64_tr_b16 a[184:185], v22 offset:0\n"
    "v_mfma_f32_16x16x32_bf16 v[176:179], a[208:211], a[108:111], v[176:179]\n"
    "ds_read_b128 a[144:147], v33 offset:40960\n"
    "v_mfma_f32_16x16x32_bf16 v[180:183], a[212:215], a[108:111], v[180:183]\n"
    "v_mfma_f32_16x16x32_bf16 v[148:151], a[212:215], a[104:107], v[148:151]\n"
    "ds_read_b64_tr_b16 a[186:187], v22 offset:1024\n"
    "v_mfma_f32_16x16x32_bf16 v[208:211], a[208:211], a[112:115], v[208:211]\n"
    "ds_read_b128 a[148:151], v33 offset:43008\n"
    "v_mfma_f32_16x16x32_bf16 v[240:243], a[208:211], a[116:119], v[240:243]\n"
    "buffer_load_dwordx4 v59, s[12:15], s73 offen offset:3072  lds\n"
    "v_mfma_f32_16x16x32_bf16 v[244:247], a[212:215], a[116:119], v[244:247]\n"
    "ds_read_b64_tr_b16 a[188:189], v23 offset:0\n"
    "v_mfma_f32_16x16x32_bf16 v[212:215], a[212:215], a[112:115], v[212:215]\n"
    "ds_read_b128 a[152:155], v33 offset:45056\n"
    "v_mfma_f32_16x16x32_bf16 a[16:19], a[208:211], a[120:123], a[16:19]\n"
    "v_mfma_f32_16x16x32_bf16 a[48:51], a[208:211], a[124:127], a[48:51]\n"
    "ds_read_b64_tr_b16 a[190:191], v23 offset:1024\n"
    "v_mfma_f32_16x16x32_bf16 a[52:55], a[212:215], a[124:127], a[52:55]\n"
    "ds_read_b128 a[156:159], v33 offset:47104\n"
    "v_mfma_f32_16x16x32_bf16 a[20:23], a[212:215], a[120:123], a[20:23]\n"
    "v_mfma_f32_16x16x32_bf16 a[24:27], a[216:219], a[120:123], a[24:27]\n"
    "s_add_u32 s76, s76, 1\n"
    "v_mfma_f32_16x16x32_bf16 a[56:59], a[216:219], a[124:127], a[56:59]\n"
    "v_readlane_b32 s72, v40, s75\n"
    "v_mfma_f32_16x16x32_bf16 a[60:63], a[220:223], a[124:127], a[60:63]\n"
    "v_mfma_f32_16x16x32_bf16 a[28:31], a[220:223], a[120:123], a[28:31]\n"
    "v_readlane_b32 s73, v41, s76\n"
    "v_mfma_f32_16x16x32_bf16 v[216:219], a[216:219], a[112:115], v[216:219]\n"
    "s_cmp_eq_u32 s75, 8\n"
    "v_mfma_f32_16x16x32_bf16 v[248:251], a[216:219], a[116:119], v[248:251]\n"
    "s_cselect_b32 s84, s88, s84\n"
    "s_cselect_b32 s85, s89, s85\n"
    "v_mfma_f32_16x16x32_bf16 v[252:255], a[220:223], a[116:119], v[252:255]\n"
    "v_mfma_f32_16x16x32_bf16 v[220:223], a[220:223], a[112:115], v[220:223]\n"
    "s_cmp_ge_u32 s75, 64\n"
    "s_cselect_b32 s72, 1073741824, s72\n"
    "v_mfma_f32_16x16x32_bf16 v[152:155], a[216:219], a[104:107], v[152:155]\n"
    "v_mfma_f32_16x16x32_bf16 v[184:187], a[216:219], a[108:111], v[184:187]\n"
    "s_cmp_ge_u32 s76, 64\n"
    "s_cselect_b32 s73, 1073741824, s73\n"
    "v_mfma_f32_16x16x32_bf16 v[188:191], a[220:223], a[108:111], v[188:191]\n"
    "v_mfma_f32_16x16x32_bf16 v[156:159], a[220:223], a[104:107], v[156:159]\n"
    "s_bitcmp1_b64 s[60:61], s79\n"
    "s_cbranch_scc1 Loe4%=\n"
    "Loe4r%=:\n"
    "v_mfma_f32_16x16x32_bf16 v[88:91], a[216:219], a[96:99], v[88:91]\n"
    "v_mfma_f32_16x16x32_bf16 v[120:123], a[216:219], a[100:103], v[120:123]\n"
    "s_add_u32 s79, s79, 1\n"
    "v_mfma_f32_16x16x32_bf16 v[124:127], a[220:223], a[100:103], v[124:127]\n"
    "v_mfma_f32_16x16x32_bf16 v[92:95], a[220:223], a[96:99], v[92:95]\n"
    "s_cmp_lg_u32 s41, 0\n"
    "s_cbranch_scc1 LW104%=\n"
    "LW4%=:\n"
    "s_waitcnt lgkmcnt(0)\n"
    "s_barrier\n"
    "LT5%=:\n"
    "v_mfma_f32_16x16x32_bf16 v[64:67], a[160:163], a[64:67], v[64:67]\n"
    "s_sub_u32 s78, s4, s56\n"
    "ds_read_b64_tr_b16 a[192:193], v16 offset:16384\n"
    "v_mfma_f32_16x16x32_bf16 v[96:99], a[160:163], a[68:71], v[96:99]\n"
    "s_memtime s[56:57]\n"
    "s_add_u32 m0, s70, 32768\n"
    "v_mfma_f32_16x16x32_bf16 v[100:103], a[164:167], a[68:71], v[100:103]\n"
    "buffer_load_dwordx4 v48, s[84:87], s72 offen offset:0 sc0 sc1 lds\n"
    "v_mfma_f32_16x16x32_bf16 v[68:71], a[164:167], a[64:67], v[68:71]\n"
    "ds_read_b64_tr_b16 a[194:195], v16 offset:17408\n"
    "v_mfma_f32_16x16x32_bf16 v[128:131], a[160:163], a[72:75], v[128:131]\n"
    "v_mfma_f32_16x16x32_bf16 v[160:163], a[160:163], a[76:79], v[160:163]\n"
    "buffer_load_dwordx4 v49, s[84:87], s72 offen offset:1024 sc0 sc1 lds\n"
    "v_mfma_f32_16x16x32_bf16 v[164:167], a[164:167], a[76:79], v[164:167]\n"
    "ds_read_b64_tr_b16 a[196:197], v17 offset:16384\n"
    "v_mfma_f32_16x16x32_bf16 v[132:135], a[164:167], a[72:75], v[132:135]\n"
    "v_mfma_f32_16x16x32_bf16 v[192:195], a[160:163], a[80:83], v[192:195]\n"
    "buffer_load_dwordx4 v50, s[84:87], s72 offen offset:2048 sc0 sc1 lds\n"
    "v_mfma_f32_16x16x32_bf16 v[224:227], a[160:163], a[84:87], v[224:227]\n"
    "ds_read_b64_tr_b16 a[198:199], v17 offset:17408\n"
    "v_mfma_f32_16x16x32_bf16 v[228:231], a[164:167], a[84:87], v[228:231]\n"
    "v_mfma_f32_16x16x32_bf16 v[196:199], a[164:167], a[80:83], v[196:199]\n"
    "buffer_load_dwordx4 v51, s[84:87], s72 offen offset:3072 sc0 sc1 lds\n"
    "v_mfma_f32_16x16x32_bf16 a[0:3], a[160:163], a[88:91], a[0:3]\n"
    "ds_read_b64_tr_b16 a[200:201], v18 offset:16384\n"
    "v_mfma_f32_16x16x32_bf16 a[32:35], a[160:163], a[92:95], a[32:35]\n"
    "s_add_u32 m0, s70, 36864\n"
    "v_mfma_f32_16x16x32_bf16 a[36:39], a[164:167], a[92:95], a[36:39]\n"
    "buffer_load_dwordx4 v52, s[84:87], s72 offen offset:0 sc0 sc1 lds\n"
    "v_mfma_f32_16x16x32_bf16 a[4:7], a[164:167], a[88:91], a[4:7]\n"
    "ds_read_b64_tr_b16 a[202:203], v18 offset:17408\n"
    "v_mfma_f32_16x16x32_bf16 a[8:11], a[168:171], a[88:91], a[8:11]\n"
    "v_mfma_f32_16x16x32_bf16 a[40:43], a[168:171], a[92:95], a[40:43]\n"
    "buffer_load_dwordx4 v53, s[84:87], s72 offen offset:1024 sc0 sc1 lds\n"
    "v_mfma_f32_16x16x32_bf16 a[44:47], a[172:175], a[92:95], a[44:47]\n"
    "ds_read_b64_tr_b16 a[204:205], v19 offset:16384\n"
    "v_mfma_f32_16x16x32_bf16 a[12:15], a[172:175], a[88:91], a[12:15]\n"
    "v_mfma_f32_16x16x32_bf16 v[200:203], a[168:171], a[80:83], v[200:203]\n"
    "buffer_load_dwordx4 v54, s[84:87], s72 offen offset:2048 sc0 sc1 lds\n"
    "v_mfma_f32_16x16x32_bf16 v[232:235], a[168:171], a[84:87], v[232:235]\n"
    "ds_read_b64_tr_b16 a[206:207], v19 offset:17408\n"
    "v_mfma_f32_16x16x32_bf16 v[236:239], a[172:175], a[84:87], v[236:239]\n"
    "v_mfma_f32_16x16x32_bf16 v[204:207], a[172:175], a[80:83], v[204:207]\n"
    "buffer_load_dwordx4 v55, s[84:87], s72 offen offset:3072 sc0 sc1 lds\n"
    "v_mfma_f32_16x16x32_bf16 v[136:139], a[168:171], a[72:75], v[136:139]\n"
    "ds_read_b64_tr_b16 a[208:209], v20 offset:16384\n"
    "v_mfma_f32_16x16x32_bf16 v[168:171], a[168:171], a[76:79], v[168:171]\n"
    "v_mfma_f32_16x16x32_bf16 v[172:175], a[172:175], a[76:79], v[172:175]\n"
    "v_mfma_f32_16x16x32_bf16 v[140:143], a[172:175], a[72:75], v[140:143]\n"
    "ds_read_b64_tr_b16 a[210:211], v20 offset:17408\n"
    "v_mfma_f32_16x16x32_bf16 v[72:75], a[168:171], a[64:67], v[72:75]\n"
    "s_add_u32 m0, s71, 65536\n"
    "v_mfma_f32_16x16x32_bf16 v[104:107], a[168:171], a[68:71], v[104:107]\n"
    "buffer_load_dwordx4 v56, s[12:15], s73 offen offset:0  lds\n"
    "v_mfma_f32_16x16x32_bf16 v[108:111], a[172:175], a[68:71], v[108:111]\n"
    "ds_read_b64_tr_b16 a[212:213], v21 offset:16384\n"
    "v_mfma_f32_16x16x32_bf16 v[76:79], a[172:175], a[64:67], v[76:79]\n"
    "v_mfma_f32_16x16x32_bf16 v[80:83], a[176:179], a[64:67], v[80:83]\n"
    "v_mfma_f32_16x16x32_bf16 v[112:115], a[176:179], a[68:71], v[112:115]\n"
    "ds_read_b64_tr_b16 a[214:215], v21 offset:17408\n"
    "v_mfma_f32_16x16x32_bf16 v[116:119], a[180:183], a[68:71], v[116:119]\n"
    "v_mfma_f32_16x16x32_bf16 v[84:87], a[180:183], a[64:67], v[84:87]\n"
    "buffer_load_dwordx4 v57, s[12:15], s73 offen offset:1024  lds\n"
    "v_mfma_f32_16x16x32_bf16 v[144:147], a[176:179], a[72:75], v[144:147]\n"
    "ds_read_b64_tr_b16 a[216:217], v22 offset:16384\n"
    "v_mfma_f32_16x16x32_bf16 v[176:179], a[176:179], a[76:79], v[176:179]\n"
    "v_mfma_f32_16x16x32_bf16 v[180:183], a[180:183], a[76:79], v[180:183]\n"
    "v_mfma_f32_16x16x32_bf16 v[148:151], a[180:183], a[72:75], v[148:151]\n"
    "ds_read_b64_tr_b16 a[218:219], v22 offset:17408\n"
    "v_mfma_f32_16x16x32_bf16 v[208:211], a[176:179], a[80:83], v[208:211]\n"
    "v_mfma_f32_16x16x32_bf16 v[240:243], a[176:179], a[84:87], v[240:243]\n"
    "buffer_load_dwordx4 v58, s[12:15], s73 offen offset:2048  lds\n"
    "v_mfma_f32_16x16x32_bf16 v[244:247], a[180:183], a[84:87], v[244:247]\n"
    "ds_read_b64_tr_b16 a[220:221], v23 offset:16384\n"
    "v_mfma_f32_16x16x32_bf16 v[212:215], a[180:183], a[80:83], v[212:215]\n"
    "v_mfma_f32_16x16x32_bf16 a[16:19], a[176:179], a[88:91], a[16:19]\n"
    "v_mfma_f32_16x16x32_bf16 a[48:51], a[176:179], a[92:95], a[48:51]\n"
    "ds_read_b64_tr_b16 a[222:223], v23 offset:17408\n"
    "v_mfma_f32_16x16x32_bf16 a[52:55], a[180:183], a[92:95], a[52:55]\n"
    "v_mfma_f32_16x16x32_bf16 a[20:23], a[180:183], a[88:91], a[20:23]\n"
    "buffer_load_dwordx4 v59, s[12:15], s73 offen offset:3072  lds\n"
    "v_mfma_f32_16x16x32_bf16 a[24:27], a[184:187], a[88:91], a[24:27]\n"
    "v_mfma_f32_16x16x32_bf16 a[56:59], a[184:187], a[92:95], a[56:59]\n"
    "s_add_u32 s73, s73, 0x200000\n"
    "v_mfma_f32_16x16x32_bf16 a[60:63], a[188:191], a[92:95], a[60:63]\n"
    "s_add_u32 s75, s75, 1\n"
    "v_mfma_f32_16x16x32_bf16 a[28:31], a[188:191], a[88:91], a[28:31]\n"
    "v_mfma_f32_16x16x32_bf16 v[216:219], a[184:187], a[80:83], v[216:219]\n"
    "s_cmp_gt_u32 s78, s30\n"
    "s_cbranch_scc1 Lad5%=\n"
    "Lad5r%=:\n"
    "v_mfma_f32_16x16x32_bf16 v[248:251], a[184:187], a[84:87], v[248:251]\n"
    "v_mfma_f32_16x16x32_bf16 v[252:255], a[188:191], a[84:87], v[252:255]\n"
    "v_mfma_f32_16x16x32_bf16 v[220:223], a[188:191], a[80:83], v[220:223]\n"
    "v_mfma_f32_16x16x32_bf16 v[152:155], a[184:187], a[72:75], v[152:155]\n"
    "v_mfma_f32_16x16x32_bf16 v[184:187], a[184:187], a[76:79], v[184:187]\n"
    "v_mfma_f32_16x16x32_bf16 v[188:191], a[188:191], a[76:79], v[188:191]\n"
    "v_mfma_f32_16x16x32_bf16 v[156:159], a[188:191], a[72:75], v[156:159]\n"
    "v_mfma_f32_16x16x32_bf16 v[88:91], a[184:187], a[64:67], v[88:91]\n"
    "v_mfma_f32_16x16x32_bf16 v[120:123], a[184:187], a[68:71], v[120:123]\n"
    "v_mfma_f32_16x16x32_bf16 v[124:127], a[188:191], a[68:71], v[124:127]\n"
    "v_mfma_f32_16x16x32_bf16 v[92:95], a[188:191], a[64:67], v[92:95]\n"
    "s_waitcnt vmcnt(20) lgkmcnt(0)\n"
    "s_barrier\n"
    "v_mfma_f32_16x16x32_bf16 v[64:67], a[192:195], a[128:131], v[64:67]\n"
    "ds_read_b64_tr_b16 a[160:161], v24 offset:0\n"
    "s_memtime s[4:5]\n"
    "v_mfma_f32_16x16x32_bf16 v[96:99], a[192:195], a[132:135], v[96:99]\n"
    "ds_read_b128 a[64:67], v32 offset:0\n"
    "v_mfma_f32_16x16x32_bf16 v[100:103], a[196:199], a[132:135], v[100:103]\n"
    "s_bitcmp1_b64 s[64:65], s79\n"
    "s_cbranch_scc1 Lfl5%=\n"
    "Lfl5r%=:\n"
    "v_mfma_f32_16x16x32_bf16 v[68:71], a[196:199], a[128:131], v[68:71]\n"
    "ds_read_b64_tr_b16 a[162:163], v24 offset:1024\n"
    "v_mfma_f32_16x16x32_bf16 v[128:131], a[192:195], a[136:139], v[128:131]\n"
    "ds_read_b128 a[68:71], v32 offset:2048\n"
    "s_add_u32 m0, s71, 81920\n"
    "v_mfma_f32_16x16x32_bf16 v[160:163], a[192:195], a[140:143], v[160:163]\n"
    "buffer_load_dwordx4 v56, s[12:15], s73 offen offset:0  lds\n"
    "v_mfma_f32_16x16x32_bf16 v[164:167], a[196:199], a[140:143], v[164:167]\n"
    "ds_read_b64_tr_b16 a[164:165], v25 offset:0\n"
    "v_mfma_f32_16x16x32_bf16 v[132:135], a[196:199], a[136:139], v[132:135]\n"
    "ds_read_b128 a[72:75], v32 offset:4096\n"
    "v_mfma_f32_16x16x32_bf16 v[192:195], a[192:195], a[144:147], v[192:195]\n"
    "v_mfma_f32_16x16x32_bf16 v[224:227], a[192:195], a[148:151], v[224:227]\n"
    "ds_read_b64_tr_b16 a[166:167], v25 offset:1024\n"
    "v_mfma_f32_16x16x32_bf16 v[228:231], a[196:199], a[148:151], v[228:231]\n"
    "ds_read_b128 a[76:79], v32 offset:6144\n"
    "v_mfma_f32_16x16x32_bf16 v[196:199], a[196:199], a[144:147], v[196:199]\n"
    "v_mfma_f32_16x16x32_bf16 a[0:3], a[192:195], a[152:155], a[0:3]\n"
    "ds_read_b64_tr_b16 a[168:169], v26 offset:0\n"
    "v_mfma_f32_16x16x32_bf16 a[32:35], a[192:195], a[156:159], a[32:35]\n"
    "ds_read_b128 a[80:83], v32 offset:8192\n"
    "v_mfma_f32_16x16x32_bf16 a[36:39], a[196:199], a[156:159], a[36:39]\n"
    "v_mfma_f32_16x16x32_bf16 a[4:7], a[196:199], a[152:155], a[4:7]\n"
    "ds_read_b64_tr_b16 a[170:171], v26 offset:1024\n"
    "v_mfma_f32_16x16x32_bf16 a[8:11], a[200:203], a[152:155], a[8:11]\n"
    "ds_read_b128 a[84:87], v32 offset:10240\n"
    "v_mfma_f32_16x16x32_bf16 a[40:43], a[200:203], a[156:159], a[40:43]\n"
    "buffer_load_dwordx4 v57, s[12:15], s73 offen offset:1024  lds\n"
    "v_mfma_f32_16x16x32_bf16 a[44:47], a[204:207], a[156:159], a[44:47]\n"
    "ds_read_b64_tr_b16 a[172:173], v27 offset:0\n"
    "v_mfma_f32_16x16x32_bf16 a[12:15], a[204:207], a[152:155], a[12:15]\n"
    "ds_read_b128 a[88:91], v32 offset:12288\n"
    "v_mfma_f32_16x16x32_bf16 v[200:203], a[200:203], a[144:147], v[200:203]\n"
    "v_mfma_f32_16x16x32_bf16 v[232:235], a[200:203], a[148:151], v[232:235]\n"
    "ds_read_b64_tr_b16 a[174:175], v27 offset:1024\n"
    "v_mfma_f32_16x16x32_bf16 v[236:239], a[204:207], a[148:151], v[236:239]\n"
    "ds_read_b128 a[92:95], v32 offset:14336\n"
    "v_mfma_f32_16x16x32_bf16 v[204:207], a[204:207], a[144:147], v[204:207]\n"
    "v_mfma_f32_16x16x32_bf16 v[136:139], a[200:203], a[136:139], v[136:139]\n"
    "ds_read_b64_tr_b16 a[176:177], v28 offset:0\n"
    "v_mfma_f32_16x16x32_bf16 v[168:171], a[200:203], a[140:143], v[168:171]\n"
    "ds_read_b128 a[96:99], v33 offset:0\n"
    "v_mfma_f32_16x16x32_bf16 v[172:175], a[204:207], a[140:143], v[172:175]\n"
    "v_mfma_f32_16x16x32_bf16 v[140:143], a[204:207], a[136:139], v[140:143]\n"
    "ds_read_b64_tr_b16 a[178:179], v28 offset:1024\n"
    "v_mfma_f32_16x16x32_bf16 v[72:75], a[200:203], a[128:131], v[72:75]\n"
    "ds_read_b128 a[100:103], v33 offset:2048\n"
    "v_mfma_f32_16x16x32_bf16 v[104:107], a[200:203], a[132:135], v[104:107]\n"
    "buffer_load_dwordx4 v58, s[12:15], s73 offen offset:2048  lds\n"
    "v_mfma_f32_16x16x32_bf16 v[108:111], a[204:207], a[132:135], v[108:111]\n"
    "ds_read_b64_tr_b16 a[180:181], v29 offset:0\n"
    "v_mfma_f32_16x16x32_bf16 v[76:79], a[204:207], a[128:131], v[76:79]\n"
    "ds_read_b128 a[104:107], v33 offset:4096\n"
    "v_mfma_f32_16x16x32_bf16 v[80:83], a[208:211], a[128:131], v[80:83]\n"
    "v_mfma_f32_16x16x32_bf16 v[112:115], a[208:211], a[132:135], v[112:115]\n"
    "ds_read_b64_tr_b16 a[182:183], v29 offset:1024\n"
    "v_mfma_f32_16x16x32_bf16 v[116:119], a[212:215], a[132:135], v[116:119]\n"
    "ds_read_b128 a[108:111], v33 offset:6144\n"
    "v_mfma_f32_16x16x32_bf16 v[84:87], a[212:215], a[128:131], v[84:87]\n"
    "v_mfma_f32_16x16x32_bf16 v[144:147], a[208:211], a[136:139], v[144:147]\n"
    "ds_read_b64_tr_b16 a[184:185], v30 offset:0\n"
    "v_mfma_f32_16x16x32_bf16 v[176:179], a[208:211], a[140:143], v[176:179]\n"
    "ds_read_b128 a[112:115], v33 offset:8192\n"
    "v_mfma_f32_16x16x32_bf16 v[180:183], a[212:215], a[140:143], v[180:183]\n"
    "v_mfma_f32_16x16x32_bf16 v[148:151], a[212:215], a[136:139], v[148:151]\n"
    "ds_read_b64_tr_b16 a[186:187], v30 offset:1024\n"
    "v_mfma_f32_16x16x32_bf16 v[208:211], a[208:211], a[144:147], v[208:211]\n"
    "ds_read_b128 a[116:119], v33 offset:10240\n"
    "v_mfma_f32_16x16x32_bf16 v[240:243], a[208:211], a[148:151], v[240:243]\n"
    "buffer_load_dwordx4 v59, s[12:15], s73 offen offset:3072  lds\n"
    "v_mfma_f32_16x16x32_bf16 v[244:247], a[212:215], a[148:151], v[244:247]\n"
    "ds_read_b64_tr_b16 a[188:189], v31 offset:0\n"
    "v_mfma_f32_16x16x32_bf16 v[212:215], a[212:215], a[144:147], v[212:215]\n"
    "ds_read_b128 a[120:123], v33 offset:12288\n"
    "v_mfma_f32_16x16x32_bf16 a[16:19], a[208:211], a[152:155], a[16:19]\n"
    "v_mfma_f32_16x16x32_bf16 a[48:51], a[208:211], a[156:159], a[48:51]\n"
    "ds_read_b64_tr_b16 a[190:191], v31 offset:1024\n"
    "v_mfma_f32_16x16x32_bf16 a[52:55], a[212:215], a[156:159], a[52:55]\n"
    "ds_read_b128 a[124:127], v33 offset:14336\n"
    "v_mfma_f32_16x16x32_bf16 a[20:23], a[212:215], a[152:155], a[20:23]\n"
    "v_mfma_f32_16x16x32_bf16 a[24:27], a[216:219], a[152:155], a[24:27]\n"
    "s_add_u32 s76, s76, 1\n"
    "v_mfma_f32_16x16x32_bf16 a[56:59], a[216:219], a[156:159], a[56:59]\n"
    "v_readlane_b32 s72, v40, s75\n"
    "v_mfma_f32_16x16x32_bf16 a[60:63], a[220:223], a[156:159], a[60:63]\n"
    "v_mfma_f32_16x16x32_bf16 a[28:31], a[220:223], a[152:155], a[28:31]\n"
    "v_readlane_b32 s73, v41, s76\n"
    "v_mfma_f32_16x16x32_bf16 v[216:219], a[216:219], a[144:147], v[216:219]\n"
    "s_cmp_eq_u32 s75, 8\n"
    "v_mfma_f32_16x16x32_bf16 v[248:251], a[216:219], a[148:151], v[248:251]\n"
    "s_cselect_b32 s84, s88, s84\n"
    "s_cselect_b32 s85, s89, s85\n"
    "v_mfma_f32_16x16x32_bf16 v[252:255], a[220:223], a[148:151], v[252:255]\n"
    "v_mfma_f32_16x16x32_bf16 v[220:223], a[220:223], a[144:147], v[220:223]\n"
    "s_cmp_ge_u32 s75, 64\n"
    "s_cselect_b32 s72, 1073741824, s72\n"
    "v_mfma_f32_16x16x32_bf16 v[152:155], a[216:219], a[136:139], v[152:155]\n"
    "v_mfma_f32_16x16x32_bf16 v[184:187], a[216:219], a[140:143], v[184:187]\n"
    "s_cmp_ge_u32 s76, 64\n"
    "s_cselect_b32 s73, 1073741824, s73\n"
    "v_mfma_f32_16x16x32_bf16 v[188:191], a[220:223], a[140:143], v[188:191]\n"
    "v_mfma_f32_16x16x32_bf16 v[156:159], a[220:223], a[136:139], v[156:159]\n"
    "s_bitcmp1_b64 s[60:61], s79\n"
    "s_cbranch_scc1 Loe5%=\n"
    "Loe5r%=:\n"
    "v_mfma_f32_16x16x32_bf16 v[88:91], a[216:219], a[128:131], v[88:91]\n"
    "v_mfma_f32_16x16x32_bf16 v[120:123], a[216:219], a[132:135], v[120:123]\n"
    "s_add_u32 s79, s79, 1\n"
    "v_mfma_f32_16x16x32_bf16 v[124:127], a[220:223], a[132:135], v[124:127]\n"
    "v_mfma_f32_16x16x32_bf16 v[92:95], a[220:223], a[128:131], v[92:95]\n"
    "s_cmp_lg_u32 s41, 0\n"
    "s_cbranch_scc1 LW105%=\n"
    "LW5%=:\n"
    "s_waitcnt lgkmcnt(0)\n"
    "s_barrier\n"
    "LT6%=:\n"
    "v_mfma_f32_16x16x32_bf16 v[64:67], a[160:163], a[64:67], v[64:67]\n"
    "s_sub_u32 s78, s4, s56\n"
    "ds_read_b64_tr_b16 a[192:193], v24 offset:16384\n"
    "v_mfma_f32_16x16x32_bf16 v[96:99], a[160:163], a[68:71], v[96:99]\n"
    "s_memtime s[56:57]\n"
    "s_add_u32 m0, s70, 0\n"
    "v_mfma_f32_16x16x32_bf16 v[100:103], a[164:167], a[68:71], v[100:103]\n"
    "buffer_load_dwordx4 v48, s[84:87], s72 offen offset:0 sc0 sc1 lds\n"
    "v_mfma_f32_16x16x32_bf16 v[68:71], a[164:167], a[64:67], v[68:71]\n"
    "ds_read_b64_tr_b16 a[194:195], v24 offset:17408\n"
    "v_mfma_f32_16x16x32_bf16 v[128:131], a[160:163], a[72:75], v[128:131]\n"
    "v_mfma_f32_16x16x32_bf16 v[160:163], a[160:163], a[76:79], v[160:163]\n"
    "buffer_load_dwordx4 v49, s[84:87], s72 offen offset:1024 sc0 sc1 lds\n"
    "v_mfma_f32_16x16x32_bf16 v[164:167], a[164:167], a[76:79], v[164:167]\n"
    "ds_read_b64_tr_b16 a[196:197], v25 offset:16384\n"
    "v_mfma_f32_16x16x32_bf16 v[132:135], a[164:167], a[72:75], v[132:135]\n"
    "v_mfma_f32_16x16x32_bf16 v[192:195], a[160:163], a[80:83], v[192:195]\n"
    "buffer_load_dwordx4 v50, s[84:87], s72 offen offset:2048 sc0 sc1 lds\n"
    "v_mfma_f32_16x16x32_bf16 v[224:227], a[160:163], a[84:87], v[224:227]\n"
    "ds_read_b64_tr_b16 a[198:199], v25 offset:17408\n"
    "v_mfma_f32_16x16x32_bf16 v[228:231], a[164:167], a[84:87], v[228:231]\n"
    "v_mfma_f32_16x16x32_bf16 v[196:199], a[164:167], a[80:83], v[196:199]\n"
    "buffer_load_dwordx4 v51, s[84:87], s72 offen offset:3072 sc0 sc1 lds\n"
    "v_mfma_f32_16x16x32_bf16 a[0:3], a[160:163], a[88:91], a[0:3]\n"
    "ds_read_b64_tr_b16 a[200:201], v26 offset:16384\n"
    "v_mfma_f32_16x16x32_bf16 a[32:35], a[160:163], a[92:95], a[32:35]\n"
    "s_add_u32 m0, s70, 4096\n"
    "v_mfma_f32_16x16x32_bf16 a[36:39], a[164:167], a[92:95], a[36:39]\n"
    "buffer_load_dwordx4 v52, s[84:87], s72 offen offset:0 sc0 sc1 lds\n"
    "v_mfma_f32_16x16x32_bf16 a[4:7], a[164:167], a[88:91], a[4:7]\n"
    "ds_read_b64_tr_b16 a[202:203], v26 offset:17408\n"
    "v_mfma_f32_16x16x32_bf16 a[8:11], a[168:171], a[88:91], a[8:11]\n"
    "v_mfma_f32_16x16x32_bf16 a[40:43], a[168:171], a[92:95], a[40:43]\n"
    "buffer_load_dwordx4 v53, s[84:87], s72 offen offset:1024 sc0 sc1 lds\n"
    "v_mfma_f32_16x16x32_bf16 a[44:47], a[172:175], a[92:95], a[44:47]\n"
    "ds_read_b64_tr_b16 a[204:205], v27 offset:16384\n"
    "v_mfma_f32_16x16x32_bf16 a[12:15], a[172:175], a[88:91], a[12:15]\n"
    "v_mfma_f32_16x16x32_bf16 v[200:203], a[168:171], a[80:83], v[200:203]\n"
    "buffer_load_dwordx4 v54, s[84:87], s72 offen offset:2048 sc0 sc1 lds\n"
    "v_mfma_f32_16x16x32_bf16 v[232:235], a[168:171], a[84:87], v[232:235]\n"
    "ds_read_b64_tr_b16 a[206:207], v27 offset:17408\n"
    "v_mfma_f32_16x16x32_bf16 v[236:239], a[172:175], a[84:87], v[236:239]\n"
    "v_mfma_f32_16x16x32_bf16 v[204:207], a[172:175], a[80:83], v[204:207]\n"
    "buffer_load_dwordx4 v55, s[84:87], s72 offen offset:3072 sc0 sc1 lds\n"
    "v_mfma_f32_16x16x32_bf16 v[136:139], a[168:171], a[72:75], v[136:139]\n"
    "ds_read_b64_tr_b16 a[208:209], v28 offset:16384\n"
    "v_mfma_f32_16x16x32_bf16 v[168:171], a[168:171], a[76:79], v[168:171]\n"
    "v_mfma_f32_16x16x32_bf16 v[172:175], a[172:175], a[76:79], v[172:175]\n"
    "v_mfma_f32_16x16x32_bf16 v[140:143], a[172:175], a[72:75], v[140:143]\n"
    "ds_read_b64_tr_b16 a[210:211], v28 offset:17408\n"
    "v_mfma_f32_16x16x32_bf16 v[72:75], a[168:171], a[64:67], v[72:75]\n"
    "s_add_u32 m0, s71, 0\n"
    "v_mfma_f32_16x16x32_bf16 v[104:107], a[168:171], a[68:71], v[104:107]\n"
    "buffer_load_dwordx4 v56, s[12:15], s73 offen offset:0  lds\n"
    "v_mfma_f32_16x16x32_bf16 v[108:111], a[172:175], a[68:71], v[108:111]\n"
    "ds_read_b64_tr_b16 a[212:213], v29 offset:16384\n"
    "v_mfma_f32_16x16x32_bf16 v[76:79], a[172:175], a[64:67], v[76:79]\n"
    "v_mfma_f32_16x16x32_bf16 v[80:83], a[176:179], a[64:67], v[80:83]\n"
    "v_mfma_f32_16x16x32_bf16 v[112:115], a[176:179], a[68:71], v[112:115]\n"
    "ds_read_b64_tr_b16 a[214:215], v29 offset:17408\n"
    "v_mfma_f32_16x16x32_bf16 v[116:119], a[180:183], a[68:71], v[116:119]\n"
    "v_mfma_f32_16x16x32_bf16 v[84:87], a[180:183], a[64:67], v[84:87]\n"
    "buffer_load_dwordx4 v57, s[12:15], s73 offen offset:1024  lds\n"
    "v_mfma_f32_16x16x32_bf16 v[144:147], a[176:179], a[72:75], v[144:147]\n"
    "ds_read_b64_tr_b16 a[216:217], v30 offset:16384\n"
    "v_mfma_f32_16x16x32_bf16 v[176:179], a[176:179], a[76:79], v[176:179]\n"
    "v_mfma_f32_16x16x32_bf16 v[180:183], a[180:183], a[76:79], v[180:183]\n"
    "v_mfma_f32_16x16x32_bf16 v[148:151], a[180:183], a[72:75], v[148:151]\n"
    "ds_read_b64_tr_b16 a[218:219], v30 offset:17408\n"
    "v_mfma_f32_16x16x32_bf16 v[208:211], a[176:179], a[80:83], v[208:211]\n"
    "v_mfma_f32_16x16x32_bf16 v[240:243], a[176:179], a[84:87], v[240:243]\n"
    "buffer_load_dwordx4 v58, s[12:15], s73 offen offset:2048  lds\n"
    "v_mfma_f32_16x16x32_bf16 v[244:247], a[180:183], a[84:87], v[244:247]\n"
    "ds_read_b64_tr_b16 a[220:221], v31 offset:16384\n"
    "v_mfma_f32_16x16x32_bf16 v[212:215], a[180:183], a[80:83], v[212:215]\n"
    "v_mfma_f32_16x16x32_bf16 a[16:19], a[176:179], a[88:91], a[16:19]\n"
    "v_mfma_f32_16x16x32_bf16 a[48:51], a[176:179], a[92:95], a[48:51]\n"
    "ds_read_b64_tr_b16 a[222:223], v31 offset:17408\n"
    "v_mfma_f32_16x16x32_bf16 a[52:55], a[180:183], a[92:95], a[52:55]\n"
    "v_mfma_f32_16x16x32_bf16 a[20:23], a[180:183], a[88:91], a[20:23]\n"
    "buffer_load_dwordx4 v59, s[12:15], s73 offen offset:3072  lds\n"
    "v_mfma_f32_16x16x32_bf16 a[24:27], a[184:187], a[88:91], a[24:27]\n"
    "v_mfma_f32_16x16x32_bf16 a[56:59], a[184:187], a[92:95], a[56:59]\n"
    "s_add_u32 s73, s73, 0x200000\n"
    "v_mfma_f32_16x16x32_bf16 a[60:63], a[188:191], a[92:95], a[60:63]\n"
    "s_add_u32 s75, s75, 1\n"
    "v_mfma_f32_16x16x32_bf16 a[28:31], a[188:191], a[88:91], a[28:31]\n"
    "v_mfma_f32_16x16x32_bf16 v[216:219], a[184:187], a[80:83], v[216:219]\n"
    "s_cmp_gt_u32 s78, s30\n"
    "s_cbranch_scc1 Lad6%=\n"
    "Lad6r%=:\n"
    "v_mfma_f32_16x16x32_bf16 v[248:251], a[184:187], a[84:87], v[248:251]\n"
    "v_mfma_f32_16x16x32_bf16 v[252:255], a[188:191], a[84:87], v[252:255]\n"
    "v_mfma_f32_16x16x32_bf16 v[220:223], a[188:191], a[80:83], v[220:223]\n"
    "v_mfma_f32_16x16x32_bf16 v[152:155], a[184:187], a[72:75], v[152:155]\n"
    "v_mfma_f32_16x16x32_bf16 v[184:187], a[184:187], a[76:79], v[184:187]\n"
    "v_mfma_f32_16x16x32_bf16 v[188:191], a[188:191], a[76:79], v[188:191]\n"
    "v_mfma_f32_16x16x32_bf16 v[156:159], a[188:191], a[72:75], v[156:159]\n"
    "v_mfma_f32_16x16x32_bf16 v[88:91], a[184:187], a[64:67], v[88:91]\n"
    "v_mfma_f32_16x16x32_bf16 v[120:123], a[184:187], a[68:71], v[120:123]\n"
    "v_mfma_f32_16x16x32_bf16 v[124:127], a[188:191], a[68:71], v[124:127]\n"
    "v_mfma_f32_16x16x32_bf16 v[92:95], a[188:191], a[64:67], v[92:95]\n"
    "s_waitcnt vmcnt(20) lgkmcnt(0)\n"
    "s_barrier\n"
    "v_mfma_f32_16x16x32_bf16 v[64:67], a[192:195], a[96:99], v[64:67]\n"
    "ds_read_b64_tr_b16 a[160:161], v24 offset:32768\n"
    "s_memtime s[4:5]\n"
    "v_mfma_f32_16x16x32_bf16 v[96:99], a[192:195], a[100:103], v[96:99]\n"
    "ds_read_b128 a[64:67], v32 offset:32768\n"
    "v_mfma_f32_16x16x32_bf16 v[100:103], a[196:199], a[100:103], v[100:103]\n"
    "s_bitcmp1_b64 s[64:65], s79\n"
    "s_cbranch_scc1 Lfl6%=\n"
    "Lfl6r%=:\n"
    "v_mfma_f32_16x16x32_bf16 v[68:71], a[196:199], a[96:99], v[68:71]\n"
    "ds_read_b64_tr_b16 a[162:163], v24 offset:33792\n"
    "v_mfma_f32_16x16x32_bf16 v[128:131], a[192:195], a[104:107], v[128:131]\n"
    "ds_read_b128 a[68:71], v32 offset:34816\n"
    "s_add_u32 m0, s71, 16384\n"
    "v_mfma_f32_16x16x32_bf16 v[160:163], a[192:195], a[108:111], v[160:163]\n"
    "buffer_load_dwordx4 v56, s[12:15], s73 offen offset:0  lds\n"
    "v_mfma_f32_16x16x32_bf16 v[164:167], a[196:199], a[108:111], v[164:167]\n"
    "ds_read_b64_tr_b16 a[164:165], v25 offset:32768\n"
    "v_mfma_f32_16x16x32_bf16 v[132:135], a[196:199], a[104:107], v[132:135]\n"
    "ds_read_b128 a[72:75], v32 offset:36864\n"
    "v_mfma_f32_16x16x32_bf16 v[192:195], a[192:195], a[112:115], v[192:195]\n"
    "v_mfma_f32_16x16x32_bf16 v[224:227], a[192:195], a[116:119], v[224:227]\n"
    "ds_read_b64_tr_b16 a[166:167], v25 offset:33792\n"
    "v_mfma_f32_16x16x32_bf16 v[228:231], a[196:199], a[116:119], v[228:231]\n"
    "ds_read_b128 a[76:79], v32 offset:38912\n"
    "v_mfma_f32_16x16x32_bf16 v[196:199], a[196:199], a[112:115], v[196:199]\n"
    "v_mfma_f32_16x16x32_bf16 a[0:3], a[192:195], a[120:123], a[0:3]\n"
    "ds_read_b64_tr_b16 a[168:169], v26 offset:32768\n"
    "v_mfma_f32_16x16x32_bf16 a[32:35], a[192:195], a[124:127], a[32:35]\n"
    "ds_read_b128 a[80:83], v32 offset:40960\n"
    "v_mfma_f32_16x16x32_bf16 a[36:39], a[196:199], a[124:127], a[36:39]\n"
    "v_mfma_f32_16x16x32_bf16 a[4:7], a[196:199], a[120:123], a[4:7]\n"
    "ds_read_b64_tr_b16 a[170:171], v26 offset:33792\n"
    "v_mfma_f32_16x16x32_bf16 a[8:11], a[200:203], a[120:123], a[8:11]\n"
    "ds_read_b128 a[84:87], v32 offset:43008\n"
    "v_mfma_f32_16x16x32_bf16 a[40:43], a[200:203], a[124:127], a[40:43]\n"
    "buffer_load_dwordx4 v57, s[12:15], s73 offen offset:1024  lds\n"
    "v_mfma_f32_16x16x32_bf16 a[44:47], a[204:207], a[124:127], a[44:47]\n"
    "ds_read_b64_tr_b16 a[172:173], v27 offset:32768\n"
    "v_mfma_f32_16x16x32_bf16 a[12:15], a[204:207], a[120:123], a[12:15]\n"
    "ds_read_b128 a[88:91], v32 offset:45056\n"
    "v_mfma_f32_16x16x32_bf16 v[200:203], a[200:203], a[112:115], v[200:203]\n"
    "v_mfma_f32_16x16x32_bf16 v[232:235], a[200:203], a[116:119], v[232:235]\n"
    "ds_read_b64_tr_b16 a[174:175], v27 offset:33792\n"
    "v_mfma_f32_16x16x32_bf16 v[236:239], a[204:207], a[116:119], v[236:239]\n"
    "ds_read_b128 a[92:95], v32 offset:47104\n"
    "v_mfma_f32_16x16x32_bf16 v[204:207], a[204:207], a[112:115], v[204:207]\n"
    "v_mfma_f32_16x16x32_bf16 v[136:139], a[200:203], a[104:107], v[136:139]\n"
    "ds_read_b64_tr_b16 a[176:177], v28 offset:32768\n"
    "v_mfma_f32_16x16x32_bf16 v[168:171], a[200:203], a[108:111], v[168:171]\n"
    "ds_read_b128 a[128:131], v33 offset:32768\n"
    "v_mfma_f32_16x16x32_bf16 v[172:175], a[204:207], a[108:111], v[172:175]\n"
    "v_mfma_f32_16x16x32_bf16 v[140:143], a[204:207], a[104:107], v[140:143]\n"
    "ds_read_b64_tr_b16 a[178:179], v28 offset:33792\n"
    "v_mfma_f32_16x16x32_bf16 v[72:75], a[200:203], a[96:99], v[72:75]\n"
    "ds_read_b128 a[132:135], v33 offset:34816\n"
    "v_mfma_f32_16x16x32_bf16 v[104:107], a[200:203], a[100:103], v[104:107]\n"
    "buffer_load_dwordx4 v58, s[12:15], s73 offen offset:2048  lds\n"
    "v_mfma_f32_16x16x32_bf16 v[108:111], a[204:207], a[100:103], v[108:111]\n"
    "ds_read_b64_tr_b16 a[180:181], v29 offset:32768\n"
    "v_mfma_f32_16x16x32_bf16 v[76:79], a[204:207], a[96:99], v[76:79]\n"
    "ds_read_b128 a[136:139], v33 offset:36864\n"
    "v_mfma_f32_16x16x32_bf16 v[80:83], a[208:211], a[96:99], v[80:83]\n"
    "v_mfma_f32_16x16x32_bf16 v[112:115], a[208:211], a[100:103], v[112:115]\n"
    "ds_read_b64_tr_b16 a[182:183], v29 offset:33792\n"
    "v_mfma_f32_16x16x32_bf16 v[116:119], a[212:215], a[100:103], v[116:119]\n"
    "ds_read_b128 a[140:143], v33 offset:38912\n"
    "v_mfma_f32_16x16x32_bf16 v[84:87], a[212:215], a[96:99], v[84:87]\n"
    "v_mfma_f32_16x16x32_bf16 v[144:147], a[208:211], a[104:107], v[144:147]\n"
    "ds_read_b64_tr_b16 a[184:185], v30 offset:32768\n"
    "v_mfma_f32_16x16x32_bf16 v[176:179], a[208:211], a[108:111], v[176:179]\n"
    "ds_read_b128 a[144:147], v33 offset:40960\n"
    "v_mfma_f32_16x16x32_bf16 v[180:183], a[212:215], a[108:111], v[180:183]\n"
    "v_mfma_f32_16x16x32_bf16 v[148:151], a[212:215], a[104:107], v[148:151]\n"
    "ds_read_b64_tr_b16 a[186:187], v30 offset:33792\n"
    "v_mfma_f32_16x16x32_bf16 v[208:211], a[208:211], a[112:115], v[208:211]\n"
    "ds_read_b128 a[148:151], v33 offset:43008\n"
    "v_mfma_f32_16x16x32_bf16 v[240:243], a[208:211], a[116:119], v[240:243]\n"
    "buffer_load_dwordx4 v59, s[12:15], s73 offen offset:3072  lds\n"
    "v_mfma_f32_16x16x32_bf16 v[244:247], a[212:215], a[116:119], v[244:247]\n"
    "ds_read_b64_tr_b16 a[188:189], v31 offset:32768\n"
    "v_mfma_f32_16x16x32_bf16 v[212:215], a[212:215], a[112:115], v[212:215]\n"
    "ds_read_b128 a[152:155], v33 offset:45056\n"
    "v_mfma_f32_16x16x32_bf16 a[16:19], a[208:211], a[120:123], a[16:19]\n"
    "v_mfma_f32_16x16x32_bf16 a[48:51], a[208:211], a[124:127], a[48:51]\n"
    "ds_read_b64_tr_b16 a[190:191], v31 offset:33792\n"
    "v_mfma_f32_16x16x32_bf16 a[52:55], a[212:215], a[124:127], a[52:55]\n"
    "ds_read_b128 a[156:159], v33 offset:47104\n"
    "v_mfma_f32_16x16x32_bf16 a[20:23], a[212:215], a[120:123], a[20:23]\n"
    "v_mfma_f32_16x16x32_bf16 a[24:27], a[216:219], a[120:123], a[24:27]\n"
    "s_add_u32 s76, s76, 1\n"
    "v_mfma_f32_16x16x32_bf16 a[56:59], a[216:219], a[124:127], a[56:59]\n"
    "v_readlane_b32 s72, v40, s75\n"
    "v_mfma_f32_16x16x32_bf16 a[60:63], a[220:223], a[124:127], a[60:63]\n"
    "v_mfma_f32_16x16x32_bf16 a[28:31], a[220:223], a[120:123], a[28:31]\n"
    "v_readlane_b32 s73, v41, s76\n"
    "v_mfma_f32_16x16x32_bf16 v[216:219], a[216:219], a[112:115], v[216:219]\n"
    "s_cmp_eq_u32 s75, 8\n"
    "v_mfma_f32_16x16x32_bf16 v[248:251], a[216:219], a[116:119], v[248:251]\n"
    "s_cselect_b32 s84, s88, s84\n"
    "s_cselect_b32 s85, s89, s85\n"
    "v_mfma_f32_16x16x32_bf16 v[252:255], a[220:223], a[116:119], v[252:255]\n"
    "v_mfma_f32_16x16x32_bf16 v[220:223], a[220:223], a[112:115], v[220:223]\n"
    "s_cmp_ge_u32 s75, 64\n"
    "s_cselect_b32 s72, 1073741824, s72\n"
    "v_mfma_f32_16x16x32_bf16 v[152:155], a[216:219], a[104:107], v[152:155]\n"
    "v_mfma_f32_16x16x32_bf16 v[184:187], a[216:219], a[108:111], v[184:187]\n"
    "s_cmp_ge_u32 s76, 64\n"
    "s_cselect_b32 s73, 1073741824, s73\n"
    "v_mfma_f32_16x16x32_bf16 v[188:191], a[220:223], a[108:111], v[188:191]\n"
    "v_mfma_f32_16x16x32_bf16 v[156:159], a[220:223], a[104:107], v[156:159]\n"
    "s_bitcmp1_b64 s[60:61], s79\n"
    "s_cbranch_scc1 Loe6%=\n"
    "Loe6r%=:\n"
    "v_mfma_f32_16x16x32_bf16 v[88:91], a[216:219], a[96:99], v[88:91]\n"
    "v_mfma_f32_16x16x32_bf16 v[120:123], a[216:219], a[100:103], v[120:123]\n"
    "s_add_u32 s79, s79, 1\n"
    "v_mfma_f32_16x16x32_bf16 v[124:127], a[220:223], a[100:103], v[124:127]\n"
    "v_mfma_f32_16x16x32_bf16 v[92:95], a[220:223], a[96:99], v[92:95]\n"
    "s_cmp_lg_u32 s41, 0\n"
    "s_cbranch_scc1 LW106%=\n"
    "LW6%=:\n"
    "s_waitcnt lgkmcnt(0)\n"
    "s_barrier\n"
    "LT7%=:\n"
    "v_mfma_f32_16x16x32_bf16 v[64:67], a[160:163], a[64:67], v[64:67]\n"
    "s_sub_u32 s78, s4, s56\n"
    "ds_read_b64_tr_b16 a[192:193], v24 offset:49152\n"
    "v_mfma_f32_16x16x32_bf16 v[96:99], a[160:163], a[68:71], v[96:99]\n"
    "s_memtime s[56:57]\n"
    "s_add_u32 m0, s70, 32768\n"
    "v_mfma_f32_16x16x32_bf16 v[100:103], a[164:167], a[68:71], v[100:103]\n"
    "buffer_load_dwordx4 v48, s[84:87], s72 offen offset:0 sc0 sc1 lds\n"
    "v_mfma_f32_16x16x32_bf16 v[68:71], a[164:167], a[64:67], v[68:71]\n"
    "ds_read_b64_tr_b16 a[194:195], v24 offset:50176\n"
    "v_mfma_f32_16x16x32_bf16 v[128:131], a[160:163], a[72:75], v[128:131]\n"
    "v_mfma_f32_16x16x32_bf16 v[160:163], a[160:163], a[76:79], v[160:163]\n"
    "buffer_load_dwordx4 v49, s[84:87], s72 offen offset:1024 sc0 sc1 lds\n"
    "v_mfma_f32_16x16x32_bf16 v[164:167], a[164:167], a[76:79], v[164:167]\n"
    "ds_read_b64_tr_b16 a[196:197], v25 offset:49152\n"
    "v_mfma_f32_16x16x32_bf16 v[132:135], a[164:167], a[72:75], v[132:135]\n"
    "v_mfma_f32_16x16x32_bf16 v[192:195], a[160:163], a[80:83], v[192:195]\n"
    "buffer_load_dwordx4 v50, s[84:87], s72 offen offset:2048 sc0 sc1 lds\n"
    "v_mfma_f32_16x16x32_bf16 v[224:227], a[160:163], a[84:87], v[224:227]\n"
    "ds_read_b64_tr_b16 a[198:199], v25 offset:50176\n"
    "v_mfma_f32_16x16x32_bf16 v[228:231], a[164:167], a[84:87], v[228:231]\n"
    "v_mfma_f32_16x16x32_bf16 v[196:199], a[164:167], a[80:83], v[196:199]\n"
    "buffer_load_dwordx4 v51, s[84:87], s72 offen offset:3072 sc0 sc1 lds\n"
    "v_mfma_f32_16x16x32_bf16 a[0:3], a[160:163], a[88:91], a[0:3]\n"
    "ds_read_b64_tr_b16 a[200:201], v26 offset:49152\n"
    "v_mfma_f32_16x16x32_bf16 a[32:35], a[160:163], a[92:95], a[32:35]\n"
    "s_add_u32 m0, s70, 36864\n"
    "v_mfma_f32_16x16x32_bf16 a[36:39], a[164:167], a[92:95], a[36:39]\n"
    "buffer_load_dwordx4 v52, s[84:87], s72 offen offset:0 sc0 sc1 lds\n"
    "v_mfma_f32_16x16x32_bf16 a[4:7], a[164:167], a[88:91], a[4:7]\n"
    "ds_read_b64_tr_b16 a[202:203], v26 offset:50176\n"
    "v_mfma_f32_16x16x32_bf16 a[8:11], a[168:171], a[88:91], a[8:11]\n"
    "v_mfma_f32_16x16x32_bf16 a[40:43], a[168:171], a[92:95], a[40:43]\n"
    "buffer_load_dwordx4 v53, s[84:87], s72 offen offset:1024 sc0 sc1 lds\n"
    "v_mfma_f32_16x16x32_bf16 a[44:47], a[172:175], a[92:95], a[44:47]\n"
    "ds_read_b64_tr_b16 a[204:205], v27 offset:49152\n"
    "v_mfma_f32_16x16x32_bf16 a[12:15], a[172:175], a[88:91], a[12:15]\n"
    "v_mfma_f32_16x16x32_bf16 v[200:203], a[168:171], a[80:83], v[200:203]\n"
    "buffer_load_dwordx4 v54, s[84:87], s72 offen offset:2048 sc0 sc1 lds\n"
    "v_mfma_f32_16x16x32_bf16 v[232:235], a[168:171], a[84:87], v[232:235]\n"
    "ds_read_b64_tr_b16 a[206:207], v27 offset:50176\n"
    "v_mfma_f32_16x16x32_bf16 v[236:239], a[172:175], a[84:87], v[236:239]\n"
    "v_mfma_f32_16x16x32_bf16 v[204:207], a[172:175], a[80:83], v[204:207]\n"
    "buffer_load_dwordx4 v55, s[84:87], s72 offen offset:3072 sc0 sc1 lds\n"
    "v_mfma_f32_16x16x32_bf16 v[136:139], a[168:171], a[72:75], v[136:139]\n"
    "ds_read_b64_tr_b16 a[208:209], v28 offset:49152\n"
    "v_mfma_f32_16x16x32_bf16 v[168:171], a[168:171], a[76:79], v[168:171]\n"
    "v_mfma_f32_16x16x32_bf16 v[172:175], a[172:175], a[76:79], v[172:175]\n"
    "v_mfma_f32_16x16x32_bf16 v[140:143], a[172:175], a[72:75], v[140:143]\n"
    "ds_read_b64_tr_b16 a[210:211], v28 offset:50176\n"
    "v_mfma_f32_16x16x32_bf16 v[72:75], a[168:171], a[64:67], v[72:75]\n"
    "s_add_u32 m0, s71, 32768\n"
    "v_mfma_f32_16x16x32_bf16 v[104:107], a[168:171], a[68:71], v[104:107]\n"
    "buffer_load_dwordx4 v56, s[12:15], s73 offen offset:0  lds\n"
    "v_mfma_f32_16x16x32_bf16 v[108:111], a[172:175], a[68:71], v[108:111]\n"
    "ds_read_b64_tr_b16 a[212:213], v29 offset:49152\n"
    "v_mfma_f32_16x16x32_bf16 v[76:79], a[172:175], a[64:67], v[76:79]\n"
    "v_mfma_f32_16x16x32_bf16 v[80:83], a[176:179], a[64:67], v[80:83]\n"
    "v_mfma_f32_16x16x32_bf16 v[112:115], a[176:179], a[68:71], v[112:115]\n"
    "ds_read_b64_tr_b16 a[214:215], v29 offset:50176\n"
    "v_mfma_f32_16x16x32_bf16 v[116:119], a[180:183], a[68:71], v[116:119]\n"
    "v_mfma_f32_16x16x32_bf16 v[84:87], a[180:183], a[64:67], v[84:87]\n"
    "buffer_load_dwordx4 v57, s[12:15], s73 offen offset:1024  lds\n"
    "v_mfma_f32_16x16x32_bf16 v[144:147], a[176:179], a[72:75], v[144:147]\n"
    "ds_read_b64_tr_b16 a[216:217], v30 offset:49152\n"
    "v_mfma_f32_16x16x32_bf16 v[176:179], a[176:179], a[76:79], v[176:179]\n"
    "v_mfma_f32_16x16x32_bf16 v[180:183], a[180:183], a[76:79], v[180:183]\n"
    "v_mfma_f32_16x16x32_bf16 v[148:151], a[180:183], a[72:75], v[148:151]\n"
    "ds_read_b64_tr_b16 a[218:219], v30 offset:50176\n"
    "v_mfma_f32_16x16x32_bf16 v[208:211], a[176:179], a[80:83], v[208:211]\n"
    "v_mfma_f32_16x16x32_bf16 v[240:243], a[176:179], a[84:87], v[240:243]\n"
    "buffer_load_dwordx4 v58, s[12:15], s73 offen offset:2048  lds\n"
    "v_mfma_f32_16x16x32_bf16 v[244:247], a[180:183], a[84:87], v[244:247]\n"
    "ds_read_b64_tr_b16 a[220:221], v31 offset:49152\n"
    "v_mfma_f32_16x16x32_bf16 v[212:215], a[180:183], a[80:83], v[212:215]\n"
    "v_mfma_f32_16x16x32_bf16 a[16:19], a[176:179], a[88:91], a[16:19]\n"
    "v_mfma_f32_16x16x32_bf16 a[48:51], a[176:179], a[92:95], a[48:51]\n"
    "ds_read_b64_tr_b16 a[222:223], v31 offset:50176\n"
    "v_mfma_f32_16x16x32_bf16 a[52:55], a[180:183], a[92:95], a[52:55]\n"
    "v_mfma_f32_16x16x32_bf16 a[20:23], a[180:183], a[88:91], a[20:23]\n"
    "buffer_load_dwordx4 v59, s[12:15], s73 offen offset:3072  lds\n"
    "v_mfma_f32_16x16x32_bf16 a[24:27], a[184:187], a[88:91], a[24:27]\n"
    "v_mfma_f32_16x16x32_bf16 a[56:59], a[184:187], a[92:95], a[56:59]\n"
    "s_add_u32 s73, s73, 0x200000\n"
    "v_mfma_f32_16x16x32_bf16 a[60:63], a[188:191], a[92:95], a[60:63]\n"
    "s_add_u32 s75, s75, 1\n"
    "v_mfma_f32_16x16x32_bf16 a[28:31], a[188:191], a[88:91], a[28:31]\n"
    "v_mfma_f32_16x16x32_bf16 v[216:219], a[184:187], a[80:83], v[216:219]\n"
    "s_cmp_gt_u32 s78, s30\n"
    "s_cbranch_scc1 Lad7%=\n"
    "Lad7r%=:\n"
    "v_mfma_f32_16x16x32_bf16 v[248:251], a[184:187], a[84:87], v[248:251]\n"
    "v_mfma_f32_16x16x32_bf16 v[252:255], a[188:191], a[84:87], v[252:255]\n"
    "v_mfma_f32_16x16x32_bf16 v[220:223], a[188:191], a[80:83], v[220:223]\n"
    "v_mfma_f32_16x16x32_bf16 v[152:155], a[184:187], a[72:75], v[152:155]\n"
    "v_mfma_f32_16x16x32_bf16 v[184:187], a[184:187], a[76:79], v[184:187]\n"
    "v_mfma_f32_16x16x32_bf16 v[188:191], a[188:191], a[76:79], v[188:191]\n"
    "v_mfma_f32_16x16x32_bf16 v[156:159], a[188:191], a[72:75], v[156:159]\n"
    "v_mfma_f32_16x16x32_bf16 v[88:91], a[184:187], a[64:67], v[88:91]\n"
    "v_mfma_f32_16x16x32_bf16 v[120:123], a[184:187], a[68:71], v[120:123]\n"
    "v_mfma_f32_16x16x32_bf16 v[124:127], a[188:191], a[68:71], v[124:127]\n"
    "v_mfma_f32_16x16x32_bf16 v[92:95], a[188:191], a[64:67], v[92:95]\n"
    "s_waitcnt vmcnt(20) lgkmcnt(0)\n"
    "s_barrier\n"
    "v_mfma_f32_16x16x32_bf16 v[64:67], a[192:195], a[128:131], v[64:67]\n"
    "ds_read_b64_tr_b16 a[160:161], v16 offset:0\n"
    "s_memtime s[4:5]\n"
    "v_mfma_f32_16x16x32_bf16 v[96:99], a[192:195], a[132:135], v[96:99]\n"
    "ds_read_b128 a[64:67], v32 offset:0\n"
    "v_mfma_f32_16x16x32_bf16 v[100:103], a[196:199], a[132:135], v[100:103]\n"
    "s_bitcmp1_b64 s[64:65], s79\n"
    "s_cbranch_scc1 Lfl7%=\n"
    "Lfl7r%=:\n"
    "v_mfma_f32_16x16x32_bf16 v[68:71], a[196:199], a[128:131], v[68:71]\n"
    "ds_read_b64_tr_b16 a[162:163], v16 offset:1024\n"
    "v_mfma_f32_16x16x32_bf16 v[128:131], a[192:195], a[136:139], v[128:131]\n"
    "ds_read_b128 a[68:71], v32 offset:2048\n"
    "s_add_u32 m0, s71, 49152\n"
    "v_mfma_f32_16x16x32_bf16 v[160:163], a[192:195], a[140:143], v[160:163]\n"
    "buffer_load_dwordx4 v56, s[12:15], s73 offen offset:0  lds\n"
    "v_mfma_f32_16x16x32_bf16 v[164:167], a[196:199], a[140:143], v[164:167]\n"
    "ds_read_b64_tr_b16 a[164:165], v17 offset:0\n"
    "v_mfma_f32_16x16x32_bf16 v[132:135], a[196:199], a[136:139], v[132:135]\n"
    "ds_read_b128 a[72:75], v32 offset:4096\n"
    "v_mfma_f32_16x16x32_bf16 v[192:195], a[192:195], a[144:147], v[192:195]\n"
    "v_mfma_f32_16x16x32_bf16 v[224:227], a[192:195], a[148:151], v[224:227]\n"
    "ds_read_b64_tr_b16 a[166:167], v17 offset:1024\n"
    "v_mfma_f32_16x16x32_bf16 v[228:231], a[196:199], a[148:151], v[228:231]\n"
    "ds_read_b128 a[76:79], v32 offset:6144\n"
    "v_mfma_f32_16x16x32_bf16 v[196:199], a[196:199], a[144:147], v[196:199]\n"
    "v_mfma_f32_16x16x32_bf16 a[0:3], a[192:195], a[152:155], a[0:3]\n"
    "ds_read_b64_tr_b16 a[168:169], v18 offset:0\n"
    "v_mfma_f32_16x16x32_bf16 a[32:35], a[192:195], a[156:159], a[32:35]\n"
    "ds_read_b128 a[80:83], v32 offset:8192\n"
    "v_mfma_f32_16x16x32_bf16 a[36:39], a[196:199], a[156:159], a[36:39]\n"
    "v_mfma_f32_16x16x32_bf16 a[4:7], a[196:199], a[152:155], a[4:7]\n"
    "ds_read_b64_tr_b16 a[170:171], v18 offset:1024\n"
    "v_mfma_f32_16x16x32_bf16 a[8:11], a[200:203], a[152:155], a[8:11]\n"
    "ds_read_b128 a[84:87], v32 offset:10240\n"
    "v_mfma_f32_16x16x32_bf16 a[40:43], a[200:203], a[156:159], a[40:43]\n"
    "buffer_load_dwordx4 v57, s[12:15], s73 offen offset:1024  lds\n"
    "v_mfma_f32_16x16x32_bf16 a[44:47], a[204:207], a[156:159], a[44:47]\n"
    "ds_read_b64_tr_b16 a[172:173], v19 offset:0\n"
    "v_mfma_f32_16x16x32_bf16 a[12:15], a[204:207], a[152:155], a[12:15]\n"
    "ds_read_b128 a[88:91], v32 offset:12288\n"
    "v_mfma_f32_16x16x32_bf16 v[200:203], a[200:203], a[144:147], v[200:203]\n"
    "v_mfma_f32_16x16x32_bf16 v[232:235], a[200:203], a[148:151], v[232:235]\n"
    "ds_read_b64_tr_b16 a[174:175], v19 offset:1024\n"
    "v_mfma_f32_16x16x32_bf16 v[236:239], a[204:207], a[148:151], v[236:239]\n"
    "ds_read_b128 a[92:95], v32 offset:14336\n"
    "v_mfma_f32_16x16x32_bf16 v[204:207], a[204:207], a[144:147], v[204:207]\n"
    "v_mfma_f32_16x16x32_bf16 v[136:139], a[200:203], a[136:139], v[136:139]\n"
    "ds_read_b64_tr_b16 a[176:177], v20 offset:0\n"
    "v_mfma_f32_16x16x32_bf16 v[168:171], a[200:203], a[140:143], v[168:171]\n"
    "ds_read_b128 a[96:99], v33 offset:0\n"
    "v_mfma_f32_16x16x32_bf16 v[172:175], a[204:207], a[140:143], v[172:175]\n"
    "v_mfma_f32_16x16x32_bf16 v[140:143], a[204:207], a[136:139], v[140:143]\n"
    "ds_read_b64_tr_b16 a[178:179], v20 offset:1024\n"
    "v_mfma_f32_16x16x32_bf16 v[72:75], a[200:203], a[128:131], v[72:75]\n"
    "ds_read_b128 a[100:103], v33 offset:2048\n"
    "v_mfma_f32_16x16x32_bf16 v[104:107], a[200:203], a[132:135], v[104:107]\n"
    "buffer_load_dwordx4 v58, s[12:15], s73 offen offset:2048  lds\n"
    "v_mfma_f32_16x16x32_bf16 v[108:111], a[204:207], a[132:135], v[108:111]\n"
    "ds_read_b64_tr_b16 a[180:181], v21 offset:0\n"
    "v_mfma_f32_16x16x32_bf16 v[76:79], a[204:207], a[128:131], v[76:79]\n"
    "ds_read_b128 a[104:107], v33 offset:4096\n"
    "v_mfma_f32_16x16x32_bf16 v[80:83], a[208:211], a[128:131], v[80:83]\n"
    "v_mfma_f32_16x16x32_bf16 v[112:115], a[208:211], a[132:135], v[112:115]\n"
    "ds_read_b64_tr_b16 a[182:183], v21 offset:1024\n"
    "v_mfma_f32_16x16x32_bf16 v[116:119], a[212:215], a[132:135], v[116:119]\n"
    "ds_read_b128 a[108:111], v33 offset:6144\n"
    "v_mfma_f32_16x16x32_bf16 v[84:87], a[212:215], a[128:131], v[84:87]\n"
    "v_mfma_f32_16x16x32_bf16 v[144:147], a[208:211], a[136:139], v[144:147]\n"
    "ds_read_b64_tr_b16 a[184:185], v22 offset:0\n"
    "v_mfma_f32_16x16x32_bf16 v[176:179], a[208:211], a[140:143], v[176:179]\n"
    "ds_read_b128 a[112:115], v33 offset:8192\n"
    "v_mfma_f32_16x16x32_bf16 v[180:183], a[212:215], a[140:143], v[180:183]\n"
    "v_mfma_f32_16x16x32_bf16 v[148:151], a[212:215], a[136:139], v[148:151]\n"
    "ds_read_b64_tr_b16 a[186:187], v22 offset:1024\n"
    "v_mfma_f32_16x16x32_bf16 v[208:211], a[208:211], a[144:147], v[208:211]\n"
    "ds_read_b128 a[116:119], v33 offset:10240\n"
    "v_mfma_f32_16x16x32_bf16 v[240:243], a[208:211], a[148:151], v[240:243]\n"
    "buffer_load_dwordx4 v59, s[12:15], s73 offen offset:3072  lds\n"
    "v_mfma_f32_16x16x32_bf16 v[244:247], a[212:215], a[148:151], v[244:247]\n"
    "ds_read_b64_tr_b16 a[188:189], v23 offset:0\n"
    "v_mfma_f32_16x16x32_bf16 v[212:215], a[212:215], a[144:147], v[212:215]\n"
    "ds_read_b128 a[120:123], v33 offset:12288\n"
    "v_mfma_f32_16x16x32_bf16 a[16:19], a[208:211], a[152:155], a[16:19]\n"
    "v_mfma_f32_16x16x32_bf16 a[48:51], a[208:211], a[156:159], a[48:51]\n"
    "ds_read_b64_tr_b16 a[190:191], v23 offset:1024\n"
    "v_mfma_f32_16x16x32_bf16 a[52:55], a[212:215], a[156:159], a[52:55]\n"
    "ds_read_b128 a[124:127], v33 offset:14336\n"
    "v_mfma_f32_16x16x32_bf16 a[20:23], a[212:215], a[152:155], a[20:23]\n"
    "v_mfma_f32_16x16x32_bf16 a[24:27], a[216:219], a[152:155], a[24:27]\n"
    "s_add_u32 s76, s76, 1\n"
    "v_mfma_f32_16x16x32_bf16 a[56:59], a[216:219], a[156:159], a[56:59]\n"
    "v_readlane_b32 s72, v40, s75\n"
    "v_mfma_f32_16x16x32_bf16 a[60:63], a[220:223], a[156:159], a[60:63]\n"
    "v_mfma_f32_16x16x32_bf16 a[28:31], a[220:223], a[152:155], a[28:31]\n"
    "v_readlane_b32 s73, v41, s76\n"
    "v_mfma_f32_16x16x32_bf16 v[216:219], a[216:219], a[144:147], v[216:219]\n"
    "s_cmp_eq_u32 s75, 8\n"
    "v_mfma_f32_16x16x32_bf16 v[248:251], a[216:219], a[148:151], v[248:251]\n"
    "s_cselect_b32 s84, s88, s84\n"
    "s_cselect_b32 s85, s89, s85\n"
    "v_mfma_f32_16x16x32_bf16 v[252:255], a[220:223], a[148:151], v[252:255]\n"
    "v_mfma_f32_16x16x32_bf16 v[220:223], a[220:223], a[144:147], v[220:223]\n"
    "s_cmp_ge_u32 s75, 64\n"
    "s_cselect_b32 s72, 1073741824, s72\n"
    "v_mfma_f32_16x16x32_bf16 v[152:155], a[216:219], a[136:139], v[152:155]\n"
    "v_mfma_f32_16x16x32_bf16 v[184:187], a[216:219], a[140:143], v[184:187]\n"
    "s_cmp_ge_u32 s76, 64\n"
    "s_cselect_b32 s73, 1073741824, s73\n"
    "v_mfma_f32_16x16x32_bf16 v[188:191], a[220:223], a[140:143], v[188:191]\n"
    "v_mfma_f32_16x16x32_bf16 v[156:159], a[220:223], a[136:139], v[156:159]\n"
    "s_bitcmp1_b64 s[60:61], s79\n"
    "s_cbranch_scc1 Loe7%=\n"
    "Loe7r%=:\n"
    "v_mfma_f32_16x16x32_bf16 v[88:91], a[216:219], a[128:131], v[88:91]\n"
    "v_mfma_f32_16x16x32_bf16 v[120:123], a[216:219], a[132:135], v[120:123]\n"
    "s_add_u32 s79, s79, 1\n"
    "v_mfma_f32_16x16x32_bf16 v[124:127], a[220:223], a[132:135], v[124:127]\n"
    "s_add_u32 s74, s74, 1\n"
    "v_mfma_f32_16x16x32_bf16 v[92:95], a[220:223], a[128:131], v[92:95]\n"
    "s_cmp_lt_u32 s74, 10\n"
    "s_waitcnt lgkmcnt(0)\n"
    "s_barrier\n"
    "s_cbranch_scc0 Ltail%=\n"
    "s_cmp_lg_u32 s41, 0\n"
    "s_cbranch_scc1 LT102%=\n"
    "s_branch LT2%=\n"
    ".p2align 6\n"
    "LT102%=:\n"
    "v_mfma_f32_16x16x32_bf16 v[64:67], a[160:163], a[64:67], v[64:67]\n"
    "s_sub_u32 s78, s4, s56\n"
    "ds_read_b64_tr_b16 a[192:193], v16 offset:16384\n"
    "v_mfma_f32_16x16x32_bf16 v[96:99], a[160:163], a[68:71], v[96:99]\n"
    "s_memtime s[56:57]\n"
    "s_add_u32 m0, s70, 0\n"
    "v_mfma_f32_16x16x32_bf16 v[100:103], a[164:167], a[68:71], v[100:103]\n"
    "buffer_load_dwordx4 v48, s[84:87], s72 offen offset:0 sc0 sc1 lds\n"
    "v_mfma_f32_16x16x32_bf16 v[68:71], a[164:167], a[64:67], v[68:71]\n"
    "ds_read_b64_tr_b16 a[194:195], v16 offset:17408\n"
    "v_mfma_f32_16x16x32_bf16 v[128:131], a[160:163], a[72:75], v[128:131]\n"
    "v_mfma_f32_16x16x32_bf16 v[160:163], a[160:163], a[76:79], v[160:163]\n"
    "buffer_load_dwordx4 v49, s[84:87], s72 offen offset:1024 sc0 sc1 lds\n"
    "v_mfma_f32_16x16x32_bf16 v[164:167], a[164:167], a[76:79], v[164:167]\n"
    "ds_read_b64_tr_b16 a[196:197], v17 offset:16384\n"
    "v_mfma_f32_16x16x32_bf16 v[132:135], a[164:167], a[72:75], v[132:135]\n"
    "v_mfma_f32_16x16x32_bf16 v[192:195], a[160:163], a[80:83], v[192:195]\n"
    "buffer_load_dwordx4 v50, s[84:87], s72 offen offset:2048 sc0 sc1 lds\n"
    "v_mfma_f32_16x16x32_bf16 v[224:227], a[160:163], a[84:87], v[224:227]\n"
    "ds_read_b64_tr_b16 a[198:199], v17 offset:17408\n"
    "v_mfma_f32_16x16x32_bf16 v[228:231], a[164:167], a[84:87], v[228:231]\n"
    "v_mfma_f32_16x16x32_bf16 v[196:199], a[164:167], a[80:83], v[196:199]\n"
    "buffer_load_dwordx4 v51, s[84:87], s72 offen offset:3072 sc0 sc1 lds\n"
    "v_mfma_f32_16x16x32_bf16 a[0:3], a[160:163], a[88:91], a[0:3]\n"
    "ds_read_b64_tr_b16 a[200:201], v18 offset:16384\n"
    "v_mfma_f32_16x16x32_bf16 a[32:35], a[160:163], a[92:95], a[32:35]\n"
    "s_add_u32 m0, s70, 4096\n"
    "v_mfma_f32_16x16x32_bf16 a[36:39], a[164:167], a[92:95], a[36:39]\n"
    "buffer_load_dwordx4 v52, s[84:87], s72 offen offset:0 sc0 sc1 lds\n"
    "v_mfma_f32_16x16x32_bf16 a[4:7], a[164:167], a[88:91], a[4:7]\n"
    "ds_read_b64_tr_b16 a[202:203], v18 offset:17408\n"
    "v_mfma_f32_16x16x32_bf16 a[8:11], a[168:171], a[88:91], a[8:11]\n"
    "v_mfma_f32_16x16x32_bf16 a[40:43], a[168:171], a[92:95], a[40:43]\n"
    "buffer_load_dwordx4 v53, s[84:87], s72 offen offset:1024 sc0 sc1 lds\n"
    "v_mfma_f32_16x16x32_bf16 a[44:47], a[172:175], a[92:95], a[44:47]\n"
    "ds_read_b64_tr_b16 a[204:205], v19 offset:16384\n"
    "v_mfma_f32_16x16x32_bf16 a[12:15], a[172:175], a[88:91], a[12:15]\n"
    "v_mfma_f32_16x16x32_bf16 v[200:203], a[168:171], a[80:83], v[200:203]\n"
    "buffer_load_dwordx4 v54, s[84:87], s72 offen offset:2048 sc0 sc1 lds\n"
    "v_mfma_f32_16x16x32_bf16 v[232:235], a[168:171], a[84:87], v[232:235]\n"
    "ds_read_b64_tr_b16 a[206:207], v19 offset:17408\n"
    "v_mfma_f32_16x16x32_bf16 v[236:239], a[172:175], a[84:87], v[236:239]\n"
    "v_mfma_f32_16x16x32_bf16 v[204:207], a[172:175], a[80:83], v[204:207]\n"
    "buffer_load_dwordx4 v55, s[84:87], s72 offen offset:3072 sc0 sc1 lds\n"
    "v_mfma_f32_16x16x32_bf16 v[136:139], a[168:171], a[72:75], v[136:139]\n"
    "ds_read_b64_tr_b16 a[208:209], v20 offset:16384\n"
    "v_mfma_f32_16x16x32_bf16 v[168:171], a[168:171], a[76:79], v[168:171]\n"
    "v_mfma_f32_16x16x32_bf16 v[172:175], a[172:175], a[76:79], v[172:175]\n"
    "v_mfma_f32_16x16x32_bf16 v[140:143], a[172:175], a[72:75], v[140:143]\n"
    "ds_read_b64_tr_b16 a[210:211], v20 offset:17408\n"
    "v_mfma_f32_16x16x32_bf16 v[72:75], a[168:171], a[64:67], v[72:75]\n"
    "s_add_u32 m0, s71, 65536\n"
    "v_mfma_f32_16x16x32_bf16 v[104:107], a[168:171], a[68:71], v[104:107]\n"
    "buffer_load_dwordx4 v56, s[12:15], s73 offen offset:0 nt lds\n"
    "v_mfma_f32_16x16x32_bf16 v[108:111], a[172:175], a[68:71], v[108:111]\n"
    "ds_read_b64_tr_b16 a[212:213], v21 offset:16384\n"
    "v_mfma_f32_16x16x32_bf16 v[76:79], a[172:175], a[64:67], v[76:79]\n"
    "v_mfma_f32_16x16x32_bf16 v[80:83], a[176:179], a[64:67], v[80:83]\n"
    "v_mfma_f32_16x16x32_bf16 v[112:115], a[176:179], a[68:71], v[112:115]\n"
    "ds_read_b64_tr_b16 a[214:215], v21 offset:17408\n"
    "v_mfma_f32_16x16x32_bf16 v[116:119], a[180:183], a[68:71], v[116:119]\n"
    "v_mfma_f32_16x16x32_bf16 v[84:87], a[180:183], a[64:67], v[84:87]\n"
    "buffer_load_dwordx4 v57, s[12:15], s73 offen offset:1024 nt lds\n"
    "v_mfma_f32_16x16x32_bf16 v[144:147], a[176:179], a[72:75], v[144:147]\n"
    "ds_read_b64_tr_b16 a[216:217], v22 offset:16384\n"
    "v_mfma_f32_16x16x32_bf16 v[176:179], a[176:179], a[76:79], v[176:179]\n"
    "v_mfma_f32_16x16x32_bf16 v[180:183], a[180:183], a[76:79], v[180:183]\n"
    "v_mfma_f32_16x16x32_bf16 v[148:151], a[180:183], a[72:75], v[148:151]\n"
    "ds_read_b64_tr_b16 a[218:219], v22 offset:17408\n"
    "v_mfma_f32_16x16x32_bf16 v[208:211], a[176:179], a[80:83], v[208:211]\n"
    "v_mfma_f32_16x16x32_bf16 v[240:243], a[176:179], a[84:87], v[240:243]\n"
    "buffer_load_dwordx4 v58, s[12:15], s73 offen offset:2048 nt lds\n"
    "v_mfma_f32_16x16x32_bf16 v[244:247], a[180:183], a[84:87], v[244:247]\n"
    "ds_read_b64_tr_b16 a[220:221], v23 offset:16384\n"
    "v_mfma_f32_16x16x32_bf16 v[212:215], a[180:183], a[80:83], v[212:215]\n"
    "v_mfma_f32_16x16x32_bf16 a[16:19], a[176:179], a[88:91], a[16:19]\n"
    "v_mfma_f32_16x16x32_bf16 a[48:51], a[176:179], a[92:95], a[48:51]\n"
    "ds_read_b64_tr_b16 a[222:223], v23 offset:17408\n"
    "v_mfma_f32_16x16x32_bf16 a[52:55], a[180:183], a[92:95], a[52:55]\n"
    "v_mfma_f32_16x16x32_bf16 a[20:23], a[180:183], a[88:91], a[20:23]\n"
    "buffer_load_dwordx4 v59, s[12:15], s73 offen offset:3072 nt lds\n"
    "v_mfma_f32_16x16x32_bf16 a[24:27], a[184:187], a[88:91], a[24:27]\n"
    "v_mfma_f32_16x16x32_bf16 a[56:59], a[184:187], a[92:95], a[56:59]\n"
    "s_add_u32 s73, s73, 0x200000\n"
    "v_mfma_f32_16x16x32_bf16 a[60:63], a[188:191], a[92:95], a[60:63]\n"
    "s_add_u32 s75, s75, 1\n"
    "v_mfma_f32_16x16x32_bf16 a[28:31], a[188:191], a[88:91], a[28:31]\n"
    "v_mfma_f32_16x16x32_bf16 v[216:219], a[184:187], a[80:83], v[216:219]\n"
    "s_cmp_gt_u32 s78, s30\n"
    "s_cbranch_scc1 Lad102%=\n"
    "Lad102r%=:\n"
    "v_mfma_f32_16x16x32_bf16 v[248:251], a[184:187], a[84:87], v[248:251]\n"
    "v_mfma_f32_16x16x32_bf16 v[252:255], a[188:191], a[84:87], v[252:255]\n"
    "v_mfma_f32_16x16x32_bf16 v[220:223], a[188:191], a[80:83], v[220:223]\n"
    "v_mfma_f32_16x16x32_bf16 v[152:155], a[184:187], a[72:75], v[152:155]\n"
    "v_mfma_f32_16x16x32_bf16 v[184:187], a[184:187], a[76:79], v[184:187]\n"
    "v_mfma_f32_16x16x32_bf16 v[188:191], a[188:191], a[76:79], v[188:191]\n"
    "v_mfma_f32_16x16x32_bf16 v[156:159], a[188:191], a[72:75], v[156:159]\n"
    "v_mfma_f32_16x16x32_bf16 v[88:91], a[184:187], a[64:67], v[88:91]\n"
    "v_mfma_f32_16x16x32_bf16 v[120:123], a[184:187], a[68:71], v[120:123]\n"
    "v_mfma_f32_16x16x32_bf16 v[124:127], a[188:191], a[68:71], v[124:127]\n"
    "v_mfma_f32_16x16x32_bf16 v[92:95], a[188:191], a[64:67], v[92:95]\n"
    "s_waitcnt vmcnt(20) lgkmcnt(0)\n"
    "s_barrier\n"
    "v_mfma_f32_16x16x32_bf16 v[64:67], a[192:195], a[96:99], v[64:67]\n"
    "ds_read_b64_tr_b16 a[160:161], v24 offset:0\n"
    "s_memtime s[4:5]\n"
    "v_mfma_f32_16x16x32_bf16 v[96:99], a[192:195], a[100:103], v[96:99]\n"
    "ds_read_b128 a[64:67], v32 offset:32768\n"
    "v_mfma_f32_16x16x32_bf16 v[100:103], a[196:199], a[100:103], v[100:103]\n"
    "s_bitcmp1_b64 s[64:65], s79\n"
    "s_cbranch_scc1 Lfl102%=\n"
    "Lfl102r%=:\n"
    "v_mfma_f32_16x16x32_bf16 v[68:71], a[196:199], a[96:99], v[68:71]\n"
    "ds_read_b64_tr_b16 a[162:163], v24 offset:1024\n"
    "v_mfma_f32_16x16x32_bf16 v[128:131], a[192:195], a[104:107], v[128:131]\n"
    "ds_read_b128 a[68:71], v32 offset:34816\n"
    "s_add_u32 m0, s71, 81920\n"
    "v_mfma_f32_16x16x32_bf16 v[160:163], a[192:195], a[108:111], v[160:163]\n"
    "buffer_load_dwordx4 v56, s[12:15], s73 offen offset:0 nt lds\n"
    "v_mfma_f32_16x16x32_bf16 v[164:167], a[196:199], a[108:111], v[164:167]\n"
    "ds_read_b64_tr_b16 a[164:165], v25 offset:0\n"
    "v_mfma_f32_16x16x32_bf16 v[132:135], a[196:199], a[104:107], v[132:135]\n"
    "ds_read_b128 a[72:75], v32 offset:36864\n"
    "v_mfma_f32_16x16x32_bf16 v[192:195], a[192:195], a[112:115], v[192:195]\n"
    "v_mfma_f32_16x16x32_bf16 v[224:227], a[192:195], a[116:119], v[224:227]\n"
    "ds_read_b64_tr_b16 a[166:167], v25 offset:1024\n"
    "v_mfma_f32_16x16x32_bf16 v[228:231], a[196:199], a[116:119], v[228:231]\n"
    "ds_read_b128 a[76:79], v32 offset:38912\n"
    "v_mfma_f32_16x16x32_bf16 v[196:199], a[196:199], a[112:115], v[196:199]\n"
    "v_mfma_f32_16x16x32_bf16 a[0:3], a[192:195], a[120:123], a[0:3]\n"
    "ds_read_b64_tr_b16 a[168:169], v26 offset:0\n"
    "v_mfma_f32_16x16x32_bf16 a[32:35], a[192:195], a[124:127], a[32:35]\n"
    "ds_read_b128 a[80:83], v32 offset:40960\n"
    "v_mfma_f32_16x16x32_bf16 a[36:39], a[196:199], a[124:127], a[36:39]\n"
    "v_mfma_f32_16x16x32_bf16 a[4:7], a[196:199], a[120:123], a[4:7]\n"
    "ds_read_b64_tr_b16 a[170:171], v26 offset:1024\n"
    "v_mfma_f32_16x16x32_bf16 a[8:11], a[200:203], a[120:123], a[8:11]\n"
    "ds_read_b128 a[84:87], v32 offset:43008\n"
    "v_mfma_f32_16x16x32_bf16 a[40:43], a[200:203], a[124:127], a[40:43]\n"
    "buffer_load_dwordx4 v57, s[12:15], s73 offen offset:1024 nt lds\n"
    "v_mfma_f32_16x16x32_bf16 a[44:47], a[204:207], a[124:127], a[44:47]\n"
    "ds_read_b64_tr_b16 a[172:173], v27 offset:0\n"
    "v_mfma_f32_16x16x32_bf16 a[12:15], a[204:207], a[120:123], a[12:15]\n"
    "ds_read_b128 a[88:91], v32 offset:45056\n"
    "v_mfma_f32_16x16x32_bf16 v[200:203], a[200:203], a[112:115], v[200:203]\n"
    "v_mfma_f32_16x16x32_bf16 v[232:235], a[200:203], a[116:119], v[232:235]\n"
    "ds_read_b64_tr_b16 a[174:175], v27 offset:1024\n"
    "v_mfma_f32_16x16x32_bf16 v[236:239], a[204:207], a[116:119], v[236:239]\n"
    "ds_read_b128 a[92:95], v32 offset:47104\n"
    "v_mfma_f32_16x16x32_bf16 v[204:207], a[204:207], a[112:115], v[204:207]\n"
    "v_mfma_f32_16x16x32_bf16 v[136:139], a[200:203], a[104:107], v[136:139]\n"
    "ds_read_b64_tr_b16 a[176:177], v28 offset:0\n"
    "v_mfma_f32_16x16x32_bf16 v[168:171], a[200:203], a[108:111], v[168:171]\n"
    "ds_read_b128 a[128:131], v33 offset:32768\n"
    "v_mfma_f32_16x16x32_bf16 v[172:175], a[204:207], a[108:111], v[172:175]\n"
    "v_mfma_f32_16x16x32_bf16 v[140:143], a[204:207], a[104:107], v[140:143]\n"
    "ds_read_b64_tr_b16 a[178:179], v28 offset:1024\n"
    "v_mfma_f32_16x16x32_bf16 v[72:75], a[200:203], a[96:99], v[72:75]\n"
    "ds_read_b128 a[132:135], v33 offset:34816\n"
    "v_mfma_f32_16x16x32_bf16 v[104:107], a[200:203], a[100:103], v[104:107]\n"
    "buffer_load_dwordx4 v58, s[12:15], s73 offen offset:2048 nt lds\n"
    "v_mfma_f32_16x16x32_bf16 v[108:111], a[204:207], a[100:103], v[108:111]\n"
    "ds_read_b64_tr_b16 a[180:181], v29 offset:0\n"
    "v_mfma_f32_16x16x32_bf16 v[76:79], a[204:207], a[96:99], v[76:79]\n"
    "ds_read_b128 a[136:139], v33 offset:36864\n"
    "v_mfma_f32_16x16x32_bf16 v[80:83], a[208:211], a[96:99], v[80:83]\n"
    "v_mfma_f32_16x16x32_bf16 v[112:115], a[208:211], a[100:103], v[112:115]\n"
    "ds_read_b64_tr_b16 a[182:183], v29 offset:1024\n"
    "v_mfma_f32_16x16x32_bf16 v[116:119], a[212:215], a[100:103], v[116:119]\n"
    "ds_read_b128 a[140:143], v33 offset:38912\n"
    "v_mfma_f32_16x16x32_bf16 v[84:87], a[212:215], a[96:99], v[84:87]\n"
    "v_mfma_f32_16x16x32_bf16 v[144:147], a[208:211], a[104:107], v[144:147]\n"
    "ds_read_b64_tr_b16 a[184:185], v30 offset:0\n"
    "v_mfma_f32_16x16x32_bf16 v[176:179], a[208:211], a[108:111], v[176:179]\n"
    "ds_read_b128 a[144:147], v33 offset:40960\n"
    "v_mfma_f32_16x16x32_bf16 v[180:183], a[212:215], a[108:111], v[180:183]\n"
    "v_mfma_f32_16x16x32_bf16 v[148:151], a[212:215], a[104:107], v[148:151]\n"
    "ds_read_b64_tr_b16 a[186:187], v30 offset:1024\n"
    "v_mfma_f32_16x16x32_bf16 v[208:211], a[208:211], a[112:115], v[208:211]\n"
    "ds_read_b128 a[148:151], v33 offset:43008\n"
    "v_mfma_f32_16x16x32_bf16 v[240:243], a[208:211], a[116:119], v[240:243]\n"
    "buffer_load_dwordx4 v59, s[12:15], s73 offen offset:3072 nt lds\n"
    "v_mfma_f32_16x16x32_bf16 v[244:247], a[212:215], a[116:119], v[244:247]\n"
    "ds_read_b64_tr_b16 a[188:189], v31 offset:0\n"
    "v_mfma_f32_16x16x32_bf16 v[212:215], a[212:215], a[112:115], v[212:215]\n"
    "ds_read_b128 a[152:155], v33 offset:45056\n"
    "v_mfma_f32_16x16x32_bf16 a[16:19], a[208:211], a[120:123], a[16:19]\n"
    "v_mfma_f32_16x16x32_bf16 a[48:51], a[208:211], a[124:127], a[48:51]\n"
    "ds_read_b64_tr_b16 a[190:191], v31 offset:1024\n"
    "v_mfma_f32_16x16x32_bf16 a[52:55], a[212:215], a[124:127], a[52:55]\n"
    "ds_read_b128 a[156:159], v33 offset:47104\n"
    "v_mfma_f32_16x16x32_bf16 a[20:23], a[212:215], a[120:123], a[20:23]\n"
    "v_mfma_f32_16x16x32_bf16 a[24:27], a[216:219], a[120:123], a[24:27]\n"
    "s_add_u32 s76, s76, 1\n"
    "v_mfma_f32_16x16x32_bf16 a[56:59], a[216:219], a[124:127], a[56:59]\n"
    "v_readlane_b32 s72, v40, s75\n"
    "v_mfma_f32_16x16x32_bf16 a[60:63], a[220:223], a[124:127], a[60:63]\n"
    "v_mfma_f32_16x16x32_bf16 a[28:31], a[220:223], a[120:123], a[28:31]\n"
    "v_readlane_b32 s73, v41, s76\n"
    "v_mfma_f32_16x16x32_bf16 v[216:219], a[216:219], a[112:115], v[216:219]\n"
    "s_cmp_eq_u32 s75, 8\n"
    "v_mfma_f32_16x16x32_bf16 v[248:251], a[216:219], a[116:119], v[248:251]\n"
    "s_cselect_b32 s84, s88, s84\n"
    "s_cselect_b32 s85, s89, s85\n"
    "v_mfma_f32_16x16x32_bf16 v[252:255], a[220:223], a[116:119], v[252:255]\n"
    "v_mfma_f32_16x16x32_bf16 v[220:223], a[220:223], a[112:115], v[220:223]\n"
    "s_cmp_ge_u32 s75, 64\n"
    "s_cselect_b32 s72, 1073741824, s72\n"
    "v_mfma_f32_16x16x32_bf16 v[152:155], a[216:219], a[104:107], v[152:155]\n"
    "v_mfma_f32_16x16x32_bf16 v[184:187], a[216:219], a[108:111], v[184:187]\n"
    "s_cmp_ge_u32 s76, 64\n"
    "s_cselect_b32 s73, 1073741824, s73\n"
    "v_mfma_f32_16x16x32_bf16 v[188:191], a[220:223], a[108:111], v[188:191]\n"
    "v_mfma_f32_16x16x32_bf16 v[156:159], a[220:223], a[104:107], v[156:159]\n"
    "s_bitcmp1_b64 s[60:61], s79\n"
    "s_cbranch_scc1 Loe102%=\n"
    "Loe102r%=:\n"
    "v_mfma_f32_16x16x32_bf16 v[88:91], a[216:219], a[96:99], v[88:91]\n"
    "v_mfma_f32_16x16x32_bf16 v[120:123], a[216:219], a[100:103], v[120:123]\n"
    "s_add_u32 s79, s79, 1\n"
    "v_mfma_f32_16x16x32_bf16 v[124:127], a[220:223], a[100:103], v[124:127]\n"
    "v_mfma_f32_16x16x32_bf16 v[92:95], a[220:223], a[96:99], v[92:95]\n"
    "s_cmp_eq_u32 s41, 0\n"
    "s_cbranch_scc1 LW2%=\n"
    "LW102%=:\n"
    "s_waitcnt lgkmcnt(0)\n"
    "s_barrier\n"
    "LT103%=:\n"
    "v_mfma_f32_16x16x32_bf16 v[64:67], a[160:163], a[64:67], v[64:67]\n"
    "s_sub_u32 s78, s4, s56\n"
    "ds_read_b64_tr_b16 a[192:193], v24 offset:16384\n"
    "v_mfma_f32_16x16x32_bf16 v[96:99], a[160:163], a[68:71], v[96:99]\n"
    "s_memtime s[56:57]\n"
    "s_add_u32 m0, s70, 32768\n"
    "v_mfma_f32_16x16x32_bf16 v[100:103], a[164:167], a[68:71], v[100:103]\n"
    "buffer_load_dwordx4 v48, s[84:87], s72 offen offset:0 sc0 sc1 lds\n"
    "v_mfma_f32_16x16x32_bf16 v[68:71], a[164:167], a[64:67], v[68:71]\n"
    "ds_read_b64_tr_b16 a[194:195], v24 offset:17408\n"
    "v_mfma_f32_16x16x32_bf16 v[128:131], a[160:163], a[72:75], v[128:131]\n"
    "v_mfma_f32_16x16x32_bf16 v[160:163], a[160:163], a[76:79], v[160:163]\n"
    "buffer_load_dwordx4 v49, s[84:87], s72 offen offset:1024 sc0 sc1 lds\n"
    "v_mfma_f32_16x16x32_bf16 v[164:167], a[164:167], a[76:79], v[164:167]\n"
    "ds_read_b64_tr_b16 a[196:197], v25 offset:16384\n"
    "v_mfma_f32_16x16x32_bf16 v[132:135], a[164:167], a[72:75], v[132:135]\n"
    "v_mfma_f32_16x16x32_bf16 v[192:195], a[160:163], a[80:83], v[192:195]\n"
    "buffer_load_dwordx4 v50, s[84:87], s72 offen offset:2048 sc0 sc1 lds\n"
    "v_mfma_f32_16x16x32_bf16 v[224:227], a[160:163], a[84:87], v[224:227]\n"
    "ds_read_b64_tr_b16 a[198:199], v25 offset:17408\n"
    "v_mfma_f32_16x16x32_bf16 v[228:231], a[164:167], a[84:87], v[228:231]\n"
    "v_mfma_f32_16x16x32_bf16 v[196:199], a[164:167], a[80:83], v[196:199]\n"
    "buffer_load_dwordx4 v51, s[84:87], s72 offen offset:3072 sc0 sc1 lds\n"
    "v_mfma_f32_16x16x32_bf16 a[0:3], a[160:163], a[88:91], a[0:3]\n"
    "ds_read_b64_tr_b16 a[200:201], v26 offset:16384\n"
    "v_mfma_f32_16x16x32_bf16 a[32:35], a[160:163], a[92:95], a[32:35]\n"
    "s_add_u32 m0, s70, 36864\n"
    "v_mfma_f32_16x16x32_bf16 a[36:39], a[164:167], a[92:95], a[36:39]\n"
    "buffer_load_dwordx4 v52, s[84:87], s72 offen offset:0 sc0 sc1 lds\n"
    "v_mfma_f32_16x16x32_bf16 a[4:7], a[164:167], a[88:91], a[4:7]\n"
    "ds_read_b64_tr_b16 a[202:203], v26 offset:17408\n"
    "v_mfma_f32_16x16x32_bf16 a[8:11], a[168:171], a[88:91], a[8:11]\n"
    "v_mfma_f32_16x16x32_bf16 a[40:43], a[168:171], a[92:95], a[40:43]\n"
    "buffer_load_dwordx4 v53, s[84:87], s72 offen offset:1024 sc0 sc1 lds\n"
    "v_mfma_f32_16x16x32_bf16 a[44:47], a[172:175], a[92:95], a[44:47]\n"
    "ds_read_b64_tr_b16 a[204:205], v27 offset:16384\n"
    "v_mfma_f32_16x16x32_bf16 a[12:15], a[172:175], a[88:91], a[12:15]\n"
    "v_mfma_f32_16x16x32_bf16 v[200:203], a[168:171], a[80:83], v[200:203]\n"
    "buffer_load_dwordx4 v54, s[84:87], s72 offen offset:2048 sc0 sc1 lds\n"
    "v_mfma_f32_16x16x32_bf16 v[232:235], a[168:171], a[84:87], v[232:235]\n"
    "ds_read_b64_tr_b16 a[206:207], v27 offset:17408\n"
    "v_mfma_f32_16x16x32_bf16 v[236:239], a[172:175], a[84:87], v[236:239]\n"
    "v_mfma_f32_16x16x32_bf16 v[204:207], a[172:175], a[80:83], v[204:207]\n"
    "buffer_load_dwordx4 v55, s[84:87], s72 offen offset:3072 sc0 sc1 lds\n"
    "v_mfma_f32_16x16x32_bf16 v[136:139], a[168:171], a[72:75], v[136:139]\n"
    "ds_read_b64_tr_b16 a[208:209], v28 offset:16384\n"
    "v_mfma_f32_16x16x32_bf16 v[168:171], a[168:171], a[76:79], v[168:171]\n"
    "v_mfma_f32_16x16x32_bf16 v[172:175], a[172:175], a[76:79], v[172:175]\n"
    "v_mfma_f32_16x16x32_bf16 v[140:143], a[172:175], a[72:75], v[140:143]\n"
    "ds_read_b64_tr_b16 a[210:211], v28 offset:17408\n"
    "v_mfma_f32_16x16x32_bf16 v[72:75], a[168:171], a[64:67], v[72:75]\n"
    "s_add_u32 m0, s71, 0\n"
    "v_mfma_f32_16x16x32_bf16 v[104:107], a[168:171], a[68:71], v[104:107]\n"
    "buffer_load_dwordx4 v56, s[12:15], s73 offen offset:0 nt lds\n"
    "v_mfma_f32_16x16x32_bf16 v[108:111], a[172:175], a[68:71], v[108:111]\n"
    "ds_read_b64_tr_b16 a[212:213], v29 offset:16384\n"
    "v_mfma_f32_16x16x32_bf16 v[76:79], a[172:175], a[64:67], v[76:79]\n"
    "v_mfma_f32_16x16x32_bf16 v[80:83], a[176:179], a[64:67], v[80:83]\n"
    "v_mfma_f32_16x16x32_bf16 v[112:115], a[176:179], a[68:71], v[112:115]\n"
    "ds_read_b64_tr_b16 a[214:215], v29 offset:17408\n"
    "v_mfma_f32_16x16x32_bf16 v[116:119], a[180:183], a[68:71], v[116:119]\n"
    "v_mfma_f32_16x16x32_bf16 v[84:87], a[180:183], a[64:67], v[84:87]\n"
    "buffer_load_dwordx4 v57, s[12:15], s73 offen offset:1024 nt lds\n"
    "v_mfma_f32_16x16x32_bf16 v[144:147], a[176:179], a[72:75], v[144:147]\n"
    "ds_read_b64_tr_b16 a[216:217], v30 offset:16384\n"
    "v_mfma_f32_16x16x32_bf16 v[176:179], a[176:179], a[76:79], v[176:179]\n"
    "v_mfma_f32_16x16x32_bf16 v[180:183], a[180:183], a[76:79], v[180:183]\n"
    "v_mfma_f32_16x16x32_bf16 v[148:151], a[180:183], a[72:75], v[148:151]\n"
    "ds_read_b64_tr_b16 a[218:219], v30 offset:17408\n"
    "v_mfma_f32_16x16x32_bf16 v[208:211], a[176:179], a[80:83], v[208:211]\n"
    "v_mfma_f32_16x16x32_bf16 v[240:243], a[176:179], a[84:87], v[240:243]\n"
    "buffer_load_dwordx4 v58, s[12:15], s73 offen offset:2048 nt lds\n"
    "v_mfma_f32_16x16x32_bf16 v[244:247], a[180:183], a[84:87], v[244:247]\n"
    "ds_read_b64_tr_b16 a[220:221], v31 offset:16384\n"
    "v_mfma_f32_16x16x32_bf16 v[212:215], a[180:183], a[80:83], v[212:215]\n"
    "v_mfma_f32_16x16x32_bf16 a[16:19], a[176:179], a[88:91], a[16:19]\n"
    "v_mfma_f32_16x16x32_bf16 a[48:51], a[176:179], a[92:95], a[48:51]\n"
    "ds_read_b64_tr_b16 a[222:223], v31 offset:17408\n"
    "v_mfma_f32_16x16x32_bf16 a[52:55], a[180:183], a[92:95], a[52:55]\n"
    "v_mfma_f32_16x16x32_bf16 a[20:23], a[180:183], a[88:91], a[20:23]\n"
    "buffer_load_dwordx4 v59, s[12:15], s73 offen offset:3072 nt lds\n"
    "v_mfma_f32_16x16x32_bf16 a[24:27], a[184:187], a[88:91], a[24:27]\n"
    "v_mfma_f32_16x16x32_bf16 a[56:59], a[184:187], a[92:95], a[56:59]\n"
    "s_add_u32 s73, s73, 0x200000\n"
    "v_mfma_f32_16x16x32_bf16 a[60:63], a[188:191], a[92:95], a[60:63]\n"
    "s_add_u32 s75, s75, 1\n"
    "v_mfma_f32_16x16x32_bf16 a[28:31], a[188:191], a[88:91], a[28:31]\n"
    "v_mfma_f32_16x16x32_bf16 v[216:219], a[184:187], a[80:83], v[216:219]\n"
    "s_cmp_gt_u32 s78, s30\n"
    "s_cbranch_scc1 Lad103%=\n"
    "Lad103r%=:\n"
    "v_mfma_f32_16x16x32_bf16 v[248:251], a[184:187], a[84:87], v[248:251]\n"
    "v_mfma_f32_16x16x32_bf16 v[252:255], a[188:191], a[84:87], v[252:255]\n"
    "v_mfma_f32_16x16x32_bf16 v[220:223], a[188:191], a[80:83], v[220:223]\n"
    "v_mfma_f32_16x16x32_bf16 v[152:155], a[184:187], a[72:75], v[152:155]\n"
    "v_mfma_f32_16x16x32_bf16 v[184:187], a[184:187], a[76:79], v[184:187]\n"
    "v_mfma_f32_16x16x32_bf16 v[188:191], a[188:191], a[76:79], v[188:191]\n"
    "v_mfma_f32_16x16x32_bf16 v[156:159], a[188:191], a[72:75], v[156:159]\n"
    "v_mfma_f32_16x16x32_bf16 v[88:91], a[184:187], a[64:67], v[88:91]\n"
    "v_mfma_f32_16x16x32_bf16 v[120:123], a[184:187], a[68:71], v[120:123]\n"
    "v_mfma_f32_16x16x32_bf16 v[124:127], a[188:191], a[68:71], v[124:127]\n"
    "v_mfma_f32_16x16x32_bf16 v[92:95], a[188:191], a[64:67], v[92:95]\n"
    "s_waitcnt vmcnt(20) lgkmcnt(0)\n"
    "s_barrier\n"
    "v_mfma_f32_16x16x32_bf16 v[64:67], a[192:195], a[128:131], v[64:67]\n"
    "ds_read_b64_tr_b16 a[160:161], v24 offset:32768\n"
    "s_memtime s[4:5]\n"
    "v_mfma_f32_16x16x32_bf16 v[96:99], a[192:195], a[132:135], v[96:99]\n"
    "ds_read_b128 a[64:67], v32 offset:0\n"
    "v_mfma_f32_16x16x32_bf16 v[100:103], a[196:199], a[132:135], v[100:103]\n"
    "s_bitcmp1_b64 s[64:65], s79\n"
    "s_cbranch_scc1 Lfl103%=\n"
    "Lfl103r%=:\n"
    "v_mfma_f32_16x16x32_bf16 v[68:71], a[196:199], a[128:131], v[68:71]\n"
    "ds_read_b64_tr_b16 a[162:163], v24 offset:33792\n"
    "v_mfma_f32_16x16x32_bf16 v[128:131], a[192:195], a[136:139], v[128:131]\n"
    "ds_read_b128 a[68:71], v32 offset:2048\n"
    "s_add_u32 m0, s71, 16384\n"
    "v_mfma_f32_16x16x32_bf16 v[160:163], a[192:195], a[140:143], v[160:163]\n"
    "buffer_load_dwordx4 v56, s[12:15], s73 offen offset:0 nt lds\n"
    "v_mfma_f32_16x16x32_bf16 v[164:167], a[196:199], a[140:143], v[164:167]\n"
    "ds_read_b64_tr_b16 a[164:165], v25 offset:32768\n"
    "v_mfma_f32_16x16x32_bf16 v[132:135], a[196:199], a[136:139], v[132:135]\n"
    "ds_read_b128 a[72:75], v32 offset:4096\n"
    "v_mfma_f32_16x16x32_bf16 v[192:195], a[192:195], a[144:147], v[192:195]\n"
    "v_mfma_f32_16x16x32_bf16 v[224:227], a[192:195], a[148:151], v[224:227]\n"
    "ds_read_b64_tr_b16 a[166:167], v25 offset:33792\n"
    "v_mfma_f32_16x16x32_bf16 v[228:231], a[196:199], a[148:151], v[228:231]\n"
    "ds_read_b128 a[76:79], v32 offset:6144\n"
    "v_mfma_f32_16x16x32_bf16 v[196:199], a[196:199], a[144:147], v[196:199]\n"
    "v_mfma_f32_16x16x32_bf16 a[0:3], a[192:195], a[152:155], a[0:3]\n"
    "ds_read_b64_tr_b16 a[168:169], v26 offset:32768\n"
    "v_mfma_f32_16x16x32_bf16 a[32:35], a[192:195], a[156:159], a[32:35]\n"
    "ds_read_b128 a[80:83], v32 offset:8192\n"
    "v_mfma_f32_16x16x32_bf16 a[36:39], a[196:199], a[156:159], a[36:39]\n"
    "v_mfma_f32_16x16x32_bf16 a[4:7], a[196:199], a[152:155], a[4:7]\n"
    "ds_read_b64_tr_b16 a[170:171], v26 offset:33792\n"
    "v_mfma_f32_16x16x32_bf16 a[8:11], a[200:203], a[152:155], a[8:11]\n"
    "ds_read_b128 a[84:87], v32 offset:10240\n"
    "v_mfma_f32_16x16x32_bf16 a[40:43], a[200:203], a[156:159], a[40:43]\n"
    "buffer_load_dwordx4 v57, s[12:15], s73 offen offset:1024 nt lds\n"
    "v_mfma_f32_16x16x32_bf16 a[44:47], a[204:207], a[156:159], a[44:47]\n"
    "ds_read_b64_tr_b16 a[172:173], v27 offset:32768\n"
    "v_mfma_f32_16x16x32_bf16 a[12:15], a[204:207], a[152:155], a[12:15]\n"
    "ds_read_b128 a[88:91], v32 offset:12288\n"
    "v_mfma_f32_16x16x32_bf16 v[200:203], a[200:203], a[144:147], v[200:203]\n"
    "v_mfma_f32_16x16x32_bf16 v[232:235], a[200:203], a[148:151], v[232:235]\n"
    "ds_read_b64_tr_b16 a[174:175], v27 offset:33792\n"
    "v_mfma_f32_16x16x32_bf16 v[236:239], a[204:207], a[148:151], v[236:239]\n"
    "ds_read_b128 a[92:95], v32 offset:14336\n"
    "v_mfma_f32_16x16x32_bf16 v[204:207], a[204:207], a[144:147], v[204:207]\n"
    "v_mfma_f32_16x16x32_bf16 v[136:139], a[200:203], a[136:139], v[136:139]\n"
    "ds_read_b64_tr_b16 a[176:177], v28 offset:32768\n"
    "v_mfma_f32_16x16x32_bf16 v[168:171], a[200:203], a[140:143], v[168:171]\n"
    "ds_read_b128 a[96:99], v33 offset:0\n"
    "v_mfma_f32_16x16x32_bf16 v[172:175], a[204:207], a[140:143], v[172:175]\n"
    "v_mfma_f32_16x16x32_bf16 v[140:143], a[204:207], a[136:139], v[140:143]\n"
    "ds_read_b64_tr_b16 a[178:179], v28 offset:33792\n"
    "v_mfma_f32_16x16x32_bf16 v[72:75], a[200:203], a[128:131], v[72:75]\n"
    "ds_read_b128 a[100:103], v33 offset:2048\n"
    "v_mfma_f32_16x16x32_bf16 v[104:107], a[200:203], a[132:135], v[104:107]\n"
    "buffer_load_dwordx4 v58, s[12:15], s73 offen offset:2048 nt lds\n"
    "v_mfma_f32_16x16x32_bf16 v[108:111], a[204:207], a[132:135], v[108:111]\n"
    "ds_read_b64_tr_b16 a[180:181], v29 offset:32768\n"
    "v_mfma_f32_16x16x32_bf16 v[76:79], a[204:207], a[128:131], v[76:79]\n"
    "ds_read_b128 a[104:107], v33 offset:4096\n"
    "v_mfma_f32_16x16x32_bf16 v[80:83], a[208:211], a[128:131], v[80:83]\n"
    "v_mfma_f32_16x16x32_bf16 v[112:115], a[208:211], a[132:135], v[112:115]\n"
    "ds_read_b64_tr_b16 a[182:183], v29 offset:33792\n"
    "v_mfma_f32_16x16x32_bf16 v[116:119], a[212:215], a[132:135], v[116:119]\n"
    "ds_read_b128 a[108:111], v33 offset:6144\n"
    "v_mfma_f32_16x16x32_bf16 v[84:87], a[212:215], a[128:131], v[84:87]\n"
    "v_mfma_f32_16x16x32_bf16 v[144:147], a[208:211], a[136:139], v[144:147]\n"
    "ds_read_b64_tr_b16 a[184:185], v30 offset:32768\n"
    "v_mfma_f32_16x16x32_bf16 v[176:179], a[208:211], a[140:143], v[176:179]\n"
    "ds_read_b128 a[112:115], v33 offset:8192\n"
    "v_mfma_f32_16x16x32_bf16 v[180:183], a[212:215], a[140:143], v[180:183]\n"
    "v_mfma_f32_16x16x32_bf16 v[148:151], a[212:215], a[136:139], v[148:151]\n"
    "ds_read_b64_tr_b16 a[186:187], v30 offset:33792\n"
    "v_mfma_f32_16x16x32_bf16 v[208:211], a[208:211], a[144:147], v[208:211]\n"
    "ds_read_b128 a[116:119], v33 offset:10240\n"
    "v_mfma_f32_16x16x32_bf16 v[240:243], a[208:211], a[148:151], v[240:243]\n"
    "buffer_load_dwordx4 v59, s[12:15], s73 offen offset:3072 nt lds\n"
    "v_mfma_f32_16x16x32_bf16 v[244:247], a[212:215], a[148:151], v[244:247]\n"
    "ds_read_b64_tr_b16 a[188:189], v31 offset:32768\n"
    "v_mfma_f32_16x16x32_bf16 v[212:215], a[212:215], a[144:147], v[212:215]\n"
    "ds_read_b128 a[120:123], v33 offset:12288\n"
    "v_mfma_f32_16x16x32_bf16 a[16:19], a[208:211], a[152:155], a[16:19]\n"
    "v_mfma_f32_16x16x32_bf16 a[48:51], a[208:211], a[156:159], a[48:51]\n"
    "ds_read_b64_tr_b16 a[190:191], v31 offset:33792\n"
    "v_mfma_f32_16x16x32_bf16 a[52:55], a[212:215], a[156:159], a[52:55]\n"
    "ds_read_b128 a[124:127], v33 offset:14336\n"
    "v_mfma_f32_16x16x32_bf16 a[20:23], a[212:215], a[152:155], a[20:23]\n"
    "v_mfma_f32_16x16x32_bf16 a[24:27], a[216:219], a[152:155], a[24:27]\n"
    "s_add_u32 s76, s76, 1\n"
    "v_mfma_f32_16x16x32_bf16 a[56:59], a[216:219], a[156:159], a[56:59]\n"
    "v_readlane_b32 s72, v40, s75\n"
    "v_mfma_f32_16x16x32_bf16 a[60:63], a[220:223], a[156:159], a[60:63]\n"
    "v_mfma_f32_16x16x32_bf16 a[28:31], a[220:223], a[152:155], a[28:31]\n"
    "v_readlane_b32 s73, v41, s76\n"
    "v_mfma_f32_16x16x32_bf16 v[216:219], a[216:219], a[144:147], v[216:219]\n"
    "s_cmp_eq_u32 s75, 8\n"
    "v_mfma_f32_16x16x32_bf16 v[248:251], a[216:219], a[148:151], v[248:251]\n"
    "s_cselect_b32 s84, s88, s84\n"
    "s_cselect_b32 s85, s89, s85\n"
    "v_mfma_f32_16x16x32_bf16 v[252:255], a[220:223], a[148:151], v[252:255]\n"
    "v_mfma_f32_16x16x32_bf16 v[220:223], a[220:223], a[144:147], v[220:223]\n"
    "s_cmp_ge_u32 s75, 64\n"
    "s_cselect_b32 s72, 1073741824, s72\n"
    "v_mfma_f32_16x16x32_bf16 v[152:155], a[216:219], a[136:139], v[152:155]\n"
    "v_mfma_f32_16x16x32_bf16 v[184:187], a[216:219], a[140:143], v[184:187]\n"
    "s_cmp_ge_u32 s76, 64\n"
    "s_cselect_b32 s73, 1073741824, s73\n"
    "v_mfma_f32_16x16x32_bf16 v[188:191], a[220:223], a[140:143], v[188:191]\n"
    "v_mfma_f32_16x16x32_bf16 v[156:159], a[220:223], a[136:139], v[156:159]\n"
    "s_bitcmp1_b64 s[60:61], s79\n"
    "s_cbranch_scc1 Loe103%=\n"
    "Loe103r%=:\n"
    "v_mfma_f32_16x16x32_bf16 v[88:91], a[216:219], a[128:131], v[88:91]\n"
    "v_mfma_f32_16x16x32_bf16 v[120:123], a[216:219], a[132:135], v[120:123]\n"
    "s_add_u32 s79, s79, 1\n"
    "v_mfma_f32_16x16x32_bf16 v[124:127], a[220:223], a[132:135], v[124:127]\n"
    "v_mfma_f32_16x16x32_bf16 v[92:95], a[220:223], a[128:131], v[92:95]\n"
    "s_cmp_eq_u32 s41, 0\n"
    "s_cbranch_scc1 LW3%=\n"
    "LW103%=:\n"
    "s_waitcnt lgkmcnt(0)\n"
    "s_barrier\n"
    "LT104%=:\n"
    "v_mfma_f32_16x16x32_bf16 v[64:67], a[160:163], a[64:67], v[64:67]\n"
    "s_sub_u32 s78, s4, s56\n"
    "ds_read_b64_tr_b16 a[192:193], v24 offset:49152\n"
    "v_mfma_f32_16x16x32_bf16 v[96:99], a[160:163], a[68:71], v[96:99]\n"
    "s_memtime s[56:57]\n"
    "s_add_u32 m0, s70, 0\n"
    "v_mfma_f32_16x16x32_bf16 v[100:103], a[164:167], a[68:71], v[100:103]\n"
    "buffer_load_dwordx4 v48, s[84:87], s72 offen offset:0 sc0 sc1 lds\n"
    "v_mfma_f32_16x16x32_bf16 v[68:71], a[164:167], a[64:67], v[68:71]\n"
    "ds_read_b64_tr_b16 a[194:195], v24 offset:50176\n"
    "v_mfma_f32_16x16x32_bf16 v[128:131], a[160:163], a[72:75], v[128:131]\n"
    "v_mfma_f32_16x16x32_bf16 v[160:163], a[160:163], a[76:79], v[160:163]\n"
    "buffer_load_dwordx4 v49, s[84:87], s72 offen offset:1024 sc0 sc1 lds\n"
    "v_mfma_f32_16x16x32_bf16 v[164:167], a[164:167], a[76:79], v[164:167]\n"
    "ds_read_b64_tr_b16 a[196:197], v25 offset:49152\n"
    "v_mfma_f32_16x16x32_bf16 v[132:135], a[164:167], a[72:75], v[132:135]\n"
    "v_mfma_f32_16x16x32_bf16 v[192:195], a[160:163], a[80:83], v[192:195]\n"
    "buffer_load_dwordx4 v50, s[84:87], s72 offen offset:2048 sc0 sc1 lds\n"
    "v_mfma_f32_16x16x32_bf16 v[224:227], a[160:163], a[84:87], v[224:227]\n"
    "ds_read_b64_tr_b16 a[198:199], v25 offset:50176\n"
    "v_mfma_f32_16x16x32_bf16 v[228:231], a[164:167], a[84:87], v[228:231]\n"
    "v_mfma_f32_16x16x32_bf16 v[196:199], a[164:167], a[80:83], v[196:199]\n"
    "buffer_load_dwordx4 v51, s[84:87], s72 offen offset:3072 sc0 sc1 lds\n"
    "v_mfma_f32_16x16x32_bf16 a[0:3], a[160:163], a[88:91], a[0:3]\n"
    "ds_read_b64_tr_b16 a[200:201], v26 offset:49152\n"
    "v_mfma_f32_16x16x32_bf16 a[32:35], a[160:163], a[92:95], a[32:35]\n"
    "s_add_u32 m0, s70, 4096\n"
    "v_mfma_f32_16x16x32_bf16 a[36:39], a[164:167], a[92:95], a[36:39]\n"
    "buffer_load_dwordx4 v52, s[84:87], s72 offen offset:0 sc0 sc1 lds\n"
    "v_mfma_f32_16x16x32_bf16 a[4:7], a[164:167], a[88:91], a[4:7]\n"
    "ds_read_b64_tr_b16 a[202:203], v26 offset:50176\n"
    "v_mfma_f32_16x16x32_bf16 a[8:11], a[168:171], a[88:91], a[8:11]\n"
    "v_mfma_f32_16x16x32_bf16 a[40:43], a[168:171], a[92:95], a[40:43]\n"
    "buffer_load_dwordx4 v53, s[84:87], s72 offen offset:1024 sc0 sc1 lds\n"
    "v_mfma_f32_16x16x32_bf16 a[44:47], a[172:175], a[92:95], a[44:47]\n"
    "ds_read_b64_tr_b16 a[204:205], v27 offset:49152\n"
    "v_mfma_f32_16x16x32_bf16 a[12:15], a[172:175], a[88:91], a[12:15]\n"
    "v_mfma_f32_16x16x32_bf16 v[200:203], a[168:171], a[80:83], v[200:203]\n"
    "buffer_load_dwordx4 v54, s[84:87], s72 offen offset:2048 sc0 sc1 lds\n"
    "v_mfma_f32_16x16x32_bf16 v[232:235], a[168:171], a[84:87], v[232:235]\n"
    "ds_read_b64_tr_b16 a[206:207], v27 offset:50176\n"
    "v_mfma_f32_16x16x32_bf16 v[236:239], a[172:175], a[84:87], v[236:239]\n"
    "v_mfma_f32_16x16x32_bf16 v[204:207], a[172:175], a[80:83], v[204:207]\n"
    "buffer_load_dwordx4 v55, s[84:87], s72 offen offset:3072 sc0 sc1 lds\n"
    "v_mfma_f32_16x16x32_bf16 v[136:139], a[168:171], a[72:75], v[136:139]\n"
    "ds_read_b64_tr_b16 a[208:209], v28 offset:49152\n"
    "v_mfma_f32_16x16x32_bf16 v[168:171], a[168:171], a[76:79], v[168:171]\n"
    "v_mfma_f32_16x16x32_bf16 v[172:175], a[172:175], a[76:79], v[172:175]\n"
    "v_mfma_f32_16x16x32_bf16 v[140:143], a[172:175], a[72:75], v[140:143]\n"
    "ds_read_b64_tr_b16 a[210:211], v28 offset:50176\n"
    "v_mfma_f32_16x16x32_bf16 v[72:75], a[168:171], a[64:67], v[72:75]\n"
    "s_add_u32 m0, s71, 32768\n"
    "v_mfma_f32_16x16x32_bf16 v[104:107], a[168:171], a[68:71], v[104:107]\n"
    "buffer_load_dwordx4 v56, s[12:15], s73 offen offset:0 nt lds\n"
    "v_mfma_f32_16x16x32_bf16 v[108:111], a[172:175], a[68:71], v[108:111]\n"
    "ds_read_b64_tr_b16 a[212:213], v29 offset:49152\n"
    "v_mfma_f32_16x16x32_bf16 v[76:79], a[172:175], a[64:67], v[76:79]\n"
    "v_mfma_f32_16x16x32_bf16 v[80:83], a[176:179], a[64:67], v[80:83]\n"
    "v_mfma_f32_16x16x32_bf16 v[112:115], a[176:179], a[68:71], v[112:115]\n"
    "ds_read_b64_tr_b16 a[214:215], v29 offset:50176\n"
    "v_mfma_f32_16x16x32_bf16 v[116:119], a[180:183], a[68:71], v[116:119]\n"
    "v_mfma_f32_16x16x32_bf16 v[84:87], a[180:183], a[64:67], v[84:87]\n"
    "buffer_load_dwordx4 v57, s[12:15], s73 offen offset:1024 nt lds\n"
    "v_mfma_f32_16x16x32_bf16 v[144:147], a[176:179], a[72:75], v[144:147]\n"
    "ds_read_b64_tr_b16 a[216:217], v30 offset:49152\n"
    "v_mfma_f32_16x16x32_bf16 v[176:179], a[176:179], a[76:79], v[176:179]\n"
    "v_mfma_f32_16x16x32_bf16 v[180:183], a[180:183], a[76:79], v[180:183]\n"
    "v_mfma_f32_16x16x32_bf16 v[148:151], a[180:183], a[72:75], v[148:151]\n"
    "ds_read_b64_tr_b16 a[218:219], v30 offset:50176\n"
    "v_mfma_f32_16x16x32_bf16 v[208:211], a[176:179], a[80:83], v[208:211]\n"
    "v_mfma_f32_16x16x32_bf16 v[240:243], a[176:179], a[84:87], v[240:243]\n"
    "buffer_load_dwordx4 v58, s[12:15], s73 offen offset:2048 nt lds\n"
    "v_mfma_f32_16x16x32_bf16 v[244:247], a[180:183], a[84:87], v[244:247]\n"
    "ds_read_b64_tr_b16 a[220:221], v31 offset:49152\n"
    "v_mfma_f32_16x16x32_bf16 v[212:215], a[180:183], a[80:83], v[212:215]\n"
    "v_mfma_f32_16x16x32_bf16 a[16:19], a[176:179], a[88:91], a[16:19]\n"
    "v_mfma_f32_16x16x32_bf16 a[48:51], a[176:179], a[92:95], a[48:51]\n"
    "ds_read_b64_tr_b16 a[222:223], v31 offset:50176\n"
    "v_mfma_f32_16x16x32_bf16 a[52:55], a[180:183], a[92:95], a[52:55]\n"
    "v_mfma_f32_16x16x32_bf16 a[20:23], a[180:183], a[88:91], a[20:23]\n"
    "buffer_load_dwordx4 v59, s[12:15], s73 offen offset:3072 nt lds\n"
    "v_mfma_f32_16x16x32_bf16 a[24:27], a[184:187], a[88:91], a[24:27]\n"
    "v_mfma_f32_16x16x32_bf16 a[56:59], a[184:187], a[92:95], a[56:59]\n"
    "s_add_u32 s73, s73, 0x200000\n"
    "v_mfma_f32_16x16x32_bf16 a[60:63], a[188:191], a[92:95], a[60:63]\n"
    "s_add_u32 s75, s75, 1\n"
    "v_mfma_f32_16x16x32_bf16 a[28:31], a[188:191], a[88:91], a[28:31]\n"
    "v_mfma_f32_16x16x32_bf16 v[216:219], a[184:187], a[80:83], v[216:219]\n"
    "s_cmp_gt_u32 s78, s30\n"
    "s_cbranch_scc1 Lad104%=\n"
    "Lad104r%=:\n"
    "v_mfma_f32_16x16x32_bf16 v[248:251], a[184:187], a[84:87], v[248:251]\n"
    "v_mfma_f32_16x16x32_bf16 v[252:255], a[188:191], a[84:87], v[252:255]\n"
    "v_mfma_f32_16x16x32_bf16 v[220:223], a[188:191], a[80:83], v[220:223]\n"
    "v_mfma_f32_16x16x32_bf16 v[152:155], a[184:187], a[72:75], v[152:155]\n"
    "v_mfma_f32_16x16x32_bf16 v[184:187], a[184:187], a[76:79], v[184:187]\n"
    "v_mfma_f32_16x16x32_bf16 v[188:191], a[188:191], a[76:79], v[188:191]\n"
    "v_mfma_f32_16x16x32_bf16 v[156:159], a[188:191], a[72:75], v[156:159]\n"
    "v_mfma_f32_16x16x32_bf16 v[88:91], a[184:187], a[64:67], v[88:91]\n"
    "v_mfma_f32_16x16x32_bf16 v[120:123], a[184:187], a[68:71], v[120:123]\n"
    "v_mfma_f32_16x16x32_bf16 v[124:127], a[188:191], a[68:71], v[124:127]\n"
    "v_mfma_f32_16x16x32_bf16 v[92:95], a[188:191], a[64:67], v[92:95]\n"
    "s_waitcnt vmcnt(20) lgkmcnt(0)\n"
    "s_barrier\n"
    "v_mfma_f32_16x16x32_bf16 v[64:67], a[192:195], a[96:99], v[64:67]\n"
    "ds_read_b64_tr_b16 a[160:161], v16 offset:0\n"
    "s_memtime s[4:5]\n"
    "v_mfma_f32_16x16x32_bf16 v[96:99], a[192:195], a[100:103], v[96:99]\n"
    "ds_read_b128 a[64:67], v32 offset:32768\n"
    "v_mfma_f32_16x16x32_bf16 v[100:103], a[196:199], a[100:103], v[100:103]\n"
    "s_bitcmp1_b64 s[64:65], s79\n"
    "s_cbranch_scc1 Lfl104%=\n"
    "Lfl104r%=:\n"
    "v_mfma_f32_16x16x32_bf16 v[68:71], a[196:199], a[96:99], v[68:71]\n"
    "ds_read_b64_tr_b16 a[162:163], v16 offset:1024\n"
    "v_mfma_f32_16x16x32_bf16 v[128:131], a[192:195], a[104:107], v[128:131]\n"
    "ds_read_b128 a[68:71], v32 offset:34816\n"
    "s_add_u32 m0, s71, 49152\n"
    "v_mfma_f32_16x16x32_bf16 v[160:163], a[192:195], a[108:111], v[160:163]\n"
    "buffer_load_dwordx4 v56, s[12:15], s73 offen offset:0 nt lds\n"
    "v_mfma_f32_16x16x32_bf16 v[164:167], a[196:199], a[108:111], v[164:167]\n"
    "ds_read_b64_tr_b16 a[164:165], v17 offset:0\n"
    "v_mfma_f32_16x16x32_bf16 v[132:135], a[196:199], a[104:107], v[132:135]\n"
    "ds_read_b128 a[72:75], v32 offset:36864\n"
    "v_mfma_f32_16x16x32_bf16 v[192:195], a[192:195], a[112:115], v[192:195]\n"
    "v_mfma_f32_16x16x32_bf16 v[224:227], a[192:195], a[116:119], v[224:227]\n"
    "ds_read_b64_tr_b16 a[166:167], v17 offset:1024\n"
    "v_mfma_f32_16x16x32_bf16 v[228:231], a[196:199], a[116:119], v[228:231]\n"
    "ds_read_b128 a[76:79], v32 offset:38912\n"
    "v_mfma_f32_16x16x32_bf16 v[196:199], a[196:199], a[112:115], v[196:199]\n"
    "v_mfma_f32_16x16x32_bf16 a[0:3], a[192:195], a[120:123], a[0:3]\n"
    "ds_read_b64_tr_b16 a[168:169], v18 offset:0\n"
    "v_mfma_f32_16x16x32_bf16 a[32:35], a[192:195], a[124:127], a[32:35]\n"
    "ds_read_b128 a[80:83], v32 offset:40960\n"
    "v_mfma_f32_16x16x32_bf16 a[36:39], a[196:199], a[124:127], a[36:39]\n"
    "v_mfma_f32_16x16x32_bf16 a[4:7], a[196:199], a[120:123], a[4:7]\n"
    "ds_read_b64_tr_b16 a[170:171], v18 offset:1024\n"
    "v_mfma_f32_16x16x32_bf16 a[8:11], a[200:203], a[120:123], a[8:11]\n"
    "ds_read_b128 a[84:87], v32 offset:43008\n"
    "v_mfma_f32_16x16x32_bf16 a[40:43], a[200:203], a[124:127], a[40:43]\n"
    "buffer_load_dwordx4 v57, s[12:15], s73 offen offset:1024 nt lds\n"
    "v_mfma_f32_16x16x32_bf16 a[44:47], a[204:207], a[124:127], a[44:47]\n"
    "ds_read_b64_tr_b16 a[172:173], v19 offset:0\n"
    "v_mfma_f32_16x16x32_bf16 a[12:15], a[204:207], a[120:123], a[12:15]\n"
    "ds_read_b128 a[88:91], v32 offset:45056\n"
    "v_mfma_f32_16x16x32_bf16 v[200:203], a[200:203], a[112:115], v[200:203]\n"
    "v_mfma_f32_16x16x32_bf16 v[232:235], a[200:203], a[116:119], v[232:235]\n"
    "ds_read_b64_tr_b16 a[174:175], v19 offset:1024\n"
    "v_mfma_f32_16x16x32_bf16 v[236:239], a[204:207], a[116:119], v[236:239]\n"
    "ds_read_b128 a[92:95], v32 offset:47104\n"
    "v_mfma_f32_16x16x32_bf16 v[204:207], a[204:207], a[112:115], v[204:207]\n"
    "v_mfma_f32_16x16x32_bf16 v[136:139], a[200:203], a[104:107], v[136:139]\n"
    "ds_read_b64_tr_b16 a[176:177], v20 offset:0\n"
    "v_mfma_f32_16x16x32_bf16 v[168:171], a[200:203], a[108:111], v[168:171]\n"
    "ds_read_b128 a[128:131], v33 offset:32768\n"
    "v_mfma_f32_16x16x32_bf16 v[172:175], a[204:207], a[108:111], v[172:175]\n"
    "v_mfma_f32_16x16x32_bf16 v[140:143], a[204:207], a[104:107], v[140:143]\n"
    "ds_read_b64_tr_b16 a[178:179], v20 offset:1024\n"
    "v_mfma_f32_16x16x32_bf16 v[72:75], a[200:203], a[96:99], v[72:75]\n"
    "ds_read_b128 a[132:135], v33 offset:34816\n"
    "v_mfma_f32_16x16x32_bf16 v[104:107], a[200:203], a[100:103], v[104:107]\n"
    "buffer_load_dwordx4 v58, s[12:15], s73 offen offset:2048 nt lds\n"
    "v_mfma_f32_16x16x32_bf16 v[108:111], a[204:207], a[100:103], v[108:111]\n"
    "ds_read_b64_tr_b16 a[180:181], v21 offset:0\n"
    "v_mfma_f32_16x16x32_bf16 v[76:79], a[204:207], a[96:99], v[76:79]\n"
    "ds_read_b128 a[136:139], v33 offset:36864\n"
    "v_mfma_f32_16x16x32_bf16 v[80:83], a[208:211], a[96:99], v[80:83]\n"
    "v_mfma_f32_16x16x32_bf16 v[112:115], a[208:211], a[100:103], v[112:115]\n"
    "ds_read_b64_tr_b16 a[182:183], v21 offset:1024\n"
    "v_mfma_f32_16x16x32_bf16 v[116:119], a[212:215], a[100:103], v[116:119]\n"
    "ds_read_b128 a[140:143], v33 offset:38912\n"
    "v_mfma_f32_16x16x32_bf16 v[84:87], a[212:215], a[96:99], v[84:87]\n"
    "v_mfma_f32_16x16x32_bf16 v[144:147], a[208:211], a[104:107], v[144:147]\n"
    "ds_read_b64_tr_b16 a[184:185], v22 offset:0\n"
    "v_mfma_f32_16x16x32_bf16 v[176:179], a[208:211], a[108:111], v[176:179]\n"
    "ds_read_b128 a[144:147], v33 offset:40960\n"
    "v_mfma_f32_16x16x32_bf16 v[180:183], a[212:215], a[108:111], v[180:183]\n"
    "v_mfma_f32_16x16x32_bf16 v[148:151], a[212:215], a[104:107], v[148:151]\n"
    "ds_read_b64_tr_b16 a[186:187], v22 offset:1024\n"
    "v_mfma_f32_16x16x32_bf16 v[208:211], a[208:211], a[112:115], v[208:211]\n"
    "ds_read_b128 a[148:151], v33 offset:43008\n"
    "v_mfma_f32_16x16x32_bf16 v[240:243], a[208:211], a[116:119], v[240:243]\n"
    "buffer_load_dwordx4 v59, s[12:15], s73 offen offset:3072 nt lds\n"
    "v_mfma_f32_16x16x32_bf16 v[244:247], a[212:215], a[116:119], v[244:247]\n"
    "ds_read_b64_tr_b16 a[188:189], v23 offset:0\n"
    "v_mfma_f32_16x16x32_bf16 v[212:215], a[212:215], a[112:115], v[212:215]\n"
    "ds_read_b128 a[152:155], v33 offset:45056\n"
    "v_mfma_f32_16x16x32_bf16 a[16:19], a[208:211], a[120:123], a[16:19]\n"
    "v_mfma_f32_16x16x32_bf16 a[48:51], a[208:211], a[124:127], a[48:51]\n"
    "ds_read_b64_tr_b16 a[190:191], v23 offset:1024\n"
    "v_mfma_f32_16x16x32_bf16 a[52:55], a[212:215], a[124:127], a[52:55]\n"
    "ds_read_b128 a[156:159], v33 offset:47104\n"
    "v_mfma_f32_16x16x32_bf16 a[20:23], a[212:215], a[120:123], a[20:23]\n"
    "v_mfma_f32_16x16x32_bf16 a[24:27], a[216:219], a[120:123], a[24:27]\n"
    "s_add_u32 s76, s76, 1\n"
    "v_mfma_f32_16x16x32_bf16 a[56:59], a[216:219], a[124:127], a[56:59]\n"
    "v_readlane_b32 s72, v40, s75\n"
    "v_mfma_f32_16x16x32_bf16 a[60:63], a[220:223], a[124:127], a[60:63]\n"
    "v_mfma_f32_16x16x32_bf16 a[28:31], a[220:223], a[120:123], a[28:31]\n"
    "v_readlane_b32 s73, v41, s76\n"
    "v_mfma_f32_16x16x32_bf16 v[216:219], a[216:219], a[112:115], v[216:219]\n"
    "s_cmp_eq_u32 s75, 8\n"
    "v_mfma_f32_16x16x32_bf16 v[248:251], a[216:219], a[116:119], v[248:251]\n"
    "s_cselect_b32 s84, s88, s84\n"
    "s_cselect_b32 s85, s89, s85\n"
    "v_mfma_f32_16x16x32_bf16 v[252:255], a[220:223], a[116:119], v[252:255]\n"
    "v_mfma_f32_16x16x32_bf16 v[220:223], a[220:223], a[112:115], v[220:223]\n"
    "s_cmp_ge_u32 s75, 64\n"
    "s_cselect_b32 s72, 1073741824, s72\n"
    "v_mfma_f32_16x16x32_bf16 v[152:155], a[216:219], a[104:107], v[152:155]\n"
    "v_mfma_f32_16x16x32_bf16 v[184:187], a[216:219], a[108:111], v[184:187]\n"
    "s_cmp_ge_u32 s76, 64\n"
    "s_cselect_b32 s73, 1073741824, s73\n"
    "v_mfma_f32_16x16x32_bf16 v[188:191], a[220:223], a[108:111], v[188:191]\n"
    "v_mfma_f32_16x16x32_bf16 v[156:159], a[220:223], a[104:107], v[156:159]\n"
    "s_bitcmp1_b64 s[60:61], s79\n"
    "s_cbranch_scc1 Loe104%=\n"
    "Loe104r%=:\n"
    "v_mfma_f32_16x16x32_bf16 v[88:91], a[216:219], a[96:99], v[88:91]\n"
    "v_mfma_f32_16x16x32_bf16 v[120:123], a[216:219], a[100:103], v[120:123]\n"
    "s_add_u32 s79, s79, 1\n"
    "v_mfma_f32_16x16x32_bf16 v[124:127], a[220:223], a[100:103], v[124:127]\n"
    "v_mfma_f32_16x16x32_bf16 v[92:95], a[220:223], a[96:99], v[92:95]\n"
    "s_cmp_eq_u32 s41, 0\n"
    "s_cbranch_scc1 LW4%=\n"
    "LW104%=:\n"
    "s_waitcnt lgkmcnt(0)\n"
    "s_barrier\n"
    "LT105%=:\n"
    "v_mfma_f32_16x16x32_bf16 v[64:67], a[160:163], a[64:67], v[64:67]\n"
    "s_sub_u32 s78, s4, s56\n"
    "ds_read_b64_tr_b16 a[192:193], v16 offset:16384\n"
    "v_mfma_f32_16x16x32_bf16 v[96:99], a[160:163], a[68:71], v[96:99]\n"
    "s_memtime s[56:57]\n"
    "s_add_u32 m0, s70, 32768\n"
    "v_mfma_f32_16x16x32_bf16 v[100:103], a[164:167], a[68:71], v[100:103]\n"
    "buffer_load_dwordx4 v48, s[84:87], s72 offen offset:0 sc0 sc1 lds\n"
    "v_mfma_f32_16x16x32_bf16 v[68:71], a[164:167], a[64:67], v[68:71]\n"
    "ds_read_b64_tr_b16 a[194:195], v16 offset:17408\n"
    "v_mfma_f32_16x16x32_bf16 v[128:131], a[160:163], a[72:75], v[128:131]\n"
    "v_mfma_f32_16x16x32_bf16 v[160:163], a[160:163], a[76:79], v[160:163]\n"
    "buffer_load_dwordx4 v49, s[84:87], s72 offen offset:1024 sc0 sc1 lds\n"
    "v_mfma_f32_16x16x32_bf16 v[164:167], a[164:167], a[76:79], v[164:167]\n"
    "ds_read_b64_tr_b16 a[196:197], v17 offset:16384\n"
    "v_mfma_f32_16x16x32_bf16 v[132:135], a[164:167], a[72:75], v[132:135]\n"
    "v_mfma_f32_16x16x32_bf16 v[192:195], a[160:163], a[80:83], v[192:195]\n"
    "buffer_load_dwordx4 v50, s[84:87], s72 offen offset:2048 sc0 sc1 lds\n"
    "v_mfma_f32_16x16x32_bf16 v[224:227], a[160:163], a[84:87], v[224:227]\n"
    "ds_read_b64_tr_b16 a[198:199], v17 offset:17408\n"
    "v_mfma_f32_16x16x32_bf16 v[228:231], a[164:167], a[84:87], v[228:231]\n"
    "v_mfma_f32_16x16x32_bf16 v[196:199], a[164:167], a[80:83], v[196:199]\n"
    "buffer_load_dwordx4 v51, s[84:87], s72 offen offset:3072 sc0 sc1 lds\n"
    "v_mfma_f32_16x16x32_bf16 a[0:3], a[160:163], a[88:91], a[0:3]\n"
    "ds_read_b64_tr_b16 a[200:201], v18 offset:16384\n"
    "v_mfma_f32_16x16x32_bf16 a[32:35], a[160:163], a[92:95], a[32:35]\n"
    "s_add_u32 m0, s70, 36864\n"
    "v_mfma_f32_16x16x32_bf16 a[36:39], a[164:167], a[92:95], a[36:39]\n"
    "buffer_load_dwordx4 v52, s[84:87], s72 offen offset:0 sc0 sc1 lds\n"
    "v_mfma_f32_16x16x32_bf16 a[4:7], a[164:167], a[88:91], a[4:7]\n"
    "ds_read_b64_tr_b16 a[202:203], v18 offset:17408\n"
    "v_mfma_f32_16x16x32_bf16 a[8:11], a[168:171], a[88:91], a[8:11]\n"
    "v_mfma_f32_16x16x32_bf16 a[40:43], a[168:171], a[92:95], a[40:43]\n"
    "buffer_load_dwordx4 v53, s[84:87], s72 offen offset:1024 sc0 sc1 lds\n"
    "v_mfma_f32_16x16x32_bf16 a[44:47], a[172:175], a[92:95], a[44:47]\n"
    "ds_read_b64_tr_b16 a[204:205], v19 offset:16384\n"
    "v_mfma_f32_16x16x32_bf16 a[12:15], a[172:175], a[88:91], a[12:15]\n"
    "v_mfma_f32_16x16x32_bf16 v[200:203], a[168:171], a[80:83], v[200:203]\n"
    "buffer_load_dwordx4 v54, s[84:87], s72 offen offset:2048 sc0 sc1 lds\n"
    "v_mfma_f32_16x16x32_bf16 v[232:235], a[168:171], a[84:87], v[232:235]\n"
    "ds_read_b64_tr_b16 a[206:207], v19 offset:17408\n"
    "v_mfma_f32_16x16x32_bf16 v[236:239], a[172:175], a[84:87], v[236:239]\n"
    "v_mfma_f32_16x16x32_bf16 v[204:207], a[172:175], a[80:83], v[204:207]\n"
    "buffer_load_dwordx4 v55, s[84:87], s72 offen offset:3072 sc0 sc1 lds\n"
    "v_mfma_f32_16x16x32_bf16 v[136:139], a[168:171], a[72:75], v[136:139]\n"
    "ds_read_b64_tr_b16 a[208:209], v20 offset:16384\n"
    "v_mfma_f32_16x16x32_bf16 v[168:171], a[168:171], a[76:79], v[168:171]\n"
    "v_mfma_f32_16x16x32_bf16 v[172:175], a[172:175], a[76:79], v[172:175]\n"
    "v_mfma_f32_16x16x32_bf16 v[140:143], a[172:175], a[72:75], v[140:143]\n"
    "ds_read_b64_tr_b16 a[210:211], v20 offset:17408\n"
    "v_mfma_f32_16x16x32_bf16 v[72:75], a[168:171], a[64:67], v[72:75]\n"
    "s_add_u32 m0, s71, 65536\n"
    "v_mfma_f32_16x16x32_bf16 v[104:107], a[168:171], a[68:71], v[104:107]\n"
    "buffer_load_dwordx4 v56, s[12:15], s73 offen offset:0 nt lds\n"
    "v_mfma_f32_16x16x32_bf16 v[108:111], a[172:175], a[68:71], v[108:111]\n"
    "ds_read_b64_tr_b16 a[212:213], v21 offset:16384\n"
    "v_mfma_f32_16x16x32_bf16 v[76:79], a[172:175], a[64:67], v[76:79]\n"
    "v_mfma_f32_16x16x32_bf16 v[80:83], a[176:179], a[64:67], v[80:83]\n"
    "v_mfma_f32_16x16x32_bf16 v[112:115], a[176:179], a[68:71], v[112:115]\n"
    "ds_read_b64_tr_b16 a[214:215], v21 offset:17408\n"
    "v_mfma_f32_16x16x32_bf16 v[116:119], a[180:183], a[68:71], v[116:119]\n"
    "v_mfma_f32_16x16x32_bf16 v[84:87], a[180:183], a[64:67], v[84:87]\n"
    "buffer_load_dwordx4 v57, s[12:15], s73 offen offset:1024 nt lds\n"
    "v_mfma_f32_16x16x32_bf16 v[144:147], a[176:179], a[72:75], v[144:147]\n"
    "ds_read_b64_tr_b16 a[216:217], v22 offset:16384\n"
    "v_mfma_f32_16x16x32_bf16 v[176:179], a[176:179], a[76:79], v[176:179]\n"
    "v_mfma_f32_16x16x32_bf16 v[180:183], a[180:183], a[76:79], v[180:183]\n"
    "v_mfma_f32_16x16x32_bf16 v[148:151], a[180:183], a[72:75], v[148:151]\n"
    "ds_read_b64_tr_b16 a[218:219], v22 offset:17408\n"
    "v_mfma_f32_16x16x32_bf16 v[208:211], a[176:179], a[80:83], v[208:211]\n"
    "v_mfma_f32_16x16x32_bf16 v[240:243], a[176:179], a[84:87], v[240:243]\n"
    "buffer_load_dwordx4 v58, s[12:15], s73 offen offset:2048 nt lds\n"
    "v_mfma_f32_16x16x32_bf16 v[244:247], a[180:183], a[84:87], v[244:247]\n"
    "ds_read_b64_tr_b16 a[220:221], v23 offset:16384\n"
    "v_mfma_f32_16x16x32_bf16 v[212:215], a[180:183], a[80:83], v[212:215]\n"
    "v_mfma_f32_16x16x32_bf16 a[16:19], a[176:179], a[88:91], a[16:19]\n"
    "v_mfma_f32_16x16x32_bf16 a[48:51], a[176:179], a[92:95], a[48:51]\n"
    "ds_read_b64_tr_b16 a[222:223], v23 offset:17408\n"
    "v_mfma_f32_16x16x32_bf16 a[52:55], a[180:183], a[92:95], a[52:55]\n"
    "v_mfma_f32_16x16x32_bf16 a[20:23], a[180:183], a[88:91], a[20:23]\n"
    "buffer_load_dwordx4 v59, s[12:15], s73 offen offset:3072 nt lds\n"
    "v_mfma_f32_16x16x32_bf16 a[24:27], a[184:187], a[88:91], a[24:27]\n"
    "v_mfma_f32_16x16x32_bf16 a[56:59], a[184:187], a[92:95], a[56:59]\n"
    "s_add_u32 s73, s73, 0x200000\n"
    "v_mfma_f32_16x16x32_bf16 a[60:63], a[188:191], a[92:95], a[60:63]\n"
    "s_add_u32 s75, s75, 1\n"
    "v_mfma_f32_16x16x32_bf16 a[28:31], a[188:191], a[88:91], a[28:31]\n"
    "v_mfma_f32_16x16x32_bf16 v[216:219], a[184:187], a[80:83], v[216:219]\n"
    "s_cmp_gt_u32 s78, s30\n"
    "s_cbranch_scc1 Lad105%=\n"
    "Lad105r%=:\n"
    "v_mfma_f32_16x16x32_bf16 v[248:251], a[184:187], a[84:87], v[248:251]\n"
    "v_mfma_f32_16x16x32_bf16 v[252:255], a[188:191], a[84:87], v[252:255]\n"
    "v_mfma_f32_16x16x32_bf16 v[220:223], a[188:191], a[80:83], v[220:223]\n"
    "v_mfma_f32_16x16x32_bf16 v[152:155], a[184:187], a[72:75], v[152:155]\n"
    "v_mfma_f32_16x16x32_bf16 v[184:187], a[184:187], a[76:79], v[184:187]\n"
    "v_mfma_f32_16x16x32_bf16 v[188:191], a[188:191], a[76:79], v[188:191]\n"
    "v_mfma_f32_16x16x32_bf16 v[156:159], a[188:191], a[72:75], v[156:159]\n"
    "v_mfma_f32_16x16x32_bf16 v[88:91], a[184:187], a[64:67], v[88:91]\n"
    "v_mfma_f32_16x16x32_bf16 v[120:123], a[184:187], a[68:71], v[120:123]\n"
    "v_mfma_f32_16x16x32_bf16 v[124:127], a[188:191], a[68:71], v[124:127]\n"
    "v_mfma_f32_16x16x32_bf16 v[92:95], a[188:191], a[64:67], v[92:95]\n"
    "s_waitcnt vmcnt(20) lgkmcnt(0)\n"
    "s_barrier\n"
    "v_mfma_f32_16x16x32_bf16 v[64:67], a[192:195], a[128:131], v[64:67]\n"
    "ds_read_b64_tr_b16 a[160:161], v24 offset:0\n"
    "s_memtime s[4:5]\n"
    "v_mfma_f32_16x16x32_bf16 v[96:99], a[192:195], a[132:135], v[96:99]\n"
    "ds_read_b128 a[64:67], v32 offset:0\n"
    "v_mfma_f32_16x16x32_bf16 v[100:103], a[196:199], a[132:135], v[100:103]\n"
    "s_bitcmp1_b64 s[64:65], s79\n"
    "s_cbranch_scc1 Lfl105%=\n"
    "Lfl105r%=:\n"
    "v_mfma_f32_16x16x32_bf16 v[68:71], a[196:199], a[128:131], v[68:71]\n"
    "ds_read_b64_tr_b16 a[162:163], v24 offset:1024\n"
    "v_mfma_f32_16x16x32_bf16 v[128:131], a[192:195], a[136:139], v[128:131]\n"
    "ds_read_b128 a[68:71], v32 offset:2048\n"
    "s_add_u32 m0, s71, 81920\n"
    "v_mfma_f32_16x16x32_bf16 v[160:163], a[192:195], a[140:143], v[160:163]\n"
    "buffer_load_dwordx4 v56, s[12:15], s73 offen offset:0 nt lds\n"
    "v_mfma_f32_16x16x32_bf16 v[164:167], a[196:199], a[140:143], v[164:167]\n"
    "ds_read_b64_tr_b16 a[164:165], v25 offset:0\n"
    "v_mfma_f32_16x16x32_bf16 v[132:135], a[196:199], a[136:139], v[132:135]\n"
    "ds_read_b128 a[72:75], v32 offset:4096\n"
    "v_mfma_f32_16x16x32_bf16 v[192:195], a[192:195], a[144:147], v[192:195]\n"
    "v_mfma_f32_16x16x32_bf16 v[224:227], a[192:195], a[148:151], v[224:227]\n"
    "ds_read_b64_tr_b16 a[166:167], v25 offset:1024\n"
    "v_mfma_f32_16x16x32_bf16 v[228:231], a[196:199], a[148:151], v[228:231]\n"
    "ds_read_b128 a[76:79], v32 offset:6144\n"
    "v_mfma_f32_16x16x32_bf16 v[196:199], a[196:199], a[144:147], v[196:199]\n"
    "v_mfma_f32_16x16x32_bf16 a[0:3], a[192:195], a[152:155], a[0:3]\n"
    "ds_read_b64_tr_b16 a[168:169], v26 offset:0\n"
    "v_mfma_f32_16x16x32_bf16 a[32:35], a[192:195], a[156:159], a[32:35]\n"
    "ds_read_b128 a[80:83], v32 offset:8192\n"
    "v_mfma_f32_16x16x32_bf16 a[36:39], a[196:199], a[156:159], a[36:39]\n"
    "v_mfma_f32_16x16x32_bf16 a[4:7], a[196:199], a[152:155], a[4:7]\n"
    "ds_read_b64_tr_b16 a[170:171], v26 offset:1024\n"
    "v_mfma_f32_16x16x32_bf16 a[8:11], a[200:203], a[152:155], a[8:11]\n"
    "ds_read_b128 a[84:87], v32 offset:10240\n"
    "v_mfma_f32_16x16x32_bf16 a[40:43], a[200:203], a[156:159], a[40:43]\n"
    "buffer_load_dwordx4 v57, s[12:15], s73 offen offset:1024 nt lds\n"
    "v_mfma_f32_16x16x32_bf16 a[44:47], a[204:207], a[156:159], a[44:47]\n"
    "ds_read_b64_tr_b16 a[172:173], v27 offset:0\n"
    "v_mfma_f32_16x16x32_bf16 a[12:15], a[204:207], a[152:155], a[12:15]\n"
    "ds_read_b128 a[88:91], v32 offset:12288\n"
    "v_mfma_f32_16x16x32_bf16 v[200:203], a[200:203], a[144:147], v[200:203]\n"
    "v_mfma_f32_16x16x32_bf16 v[232:235], a[200:203], a[148:151], v[232:235]\n"
    "ds_read_b64_tr_b16 a[174:175], v27 offset:1024\n"
    "v_mfma_f32_16x16x32_bf16 v[236:239], a[204:207], a[148:151], v[236:239]\n"
    "ds_read_b128 a[92:95], v32 offset:14336\n"
    "v_mfma_f32_16x16x32_bf16 v[204:207], a[204:207], a[144:147], v[204:207]\n"
    "v_mfma_f32_16x16x32_bf16 v[136:139], a[200:203], a[136:139], v[136:139]\n"
    "ds_read_b64_tr_b16 a[176:177], v28 offset:0\n"
    "v_mfma_f32_16x16x32_bf16 v[168:171], a[200:203], a[140:143], v[168:171]\n"
    "ds_read_b128 a[96:99], v33 offset:0\n"
    "v_mfma_f32_16x16x32_bf16 v[172:175], a[204:207], a[140:143], v[172:175]\n"
    "v_mfma_f32_16x16x32_bf16 v[140:143], a[204:207], a[136:139], v[140:143]\n"
    "ds_read_b64_tr_b16 a[178:179], v28 offset:1024\n"
    "v_mfma_f32_16x16x32_bf16 v[72:75], a[200:203], a[128:131], v[72:75]\n"
    "ds_read_b128 a[100:103], v33 offset:2048\n"
    "v_mfma_f32_16x16x32_bf16 v[104:107], a[200:203], a[132:135], v[104:107]\n"
    "buffer_load_dwordx4 v58, s[12:15], s73 offen offset:2048 nt lds\n"
    "v_mfma_f32_16x16x32_bf16 v[108:111], a[204:207], a[132:135], v[108:111]\n"
    "ds_read_b64_tr_b16 a[180:181], v29 offset:0\n"
    "v_mfma_f32_16x16x32_bf16 v[76:79], a[204:207], a[128:131], v[76:79]\n"
    "ds_read_b128 a[104:107], v33 offset:4096\n"
    "v_mfma_f32_16x16x32_bf16 v[80:83], a[208:211], a[128:131], v[80:83]\n"
    "v_mfma_f32_16x16x32_bf16 v[112:115], a[208:211], a[132:135], v[112:115]\n"
    "ds_read_b64_tr_b16 a[182:183], v29 offset:1024\n"
    "v_mfma_f32_16x16x32_bf16 v[116:119], a[212:215], a[132:135], v[116:119]\n"
    "ds_read_b128 a[108:111], v33 offset:6144\n"
    "v_mfma_f32_16x16x32_bf16 v[84:87], a[212:215], a[128:131], v[84:87]\n"
    "v_mfma_f32_16x16x32_bf16 v[144:147], a[208:211], a[136:139], v[144:147]\n"
    "ds_read_b64_tr_b16 a[184:185], v30 offset:0\n"
    "v_mfma_f32_16x16x32_bf16 v[176:179], a[208:211], a[140:143], v[176:179]\n"
    "ds_read_b128 a[112:115], v33 offset:8192\n"
    "v_mfma_f32_16x16x32_bf16 v[180:183], a[212:215], a[140:143], v[180:183]\n"
    "v_mfma_f32_16x16x32_bf16 v[148:151], a[212:215], a[136:139], v[148:151]\n"
    "ds_read_b64_tr_b16 a[186:187], v30 offset:1024\n"
    "v_mfma_f32_16x16x32_bf16 v[208:211], a[208:211], a[144:147], v[208:211]\n"
    "ds_read_b128 a[116:119], v33 offset:10240\n"
    "v_mfma_f32_16x16x32_bf16 v[240:243], a[208:211], a[148:151], v[240:243]\n"
    "buffer_load_dwordx4 v59, s[12:15], s73 offen offset:3072 nt lds\n"
    "v_mfma_f32_16x16x32_bf16 v[244:247], a[212:215], a[148:151], v[244:247]\n"
    "ds_read_b64_tr_b16 a[188:189], v31 offset:0\n"
    "v_mfma_f32_16x16x32_bf16 v[212:215], a[212:215], a[144:147], v[212:215]\n"
    "ds_read_b128 a[120:123], v33 offset:12288\n"
    "v_mfma_f32_16x16x32_bf16 a[16:19], a[208:211], a[152:155], a[16:19]\n"
    "v_mfma_f32_16x16x32_bf16 a[48:51], a[208:211], a[156:159], a[48:51]\n"
    "ds_read_b64_tr_b16 a[190:191], v31 offset:1024\n"
    "v_mfma_f32_16x16x32_bf16 a[52:55], a[212:215], a[156:159], a[52:55]\n"
    "ds_read_b128 a[124:127], v33 offset:14336\n"
    "v_mfma_f32_16x16x32_bf16 a[20:23], a[212:215], a[152:155], a[20:23]\n"
    "v_mfma_f32_16x16x32_bf16 a[24:27], a[216:219], a[152:155], a[24:27]\n"
    "s_add_u32 s76, s76, 1\n"
    "v_mfma_f32_16x16x32_bf16 a[56:59], a[216:219], a[156:159], a[56:59]\n"
    "v_readlane_b32 s72, v40, s75\n"
    "v_mfma_f32_16x16x32_bf16 a[60:63], a[220:223], a[156:159], a[60:63]\n"
    "v_mfma_f32_16x16x32_bf16 a[28:31], a[220:223], a[152:155], a[28:31]\n"
    "v_readlane_b32 s73, v41, s76\n"
    "v_mfma_f32_16x16x32_bf16 v[216:219], a[216:219], a[144:147], v[216:219]\n"
    "s_cmp_eq_u32 s75, 8\n"
    "v_mfma_f32_16x16x32_bf16 v[248:251], a[216:219], a[148:151], v[248:251]\n"
    "s_cselect_b32 s84, s88, s84\n"
    "s_cselect_b32 s85, s89, s85\n"
    "v_mfma_f32_16x16x32_bf16 v[252:255], a[220:223], a[148:151], v[252:255]\n"
    "v_mfma_f32_16x16x32_bf16 v[220:223], a[220:223], a[144:147], v[220:223]\n"
    "s_cmp_ge_u32 s75, 64\n"
    "s_cselect_b32 s72, 1073741824, s72\n"
    "v_mfma_f32_16x16x32_bf16 v[152:155], a[216:219], a[136:139], v[152:155]\n"
    "v_mfma_f32_16x16x32_bf16 v[184:187], a[216:219], a[140:143], v[184:187]\n"
    "s_cmp_ge_u32 s76, 64\n"
    "s_cselect_b32 s73, 1073741824, s73\n"
    "v_mfma_f32_16x16x32_bf16 v[188:191], a[220:223], a[140:143], v[188:191]\n"
    "v_mfma_f32_16x16x32_bf16 v[156:159], a[220:223], a[136:139], v[156:159]\n"
    "s_bitcmp1_b64 s[60:61], s79\n"
    "s_cbranch_scc1 Loe105%=\n"
    "Loe105r%=:\n"
    "v_mfma_f32_16x16x32_bf16 v[88:91], a[216:219], a[128:131], v[88:91]\n"
    "v_mfma_f32_16x16x32_bf16 v[120:123], a[216:219], a[132:135], v[120:123]\n"
    "s_add_u32 s79, s79, 1\n"
    "v_mfma_f32_16x16x32_bf16 v[124:127], a[220:223], a[132:135], v[124:127]\n"
    "v_mfma_f32_16x16x32_bf16 v[92:95], a[220:223], a[128:131], v[92:95]\n"
    "s_cmp_eq_u32 s41, 0\n"
    "s_cbranch_scc1 LW5%=\n"
    "LW105%=:\n"
    "s_waitcnt lgkmcnt(0)\n"
    "s_barrier\n"
    "LT106%=:\n"
    "v_mfma_f32_16x16x32_bf16 v[64:67], a[160:163], a[64:67], v[64:67]\n"
    "s_sub_u32 s78, s4, s56\n"
    "ds_read_b64_tr_b16 a[192:193], v24 offset:16384\n"
    "v_mfma_f32_16x16x32_bf16 v[96:99], a[160:163], a[68:71], v[96:99]\n"
    "s_memtime s[56:57]\n"
    "s_add_u32 m0, s70, 0\n"
    "v_mfma_f32_16x16x32_bf16 v[100:103], a[164:167], a[68:71], v[100:103]\n"
    "buffer_load_dwordx4 v48, s[84:87], s72 offen offset:0 sc0 sc1 lds\n"
    "v_mfma_f32_16x16x32_bf16 v[68:71], a[164:167], a[64:67], v[68:71]\n"
    "ds_read_b64_tr_b16 a[194:195], v24 offset:17408\n"
    "v_mfma_f32_16x16x32_bf16 v[128:131], a[160:163], a[72:75], v[128:131]\n"
    "v_mfma_f32_16x16x32_bf16 v[160:163], a[160:163], a[76:79], v[160:163]\n"
    "buffer_load_dwordx4 v49, s[84:87], s72 offen offset:1024 sc0 sc1 lds\n"
    "v_mfma_f32_16x16x32_bf16 v[164:167], a[164:167], a[76:79], v[164:167]\n"
    "ds_read_b64_tr_b16 a[196:197], v25 offset:16384\n"
    "v_mfma_f32_16x16x32_bf16 v[132:135], a[164:167], a[72:75], v[132:135]\n"
    "v_mfma_f32_16x16x32_bf16 v[192:195], a[160:163], a[80:83], v[192:195]\n"
    "buffer_load_dwordx4 v50, s[84:87], s72 offen offset:2048 sc0 sc1 lds\n"
    "v_mfma_f32_16x16x32_bf16 v[224:227], a[160:163], a[84:87], v[224:227]\n"
    "ds_read_b64_tr_b16 a[198:199], v25 offset:17408\n"
    "v_mfma_f32_16x16x32_bf16 v[228:231], a[164:167], a[84:87], v[228:231]\n"
    "v_mfma_f32_16x16x32_bf16 v[196:199], a[164:167], a[80:83], v[196:199]\n"
    "buffer_load_dwordx4 v51, s[84:87], s72 offen offset:3072 sc0 sc1 lds\n"
    "v_mfma_f32_16x16x32_bf16 a[0:3], a[160:163], a[88:91], a[0:3]\n"
    "ds_read_b64_tr_b16 a[200:201], v26 offset:16384\n"
    "v_mfma_f32_16x16x32_bf16 a[32:35], a[160:163], a[92:95], a[32:35]\n"
    "s_add_u32 m0, s70, 4096\n"
    "v_mfma_f32_16x16x32_bf16 a[36:39], a[164:167], a[92:95], a[36:39]\n"
    "buffer_load_dwordx4 v52, s[84:87], s72 offen offset:0 sc0 sc1 lds\n"
    "v_mfma_f32_16x16x32_bf16 a[4:7], a[164:167], a[88:91], a[4:7]\n"
    "ds_read_b64_tr_b16 a[202:203], v26 offset:17408\n"
    "v_mfma_f32_16x16x32_bf16 a[8:11], a[168:171], a[88:91], a[8:11]\n"
    "v_mfma_f32_16x16x32_bf16 a[40:43], a[168:171], a[92:95], a[40:43]\n"
    "buffer_load_dwordx4 v53, s[84:87], s72 offen offset:1024 sc0 sc1 lds\n"
    "v_mfma_f32_16x16x32_bf16 a[44:47], a[172:175], a[92:95], a[44:47]\n"
    "ds_read_b64_tr_b16 a[204:205], v27 offset:16384\n"
    "v_mfma_f32_16x16x32_bf16 a[12:15], a[172:175], a[88:91], a[12:15]\n"
    "v_mfma_f32_16x16x32_bf16 v[200:203], a[168:171], a[80:83], v[200:203]\n"
    "buffer_load_dwordx4 v54, s[84:87], s72 offen offset:2048 sc0 sc1 lds\n"
    "v_mfma_f32_16x16x32_bf16 v[232:235], a[168:171], a[84:87], v[232:235]\n"
    "ds_read_b64_tr_b16 a[206:207], v27 offset:17408\n"
    "v_mfma_f32_16x16x32_bf16 v[236:239], a[172:175], a[84:87], v[236:239]\n"
    "v_mfma_f32_16x16x32_bf16 v[204:207], a[172:175], a[80:83], v[204:207]\n"
    "buffer_load_dwordx4 v55, s[84:87], s72 offen offset:3072 sc0 sc1 lds\n"
    "v_mfma_f32_16x16x32_bf16 v[136:139], a[168:171], a[72:75], v[136:139]\n"
    "ds_read_b64_tr_b16 a[208:209], v28 offset:16384\n"
    "v_mfma_f32_16x16x32_bf16 v[168:171], a[168:171], a[76:79], v[168:171]\n"
    "v_mfma_f32_16x16x32_bf16 v[172:175], a[172:175], a[76:79], v[172:175]\n"
    "v_mfma_f32_16x16x32_bf16 v[140:143], a[172:175], a[72:75], v[140:143]\n"
    "ds_read_b64_tr_b16 a[210:211], v28 offset:17408\n"
    "v_mfma_f32_16x16x32_bf16 v[72:75], a[168:171], a[64:67], v[72:75]\n"
    "s_add_u32 m0, s71, 0\n"
    "v_mfma_f32_16x16x32_bf16 v[104:107], a[168:171], a[68:71], v[104:107]\n"
    "buffer_load_dwordx4 v56, s[12:15], s73 offen offset:0 nt lds\n"
    "v_mfma_f32_16x16x32_bf16 v[108:111], a[172:175], a[68:71], v[108:111]\n"
    "ds_read_b64_tr_b16 a[212:213], v29 offset:16384\n"
    "v_mfma_f32_16x16x32_bf16 v[76:79], a[172:175], a[64:67], v[76:79]\n"
    "v_mfma_f32_16x16x32_bf16 v[80:83], a[176:179], a[64:67], v[80:83]\n"
    "v_mfma_f32_16x16x32_bf16 v[112:115], a[176:179], a[68:71], v[112:115]\n"
    "ds_read_b64_tr_b16 a[214:215], v29 offset:17408\n"
    "v_mfma_f32_16x16x32_bf16 v[116:119], a[180:183], a[68:71], v[116:119]\n"
    "v_mfma_f32_16x16x32_bf16 v[84:87], a[180:183], a[64:67], v[84:87]\n"
    "buffer_load_dwordx4 v57, s[12:15], s73 offen offset:1024 nt lds\n"
    "v_mfma_f32_16x16x32_bf16 v[144:147], a[176:179], a[72:75], v[144:147]\n"
    "ds_read_b64_tr_b16 a[216:217], v30 offset:16384\n"
    "v_mfma_f32_16x16x32_bf16 v[176:179], a[176:179], a[76:79], v[176:179]\n"
    "v_mfma_f32_16x16x32_bf16 v[180:183], a[180:183], a[76:79], v[180:183]\n"
    "v_mfma_f32_16x16x32_bf16 v[148:151], a[180:183], a[72:75], v[148:151]\n"
    "ds_read_b64_tr_b16 a[218:219], v30 offset:17408\n"
    "v_mfma_f32_16x16x32_bf16 v[208:211], a[176:179], a[80:83], v[208:211]\n"
    "v_mfma_f32_16x16x32_bf16 v[240:243], a[176:179], a[84:87], v[240:243]\n"
    "buffer_load_dwordx4 v58, s[12:15], s73 offen offset:2048 nt lds\n"
    "v_mfma_f32_16x16x32_bf16 v[244:247], a[180:183], a[84:87], v[244:247]\n"
    "ds_read_b64_tr_b16 a[220:221], v31 offset:16384\n"
    "v_mfma_f32_16x16x32_bf16 v[212:215], a[180:183], a[80:83], v[212:215]\n"
    "v_mfma_f32_16x16x32_bf16 a[16:19], a[176:179], a[88:91], a[16:19]\n"
    "v_mfma_f32_16x16x32_bf16 a[48:51], a[176:179], a[92:95], a[48:51]\n"
    "ds_read_b64_tr_b16 a[222:223], v31 offset:17408\n"
    "v_mfma_f32_16x16x32_bf16 a[52:55], a[180:183], a[92:95], a[52:55]\n"
    "v_mfma_f32_16x16x32_bf16 a[20:23], a[180:183], a[88:91], a[20:23]\n"
    "buffer_load_dwordx4 v59, s[12:15], s73 offen offset:3072 nt lds\n"
    "v_mfma_f32_16x16x32_bf16 a[24:27], a[184:187], a[88:91], a[24:27]\n"
    "v_mfma_f32_16x16x32_bf16 a[56:59], a[184:187], a[92:95], a[56:59]\n"
    "s_add_u32 s73, s73, 0x200000\n"
    "v_mfma_f32_16x16x32_bf16 a[60:63], a[188:191], a[92:95], a[60:63]\n"
    "s_add_u32 s75, s75, 1\n"
    "v_mfma_f32_16x16x32_bf16 a[28:31], a[188:191], a[88:91], a[28:31]\n"
    "v_mfma_f32_16x16x32_bf16 v[216:219], a[184:187], a[80:83], v[216:219]\n"
    "s_cmp_gt_u32 s78, s30\n"
    "s_cbranch_scc1 Lad106%=\n"
    "Lad106r%=:\n"
    "v_mfma_f32_16x16x32_bf16 v[248:251], a[184:187], a[84:87], v[248:251]\n"
    "v_mfma_f32_16x16x32_bf16 v[252:255], a[188:191], a[84:87], v[252:255]\n"
    "v_mfma_f32_16x16x32_bf16 v[220:223], a[188:191], a[80:83], v[220:223]\n"
    "v_mfma_f32_16x16x32_bf16 v[152:155], a[184:187], a[72:75], v[152:155]\n"
    "v_mfma_f32_16x16x32_bf16 v[184:187], a[184:187], a[76:79], v[184:187]\n"
    "v_mfma_f32_16x16x32_bf16 v[188:191], a[188:191], a[76:79], v[188:191]\n"
    "v_mfma_f32_16x16x32_bf16 v[156:159], a[188:191], a[72:75], v[156:159]\n"
    "v_mfma_f32_16x16x32_bf16 v[88:91], a[184:187], a[64:67], v[88:91]\n"
    "v_mfma_f32_16x16x32_bf16 v[120:123], a[184:187], a[68:71], v[120:123]\n"
    "v_mfma_f32_16x16x32_bf16 v[124:127], a[188:191], a[68:71], v[124:127]\n"
    "v_mfma_f32_16x16x32_bf16 v[92:95], a[188:191], a[64:67], v[92:95]\n"
    "s_waitcnt vmcnt(20) lgkmcnt(0)\n"
    "s_barrier\n"
    "v_mfma_f32_16x16x32_bf16 v[64:67], a[192:195], a[96:99], v[64:67]\n"
    "ds_read_b64_tr_b16 a[160:161], v24 offset:32768\n"
    "s_memtime s[4:5]\n"
    "v_mfma_f32_16x16x32_bf16 v[96:99], a[192:195], a[100:103], v[96:99]\n"
    "ds_read_b128 a[64:67], v32 offset:32768\n"
    "v_mfma_f32_16x16x32_bf16 v[100:103], a[196:199], a[100:103], v[100:103]\n"
    "s_bitcmp1_b64 s[64:65], s79\n"
    "s_cbranch_scc1 Lfl106%=\n"
    "Lfl106r%=:\n"
    "v_mfma_f32_16x16x32_bf16 v[68:71], a[196:199], a[96:99], v[68:71]\n"
    "ds_read_b64_tr_b16 a[162:163], v24 offset:33792\n"
    "v_mfma_f32_16x16x32_bf16 v[128:131], a[192:195], a[104:107], v[128:131]\n"
    "ds_read_b128 a[68:71], v32 offset:34816\n"
    "s_add_u32 m0, s71, 16384\n"
    "v_mfma_f32_16x16x32_bf16 v[160:163], a[192:195], a[108:111], v[160:163]\n"
    "buffer_load_dwordx4 v56, s[12:15], s73 offen offset:0 nt lds\n"
    "v_mfma_f32_16x16x32_bf16 v[164:167], a[196:199], a[108:111], v[164:167]\n"
    "ds_read_b64_tr_b16 a[164:165], v25 offset:32768\n"
    "v_mfma_f32_16x16x32_bf16 v[132:135], a[196:199], a[104:107], v[132:135]\n"
    "ds_read_b128 a[72:75], v32 offset:36864\n"
    "v_mfma_f32_16x16x32_bf16 v[192:195], a[192:195], a[112:115], v[192:195]\n"
    "v_mfma_f32_16x16x32_bf16 v[224:227], a[192:195], a[116:119], v[224:227]\n"
    "ds_read_b64_tr_b16 a[166:167], v25 offset:33792\n"
    "v_mfma_f32_16x16x32_bf16 v[228:231], a[196:199], a[116:119], v[228:231]\n"
    "ds_read_b128 a[76:79], v32 offset:38912\n"
    "v_mfma_f32_16x16x32_bf16 v[196:199], a[196:199], a[112:115], v[196:199]\n"
    "v_mfma_f32_16x16x32_bf16 a[0:3], a[192:195], a[120:123], a[0:3]\n"
    "ds_read_b64_tr_b16 a[168:169], v26 offset:32768\n"
    "v_mfma_f32_16x16x32_bf16 a[32:35], a[192:195], a[124:127], a[32:35]\n"
    "ds_read_b128 a[80:83], v32 offset:40960\n"
    "v_mfma_f32_16x16x32_bf16 a[36:39], a[196:199], a[124:127], a[36:39]\n"
    "v_mfma_f32_16x16x32_bf16 a[4:7], a[196:199], a[120:123], a[4:7]\n"
    "ds_read_b64_tr_b16 a[170:171], v26 offset:33792\n"
    "v_mfma_f32_16x16x32_bf16 a[8:11], a[200:203], a[120:123], a[8:11]\n"
    "ds_read_b128 a[84:87], v32 offset:43008\n"
    "v_mfma_f32_16x16x32_bf16 a[40:43], a[200:203], a[124:127], a[40:43]\n"
    "buffer_load_dwordx4 v57, s[12:15], s73 offen offset:1024 nt lds\n"
    "v_mfma_f32_16x16x32_bf16 a[44:47], a[204:207], a[124:127], a[44:47]\n"
    "ds_read_b64_tr_b16 a[172:173], v27 offset:32768\n"
    "v_mfma_f32_16x16x32_bf16 a[12:15], a[204:207], a[120:123], a[12:15]\n"
    "ds_read_b128 a[88:91], v32 offset:45056\n"
    "v_mfma_f32_16x16x32_bf16 v[200:203], a[200:203], a[112:115], v[200:203]\n"
    "v_mfma_f32_16x16x32_bf16 v[232:235], a[200:203], a[116:119], v[232:235]\n"
    "ds_read_b64_tr_b16 a[174:175], v27 offset:33792\n"
    "v_mfma_f32_16x16x32_bf16 v[236:239], a[204:207], a[116:119], v[236:239]\n"
    "ds_read_b128 a[92:95], v32 offset:47104\n"
    "v_mfma_f32_16x16x32_bf16 v[204:207], a[204:207], a[112:115], v[204:207]\n"
    "v_mfma_f32_16x16x32_bf16 v[136:139], a[200:203], a[104:107], v[136:139]\n"
    "ds_read_b64_tr_b16 a[176:177], v28 offset:32768\n"
    "v_mfma_f32_16x16x32_bf16 v[168:171], a[200:203], a[108:111], v[168:171]\n"
    "ds_read_b128 a[128:131], v33 offset:32768\n"
    "v_mfma_f32_16x16x32_bf16 v[172:175], a[204:207], a[108:111], v[172:175]\n"
    "v_mfma_f32_16x16x32_bf16 v[140:143], a[204:207], a[104:107], v[140:143]\n"
    "ds_read_b64_tr_b16 a[178:179], v28 offset:33792\n"
    "v_mfma_f32_16x16x32_bf16 v[72:75], a[200:203], a[96:99], v[72:75]\n"
    "ds_read_b128 a[132:135], v33 offset:34816\n"
    "v_mfma_f32_16x16x32_bf16 v[104:107], a[200:203], a[100:103], v[104:107]\n"
    "buffer_load_dwordx4 v58, s[12:15], s73 offen offset:2048 nt lds\n"
    "v_mfma_f32_16x16x32_bf16 v[108:111], a[204:207], a[100:103], v[108:111]\n"
    "ds_read_b64_tr_b16 a[180:181], v29 offset:32768\n"
    "v_mfma_f32_16x16x32_bf16 v[76:79], a[204:207], a[96:99], v[76:79]\n"
    "ds_read_b128 a[136:139], v33 offset:36864\n"
    "v_mfma_f32_16x16x32_bf16 v[80:83], a[208:211], a[96:99], v[80:83]\n"
    "v_mfma_f32_16x16x32_bf16 v[112:115], a[208:211], a[100:103], v[112:115]\n"
    "ds_read_b64_tr_b16 a[182:183], v29 offset:33792\n"
    "v_mfma_f32_16x16x32_bf16 v[116:119], a[212:215], a[100:103], v[116:119]\n"
    "ds_read_b128 a[140:143], v33 offset:38912\n"
    "v_mfma_f32_16x16x32_bf16 v[84:87], a[212:215], a[96:99], v[84:87]\n"
    "v_mfma_f32_16x16x32_bf16 v[144:147], a[208:211], a[104:107], v[144:147]\n"
    "ds_read_b64_tr_b16 a[184:185], v30 offset:32768\n"
    "v_mfma_f32_16x16x32_bf16 v[176:179], a[208:211], a[108:111], v[176:179]\n"
    "ds_read_b128 a[144:147], v33 offset:40960\n"
    "v_mfma_f32_16x16x32_bf16 v[180:183], a[212:215], a[108:111], v[180:183]\n"
    "v_mfma_f32_16x16x32_bf16 v[148:151], a[212:215], a[104:107], v[148:151]\n"
    "ds_read_b64_tr_b16 a[186:187], v30 offset:33792\n"
    "v_mfma_f32_16x16x32_bf16 v[208:211], a[208:211], a[112:115], v[208:211]\n"
    "ds_read_b128 a[148:151], v33 offset:43008\n"
    "v_mfma_f32_16x16x32_bf16 v[240:243], a[208:211], a[116:119], v[240:243]\n"
    "buffer_load_dwordx4 v59, s[12:15], s73 offen offset:3072 nt lds\n"
    "v_mfma_f32_16x16x32_bf16 v[244:247], a[212:215], a[116:119], v[244:247]\n"
    "ds_read_b64_tr_b16 a[188:189], v31 offset:32768\n"
    "v_mfma_f32_16x16x32_bf16 v[212:215], a[212:215], a[112:115], v[212:215]\n"
    "ds_read_b128 a[152:155], v33 offset:45056\n"
    "v_mfma_f32_16x16x32_bf16 a[16:19], a[208:211], a[120:123], a[16:19]\n"
    "v_mfma_f32_16x16x32_bf16 a[48:51], a[208:211], a[124:127], a[48:51]\n"
    "ds_read_b64_tr_b16 a[190:191], v31 offset:33792\n"
    "v_mfma_f32_16x16x32_bf16 a[52:55], a[212:215], a[124:127], a[52:55]\n"
    "ds_read_b128 a[156:159], v33 offset:47104\n"
    "v_mfma_f32_16x16x32_bf16 a[20:23], a[212:215], a[120:123], a[20:23]\n"
    "v_mfma_f32_16x16x32_bf16 a[24:27], a[216:219], a[120:123], a[24:27]\n"
    "s_add_u32 s76, s76, 1\n"
    "v_mfma_f32_16x16x32_bf16 a[56:59], a[216:219], a[124:127], a[56:59]\n"
    "v_readlane_b32 s72, v40, s75\n"
    "v_mfma_f32_16x16x32_bf16 a[60:63], a[220:223], a[124:127], a[60:63]\n"
    "v_mfma_f32_16x16x32_bf16 a[28:31], a[220:223], a[120:123], a[28:31]\n"
    "v_readlane_b32 s73, v41, s76\n"
    "v_mfma_f32_16x16x32_bf16 v[216:219], a[216:219], a[112:115], v[216:219]\n"
    "s_cmp_eq_u32 s75, 8\n"
    "v_mfma_f32_16x16x32_bf16 v[248:251], a[216:219], a[116:119], v[248:251]\n"
    "s_cselect_b32 s84, s88, s84\n"
    "s_cselect_b32 s85, s89, s85\n"
    "v_mfma_f32_16x16x32_bf16 v[252:255], a[220:223], a[116:119], v[252:255]\n"
    "v_mfma_f32_16x16x32_bf16 v[220:223], a[220:223], a[112:115], v[220:223]\n"
    "s_cmp_ge_u32 s75, 64\n"
    "s_cselect_b32 s72, 1073741824, s72\n"
    "v_mfma_f32_16x16x32_bf16 v[152:155], a[216:219], a[104:107], v[152:155]\n"
    "v_mfma_f32_16x16x32_bf16 v[184:187], a[216:219], a[108:111], v[184:187]\n"
    "s_cmp_ge_u32 s76, 64\n"
    "s_cselect_b32 s73, 1073741824, s73\n"
    "v_mfma_f32_16x16x32_bf16 v[188:191], a[220:223], a[108:111], v[188:191]\n"
    "v_mfma_f32_16x16x32_bf16 v[156:159], a[220:223], a[104:107], v[156:159]\n"
    "s_bitcmp1_b64 s[60:61], s79\n"
    "s_cbranch_scc1 Loe106%=\n"
    "Loe106r%=:\n"
    "v_mfma_f32_16x16x32_bf16 v[88:91], a[216:219], a[96:99], v[88:91]\n"
    "v_mfma_f32_16x16x32_bf16 v[120:123], a[216:219], a[100:103], v[120:123]\n"
    "s_add_u32 s79, s79, 1\n"
    "v_mfma_f32_16x16x32_bf16 v[124:127], a[220:223], a[100:103], v[124:127]\n"
    "v_mfma_f32_16x16x32_bf16 v[92:95], a[220:223], a[96:99], v[92:95]\n"
    "s_cmp_eq_u32 s41, 0\n"
    "s_cbranch_scc1 LW6%=\n"
    "LW106%=:\n"
    "s_waitcnt lgkmcnt(0)\n"
    "s_barrier\n"
    "LT107%=:\n"
    "v_mfma_f32_16x16x32_bf16 v[64:67], a[160:163], a[64:67], v[64:67]\n"
    "s_sub_u32 s78, s4, s56\n"
    "ds_read_b64_tr_b16 a[192:193], v24 offset:49152\n"
    "v_mfma_f32_16x16x32_bf16 v[96:99], a[160:163], a[68:71], v[96:99]\n"
    "s_memtime s[56:57]\n"
    "s_add_u32 m0, s70, 32768\n"
    "v_mfma_f32_16x16x32_bf16 v[100:103], a[164:167], a[68:71], v[100:103]\n"
    "buffer_load_dwordx4 v48, s[84:87], s72 offen offset:0 sc0 sc1 lds\n"
    "v_mfma_f32_16x16x32_bf16 v[68:71], a[164:167], a[64:67], v[68:71]\n"
    "ds_read_b64_tr_b16 a[194:195], v24 offset:50176\n"
    "v_mfma_f32_16x16x32_bf16 v[128:131], a[160:163], a[72:75], v[128:131]\n"
    "v_mfma_f32_16x16x32_bf16 v[160:163], a[160:163], a[76:79], v[160:163]\n"
    "buffer_load_dwordx4 v49, s[84:87], s72 offen offset:1024 sc0 sc1 lds\n"
    "v_mfma_f32_16x16x32_bf16 v[164:167], a[164:167], a[76:79], v[164:167]\n"
    "ds_read_b64_tr_b16 a[196:197], v25 offset:49152\n"
    "v_mfma_f32_16x16x32_bf16 v[132:135], a[164:167], a[72:75], v[132:135]\n"
    "v_mfma_f32_16x16x32_bf16 v[192:195], a[160:163], a[80:83], v[192:195]\n"
    "buffer_load_dwordx4 v50, s[84:87], s72 offen offset:2048 sc0 sc1 lds\n"
    "v_mfma_f32_16x16x32_bf16 v[224:227], a[160:163], a[84:87], v[224:227]\n"
    "ds_read_b64_tr_b16 a[198:199], v25 offset:50176\n"
    "v_mfma_f32_16x16x32_bf16 v[228:231], a[164:167], a[84:87], v[228:231]\n"
    "v_mfma_f32_16x16x32_bf16 v[196:199], a[164:167], a[80:83], v[196:199]\n"
    "buffer_load_dwordx4 v51, s[84:87], s72 offen offset:3072 sc0 sc1 lds\n"
    "v_mfma_f32_16x16x32_bf16 a[0:3], a[160:163], a[88:91], a[0:3]\n"
    "ds_read_b64_tr_b16 a[200:201], v26 offset:49152\n"
    "v_mfma_f32_16x16x32_bf16 a[32:35], a[160:163], a[92:95], a[32:35]\n"
    "s_add_u32 m0, s70, 36864\n"
    "v_mfma_f32_16x16x32_bf16 a[36:39], a[164:167], a[92:95], a[36:39]\n"
    "buffer_load_dwordx4 v52, s[84:87], s72 offen offset:0 sc0 sc1 lds\n"
    "v_mfma_f32_16x16x32_bf16 a[4:7], a[164:167], a[88:91], a[4:7]\n"
    "ds_read_b64_tr_b16 a[202:203], v26 offset:50176\n"
    "v_mfma_f32_16x16x32_bf16 a[8:11], a[168:171], a[88:91], a[8:11]\n"
    "v_mfma_f32_16x16x32_bf16 a[40:43], a[168:171], a[92:95], a[40:43]\n"
    "buffer_load_dwordx4 v53, s[84:87], s72 offen offset:1024 sc0 sc1 lds\n"
    "v_mfma_f32_16x16x32_bf16 a[44:47], a[172:175], a[92:95], a[44:47]\n"
    "ds_read_b64_tr_b16 a[204:205], v27 offset:49152\n"
    "v_mfma_f32_16x16x32_bf16 a[12:15], a[172:175], a[88:91], a[12:15]\n"
    "v_mfma_f32_16x16x32_bf16 v[200:203], a[168:171], a[80:83], v[200:203]\n"
    "buffer_load_dwordx4 v54, s[84:87], s72 offen offset:2048 sc0 sc1 lds\n"
    "v_mfma_f32_16x16x32_bf16 v[232:235], a[168:171], a[84:87], v[232:235]\n"
    "ds_read_b64_tr_b16 a[206:207], v27 offset:50176\n"
    "v_mfma_f32_16x16x32_bf16 v[236:239], a[172:175], a[84:87], v[236:239]\n"
    "v_mfma_f32_16x16x32_bf16 v[204:207], a[172:175], a[80:83], v[204:207]\n"
    "buffer_load_dwordx4 v55, s[84:87], s72 offen offset:3072 sc0 sc1 lds\n"
    "v_mfma_f32_16x16x32_bf16 v[136:139], a[168:171], a[72:75], v[136:139]\n"
    "ds_read_b64_tr_b16 a[208:209], v28 offset:49152\n"
    "v_mfma_f32_16x16x32_bf16 v[168:171], a[168:171], a[76:79], v[168:171]\n"
    "v_mfma_f32_16x16x32_bf16 v[172:175], a[172:175], a[76:79], v[172:175]\n"
    "v_mfma_f32_16x16x32_bf16 v[140:143], a[172:175], a[72:75], v[140:143]\n"
    "ds_read_b64_tr_b16 a[210:211], v28 offset:50176\n"
    "v_mfma_f32_16x16x32_bf16 v[72:75], a[168:171], a[64:67], v[72:75]\n"
    "s_add_u32 m0, s71, 32768\n"
    "v_mfma_f32_16x16x32_bf16 v[104:107], a[168:171], a[68:71], v[104:107]\n"
    "buffer_load_dwordx4 v56, s[12:15], s73 offen offset:0 nt lds\n"
    "v_mfma_f32_16x16x32_bf16 v[108:111], a[172:175], a[68:71], v[108:111]\n"
    "ds_read_b64_tr_b16 a[212:213], v29 offset:49152\n"
    "v_mfma_f32_16x16x32_bf16 v[76:79], a[172:175], a[64:67], v[76:79]\n"
    "v_mfma_f32_16x16x32_bf16 v[80:83], a[176:179], a[64:67], v[80:83]\n"
    "v_mfma_f32_16x16x32_bf16 v[112:115], a[176:179], a[68:71], v[112:115]\n"
    "ds_read_b64_tr_b16 a[214:215], v29 offset:50176\n"
    "v_mfma_f32_16x16x32_bf16 v[116:119], a[180:183], a[68:71], v[116:119]\n"
    "v_mfma_f32_16x16x32_bf16 v[84:87], a[180:183], a[64:67], v[84:87]\n"
    "buffer_load_dwordx4 v57, s[12:15], s73 offen offset:1024 nt lds\n"
    "v_mfma_f32_16x16x32_bf16 v[144:147], a[176:179], a[72:75], v[144:147]\n"
    "ds_read_b64_tr_b16 a[216:217], v30 offset:49152\n"
    "v_mfma_f32_16x16x32_bf16 v[176:179], a[176:179], a[76:79], v[176:179]\n"
    "v_mfma_f32_16x16x32_bf16 v[180:183], a[180:183], a[76:79], v[180:183]\n"
    "v_mfma_f32_16x16x32_bf16 v[148:151], a[180:183], a[72:75], v[148:151]\n"
    "ds_read_b64_tr_b16 a[218:219], v30 offset:50176\n"
    "v_mfma_f32_16x16x32_bf16 v[208:211], a[176:179], a[80:83], v[208:211]\n"
    "v_mfma_f32_16x16x32_bf16 v[240:243], a[176:179], a[84:87], v[240:243]\n"
    "buffer_load_dwordx4 v58, s[12:15], s73 offen offset:2048 nt lds\n"
    "v_mfma_f32_16x16x32_bf16 v[244:247], a[180:183], a[84:87], v[244:247]\n"
    "ds_read_b64_tr_b16 a[220:221], v31 offset:49152\n"
    "v_mfma_f32_16x16x32_bf16 v[212:215], a[180:183], a[80:83], v[212:215]\n"
    "v_mfma_f32_16x16x32_bf16 a[16:19], a[176:179], a[88:91], a[16:19]\n"
    "v_mfma_f32_16x16x32_bf16 a[48:51], a[176:179], a[92:95], a[48:51]\n"
    "ds_read_b64_tr_b16 a[222:223], v31 offset:50176\n"
    "v_mfma_f32_16x16x32_bf16 a[52:55], a[180:183], a[92:95], a[52:55]\n"
    "v_mfma_f32_16x16x32_bf16 a[20:23], a[180:183], a[88:91], a[20:23]\n"
    "buffer_load_dwordx4 v59, s[12:15], s73 offen offset:3072 nt lds\n"
    "v_mfma_f32_16x16x32_bf16 a[24:27], a[184:187], a[88:91], a[24:27]\n"
    "v_mfma_f32_16x16x32_bf16 a[56:59], a[184:187], a[92:95], a[56:59]\n"
    "s_add_u32 s73, s73, 0x200000\n"
    "v_mfma_f32_16x16x32_bf16 a[60:63], a[188:191], a[92:95], a[60:63]\n"
    "s_add_u32 s75, s75, 1\n"
    "v_mfma_f32_16x16x32_bf16 a[28:31], a[188:191], a[88:91], a[28:31]\n"
    "v_mfma_f32_16x16x32_bf16 v[216:219], a[184:187], a[80:83], v[216:219]\n"
    "s_cmp_gt_u32 s78, s30\n"
    "s_cbranch_scc1 Lad107%=\n"
    "Lad107r%=:\n"
    "v_mfma_f32_16x16x32_bf16 v[248:251], a[184:187], a[84:87], v[248:251]\n"
    "v_mfma_f32_16x16x32_bf16 v[252:255], a[188:191], a[84:87], v[252:255]\n"
    "v_mfma_f32_16x16x32_bf16 v[220:223], a[188:191], a[80:83], v[220:223]\n"
    "v_mfma_f32_16x16x32_bf16 v[152:155], a[184:187], a[72:75], v[152:155]\n"
    "v_mfma_f32_16x16x32_bf16 v[184:187], a[184:187], a[76:79], v[184:187]\n"
    "v_mfma_f32_16x16x32_bf16 v[188:191], a[188:191], a[76:79], v[188:191]\n"
    "v_mfma_f32_16x16x32_bf16 v[156:159], a[188:191], a[72:75], v[156:159]\n"
    "v_mfma_f32_16x16x32_bf16 v[88:91], a[184:187], a[64:67], v[88:91]\n"
    "v_mfma_f32_16x16x32_bf16 v[120:123], a[184:187], a[68:71], v[120:123]\n"
    "v_mfma_f32_16x16x32_bf16 v[124:127], a[188:191], a[68:71], v[124:127]\n"
    "v_mfma_f32_16x16x32_bf16 v[92:95], a[188:191], a[64:67], v[92:95]\n"
    "s_waitcnt vmcnt(20) lgkmcnt(0)\n"
    "s_barrier\n"
    "v_mfma_f32_16x16x32_bf16 v[64:67], a[192:195], a[128:131], v[64:67]\n"
    "ds_read_b64_tr_b16 a[160:161], v16 offset:0\n"
    "s_memtime s[4:5]\n"
    "v_mfma_f32_16x16x32_bf16 v[96:99], a[192:195], a[132:135], v[96:99]\n"
    "ds_read_b128 a[64:67], v32 offset:0\n"
    "v_mfma_f32_16x16x32_bf16 v[100:103], a[196:199], a[132:135], v[100:103]\n"
    "s_bitcmp1_b64 s[64:65], s79\n"
    "s_cbranch_scc1 Lfl107%=\n"
    "Lfl107r%=:\n"
    "v_mfma_f32_16x16x32_bf16 v[68:71], a[196:199], a[128:131], v[68:71]\n"
    "ds_read_b64_tr_b16 a[162:163], v16 offset:1024\n"
    "v_mfma_f32_16x16x32_bf16 v[128:131], a[192:195], a[136:139], v[128:131]\n"
    "ds_read_b128 a[68:71], v32 offset:2048\n"
    "s_add_u32 m0, s71, 49152\n"
    "v_mfma_f32_16x16x32_bf16 v[160:163], a[192:195], a[140:143], v[160:163]\n"
    "buffer_load_dwordx4 v56, s[12:15], s73 offen offset:0 nt lds\n"
    "v_mfma_f32_16x16x32_bf16 v[164:167], a[196:199], a[140:143], v[164:167]\n"
    "ds_read_b64_tr_b16 a[164:165], v17 offset:0\n"
    "v_mfma_f32_16x16x32_bf16 v[132:135], a[196:199], a[136:139], v[132:135]\n"
    "ds_read_b128 a[72:75], v32 offset:4096\n"
    "v_mfma_f32_16x16x32_bf16 v[192:195], a[192:195], a[144:147], v[192:195]\n"
    "v_mfma_f32_16x16x32_bf16 v[224:227], a[192:195], a[148:151], v[224:227]\n"
    "ds_read_b64_tr_b16 a[166:167], v17 offset:1024\n"
    "v_mfma_f32_16x16x32_bf16 v[228:231], a[196:199], a[148:151], v[228:231]\n"
    "ds_read_b128 a[76:79], v32 offset:6144\n"
    "v_mfma_f32_16x16x32_bf16 v[196:199], a[196:199], a[144:147], v[196:199]\n"
    "v_mfma_f32_16x16x32_bf16 a[0:3], a[192:195], a[152:155], a[0:3]\n"
    "ds_read_b64_tr_b16 a[168:169], v18 offset:0\n"
    "v_mfma_f32_16x16x32_bf16 a[32:35], a[192:195], a[156:159], a[32:35]\n"
    "ds_read_b128 a[80:83], v32 offset:8192\n"
    "v_mfma_f32_16x16x32_bf16 a[36:39], a[196:199], a[156:159], a[36:39]\n"
    "v_mfma_f32_16x16x32_bf16 a[4:7], a[196:199], a[152:155], a[4:7]\n"
    "ds_read_b64_tr_b16 a[170:171], v18 offset:1024\n"
    "v_mfma_f32_16x16x32_bf16 a[8:11], a[200:203], a[152:155], a[8:11]\n"
    "ds_read_b128 a[84:87], v32 offset:10240\n"
    "v_mfma_f32_16x16x32_bf16 a[40:43], a[200:203], a[156:159], a[40:43]\n"
    "buffer_load_dwordx4 v57, s[12:15], s73 offen offset:1024 nt lds\n"
    "v_mfma_f32_16x16x32_bf16 a[44:47], a[204:207], a[156:159], a[44:47]\n"
    "ds_read_b64_tr_b16 a[172:173], v19 offset:0\n"
    "v_mfma_f32_16x16x32_bf16 a[12:15], a[204:207], a[152:155], a[12:15]\n"
    "ds_read_b128 a[88:91], v32 offset:12288\n"
    "v_mfma_f32_16x16x32_bf16 v[200:203], a[200:203], a[144:147], v[200:203]\n"
    "v_mfma_f32_16x16x32_bf16 v[232:235], a[200:203], a[148:151], v[232:235]\n"
    "ds_read_b64_tr_b16 a[174:175], v19 offset:1024\n"
    "v_mfma_f32_16x16x32_bf16 v[236:239], a[204:207], a[148:151], v[236:239]\n"
    "ds_read_b128 a[92:95], v32 offset:14336\n"
    "v_mfma_f32_16x16x32_bf16 v[204:207], a[204:207], a[144:147], v[204:207]\n"
    "v_mfma_f32_16x16x32_bf16 v[136:139], a[200:203], a[136:139], v[136:139]\n"
    "ds_read_b64_tr_b16 a[176:177], v20 offset:0\n"
    "v_mfma_f32_16x16x32_bf16 v[168:171], a[200:203], a[140:143], v[168:171]\n"
    "ds_read_b128 a[96:99], v33 offset:0\n"
    "v_mfma_f32_16x16x32_bf16 v[172:175], a[204:207], a[140:143], v[172:175]\n"
    "v_mfma_f32_16x16x32_bf16 v[140:143], a[204:207], a[136:139], v[140:143]\n"
    "ds_read_b64_tr_b16 a[178:179], v20 offset:1024\n"
    "v_mfma_f32_16x16x32_bf16 v[72:75], a[200:203], a[128:131], v[72:75]\n"
    "ds_read_b128 a[100:103], v33 offset:2048\n"
    "v_mfma_f32_16x16x32_bf16 v[104:107], a[200:203], a[132:135], v[104:107]\n"
    "buffer_load_dwordx4 v58, s[12:15], s73 offen offset:2048 nt lds\n"
    "v_mfma_f32_16x16x32_bf16 v[108:111], a[204:207], a[132:135], v[108:111]\n"
    "ds_read_b64_tr_b16 a[180:181], v21 offset:0\n"
    "v_mfma_f32_16x16x32_bf16 v[76:79], a[204:207], a[128:131], v[76:79]\n"
    "ds_read_b128 a[104:107], v33 offset:4096\n"
    "v_mfma_f32_16x16x32_bf16 v[80:83], a[208:211], a[128:131], v[80:83]\n"
    "v_mfma_f32_16x16x32_bf16 v[112:115], a[208:211], a[132:135], v[112:115]\n"
    "ds_read_b64_tr_b16 a[182:183], v21 offset:1024\n"
    "v_mfma_f32_16x16x32_bf16 v[116:119], a[212:215], a[132:135], v[116:119]\n"
    "ds_read_b128 a[108:111], v33 offset:6144\n"
    "v_mfma_f32_16x16x32_bf16 v[84:87], a[212:215], a[128:131], v[84:87]\n"
    "v_mfma_f32_16x16x32_bf16 v[144:147], a[208:211], a[136:139], v[144:147]\n"
    "ds_read_b64_tr_b16 a[184:185], v22 offset:0\n"
    "v_mfma_f32_16x16x32_bf16 v[176:179], a[208:211], a[140:143], v[176:179]\n"
    "ds_read_b128 a[112:115], v33 offset:8192\n"
    "v_mfma_f32_16x16x32_bf16 v[180:183], a[212:215], a[140:143], v[180:183]\n"
    "v_mfma_f32_16x16x32_bf16 v[148:151], a[212:215], a[136:139], v[148:151]\n"
    "ds_read_b64_tr_b16 a[186:187], v22 offset:1024\n"
    "v_mfma_f32_16x16x32_bf16 v[208:211], a[208:211], a[144:147], v[208:211]\n"
    "ds_read_b128 a[116:119], v33 offset:10240\n"
    "v_mfma_f32_16x16x32_bf16 v[240:243], a[208:211], a[148:151], v[240:243]\n"
    "buffer_load_dwordx4 v59, s[12:15], s73 offen offset:3072 nt lds\n"
    "v_mfma_f32_16x16x32_bf16 v[244:247], a[212:215], a[148:151], v[244:247]\n"
    "ds_read_b64_tr_b16 a[188:189], v23 offset:0\n"
    "v_mfma_f32_16x16x32_bf16 v[212:215], a[212:215], a[144:147], v[212:215]\n"
    "ds_read_b128 a[120:123], v33 offset:12288\n"
    "v_mfma_f32_16x16x32_bf16 a[16:19], a[208:211], a[152:155], a[16:19]\n"
    "v_mfma_f32_16x16x32_bf16 a[48:51], a[208:211], a[156:159], a[48:51]\n"
    "ds_read_b64_tr_b16 a[190:191], v23 offset:1024\n"
    "v_mfma_f32_16x16x32_bf16 a[52:55], a[212:215], a[156:159], a[52:55]\n"
    "ds_read_b128 a[124:127], v33 offset:14336\n"
    "v_mfma_f32_16x16x32_bf16 a[20:23], a[212:215], a[152:155], a[20:23]\n"
    "v_mfma_f32_16x16x32_bf16 a[24:27], a[216:219], a[152:155], a[24:27]\n"
    "s_add_u32 s76, s76, 1\n"
    "v_mfma_f32_16x16x32_bf16 a[56:59], a[216:219], a[156:159], a[56:59]\n"
    "v_readlane_b32 s72, v40, s75\n"
    "v_mfma_f32_16x16x32_bf16 a[60:63], a[220:223], a[156:159], a[60:63]\n"
    "v_mfma_f32_16x16x32_bf16 a[28:31], a[220:223], a[152:155], a[28:31]\n"
    "v_readlane_b32 s73, v41, s76\n"
    "v_mfma_f32_16x16x32_bf16 v[216:219], a[216:219], a[144:147], v[216:219]\n"
    "s_cmp_eq_u32 s75, 8\n"
    "v_mfma_f32_16x16x32_bf16 v[248:251], a[216:219], a[148:151], v[248:251]\n"
    "s_cselect_b32 s84, s88, s84\n"
    "s_cselect_b32 s85, s89, s85\n"
    "v_mfma_f32_16x16x32_bf16 v[252:255], a[220:223], a[148:151], v[252:255]\n"
    "v_mfma_f32_16x16x32_bf16 v[220:223], a[220:223], a[144:147], v[220:223]\n"
    "s_cmp_ge_u32 s75, 64\n"
    "s_cselect_b32 s72, 1073741824, s72\n"
    "v_mfma_f32_16x16x32_bf16 v[152:155], a[216:219], a[136:139], v[152:155]\n"
    "v_mfma_f32_16x16x32_bf16 v[184:187], a[216:219], a[140:143], v[184:187]\n"
    "s_cmp_ge_u32 s76, 64\n"
    "s_cselect_b32 s73, 1073741824, s73\n"
    "v_mfma_f32_16x16x32_bf16 v[188:191], a[220:223], a[140:143], v[188:191]\n"
    "v_mfma_f32_16x16x32_bf16 v[156:159], a[220:223], a[136:139], v[156:159]\n"
    "s_bitcmp1_b64 s[60:61], s79\n"
    "s_cbranch_scc1 Loe107%=\n"
    "Loe107r%=:\n"
    "v_mfma_f32_16x16x32_bf16 v[88:91], a[216:219], a[128:131], v[88:91]\n"
    "v_mfma_f32_16x16x32_bf16 v[120:123], a[216:219], a[132:135], v[120:123]\n"
    "s_add_u32 s79, s79, 1\n"
    "v_mfma_f32_16x16x32_bf16 v[124:127], a[220:223], a[132:135], v[124:127]\n"
    "s_add_u32 s74, s74, 1\n"
    "v_mfma_f32_16x16x32_bf16 v[92:95], a[220:223], a[128:131], v[92:95]\n"
    "s_cmp_lt_u32 s74, 10\n"
    "s_waitcnt lgkmcnt(0)\n"
    "s_barrier\n"
    "s_cbranch_scc0 Ltail%=\n"
    "s_cmp_eq_u32 s41, 0\n"
    "s_cbranch_scc1 LT2%=\n"
    "s_branch LT102%=\n"
    "Ltail%=:\n"
    "v_mfma_f32_16x16x32_bf16 v[64:67], a[160:163], a[64:67], v[64:67]\n"
    "ds_read_b64_tr_b16 a[192:193], v16 offset:16384\n"
    "v_mfma_f32_16x16x32_bf16 v[96:99], a[160:163], a[68:71], v[96:99]\n"
    "v_mov_b32 v46, 66688\n"
    "v_mfma_f32_16x16x32_bf16 v[100:103], a[164:167], a[68:71], v[100:103]\n"
    "global_load_dword v46, v46, s[18:19] sc0 sc1\n"
    "v_mfma_f32_16x16x32_bf16 v[68:71], a[164:167], a[64:67], v[68:71]\n"
    "ds_read_b64_tr_b16 a[194:195], v16 offset:17408\n"
    "v_mfma_f32_16x16x32_bf16 v[128:131], a[160:163], a[72:75], v[128:131]\n"
    "v_mfma_f32_16x16x32_bf16 v[160:163], a[160:163], a[76:79], v[160:163]\n"
    "v_mfma_f32_16x16x32_bf16 v[164:167], a[164:167], a[76:79], v[164:167]\n"
    "ds_read_b64_tr_b16 a[196:197], v17 offset:16384\n"
    "v_mfma_f32_16x16x32_bf16 v[132:135], a[164:167], a[72:75], v[132:135]\n"
    "v_mfma_f32_16x16x32_bf16 v[192:195], a[160:163], a[80:83], v[192:195]\n"
    "v_mfma_f32_16x16x32_bf16 v[224:227], a[160:163], a[84:87], v[224:227]\n"
    "ds_read_b64_tr_b16 a[198:199], v17 offset:17408\n"
    "v_mfma_f32_16x16x32_bf16 v[228:231], a[164:167], a[84:87], v[228:231]\n"
    "v_mfma_f32_16x16x32_bf16 v[196:199], a[164:167], a[80:83], v[196:199]\n"
    "v_mfma_f32_16x16x32_bf16 a[0:3], a[160:163], a[88:91], a[0:3]\n"
    "ds_read_b64_tr_b16 a[200:201], v18 offset:16384\n"
    "v_mfma_f32_16x16x32_bf16 a[32:35], a[160:163], a[92:95], a[32:35]\n"
    "v_mfma_f32_16x16x32_bf16 a[36:39], a[164:167], a[92:95], a[36:39]\n"
    "v_mfma_f32_16x16x32_bf16 a[4:7], a[164:167], a[88:91], a[4:7]\n"
    "ds_read_b64_tr_b16 a[202:203], v18 offset:17408\n"
    "v_mfma_f32_16x16x32_bf16 a[8:11], a[168:171], a[88:91], a[8:11]\n"
    "v_mfma_f32_16x16x32_bf16 a[40:43], a[168:171], a[92:95], a[40:43]\n"
    "v_mfma_f32_16x16x32_bf16 a[44:47], a[172:175], a[92:95], a[44:47]\n"
    "ds_read_b64_tr_b16 a[204:205], v19 offset:16384\n"
    "v_mfma_f32_16x16x32_bf16 a[12:15], a[172:175], a[88:91], a[12:15]\n"
    "v_mfma_f32_16x16x32_bf16 v[200:203], a[168:171], a[80:83], v[200:203]\n"
    "v_mfma_f32_16x16x32_bf16 v[232:235], a[168:171], a[84:87], v[232:235]\n"
    "ds_read_b64_tr_b16 a[206:207], v19 offset:17408\n"
    "v_mfma_f32_16x16x32_bf16 v[236:239], a[172:175], a[84:87], v[236:239]\n"
    "v_mfma_f32_16x16x32_bf16 v[204:207], a[172:175], a[80:83], v[204:207]\n"
    "v_mfma_f32_16x16x32_bf16 v[136:139], a[168:171], a[72:75], v[136:139]\n"
    "ds_read_b64_tr_b16 a[208:209], v20 offset:16384\n"
    "v_mfma_f32_16x16x32_bf16 v[168:171], a[168:171], a[76:79], v[168:171]\n"
    "v_mfma_f32_16x16x32_bf16 v[172:175], a[172:175], a[76:79], v[172:175]\n"
    "v_mfma_f32_16x16x32_bf16 v[140:143], a[172:175], a[72:75], v[140:143]\n"
    "ds_read_b64_tr_b16 a[210:211], v20 offset:17408\n"
    "v_mfma_f32_16x16x32_bf16 v[72:75], a[168:171], a[64:67], v[72:75]\n"
    "v_mfma_f32_16x16x32_bf16 v[104:107], a[168:171], a[68:71], v[104:107]\n"
    "v_mfma_f32_16x16x32_bf16 v[108:111], a[172:175], a[68:71], v[108:111]\n"
    "ds_read_b64_tr_b16 a[212:213], v21 offset:16384\n"
    "v_mfma_f32_16x16x32_bf16 v[76:79], a[172:175], a[64:67], v[76:79]\n"
    "v_mfma_f32_16x16x32_bf16 v[80:83], a[176:179], a[64:67], v[80:83]\n"
    "v_mfma_f32_16x16x32_bf16 v[112:115], a[176:179], a[68:71], v[112:115]\n"
    "ds_read_b64_tr_b16 a[214:215], v21 offset:17408\n"
    "v_mfma_f32_16x16x32_bf16 v[116:119], a[180:183], a[68:71], v[116:119]\n"
    "v_mfma_f32_16x16x32_bf16 v[84:87], a[180:183], a[64:67], v[84:87]\n"
    "v_mfma_f32_16x16x32_bf16 v[144:147], a[176:179], a[72:75], v[144:147]\n"
    "ds_read_b64_tr_b16 a[216:217], v22 offset:16384\n"
    "v_mfma_f32_16x16x32_bf16 v[176:179], a[176:179], a[76:79], v[176:179]\n"
    "v_mfma_f32_16x16x32_bf16 v[180:183], a[180:183], a[76:79], v[180:183]\n"
    "v_mfma_f32_16x16x32_bf16 v[148:151], a[180:183], a[72:75], v[148:151]\n"
    "ds_read_b64_tr_b16 a[218:219], v22 offset:17408\n"
    "v_mfma_f32_16x16x32_bf16 v[208:211], a[176:179], a[80:83], v[208:211]\n"
    "v_mfma_f32_16x16x32_bf16 v[240:243], a[176:179], a[84:87], v[240:243]\n"
    "v_mfma_f32_16x16x32_bf16 v[244:247], a[180:183], a[84:87], v[244:247]\n"
    "ds_read_b64_tr_b16 a[220:221], v23 offset:16384\n"
    "v_mfma_f32_16x16x32_bf16 v[212:215], a[180:183], a[80:83], v[212:215]\n"
    "v_mfma_f32_16x16x32_bf16 a[16:19], a[176:179], a[88:91], a[16:19]\n"
    "v_mfma_f32_16x16x32_bf16 a[48:51], a[176:179], a[92:95], a[48:51]\n"
    "ds_read_b64_tr_b16 a[222:223], v23 offset:17408\n"
    "v_mfma_f32_16x16x32_bf16 a[52:55], a[180:183], a[92:95], a[52:55]\n"
    "v_mfma_f32_16x16x32_bf16 a[20:23], a[180:183], a[88:91], a[20:23]\n"
    "v_mfma_f32_16x16x32_bf16 a[24:27], a[184:187], a[88:91], a[24:27]\n"
    "v_mfma_f32_16x16x32_bf16 a[56:59], a[184:187], a[92:95], a[56:59]\n"
    "v_mfma_f32_16x16x32_bf16 a[60:63], a[188:191], a[92:95], a[60:63]\n"
    "v_mfma_f32_16x16x32_bf16 a[28:31], a[188:191], a[88:91], a[28:31]\n"
    "v_mfma_f32_16x16x32_bf16 v[216:219], a[184:187], a[80:83], v[216:219]\n"
    "v_mfma_f32_16x16x32_bf16 v[248:251], a[184:187], a[84:87], v[248:251]\n"
    "v_mfma_f32_16x16x32_bf16 v[252:255], a[188:191], a[84:87], v[252:255]\n"
    "v_mfma_f32_16x16x32_bf16 v[220:223], a[188:191], a[80:83], v[220:223]\n"
    "v_mfma_f32_16x16x32_bf16 v[152:155], a[184:187], a[72:75], v[152:155]\n"
    "v_mfma_f32_16x16x32_bf16 v[184:187], a[184:187], a[76:79], v[184:187]\n"
    "v_mfma_f32_16x16x32_bf16 v[188:191], a[188:191], a[76:79], v[188:191]\n"
    "v_mfma_f32_16x16x32_bf16 v[156:159], a[188:191], a[72:75], v[156:159]\n"
    "v_mfma_f32_16x16x32_bf16 v[88:91], a[184:187], a[64:67], v[88:91]\n"
    "v_mfma_f32_16x16x32_bf16 v[120:123], a[184:187], a[68:71], v[120:123]\n"
    "v_mfma_f32_16x16x32_bf16 v[124:127], a[188:191], a[68:71], v[124:127]\n"
    "v_mfma_f32_16x16x32_bf16 v[92:95], a[188:191], a[64:67], v[92:95]\n"
    "s_waitcnt vmcnt(0) lgkmcnt(0)\n"
    "s_barrier\n"
    "v_mfma_f32_16x16x32_bf16 v[64:67], a[192:195], a[96:99], v[64:67]\n"
    "ds_read_b64_tr_b16 a[160:161], v24 offset:0\n"
    "v_mfma_f32_16x16x32_bf16 v[96:99], a[192:195], a[100:103], v[96:99]\n"
    "ds_read_b128 a[64:67], v32 offset:32768\n"
    "v_mfma_f32_16x16x32_bf16 v[100:103], a[196:199], a[100:103], v[100:103]\n"
    "ds_read_b64_tr_b16 a[224:225], v24 offset:16384\n"
    "v_mfma_f32_16x16x32_bf16 v[68:71], a[196:199], a[96:99], v[68:71]\n"
    "ds_read_b64_tr_b16 a[162:163], v24 offset:1024\n"
    "v_mfma_f32_16x16x32_bf16 v[128:131], a[192:195], a[104:107], v[128:131]\n"
    "ds_read_b128 a[68:71], v32 offset:34816\n"
    "v_mfma_f32_16x16x32_bf16 v[160:163], a[192:195], a[108:111], v[160:163]\n"
    "ds_read_b64_tr_b16 a[226:227], v24 offset:17408\n"
    "v_mfma_f32_16x16x32_bf16 v[164:167], a[196:199], a[108:111], v[164:167]\n"
    "ds_read_b64_tr_b16 a[164:165], v25 offset:0\n"
    "v_mfma_f32_16x16x32_bf16 v[132:135], a[196:199], a[104:107], v[132:135]\n"
    "ds_read_b128 a[72:75], v32 offset:36864\n"
    "v_mfma_f32_16x16x32_bf16 v[192:195], a[192:195], a[112:115], v[192:195]\n"
    "ds_read_b64_tr_b16 a[228:229], v25 offset:16384\n"
    "v_mfma_f32_16x16x32_bf16 v[224:227], a[192:195], a[116:119], v[224:227]\n"
    "ds_read_b64_tr_b16 a[166:167], v25 offset:1024\n"
    "v_mfma_f32_16x16x32_bf16 v[228:231], a[196:199], a[116:119], v[228:231]\n"
    "ds_read_b128 a[76:79], v32 offset:38912\n"
    "v_mfma_f32_16x16x32_bf16 v[196:199], a[196:199], a[112:115], v[196:199]\n"
    "ds_read_b64_tr_b16 a[230:231], v25 offset:17408\n"
    "v_mfma_f32_16x16x32_bf16 a[0:3], a[192:195], a[120:123], a[0:3]\n"
    "ds_read_b64_tr_b16 a[168:169], v26 offset:0\n"
    "v_mfma_f32_16x16x32_bf16 a[32:35], a[192:195], a[124:127], a[32:35]\n"
    "ds_read_b128 a[80:83], v32 offset:40960\n"
    "v_mfma_f32_16x16x32_bf16 a[36:39], a[196:199], a[124:127], a[36:39]\n"
    "ds_read_b64_tr_b16 a[232:233], v26 offset:16384\n"
    "v_mfma_f32_16x16x32_bf16 a[4:7], a[196:199], a[120:123], a[4:7]\n"
    "ds_read_b64_tr_b16 a[170:171], v26 offset:1024\n"
    "v_mfma_f32_16x16x32_bf16 a[8:11], a[200:203], a[120:123], a[8:11]\n"
    "ds_read_b128 a[84:87], v32 offset:43008\n"
    "v_mfma_f32_16x16x32_bf16 a[40:43], a[200:203], a[124:127], a[40:43]\n"
    "ds_read_b64_tr_b16 a[234:235], v26 offset:17408\n"
    "v_mfma_f32_16x16x32_bf16 a[44:47], a[204:207], a[124:127], a[44:47]\n"
    "ds_read_b64_tr_b16 a[172:173], v27 offset:0\n"
    "v_mfma_f32_16x16x32_bf16 a[12:15], a[204:207], a[120:123], a[12:15]\n"
    "ds_read_b128 a[88:91], v32 offset:45056\n"
    "v_mfma_f32_16x16x32_bf16 v[200:203], a[200:203], a[112:115], v[200:203]\n"
    "ds_read_b64_tr_b16 a[236:237], v27 offset:16384\n"
    "v_mfma_f32_16x16x32_bf16 v[232:235], a[200:203], a[116:119], v[232:235]\n"
    "ds_read_b64_tr_b16 a[174:175], v27 offset:1024\n"
    "v_mfma_f32_16x16x32_bf16 v[236:239], a[204:207], a[116:119], v[236:239]\n"
    "ds_read_b128 a[92:95], v32 offset:47104\n"
    "v_mfma_f32_16x16x32_bf16 v[204:207], a[204:207], a[112:115], v[204:207]\n"
    "ds_read_b64_tr_b16 a[238:239], v27 offset:17408\n"
    "v_mfma_f32_16x16x32_bf16 v[136:139], a[200:203], a[104:107], v[136:139]\n"
    "ds_read_b64_tr_b16 a[176:177], v28 offset:0\n"
    "v_mfma_f32_16x16x32_bf16 v[168:171], a[200:203], a[108:111], v[168:171]\n"
    "ds_read_b128 a[128:131], v33 offset:32768\n"
    "v_mfma_f32_16x16x32_bf16 v[172:175], a[204:207], a[108:111], v[172:175]\n"
    "ds_read_b64_tr_b16 a[240:241], v28 offset:16384\n"
    "v_mfma_f32_16x16x32_bf16 v[140:143], a[204:207], a[104:107], v[140:143]\n"
    "ds_read_b64_tr_b16 a[178:179], v28 offset:1024\n"
    "v_mfma_f32_16x16x32_bf16 v[72:75], a[200:203], a[96:99], v[72:75]\n"
    "ds_read_b128 a[132:135], v33 offset:34816\n"
    "v_mfma_f32_16x16x32_bf16 v[104:107], a[200:203], a[100:103], v[104:107]\n"
    "ds_read_b64_tr_b16 a[242:243], v28 offset:17408\n"
    "v_mfma_f32_16x16x32_bf16 v[108:111], a[204:207], a[100:103], v[108:111]\n"
    "ds_read_b64_tr_b16 a[180:181], v29 offset:0\n"
    "v_mfma_f32_16x16x32_bf16 v[76:79], a[204:207], a[96:99], v[76:79]\n"
    "ds_read_b128 a[136:139], v33 offset:36864\n"
    "v_mfma_f32_16x16x32_bf16 v[80:83], a[208:211], a[96:99], v[80:83]\n"
    "ds_read_b64_tr_b16 a[244:245], v29 offset:16384\n"
    "v_mfma_f32_16x16x32_bf16 v[112:115], a[208:211], a[100:103], v[112:115]\n"
    "ds_read_b64_tr_b16 a[182:183], v29 offset:1024\n"
    "v_mfma_f32_16x16x32_bf16 v[116:119], a[212:215], a[100:103], v[116:119]\n"
    "ds_read_b128 a[140:143], v33 offset:38912\n"
    "v_mfma_f32_16x16x32_bf16 v[84:87], a[212:215], a[96:99], v[84:87]\n"
    "ds_read_b64_tr_b16 a[246:247], v29 offset:17408\n"
    "v_mfma_f32_16x16x32_bf16 v[144:147], a[208:211], a[104:107], v[144:147]\n"
    "ds_read_b64_tr_b16 a[184:185], v30 offset:0\n"
    "v_mfma_f32_16x16x32_bf16 v[176:179], a[208:211], a[108:111], v[176:179]\n"
    "ds_read_b128 a[144:147], v33 offset:40960\n"
    "v_mfma_f32_16x16x32_bf16 v[180:183], a[212:215], a[108:111], v[180:183]\n"
    "ds_read_b64_tr_b16 a[248:249], v30 offset:16384\n"
    "v_mfma_f32_16x16x32_bf16 v[148:151], a[212:215], a[104:107], v[148:151]\n"
    "ds_read_b64_tr_b16 a[186:187], v30 offset:1024\n"
    "v_mfma_f32_16x16x32_bf16 v[208:211], a[208:211], a[112:115], v[208:211]\n"
    "ds_read_b128 a[148:151], v33 offset:43008\n"
    "v_mfma_f32_16x16x32_bf16 v[240:243], a[208:211], a[116:119], v[240:243]\n"
    "ds_read_b64_tr_b16 a[250:251], v30 offset:17408\n"
    "v_mfma_f32_16x16x32_bf16 v[244:247], a[212:215], a[116:119], v[244:247]\n"
    "ds_read_b64_tr_b16 a[188:189], v31 offset:0\n"
    "v_mfma_f32_16x16x32_bf16 v[212:215], a[212:215], a[112:115], v[212:215]\n"
    "ds_read_b128 a[152:155], v33 offset:45056\n"
    "v_mfma_f32_16x16x32_bf16 a[16:19], a[208:211], a[120:123], a[16:19]\n"
    "ds_read_b64_tr_b16 a[252:253], v31 offset:16384\n"
    "v_mfma_f32_16x16x32_bf16 a[48:51], a[208:211], a[124:127], a[48:51]\n"
    "ds_read_b64_tr_b16 a[190:191], v31 offset:1024\n"
    "v_mfma_f32_16x16x32_bf16 a[52:55], a[212:215], a[124:127], a[52:55]\n"
    "ds_read_b128 a[156:159], v33 offset:47104\n"
    "v_mfma_f32_16x16x32_bf16 a[20:23], a[212:215], a[120:123], a[20:23]\n"
    "ds_read_b64_tr_b16 a[254:255], v31 offset:17408\n"
    "v_mfma_f32_16x16x32_bf16 a[24:27], a[216:219], a[120:123], a[24:27]\n"
    "v_mfma_f32_16x16x32_bf16 a[56:59], a[216:219], a[124:127], a[56:59]\n"
    "v_mfma_f32_16x16x32_bf16 a[60:63], a[220:223], a[124:127], a[60:63]\n"
    "v_mfma_f32_16x16x32_bf16 a[28:31], a[220:223], a[120:123], a[28:31]\n"
    "v_mfma_f32_16x16x32_bf16 v[216:219], a[216:219], a[112:115], v[216:219]\n"
    "v_mfma_f32_16x16x32_bf16 v[248:251], a[216:219], a[116:119], v[248:251]\n"
    "v_mfma_f32_16x16x32_bf16 v[252:255], a[220:223], a[116:119], v[252:255]\n"
    "v_mfma_f32_16x16x32_bf16 v[220:223], a[220:223], a[112:115], v[220:223]\n"
    "v_mfma_f32_16x16x32_bf16 v[152:155], a[216:219], a[104:107], v[152:155]\n"
    "v_mfma_f32_16x16x32_bf16 v[184:187], a[216:219], a[108:111], v[184:187]\n"
    "v_mfma_f32_16x16x32_bf16 v[188:191], a[220:223], a[108:111], v[188:191]\n"
    "v_mfma_f32_16x16x32_bf16 v[156:159], a[220:223], a[104:107], v[156:159]\n"
    "v_mfma_f32_16x16x32_bf16 v[88:91], a[216:219], a[96:99], v[88:91]\n"
    "v_mfma_f32_16x16x32_bf16 v[120:123], a[216:219], a[100:103], v[120:123]\n"
    "s_add_u32 s79, s79, 1\n"
    "v_mfma_f32_16x16x32_bf16 v[124:127], a[220:223], a[100:103], v[124:127]\n"
    "v_mfma_f32_16x16x32_bf16 v[92:95], a[220:223], a[96:99], v[92:95]\n"
    "s_waitcnt lgkmcnt(0)\n"
    "v_readfirstlane_b32 s67, v46\n"
    "s_mov_b32 s12, s16\n"
    "s_and_b32 s13, s17, 0xffff\n"
    "s_mov_b32 s14, 0x2000000\n"
    "s_mov_b32 s15, 0x20000\n"
    "s_lshl_b32 s72, s25, 24\n"
    "s_add_u32 s72, s72, s35\n"
    "s_nop 4\n"
    "s_cmp_eq_u32 s67, s66\n"
    "s_cbranch_scc1 Ldead%=\n"
    "s_mov_b32 s90, 0xbdd2d3e7\n"
    "s_mov_b32 s91, 0xbdd2d3e7\n"
    "v_mov_b32 v62, 0xc0135761\n"
    "v_mov_b32 v63, 0xc0135761\n"
    "s_mov_b32 vcc_lo, 0xffff0000\n"
    "s_mov_b32 vcc_hi, 0xffff0000\n"
    "v_mfma_f32_16x16x32_bf16 v[64:67], a[160:163], a[64:67], v[64:67]\n"
    "v_mfma_f32_16x16x32_bf16 v[68:71], a[164:167], a[64:67], v[68:71]\n"
    "v_mfma_f32_16x16x32_bf16 v[72:75], a[168:171], a[64:67], v[72:75]\n"
    "v_mfma_f32_16x16x32_bf16 v[76:79], a[172:175], a[64:67], v[76:79]\n"
    "v_mfma_f32_16x16x32_bf16 v[80:83], a[176:179], a[64:67], v[80:83]\n"
    "v_mfma_f32_16x16x32_bf16 v[84:87], a[180:183], a[64:67], v[84:87]\n"
    "v_mfma_f32_16x16x32_bf16 v[88:91], a[184:187], a[64:67], v[88:91]\n"
    "v_mfma_f32_16x16x32_bf16 v[92:95], a[188:191], a[64:67], v[92:95]\n"
    "v_mfma_f32_16x16x32_bf16 v[64:67], a[224:227], a[128:131], v[64:67]\n"
    "v_mfma_f32_16x16x32_bf16 v[68:71], a[228:231], a[128:131], v[68:71]\n"
    "v_mfma_f32_16x16x32_bf16 v[72:75], a[232:235], a[128:131], v[72:75]\n"
    "v_mfma_f32_16x16x32_bf16 v[76:79], a[236:239], a[128:131], v[76:79]\n"
    "v_mfma_f32_16x16x32_bf16 v[80:83], a[240:243], a[128:131], v[80:83]\n"
    "v_mfma_f32_16x16x32_bf16 v[84:87], a[244:247], a[128:131], v[84:87]\n"
    "v_mfma_f32_16x16x32_bf16 v[88:91], a[248:251], a[128:131], v[88:91]\n"
    "v_mfma_f32_16x16x32_bf16 v[92:95], a[252:255], a[128:131], v[92:95]\n"
    "v_mfma_f32_16x16x32_bf16 v[96:99], a[160:163], a[68:71], v[96:99]\n"
    "v_mfma_f32_16x16x32_bf16 v[100:103], a[164:167], a[68:71], v[100:103]\n"
    "v_mfma_f32_16x16x32_bf16 v[128:131], a[160:163], a[72:75], v[128:131]\n"
    "v_mfma_f32_16x16x32_bf16 v[160:163], a[160:163], a[76:79], v[160:163]\n"
    "v_mfma_f32_16x16x32_bf16 v[164:167], a[164:167], a[76:79], v[164:167]\n"
    "v_mfma_f32_16x16x32_bf16 v[132:135], a[164:167], a[72:75], v[132:135]\n"
    "v_mfma_f32_16x16x32_bf16 v[192:195], a[160:163], a[80:83], v[192:195]\n"
    "v_mfma_f32_16x16x32_bf16 v[224:227], a[160:163], a[84:87], v[224:227]\n"
    "v_mfma_f32_16x16x32_bf16 v[228:231], a[164:167], a[84:87], v[228:231]\n"
    "v_pk_mul_f32 v[0:1], v[64:65], v[64:65]\n"
    "v_pk_mul_f32 v[2:3], v[66:67], v[66:67]\n"
    "v_mfma_f32_16x16x32_bf16 v[196:199], a[164:167], a[80:83], v[196:199]\n"
    "v_pk_mul_f32 v[4:5], v[68:69], v[68:69]\n"
    "v_pk_mul_f32 v[6:7], v[70:71], v[70:71]\n"
    "v_mfma_f32_16x16x32_bf16 a[0:3], a[160:163], a[88:91], a[0:3]\n"
    "v_pk_mul_f32 v[8:9], v[72:73], v[72:73]\n"
    "v_pk_mul_f32 v[10:11], v[74:75], v[74:75]\n"
    "v_mfma_f32_16x16x32_bf16 a[32:35], a[160:163], a[92:95], a[32:35]\n"
    "v_pk_mul_f32 v[12:13], v[76:77], v[76:77]\n"
    "v_pk_mul_f32 v[14:15], v[78:79], v[78:79]\n"
    "v_mfma_f32_16x16x32_bf16 a[36:39], a[164:167], a[92:95], a[36:39]\n"
    "v_pk_fma_f32 v[0:1], v[0:1], s[90:91], v[62:63]\n"
    "v_pk_fma_f32 v[2:3], v[2:3], s[90:91], v[62:63]\n"
    "v_mfma_f32_16x16x32_bf16 a[4:7], a[164:167], a[88:91], a[4:7]\n"
    "v_pk_fma_f32 v[4:5], v[4:5], s[90:91], v[62:63]\n"
    "v_pk_fma_f32 v[6:7], v[6:7], s[90:91], v[62:63]\n"
    "v_mfma_f32_16x16x32_bf16 a[8:11], a[168:171], a[88:91], a[8:11]\n"
    "v_pk_fma_f32 v[8:9], v[8:9], s[90:91], v[62:63]\n"
    "v_pk_fma_f32 v[10:11], v[10:11], s[90:91], v[62:63]\n"
    "v_mfma_f32_16x16x32_bf16 a[40:43], a[168:171], a[92:95], a[40:43]\n"
    "v_pk_fma_f32 v[12:13], v[12:13], s[90:91], v[62:63]\n"
    "v_pk_fma_f32 v[14:15], v[14:15], s[90:91], v[62:63]\n"
    "v_mfma_f32_16x16x32_bf16 a[44:47], a[172:175], a[92:95], a[44:47]\n"
    "v_pk_mul_f32 v[0:1], v[64:65], v[0:1]\n"
    "v_pk_mul_f32 v[2:3], v[66:67], v[2:3]\n"
    "v_mfma_f32_16x16x32_bf16 a[12:15], a[172:175], a[88:91], a[12:15]\n"
    "v_pk_mul_f32 v[4:5], v[68:69], v[4:5]\n"
    "v_pk_mul_f32 v[6:7], v[70:71], v[6:7]\n"
    "v_mfma_f32_16x16x32_bf16 v[200:203], a[168:171], a[80:83], v[200:203]\n"
    "v_pk_mul_f32 v[8:9], v[72:73], v[8:9]\n"
    "v_pk_mul_f32 v[10:11], v[74:75], v[10:11]\n"
    "v_mfma_f32_16x16x32_bf16 v[232:235], a[168:171], a[84:87], v[232:235]\n"
    "v_pk_mul_f32 v[12:13], v[76:77], v[12:13]\n"
    "v_pk_mul_f32 v[14:15], v[78:79], v[14:15]\n"
    "v_mfma_f32_16x16x32_bf16 v[236:239], a[172:175], a[84:87], v[236:239]\n"
    "v_exp_f32 v0, v0\n"
    "v_mfma_f32_16x16x32_bf16 v[204:207], a[172:175], a[80:83], v[204:207]\n"
    "v_exp_f32 v1, v1\n"
    "v_mfma_f32_16x16x32_bf16 v[136:139], a[168:171], a[72:75], v[136:139]\n"
    "v_exp_f32 v2, v2\n"
    "v_mfma_f32_16x16x32_bf16 v[168:171], a[168:171], a[76:79], v[168:171]\n"
    "v_exp_f32 v3, v3\n"
    "v_mfma_f32_16x16x32_bf16 v[172:175], a[172:175], a[76:79], v[172:175]\n"
    "v_exp_f32 v4, v4\n"
    "v_mfma_f32_16x16x32_bf16 v[140:143], a[172:175], a[72:75], v[140:143]\n"
    "v_exp_f32 v5, v5\n"
    "v_mfma_f32_16x16x32_bf16 v[104:107], a[168:171], a[68:71], v[104:107]\n"
    "v_exp_f32 v6, v6\n"
    "v_mfma_f32_16x16x32_bf16 v[108:111], a[172:175], a[68:71], v[108:111]\n"
    "v_exp_f32 v7, v7\n"
    "v_mfma_f32_16x16x32_bf16 v[112:115], a[176:179], a[68:71], v[112:115]\n"
    "v_exp_f32 v8, v8\n"
    "v_mfma_f32_16x16x32_bf16 v[116:119], a[180:183], a[68:71], v[116:119]\n"
    "v_exp_f32 v9, v9\n"
    "v_mfma_f32_16x16x32_bf16 v[144:147], a[176:179], a[72:75], v[144:147]\n"
    "v_exp_f32 v10, v10\n"
    "v_mfma_f32_16x16x32_bf16 v[176:179], a[176:179], a[76:79], v[176:179]\n"
    "v_exp_f32 v11, v11\n"
    "v_mfma_f32_16x16x32_bf16 v[180:183], a[180:183], a[76:79], v[180:183]\n"
    "v_exp_f32 v12, v12\n"
    "v_mfma_f32_16x16x32_bf16 v[148:151], a[180:183], a[72:75], v[148:151]\n"
    "v_exp_f32 v13, v13\n"
    "v_mfma_f32_16x16x32_bf16 v[208:211], a[176:179], a[80:83], v[208:211]\n"
    "v_exp_f32 v14, v14\n"
    "v_mfma_f32_16x16x32_bf16 v[240:243], a[176:179], a[84:87], v[240:243]\n"
    "v_exp_f32 v15, v15\n"
    "v_mfma_f32_16x16x32_bf16 v[244:247], a[180:183], a[84:87], v[244:247]\n"
    "v_pk_add_f32 v[0:1], v[0:1], 1.0 op_sel_hi:[1,0]\n"
    "v_pk_add_f32 v[2:3], v[2:3], 1.0 op_sel_hi:[1,0]\n"
    "v_mfma_f32_16x16x32_bf16 v[212:215], a[180:183], a[80:83], v[212:215]\n"
    "v_pk_add_f32 v[4:5], v[4:5], 1.0 op_sel_hi:[1,0]\n"
    "v_pk_add_f32 v[6:7], v[6:7], 1.0 op_sel_hi:[1,0]\n"
    "v_mfma_f32_16x16x32_bf16 a[16:19], a[176:179], a[88:91], a[16:19]\n"
    "v_pk_add_f32 v[8:9], v[8:9], 1.0 op_sel_hi:[1,0]\n"
    "v_pk_add_f32 v[10:11], v[10:11], 1.0 op_sel_hi:[1,0]\n"
    "v_mfma_f32_16x16x32_bf16 a[48:51], a[176:179], a[92:95], a[48:51]\n"
    "v_pk_add_f32 v[12:13], v[12:13], 1.0 op_sel_hi:[1,0]\n"
    "v_pk_add_f32 v[14:15], v[14:15], 1.0 op_sel_hi:[1,0]\n"
    "v_mfma_f32_16x16x32_bf16 a[52:55], a[180:183], a[92:95], a[52:55]\n"
    "v_rcp_f32 v0, v0\n"
    "v_mfma_f32_16x16x32_bf16 a[20:23], a[180:183], a[88:91], a[20:23]\n"
    "v_rcp_f32 v1, v1\n"
    "v_mfma_f32_16x16x32_bf16 a[24:27], a[184:187], a[88:91], a[24:27]\n"
    "v_rcp_f32 v2, v2\n"
    "v_mfma_f32_16x16x32_bf16 a[56:59], a[184:187], a[92:95], a[56:59]\n"
    "v_rcp_f32 v3, v3\n"
    "v_mfma_f32_16x16x32_bf16 a[60:63], a[188:191], a[92:95], a[60:63]\n"
    "v_rcp_f32 v4, v4\n"
    "v_mfma_f32_16x16x32_bf16 a[28:31], a[188:191], a[88:91], a[28:31]\n"
    "v_rcp_f32 v5, v5\n"
    "v_mfma_f32_16x16x32_bf16 v[216:219], a[184:187], a[80:83], v[216:219]\n"
    "v_rcp_f32 v6, v6\n"
    "v_mfma_f32_16x16x32_bf16 v[248:251], a[184:187], a[84:87], v[248:251]\n"
    "v_rcp_f32 v7, v7\n"
    "v_mfma_f32_16x16x32_bf16 v[252:255], a[188:191], a[84:87], v[252:255]\n"
    "v_rcp_f32 v8, v8\n"
    "v_mfma_f32_16x16x32_bf16 v[220:223], a[188:191], a[80:83], v[220:223]\n"
    "v_rcp_f32 v9, v9\n"
    "v_mfma_f32_16x16x32_bf16 v[152:155], a[184:187], a[72:75], v[152:155]\n"
    "v_rcp_f32 v10, v10\n"
    "v_mfma_f32_16x16x32_bf16 v[184:187], a[184:187], a[76:79], v[184:187]\n"
    "v_rcp_f32 v11, v11\n"
    "v_mfma_f32_16x16x32_bf16 v[188:191], a[188:191], a[76:79], v[188:191]\n"
    "v_rcp_f32 v12, v12\n"
    "v_mfma_f32_16x16x32_bf16 v[156:159], a[188:191], a[72:75], v[156:159]\n"
    "v_rcp_f32 v13, v13\n"
    "v_mfma_f32_16x16x32_bf16 v[120:123], a[184:187], a[68:71], v[120:123]\n"
    "v_rcp_f32 v14, v14\n"
    "v_mfma_f32_16x16x32_bf16 v[124:127], a[188:191], a[68:71], v[124:127]\n"
    "v_rcp_f32 v15, v15\n"
    "v_mfma_f32_16x16x32_bf16 v[96:99], a[224:227], a[132:135], v[96:99]\n"
    "v_pk_mul_f32 v[0:1], v[64:65], v[0:1]\n"
    "v_pk_mul_f32 v[2:3], v[66:67], v[2:3]\n"
    "v_mfma_f32_16x16x32_bf16 v[100:103], a[228:231], a[132:135], v[100:103]\n"
    "v_pk_mul_f32 v[4:5], v[68:69], v[4:5]\n"
    "v_pk_mul_f32 v[6:7], v[70:71], v[6:7]\n"
    "v_mfma_f32_16x16x32_bf16 v[128:131], a[224:227], a[136:139], v[128:131]\n"
    "v_pk_mul_f32 v[8:9], v[72:73], v[8:9]\n"
    "v_pk_mul_f32 v[10:11], v[74:75], v[10:11]\n"
    "v_mfma_f32_16x16x32_bf16 v[160:163], a[224:227], a[140:143], v[160:163]\n"
    "v_pk_mul_f32 v[12:13], v[76:77], v[12:13]\n"
    "v_pk_mul_f32 v[14:15], v[78:79], v[14:15]\n"
    "v_mfma_f32_16x16x32_bf16 v[164:167], a[228:231], a[140:143], v[164:167]\n"
    "v_cvt_pk_bf16_f32 v52, v0, v1\n"
    "v_cvt_pk_bf16_f32 v53, v2, v3\n"
    "v_mfma_f32_16x16x32_bf16 v[132:135], a[228:231], a[136:139], v[132:135]\n"
    "v_cvt_pk_bf16_f32 v54, v4, v5\n"
    "v_cvt_pk_bf16_f32 v55, v6, v7\n"
    "v_mfma_f32_16x16x32_bf16 v[192:195], a[224:227], a[144:147], v[192:195]\n"
    "v_cvt_pk_bf16_f32 v56, v8, v9\n"
    "v_cvt_pk_bf16_f32 v57, v10, v11\n"
    "v_mfma_f32_16x16x32_bf16 v[224:227], a[224:227], a[148:151], v[224:227]\n"
    "v_cvt_pk_bf16_f32 v58, v12, v13\n"
    "v_cvt_pk_bf16_f32 v59, v14, v15\n"
    "v_mfma_f32_16x16x32_bf16 v[228:231], a[228:231], a[148:151], v[228:231]\n"
    "v_cndmask_b32 v44, v52, v54, vcc\n"
    "v_cndmask_b32 v45, v53, v55, vcc\n"
    "v_mfma_f32_16x16x32_bf16 v[196:199], a[228:231], a[144:147], v[196:199]\n"
    "v_cndmask_b32 v46, v54, v52, vcc\n"
    "v_cndmask_b32 v47, v55, v53, vcc\n"
    "v_mfma_f32_16x16x32_bf16 a[0:3], a[224:227], a[152:155], a[0:3]\n"
    "v_cndmask_b32 v48, v56, v58, vcc\n"
    "v_cndmask_b32 v49, v57, v59, vcc\n"
    "v_mfma_f32_16x16x32_bf16 a[32:35], a[224:227], a[156:159], a[32:35]\n"
    "v_cndmask_b32 v50, v58, v56, vcc\n"
    "v_cndmask_b32 v51, v59, v57, vcc\n"
    "v_mfma_f32_16x16x32_bf16 a[36:39], a[228:231], a[156:159], a[36:39]\n"
    "buffer_store_dwordx4 v[44:47], v39, s[12:15], s72 offen offset:0 \n"
    "buffer_store_dwordx4 v[48:51], v39, s[12:15], s72 offen offset:64 \n"
    "v_mfma_f32_16x16x32_bf16 a[4:7], a[228:231], a[152:155], a[4:7]\n"
    "v_pk_mul_f32 v[0:1], v[80:81], v[80:81]\n"
    "v_pk_mul_f32 v[2:3], v[82:83], v[82:83]\n"
    "v_mfma_f32_16x16x32_bf16 a[8:11], a[232:235], a[152:155], a[8:11]\n"
    "v_pk_mul_f32 v[4:5], v[84:85], v[84:85]\n"
    "v_pk_mul_f32 v[6:7], v[86:87], v[86:87]\n"
    "v_mfma_f32_16x16x32_bf16 a[40:43], a[232:235], a[156:159], a[40:43]\n"
    "v_pk_mul_f32 v[8:9], v[88:89], v[88:89]\n"
    "v_pk_mul_f32 v[10:11], v[90:91], v[90:91]\n"
    "v_mfma_f32_16x16x32_bf16 a[44:47], a[236:239], a[156:159], a[44:47]\n"
    "v_pk_mul_f32 v[12:13], v[92:93], v[92:93]\n"
    "v_pk_mul_f32 v[14:15], v[94:95], v[94:95]\n"
    "v_mfma_f32_16x16x32_bf16 a[12:15], a[236:239], a[152:155], a[12:15]\n"
    "v_pk_fma_f32 v[0:1], v[0:1], s[90:91], v[62:63]\n"
    "v_pk_fma_f32 v[2:3], v[2:3], s[90:91], v[62:63]\n"
    "v_mfma_f32_16x16x32_bf16 v[200:203], a[232:235], a[144:147], v[200:203]\n"
    "v_pk_fma_f32 v[4:5], v[4:5], s[90:91], v[62:63]\n"
    "v_pk_fma_f32 v[6:7], v[6:7], s[90:91], v[62:63]\n"
    "v_mfma_f32_16x16x32_bf16 v[232:235], a[232:235], a[148:151], v[232:235]\n"
    "v_pk_fma_f32 v[8:9], v[8:9], s[90:91], v[62:63]\n"
    "v_pk_fma_f32 v[10:11], v[10:11], s[90:91], v[62:63]\n"
    "v_mfma_f32_16x16x32_bf16 v[236:239], a[236:239], a[148:151], v[236:239]\n"
    "v_pk_fma_f32 v[12:13], v[12:13], s[90:91], v[62:63]\n"
    "v_pk_fma_f32 v[14:15], v[14:15], s[90:91], v[62:63]\n"
    "v_mfma_f32_16x16x32_bf16 v[204:207], a[236:239], a[144:147], v[204:207]\n"
    "v_pk_mul_f32 v[0:1], v[80:81], v[0:1]\n"
    "v_pk_mul_f32 v[2:3], v[82:83], v[2:3]\n"
    "v_mfma_f32_16x16x32_bf16 v[136:139], a[232:235], a[136:139], v[136:139]\n"
    "v_pk_mul_f32 v[4:5], v[84:85], v[4:5]\n"
    "v_pk_mul_f32 v[6:7], v[86:87], v[6:7]\n"
    "v_mfma_f32_16x16x32_bf16 v[168:171], a[232:235], a[140:143], v[168:171]\n"
    "v_pk_mul_f32 v[8:9], v[88:89], v[8:9]\n"
    "v_pk_mul_f32 v[10:11], v[90:91], v[10:11]\n"
    "v_mfma_f32_16x16x32_bf16 v[172:175], a[236:239], a[140:143], v[172:175]\n"
    "v_pk_mul_f32 v[12:13], v[92:93], v[12:13]\n"
    "v_pk_mul_f32 v[14:15], v[94:95], v[14:15]\n"
    "v_mfma_f32_16x16x32_bf16 v[140:143], a[236:239], a[136:139], v[140:143]\n"
    "v_exp_f32 v0, v0\n"
    "v_mfma_f32_16x16x32_bf16 v[104:107], a[232:235], a[132:135], v[104:107]\n"
    "v_exp_f32 v1, v1\n"
    "v_mfma_f32_16x16x32_bf16 v[108:111], a[236:239], a[132:135], v[108:111]\n"
    "v_exp_f32 v2, v2\n"
    "v_mfma_f32_16x16x32_bf16 v[112:115], a[240:243], a[132:135], v[112:115]\n"
    "v_exp_f32 v3, v3\n"
    "v_mfma_f32_16x16x32_bf16 v[116:119], a[244:247], a[132:135], v[116:119]\n"
    "v_exp_f32 v4, v4\n"
    "v_mfma_f32_16x16x32_bf16 v[144:147], a[240:243], a[136:139], v[144:147]\n"
    "v_exp_f32 v5, v5\n"
    "v_mfma_f32_16x16x32_bf16 v[176:179], a[240:243], a[140:143], v[176:179]\n"
    "v_exp_f32 v6, v6\n"
    "v_mfma_f32_16x16x32_bf16 v[180:183], a[244:247], a[140:143], v[180:183]\n"
    "v_exp_f32 v7, v7\n"
    "v_mfma_f32_16x16x32_bf16 v[148:151], a[244:247], a[136:139], v[148:151]\n"
    "v_exp_f32 v8, v8\n"
    "v_mfma_f32_16x16x32_bf16 v[208:211], a[240:243], a[144:147], v[208:211]\n"
    "v_exp_f32 v9, v9\n"
    "v_mfma_f32_16x16x32_bf16 v[240:243], a[240:243], a[148:151], v[240:243]\n"
    "v_exp_f32 v10, v10\n"
    "v_mfma_f32_16x16x32_bf16 v[244:247], a[244:247], a[148:151], v[244:247]\n"
    "v_exp_f32 v11, v11\n"
    "v_mfma_f32_16x16x32_bf16 v[212:215], a[244:247], a[144:147], v[212:215]\n"
    "v_exp_f32 v12, v12\n"
    "v_mfma_f32_16x16x32_bf16 a[16:19], a[240:243], a[152:155], a[16:19]\n"
    "v_exp_f32 v13, v13\n"
    "v_mfma_f32_16x16x32_bf16 a[48:51], a[240:243], a[156:159], a[48:51]\n"
    "v_exp_f32 v14, v14\n"
    "v_mfma_f32_16x16x32_bf16 a[52:55], a[244:247], a[156:159], a[52:55]\n"
    "v_exp_f32 v15, v15\n"
    "v_mfma_f32_16x16x32_bf16 a[20:23], a[244:247], a[152:155], a[20:23]\n"
    "v_pk_add_f32 v[0:1], v[0:1], 1.0 op_sel_hi:[1,0]\n"
    "v_pk_add_f32 v[2:3], v[2:3], 1.0 op_sel_hi:[1,0]\n"
    "v_mfma_f32_16x16x32_bf16 a[24:27], a[248:251], a[152:155], a[24:27]\n"
    "v_pk_add_f32 v[4:5], v[4:5], 1.0 op_sel_hi:[1,0]\n"
    "v_pk_add_f32 v[6:7], v[6:7], 1.0 op_sel_hi:[1,0]\n"
    "v_mfma_f32_16x16x32_bf16 a[56:59], a[248:251], a[156:159], a[56:59]\n"
    "v_pk_add_f32 v[8:9], v[8:9], 1.0 op_sel_hi:[1,0]\n"
    "v_pk_add_f32 v[10:11], v[10:11], 1.0 op_sel_hi:[1,0]\n"
    "v_mfma_f32_16x16x32_bf16 a[60:63], a[252:255], a[156:159], a[60:63]\n"
    "v_pk_add_f32 v[12:13], v[12:13], 1.0 op_sel_hi:[1,0]\n"
    "v_pk_add_f32 v[14:15], v[14:15], 1.0 op_sel_hi:[1,0]\n"
    "v_mfma_f32_16x16x32_bf16 a[28:31], a[252:255], a[152:155], a[28:31]\n"
    "v_rcp_f32 v0, v0\n"
    "v_mfma_f32_16x16x32_bf16 v[216:219], a[248:251], a[144:147], v[216:219]\n"
    "v_rcp_f32 v1, v1\n"
    "v_mfma_f32_16x16x32_bf16 v[248:251], a[248:251], a[148:151], v[248:251]\n"
    "v_rcp_f32 v2, v2\n"
    "v_mfma_f32_16x16x32_bf16 v[252:255], a[252:255], a[148:151], v[252:255]\n"
    "v_rcp_f32 v3, v3\n"
    "v_mfma_f32_16x16x32_bf16 v[220:223], a[252:255], a[144:147], v[220:223]\n"
    "v_rcp_f32 v4, v4\n"
    "v_mfma_f32_16x16x32_bf16 v[152:155], a[248:251], a[136:139], v[152:155]\n"
    "v_rcp_f32 v5, v5\n"
    "v_mfma_f32_16x16x32_bf16 v[184:187], a[248:251], a[140:143], v[184:187]\n"
    "v_rcp_f32 v6, v6\n"
    "v_mfma_f32_16x16x32_bf16 v[188:191], a[252:255], a[140:143], v[188:191]\n"
    "v_rcp_f32 v7, v7\n"
    "v_mfma_f32_16x16x32_bf16 v[156:159], a[252:255], a[136:139], v[156:159]\n"
    "v_rcp_f32 v8, v8\n"
    "v_mfma_f32_16x16x32_bf16 v[120:123], a[248:251], a[132:135], v[120:123]\n"
    "v_rcp_f32 v9, v9\n"
    "v_mfma_f32_16x16x32_bf16 v[124:127], a[252:255], a[132:135], v[124:127]\n"
    "v_rcp_f32 v10, v10\n"
    "v_rcp_f32 v11, v11\n"
    "v_rcp_f32 v12, v12\n"
    "v_rcp_f32 v13, v13\n"
    "v_rcp_f32 v14, v14\n"
    "v_rcp_f32 v15, v15\n"
    "v_pk_mul_f32 v[0:1], v[80:81], v[0:1]\n"
    "v_pk_mul_f32 v[2:3], v[82:83], v[2:3]\n"
    "v_pk_mul_f32 v[4:5], v[84:85], v[4:5]\n"
    "v_pk_mul_f32 v[6:7], v[86:87], v[6:7]\n"
    "v_pk_mul_f32 v[8:9], v[88:89], v[8:9]\n"
    "v_pk_mul_f32 v[10:11], v[90:91], v[10:11]\n"
    "v_pk_mul_f32 v[12:13], v[92:93], v[12:13]\n"
    "v_pk_mul_f32 v[14:15], v[94:95], v[14:15]\n"
    "v_cvt_pk_bf16_f32 v52, v0, v1\n"
    "v_cvt_pk_bf16_f32 v53, v2, v3\n"
    "v_cvt_pk_bf16_f32 v54, v4, v5\n"
    "v_cvt_pk_bf16_f32 v55, v6, v7\n"
    "v_cvt_pk_bf16_f32 v56, v8, v9\n"
    "v_cvt_pk_bf16_f32 v57, v10, v11\n"
    "v_cvt_pk_bf16_f32 v58, v12, v13\n"
    "v_cvt_pk_bf16_f32 v59, v14, v15\n"
    "v_cndmask_b32 v44, v52, v54, vcc\n"
    "v_cndmask_b32 v45, v53, v55, vcc\n"
    "v_cndmask_b32 v46, v54, v52, vcc\n"
    "v_cndmask_b32 v47, v55, v53, vcc\n"
    "v_cndmask_b32 v48, v56, v58, vcc\n"
    "v_cndmask_b32 v49, v57, v59, vcc\n"
    "v_cndmask_b32 v50, v58, v56, vcc\n"
    "v_cndmask_b32 v51, v59, v57, vcc\n"
    "buffer_store_dwordx4 v[44:47], v39, s[12:15], s72 offen offset:128 \n"
    "buffer_store_dwordx4 v[48:51], v39, s[12:15], s72 offen offset:192 \n"
    "s_add_u32 s72, s72, 0x100000\n"
    "s_branch Lend%=\n"
    ".p2align 6\n"
    "Lhooks%=:\n"
    "Lfl1%=:\n"
    "s_add_u32 s67, s79, 1\n"
    "s_bitcmp1_b64 s[96:97], s67\n"
    "s_cbranch_scc0 Lf11%=\n"
    "global_load_dword v6, v44, s[18:19] sc0 sc1\n"
    "Lf11%=:\n"
    "s_bitcmp1_b64 s[98:99], s67\n"
    "s_cbranch_scc0 Lf21%=\n"
    "global_load_dword v8, v9, s[18:19] sc0 sc1\n"
    "Lf21%=:\n"
    "s_bitcmp1_b64 s[44:45], s79\n"
    "s_cbranch_scc0 Lf31%=\n"
    "s_mov_b64 exec, 1\n"
    "global_atomic_add v60, v62, s[20:21] sc1\n"
    "s_mov_b64 exec, -1\n"
    "v_add_u32 v60, 4, v60\n"
    "Lf31%=:\n"
    "s_bitcmp1_b64 s[42:43], s79\n"
    "s_cbranch_scc0 Lfl1r%=\n"
    "s_getpc_b64 s[68:69]\n"
    "Lgp1%=:\n"
    "s_add_u32 s68, s68, Lfl1r%=-Lgp1%=\n"
    "s_addc_u32 s69, s69, -1\n"
    "s_branch Lpush%=\n"
    "Lfl2%=:\n"
    "s_add_u32 s67, s79, 1\n"
    "s_bitcmp1_b64 s[96:97], s67\n"
    "s_cbranch_scc0 Lf12%=\n"
    "global_load_dword v61, v44, s[18:19] sc0 sc1\n"
    "Lf12%=:\n"
    "s_bitcmp1_b64 s[98:99], s67\n"
    "s_cbranch_scc0 Lf22%=\n"
    "global_load_dword v8, v9, s[18:19] sc0 sc1\n"
    "Lf22%=:\n"
    "s_bitcmp1_b64 s[44:45], s79\n"
    "s_cbranch_scc0 Lf32%=\n"
    "s_mov_b64 exec, 1\n"
    "global_atomic_add v60, v62, s[20:21] sc1\n"
    "s_mov_b64 exec, -1\n"
    "v_add_u32 v60, 4, v60\n"
    "Lf32%=:\n"
    "s_bitcmp1_b64 s[42:43], s79\n"
    "s_cbranch_scc0 Lfl2r%=\n"
    "s_getpc_b64 s[68:69]\n"
    "Lgp2%=:\n"
    "s_add_u32 s68, s68, Lfl2r%=-Lgp2%=\n"
    "s_addc_u32 s69, s69, -1\n"
    "s_branch Lpush%=\n"
    "Lfl3%=:\n"
    "s_add_u32 s67, s79, 1\n"
    "s_bitcmp1_b64 s[96:97], s67\n"
    "s_cbranch_scc0 Lf13%=\n"
    "global_load_dword v6, v44, s[18:19] sc0 sc1\n"
    "Lf13%=:\n"
    "s_bitcmp1_b64 s[98:99], s67\n"
    "s_cbranch_scc0 Lf23%=\n"
    "global_load_dword v8, v9, s[18:19] sc0 sc1\n"
    "Lf23%=:\n"
    "s_bitcmp1_b64 s[44:45], s79\n"
    "s_cbranch_scc0 Lf33%=\n"
    "s_mov_b64 exec, 1\n"
    "global_atomic_add v60, v62, s[20:21] sc1\n"
    "s_mov_b64 exec, -1\n"
    "v_add_u32 v60, 4, v60\n"
    "Lf33%=:\n"
    "s_bitcmp1_b64 s[42:43], s79\n"
    "s_cbranch_scc0 Lfl3r%=\n"
    "s_getpc_b64 s[68:69]\n"
    "Lgp3%=:\n"
    "s_add_u32 s68, s68, Lfl3r%=-Lgp3%=\n"
    "s_addc_u32 s69, s69, -1\n"
    "s_branch Lpush%=\n"
    "Lfl4%=:\n"
    "s_add_u32 s67, s79, 1\n"
    "s_bitcmp1_b64 s[96:97], s67\n"
    "s_cbranch_scc0 Lf14%=\n"
    "global_load_dword v61, v44, s[18:19] sc0 sc1\n"
    "Lf14%=:\n"
    "s_bitcmp1_b64 s[98:99], s67\n"
    "s_cbranch_scc0 Lf24%=\n"
    "global_load_dword v8, v9, s[18:19] sc0 sc1\n"
    "Lf24%=:\n"
    "s_bitcmp1_b64 s[44:45], s79\n"
    "s_cbranch_scc0 Lf34%=\n"
    "s_mov_b64 exec, 1\n"
    "global_atomic_add v60, v62, s[20:21] sc1\n"
    "s_mov_b64 exec, -1\n"
    "v_add_u32 v60, 4, v60\n"
    "Lf34%=:\n"
    "s_bitcmp1_b64 s[42:43], s79\n"
    "s_cbranch_scc0 Lfl4r%=\n"
    "s_getpc_b64 s[68:69]\n"
    "Lgp4%=:\n"
    "s_add_u32 s68, s68, Lfl4r%=-Lgp4%=\n"
    "s_addc_u32 s69, s69, -1\n"
    "s_branch Lpush%=\n"
    "Lfl5%=:\n"
    "s_add_u32 s67, s79, 1\n"
    "s_bitcmp1_b64 s[96:97], s67\n"
    "s_cbranch_scc0 Lf15%=\n"
    "global_load_dword v6, v44, s[18:19] sc0 sc1\n"
    "Lf15%=:\n"
    "s_bitcmp1_b64 s[98:99], s67\n"
    "s_cbranch_scc0 Lf25%=\n"
    "global_load_dword v8, v9, s[18:19] sc0 sc1\n"
    "Lf25%=:\n"
    "s_bitcmp1_b64 s[44:45], s79\n"
    "s_cbranch_scc0 Lf35%=\n"
    "s_mov_b64 exec, 1\n"
    "global_atomic_add v60, v62, s[20:21] sc1\n"
    "s_mov_b64 exec, -1\n"
    "v_add_u32 v60, 4, v60\n"
    "Lf35%=:\n"
    "s_bitcmp1_b64 s[42:43], s79\n"
    "s_cbranch_scc0 Lfl5r%=\n"
    "s_getpc_b64 s[68:69]\n"
    "Lgp5%=:\n"
    "s_add_u32 s68, s68, Lfl5r%=-Lgp5%=\n"
    "s_addc_u32 s69, s69, -1\n"
    "s_branch Lpush%=\n"
    "Lfl6%=:\n"
    "s_add_u32 s67, s79, 1\n"
    "s_bitcmp1_b64 s[96:97], s67\n"
    "s_cbranch_scc0 Lf16%=\n"
    "global_load_dword v61, v44, s[18:19] sc0 sc1\n"
    "Lf16%=:\n"
    "s_bitcmp1_b64 s[98:99], s67\n"
    "s_cbranch_scc0 Lf26%=\n"
    "global_load_dword v8, v9, s[18:19] sc0 sc1\n"
    "Lf26%=:\n"
    "s_bitcmp1_b64 s[44:45], s79\n"
    "s_cbranch_scc0 Lf36%=\n"
    "s_mov_b64 exec, 1\n"
    "global_atomic_add v60, v62, s[20:21] sc1\n"
    "s_mov_b64 exec, -1\n"
    "v_add_u32 v60, 4, v60\n"
    "Lf36%=:\n"
    "s_bitcmp1_b64 s[42:43], s79\n"
    "s_cbranch_scc0 Lfl6r%=\n"
    "s_getpc_b64 s[68:69]\n"
    "Lgp6%=:\n"
    "s_add_u32 s68, s68, Lfl6r%=-Lgp6%=\n"
    "s_addc_u32 s69, s69, -1\n"
    "s_branch Lpush%=\n"
    "Lfl7%=:\n"
    "s_add_u32 s67, s79, 1\n"
    "s_bitcmp1_b64 s[96:97], s67\n"
    "s_cbranch_scc0 Lf17%=\n"
    "global_load_dword v6, v44, s[18:19] sc0 sc1\n"
    "Lf17%=:\n"
    "s_bitcmp1_b64 s[98:99], s67\n"
    "s_cbranch_scc0 Lf27%=\n"
    "global_load_dword v8, v9, s[18:19] sc0 sc1\n"
    "Lf27%=:\n"
    "s_bitcmp1_b64 s[44:45], s79\n"
    "s_cbranch_scc0 Lf37%=\n"
    "s_mov_b64 exec, 1\n"
    "global_atomic_add v60, v62, s[20:21] sc1\n"
    "s_mov_b64 exec, -1\n"
    "v_add_u32 v60, 4, v60\n"
    "Lf37%=:\n"
    "s_bitcmp1_b64 s[42:43], s79\n"
    "s_cbranch_scc0 Lfl7r%=\n"
    "s_getpc_b64 s[68:69]\n"
    "Lgp7%=:\n"
    "s_add_u32 s68, s68, Lfl7r%=-Lgp7%=\n"
    "s_addc_u32 s69, s69, -1\n"
    "s_branch Lpush%=\n"
    "Lfl102%=:\n"
    "s_add_u32 s67, s79, 1\n"
    "s_bitcmp1_b64 s[96:97], s67\n"
    "s_cbranch_scc0 Lf1102%=\n"
    "global_load_dword v61, v44, s[18:19] sc0 sc1\n"
    "Lf1102%=:\n"
    "s_bitcmp1_b64 s[98:99], s67\n"
    "s_cbranch_scc0 Lf2102%=\n"
    "global_load_dword v8, v9, s[18:19] sc0 sc1\n"
    "Lf2102%=:\n"
    "s_bitcmp1_b64 s[44:45], s79\n"
    "s_cbranch_scc0 Lf3102%=\n"
    "s_mov_b64 exec, 1\n"
    "global_atomic_add v60, v62, s[20:21] sc1\n"
    "s_mov_b64 exec, -1\n"
    "v_add_u32 v60, 4, v60\n"
    "Lf3102%=:\n"
    "s_bitcmp1_b64 s[42:43], s79\n"
    "s_cbranch_scc0 Lfl102r%=\n"
    "s_getpc_b64 s[68:69]\n"
    "Lgp102%=:\n"
    "s_add_u32 s68, s68, Lfl102r%=-Lgp102%=\n"
    "s_addc_u32 s69, s69, -1\n"
    "s_branch Lpush%=\n"
    "Lfl103%=:\n"
    "s_add_u32 s67, s79, 1\n"
    "s_bitcmp1_b64 s[96:97], s67\n"
    "s_cbranch_scc0 Lf1103%=\n"
    "global_load_dword v6, v44, s[18:19] sc0 sc1\n"
    "Lf1103%=:\n"
    "s_bitcmp1_b64 s[98:99], s67\n"
    "s_cbranch_scc0 Lf2103%=\n"
    "global_load_dword v8, v9, s[18:19] sc0 sc1\n"
    "Lf2103%=:\n"
    "s_bitcmp1_b64 s[44:45], s79\n"
    "s_cbranch_scc0 Lf3103%=\n"
    "s_mov_b64 exec, 1\n"
    "global_atomic_add v60, v62, s[20:21] sc1\n"
    "s_mov_b64 exec, -1\n"
    "v_add_u32 v60, 4, v60\n"
    "Lf3103%=:\n"
    "s_bitcmp1_b64 s[42:43], s79\n"
    "s_cbranch_scc0 Lfl103r%=\n"
    "s_getpc_b64 s[68:69]\n"
    "Lgp103%=:\n"
    "s_add_u32 s68, s68, Lfl103r%=-Lgp103%=\n"
    "s_addc_u32 s69, s69, -1\n"
    "s_branch Lpush%=\n"
    "Lfl104%=:\n"
    "s_add_u32 s67, s79, 1\n"
    "s_bitcmp1_b64 s[96:97], s67\n"
    "s_cbranch_scc0 Lf1104%=\n"
    "global_load_dword v61, v44, s[18:19] sc0 sc1\n"
    "Lf1104%=:\n"
    "s_bitcmp1_b64 s[98:99], s67\n"
    "s_cbranch_scc0 Lf2104%=\n"
    "global_load_dword v8, v9, s[18:19] sc0 sc1\n"
    "Lf2104%=:\n"
    "s_bitcmp1_b64 s[44:45], s79\n"
    "s_cbranch_scc0 Lf3104%=\n"
    "s_mov_b64 exec, 1\n"
    "global_atomic_add v60, v62, s[20:21] sc1\n"
    "s_mov_b64 exec, -1\n"
    "v_add_u32 v60, 4, v60\n"
    "Lf3104%=:\n"
    "s_bitcmp1_b64 s[42:43], s79\n"
    "s_cbranch_scc0 Lfl104r%=\n"
    "s_getpc_b64 s[68:69]\n"
    "Lgp104%=:\n"
    "s_add_u32 s68, s68, Lfl104r%=-Lgp104%=\n"
    "s_addc_u32 s69, s69, -1\n"
    "s_branch Lpush%=\n"
    "Lfl105%=:\n"
    "s_add_u32 s67, s79, 1\n"
    "s_bitcmp1_b64 s[96:97], s67\n"
    "s_cbranch_scc0 Lf1105%=\n"
    "global_load_dword v6, v44, s[18:19] sc0 sc1\n"
    "Lf1105%=:\n"
    "s_bitcmp1_b64 s[98:99], s67\n"
    "s_cbranch_scc0 Lf2105%=\n"
    "global_load_dword v8, v9, s[18:19] sc0 sc1\n"
    "Lf2105%=:\n"
    "s_bitcmp1_b64 s[44:45], s79\n"
    "s_cbranch_scc0 Lf3105%=\n"
    "s_mov_b64 exec, 1\n"
    "global_atomic_add v60, v62, s[20:21] sc1\n"
    "s_mov_b64 exec, -1\n"
    "v_add_u32 v60, 4, v60\n"
    "Lf3105%=:\n"
    "s_bitcmp1_b64 s[42:43], s79\n"
    "s_cbranch_scc0 Lfl105r%=\n"
    "s_getpc_b64 s[68:69]\n"
    "Lgp105%=:\n"
    "s_add_u32 s68, s68, Lfl105r%=-Lgp105%=\n"
    "s_addc_u32 s69, s69, -1\n"
    "s_branch Lpush%=\n"
    "Lfl106%=:\n"
    "s_add_u32 s67, s79, 1\n"
    "s_bitcmp1_b64 s[96:97], s67\n"
    "s_cbranch_scc0 Lf1106%=\n"
    "global_load_dword v61, v44, s[18:19] sc0 sc1\n"
    "Lf1106%=:\n"
    "s_bitcmp1_b64 s[98:99], s67\n"
    "s_cbranch_scc0 Lf2106%=\n"
    "global_load_dword v8, v9, s[18:19] sc0 sc1\n"
    "Lf2106%=:\n"
    "s_bitcmp1_b64 s[44:45], s79\n"
    "s_cbranch_scc0 Lf3106%=\n"
    "s_mov_b64 exec, 1\n"
    "global_atomic_add v60, v62, s[20:21] sc1\n"
    "s_mov_b64 exec, -1\n"
    "v_add_u32 v60, 4, v60\n"
    "Lf3106%=:\n"
    "s_bitcmp1_b64 s[42:43], s79\n"
    "s_cbranch_scc0 Lfl106r%=\n"
    "s_getpc_b64 s[68:69]\n"
    "Lgp106%=:\n"
    "s_add_u32 s68, s68, Lfl106r%=-Lgp106%=\n"
    "s_addc_u32 s69, s69, -1\n"
    "s_branch Lpush%=\n"
    "Lfl107%=:\n"
    "s_add_u32 s67, s79, 1\n"
    "s_bitcmp1_b64 s[96:97], s67\n"
    "s_cbranch_scc0 Lf1107%=\n"
    "global_load_dword v6, v44, s[18:19] sc0 sc1\n"
    "Lf1107%=:\n"
    "s_bitcmp1_b64 s[98:99], s67\n"
    "s_cbranch_scc0 Lf2107%=\n"
    "global_load_dword v8, v9, s[18:19] sc0 sc1\n"
    "Lf2107%=:\n"
    "s_bitcmp1_b64 s[44:45], s79\n"
    "s_cbranch_scc0 Lf3107%=\n"
    "s_mov_b64 exec, 1\n"
    "global_atomic_add v60, v62, s[20:21] sc1\n"
    "s_mov_b64 exec, -1\n"
    "v_add_u32 v60, 4, v60\n"
    "Lf3107%=:\n"
    "s_bitcmp1_b64 s[42:43], s79\n"
    "s_cbranch_scc0 Lfl107r%=\n"
    "s_getpc_b64 s[68:69]\n"
    "Lgp107%=:\n"
    "s_add_u32 s68, s68, Lfl107r%=-Lgp107%=\n"
    "s_addc_u32 s69, s69, -1\n"
    "s_branch Lpush%=\n"
    "Lpush%=:\n"
    "s_cmp_lg_u32 s39, 0\n"
    "s_cbranch_scc1 Lpu1%=\n"
    "s_mov_b32 exec_lo, 0\n"
    "global_store_dwordx4 v43, a[224:227], s[82:83] sc0 sc1\n"
    "s_mov_b32 exec_lo, -1\n"
    "s_branch Lpux%=\n"
    "Lpu1%=:\n"
    "s_cmp_lg_u32 s39, 1\n"
    "s_cbranch_scc1 Lpu2%=\n"
    "global_store_dwordx4 v43, a[228:231], s[82:83] offset:256 sc0 sc1\n"
    "s_branch Lpux%=\n"
    "Lpu2%=:\n"
    "s_cmp_lg_u32 s39, 2\n"
    "s_cbranch_scc1 Lpu3%=\n"
    "global_store_dwordx4 v43, a[232:235], s[82:83] offset:512 sc0 sc1\n"
    "s_branch Lpux%=\n"
    "Lpu3%=:\n"
    "global_store_dwordx4 v43, a[236:239], s[82:83] offset:768 sc0 sc1\n"
    "Lpux%=:\n"
    "s_add_u32 s39, s39, 1\n"
    "s_setpc_b64 s[68:69]\n"
    "Loe2%=:\n"
    "s_waitcnt vmcnt(20)\n"
    "s_bitcmp1_b64 s[96:97], s79\n"
    "s_cbranch_scc0 Lgen2%=\n"
    "v_cmp_ge_u32 s[92:93], v6, v7\n"
    "v_cmp_ge_u32 vcc, v6, v13\n"
    "s_add_u32 s50, s50, 1\n"
    "v_cmp_ge_u32 s[90:91], s50, v11\n"
    "s_cmp_eq_u64 s[92:93], -1\n"
    "s_cbranch_scc1 Lall2%=\n"
    "s_cmp_eq_u64 vcc, -1\n"
    "s_cbranch_scc0 Lsp2%=\n"
    "Lok2%=:\n"
    "v_cndmask_b32 v13, 0, v12, s[90:91]\n"
    "s_bitcmp1_b64 s[62:63], s79\n"
    "s_cbranch_scc1 Lgen2%=\n"
    "s_branch Loe2r%=\n"
    "Lall2%=:\n"
    "s_mov_b64 s[96:97], 0\n"
    "s_lshr_b64 s[64:65], s[98:99], 1\n"
    "s_or_b64 s[64:65], s[64:65], s[28:29]\n"
    "s_mov_b64 s[60:61], s[62:63]\n"
    "s_branch Lok2%=\n"
    "Lsp2%=:\n"
    "s_mov_b64 s[46:47], s[90:91]\n"
    "s_getpc_b64 s[68:69]\n"
    "Lgs2%=:\n"
    "s_add_u32 s68, s68, Lok22%=-Lgs2%=\n"
    "s_addc_u32 s69, s69, 0\n"
    "s_branch Lspin0%=\n"
    "Lok22%=:\n"
    "s_mov_b64 s[90:91], s[46:47]\n"
    "s_branch Lok2%=\n"
    "Lgen2%=:\n"
    "s_getpc_b64 s[68:69]\n"
    "Lgg2%=:\n"
    "s_add_u32 s68, s68, Loe2r%=-Lgg2%=\n"
    "s_addc_u32 s69, s69, -1\n"
    "s_branch Lgen%=\n"
    "Loe3%=:\n"
    "s_waitcnt vmcnt(20)\n"
    "s_bitcmp1_b64 s[96:97], s79\n"
    "s_cbranch_scc0 Lgen3%=\n"
    "v_cmp_ge_u32 s[92:93], v61, v7\n"
    "v_cmp_ge_u32 vcc, v61, v13\n"
    "s_add_u32 s50, s50, 1\n"
    "v_cmp_ge_u32 s[90:91], s50, v11\n"
    "s_cmp_eq_u64 s[92:93], -1\n"
    "s_cbranch_scc1 Lall3%=\n"
    "s_cmp_eq_u64 vcc, -1\n"
    "s_cbranch_scc0 Lsp3%=\n"
    "Lok3%=:\n"
    "v_cndmask_b32 v13, 0, v12, s[90:91]\n"
    "s_bitcmp1_b64 s[62:63], s79\n"
    "s_cbranch_scc1 Lgen3%=\n"
    "s_branch Loe3r%=\n"
    "Lall3%=:\n"
    "s_mov_b64 s[96:97], 0\n"
    "s_lshr_b64 s[64:65], s[98:99], 1\n"
    "s_or_b64 s[64:65], s[64:65], s[28:29]\n"
    "s_mov_b64 s[60:61], s[62:63]\n"
    "s_branch Lok3%=\n"
    "Lsp3%=:\n"
    "s_mov_b64 s[46:47], s[90:91]\n"
    "s_getpc_b64 s[68:69]\n"
    "Lgs3%=:\n"
    "s_add_u32 s68, s68, Lok23%=-Lgs3%=\n"
    "s_addc_u32 s69, s69, 0\n"
    "s_branch Lspin0%=\n"
    "Lok23%=:\n"
    "s_mov_b64 s[90:91], s[46:47]\n"
    "s_branch Lok3%=\n"
    "Lgen3%=:\n"
    "s_getpc_b64 s[68:69]\n"
    "Lgg3%=:\n"
    "s_add_u32 s68, s68, Loe3r%=-Lgg3%=\n"
    "s_addc_u32 s69, s69, -1\n"
    "s_branch Lgen%=\n"
    "Loe4%=:\n"
    "s_waitcnt vmcnt(20)\n"
    "s_bitcmp1_b64 s[96:97], s79\n"
    "s_cbranch_scc0 Lgen4%=\n"
    "v_cmp_ge_u32 s[92:93], v6, v7\n"
    "v_cmp_ge_u32 vcc, v6, v13\n"
    "s_add_u32 s50, s50, 1\n"
    "v_cmp_ge_u32 s[90:91], s50, v11\n"
    "s_cmp_eq_u64 s[92:93], -1\n"
    "s_cbranch_scc1 Lall4%=\n"
    "s_cmp_eq_u64 vcc, -1\n"
    "s_cbranch_scc0 Lsp4%=\n"
    "Lok4%=:\n"
    "v_cndmask_b32 v13, 0, v12, s[90:91]\n"
    "s_bitcmp1_b64 s[62:63], s79\n"
    "s_cbranch_scc1 Lgen4%=\n"
    "s_branch Loe4r%=\n"
    "Lall4%=:\n"
    "s_mov_b64 s[96:97], 0\n"
    "s_lshr_b64 s[64:65], s[98:99], 1\n"
    "s_or_b64 s[64:65], s[64:65], s[28:29]\n"
    "s_mov_b64 s[60:61], s[62:63]\n"
    "s_branch Lok4%=\n"
    "Lsp4%=:\n"
    "s_mov_b64 s[46:47], s[90:91]\n"
    "s_getpc_b64 s[68:69]\n"
    "Lgs4%=:\n"
    "s_add_u32 s68, s68, Lok24%=-Lgs4%=\n"
    "s_addc_u32 s69, s69, 0\n"
    "s_branch Lspin0%=\n"
    "Lok24%=:\n"
    "s_mov_b64 s[90:91], s[46:47]\n"
    "s_branch Lok4%=\n"
    "Lgen4%=:\n"
    "s_getpc_b64 s[68:69]\n"
    "Lgg4%=:\n"
    "s_add_u32 s68, s68, Loe4r%=-Lgg4%=\n"
    "s_addc_u32 s69, s69, -1\n"
    "s_branch Lgen%=\n"
    "Loe5%=:\n"
    "s_waitcnt vmcnt(20)\n"
    "s_bitcmp1_b64 s[96:97], s79\n"
    "s_cbranch_scc0 Lgen5%=\n"
    "v_cmp_ge_u32 s[92:93], v61, v7\n"
    "v_cmp_ge_u32 vcc, v61, v13\n"
    "s_add_u32 s50, s50, 1\n"
    "v_cmp_ge_u32 s[90:91], s50, v11\n"
    "s_cmp_eq_u64 s[92:93], -1\n"
    "s_cbranch_scc1 Lall5%=\n"
    "s_cmp_eq_u64 vcc, -1\n"
    "s_cbranch_scc0 Lsp5%=\n"
    "Lok5%=:\n"
    "v_cndmask_b32 v13, 0, v12, s[90:91]\n"
    "s_bitcmp1_b64 s[62:63], s79\n"
    "s_cbranch_scc1 Lgen5%=\n"
    "s_branch Loe5r%=\n"
    "Lall5%=:\n"
    "s_mov_b64 s[96:97], 0\n"
    "s_lshr_b64 s[64:65], s[98:99], 1\n"
    "s_or_b64 s[64:65], s[64:65], s[28:29]\n"
    "s_mov_b64 s[60:61], s[62:63]\n"
    "s_branch Lok5%=\n"
    "Lsp5%=:\n"
    "s_mov_b64 s[46:47], s[90:91]\n"
    "s_getpc_b64 s[68:69]\n"
    "Lgs5%=:\n"
    "s_add_u32 s68, s68, Lok25%=-Lgs5%=\n"
    "s_addc_u32 s69, s69, 0\n"
    "s_branch Lspin0%=\n"
    "Lok25%=:\n"
    "s_mov_b64 s[90:91], s[46:47]\n"
    "s_branch Lok5%=\n"
    "Lgen5%=:\n"
    "s_getpc_b64 s[68:69]\n"
    "Lgg5%=:\n"
    "s_add_u32 s68, s68, Loe5r%=-Lgg5%=\n"
    "s_addc_u32 s69, s69, -1\n"
    "s_branch Lgen%=\n"
    "Loe6%=:\n"
    "s_waitcnt vmcnt(20)\n"
    "s_bitcmp1_b64 s[96:97], s79\n"
    "s_cbranch_scc0 Lgen6%=\n"
    "v_cmp_ge_u32 s[92:93], v6, v7\n"
    "v_cmp_ge_u32 vcc, v6, v13\n"
    "s_add_u32 s50, s50, 1\n"
    "v_cmp_ge_u32 s[90:91], s50, v11\n"
    "s_cmp_eq_u64 s[92:93], -1\n"
    "s_cbranch_scc1 Lall6%=\n"
    "s_cmp_eq_u64 vcc, -1\n"
    "s_cbranch_scc0 Lsp6%=\n"
    "Lok6%=:\n"
    "v_cndmask_b32 v13, 0, v12, s[90:91]\n"
    "s_bitcmp1_b64 s[62:63], s79\n"
    "s_cbranch_scc1 Lgen6%=\n"
    "s_branch Loe6r%=\n"
    "Lall6%=:\n"
    "s_mov_b64 s[96:97], 0\n"
    "s_lshr_b64 s[64:65], s[98:99], 1\n"
    "s_or_b64 s[64:65], s[64:65], s[28:29]\n"
    "s_mov_b64 s[60:61], s[62:63]\n"
    "s_branch Lok6%=\n"
    "Lsp6%=:\n"
    "s_mov_b64 s[46:47], s[90:91]\n"
    "s_getpc_b64 s[68:69]\n"
    "Lgs6%=:\n"
    "s_add_u32 s68, s68, Lok26%=-Lgs6%=\n"
    "s_addc_u32 s69, s69, 0\n"
    "s_branch Lspin0%=\n"
    "Lok26%=:\n"
    "s_mov_b64 s[90:91], s[46:47]\n"
    "s_branch Lok6%=\n"
    "Lgen6%=:\n"
    "s_getpc_b64 s[68:69]\n"
    "Lgg6%=:\n"
    "s_add_u32 s68, s68, Loe6r%=-Lgg6%=\n"
    "s_addc_u32 s69, s69, -1\n"
    "s_branch Lgen%=\n"
    "Loe7%=:\n"
    "s_waitcnt vmcnt(20)\n"
    "s_bitcmp1_b64 s[96:97], s79\n"
    "s_cbranch_scc0 Lgen7%=\n"
    "v_cmp_ge_u32 s[92:93], v61, v7\n"
    "v_cmp_ge_u32 vcc, v61, v13\n"
    "s_add_u32 s50, s50, 1\n"
    "v_cmp_ge_u32 s[90:91], s50, v11\n"
    "s_cmp_eq_u64 s[92:93], -1\n"
    "s_cbranch_scc1 Lall7%=\n"
    "s_cmp_eq_u64 vcc, -1\n"
    "s_cbranch_scc0 Lsp7%=\n"
    "Lok7%=:\n"
    "v_cndmask_b32 v13, 0, v12, s[90:91]\n"
    "s_bitcmp1_b64 s[62:63], s79\n"
    "s_cbranch_scc1 Lgen7%=\n"
    "s_branch Loe7r%=\n"
    "Lall7%=:\n"
    "s_mov_b64 s[96:97], 0\n"
    "s_lshr_b64 s[64:65], s[98:99], 1\n"
    "s_or_b64 s[64:65], s[64:65], s[28:29]\n"
    "s_mov_b64 s[60:61], s[62:63]\n"
    "s_branch Lok7%=\n"
    "Lsp7%=:\n"
    "s_mov_b64 s[46:47], s[90:91]\n"
    "s_getpc_b64 s[68:69]\n"
    "Lgs7%=:\n"
    "s_add_u32 s68, s68, Lok27%=-Lgs7%=\n"
    "s_addc_u32 s69, s69, 0\n"
    "s_branch Lspin0%=\n"
    "Lok27%=:\n"
    "s_mov_b64 s[90:91], s[46:47]\n"
    "s_branch Lok7%=\n"
    "Lgen7%=:\n"
    "s_getpc_b64 s[68:69]\n"
    "Lgg7%=:\n"
    "s_add_u32 s68, s68, Loe7r%=-Lgg7%=\n"
    "s_addc_u32 s69, s69, -1\n"
    "s_branch Lgen%=\n"
    "Loe102%=:\n"
    "s_waitcnt vmcnt(20)\n"
    "s_bitcmp1_b64 s[96:97], s79\n"
    "s_cbranch_scc0 Lgen102%=\n"
    "v_cmp_ge_u32 s[92:93], v6, v7\n"
    "v_cmp_ge_u32 vcc, v6, v13\n"
    "s_add_u32 s50, s50, 1\n"
    "v_cmp_ge_u32 s[90:91], s50, v11\n"
    "s_cmp_eq_u64 s[92:93], -1\n"
    "s_cbranch_scc1 Lall102%=\n"
    "s_cmp_eq_u64 vcc, -1\n"
    "s_cbranch_scc0 Lsp102%=\n"
    "Lok102%=:\n"
    "v_cndmask_b32 v13, 0, v12, s[90:91]\n"
    "s_bitcmp1_b64 s[62:63], s79\n"
    "s_cbranch_scc1 Lgen102%=\n"
    "s_branch Loe102r%=\n"
    "Lall102%=:\n"
    "s_mov_b64 s[96:97], 0\n"
    "s_lshr_b64 s[64:65], s[98:99], 1\n"
    "s_or_b64 s[64:65], s[64:65], s[28:29]\n"
    "s_mov_b64 s[60:61], s[62:63]\n"
    "s_branch Lok102%=\n"
    "Lsp102%=:\n"
    "s_mov_b64 s[46:47], s[90:91]\n"
    "s_getpc_b64 s[68:69]\n"
    "Lgs102%=:\n"
    "s_add_u32 s68, s68, Lok2102%=-Lgs102%=\n"
    "s_addc_u32 s69, s69, 0\n"
    "s_branch Lspin0%=\n"
    "Lok2102%=:\n"
    "s_mov_b64 s[90:91], s[46:47]\n"
    "s_branch Lok102%=\n"
    "Lgen102%=:\n"
    "s_getpc_b64 s[68:69]\n"
    "Lgg102%=:\n"
    "s_add_u32 s68, s68, Loe102r%=-Lgg102%=\n"
    "s_addc_u32 s69, s69, -1\n"
    "s_branch Lgen%=\n"
    "Loe103%=:\n"
    "s_waitcnt vmcnt(20)\n"
    "s_bitcmp1_b64 s[96:97], s79\n"
    "s_cbranch_scc0 Lgen103%=\n"
    "v_cmp_ge_u32 s[92:93], v61, v7\n"
    "v_cmp_ge_u32 vcc, v61, v13\n"
    "s_add_u32 s50, s50, 1\n"
    "v_cmp_ge_u32 s[90:91], s50, v11\n"
    "s_cmp_eq_u64 s[92:93], -1\n"
    "s_cbranch_scc1 Lall103%=\n"
    "s_cmp_eq_u64 vcc, -1\n"
    "s_cbranch_scc0 Lsp103%=\n"
    "Lok103%=:\n"
    "v_cndmask_b32 v13, 0, v12, s[90:91]\n"
    "s_bitcmp1_b64 s[62:63], s79\n"
    "s_cbranch_scc1 Lgen103%=\n"
    "s_branch Loe103r%=\n"
    "Lall103%=:\n"
    "s_mov_b64 s[96:97], 0\n"
    "s_lshr_b64 s[64:65], s[98:99], 1\n"
    "s_or_b64 s[64:65], s[64:65], s[28:29]\n"
    "s_mov_b64 s[60:61], s[62:63]\n"
    "s_branch Lok103%=\n"
    "Lsp103%=:\n"
    "s_mov_b64 s[46:47], s[90:91]\n"
    "s_getpc_b64 s[68:69]\n"
    "Lgs103%=:\n"
    "s_add_u32 s68, s68, Lok2103%=-Lgs103%=\n"
    "s_addc_u32 s69, s69, 0\n"
    "s_branch Lspin0%=\n"
    "Lok2103%=:\n"
    "s_mov_b64 s[90:91], s[46:47]\n"
    "s_branch Lok103%=\n"
    "Lgen103%=:\n"
    "s_getpc_b64 s[68:69]\n"
    "Lgg103%=:\n"
    "s_add_u32 s68, s68, Loe103r%=-Lgg103%=\n"
    "s_addc_u32 s69, s69, -1\n"
    "s_branch Lgen%=\n"
    "Loe104%=:\n"
    "s_waitcnt vmcnt(20)\n"
    "s_bitcmp1_b64 s[96:97], s79\n"
    "s_cbranch_scc0 Lgen104%=\n"
    "v_cmp_ge_u32 s[92:93], v6, v7\n"
    "v_cmp_ge_u32 vcc, v6, v13\n"
    "s_add_u32 s50, s50, 1\n"
    "v_cmp_ge_u32 s[90:91], s50, v11\n"
    "s_cmp_eq_u64 s[92:93], -1\n"
    "s_cbranch_scc1 Lall104%=\n"
    "s_cmp_eq_u64 vcc, -1\n"
    "s_cbranch_scc0 Lsp104%=\n"
    "Lok104%=:\n"
    "v_cndmask_b32 v13, 0, v12, s[90:91]\n"
    "s_bitcmp1_b64 s[62:63], s79\n"
    "s_cbranch_scc1 Lgen104%=\n"
    "s_branch Loe104r%=\n"
    "Lall104%=:\n"
    "s_mov_b64 s[96:97], 0\n"
    "s_lshr_b64 s[64:65], s[98:99], 1\n"
    "s_or_b64 s[64:65], s[64:65], s[28:29]\n"
    "s_mov_b64 s[60:61], s[62:63]\n"
    "s_branch Lok104%=\n"
    "Lsp104%=:\n"
    "s_mov_b64 s[46:47], s[90:91]\n"
    "s_getpc_b64 s[68:69]\n"
    "Lgs104%=:\n"
    "s_add_u32 s68, s68, Lok2104%=-Lgs104%=\n"
    "s_addc_u32 s69, s69, 0\n"
    "s_branch Lspin0%=\n"
    "Lok2104%=:\n"
    "s_mov_b64 s[90:91], s[46:47]\n"
    "s_branch Lok104%=\n"
    "Lgen104%=:\n"
    "s_getpc_b64 s[68:69]\n"
    "Lgg104%=:\n"
    "s_add_u32 s68, s68, Loe104r%=-Lgg104%=\n"
    "s_addc_u32 s69, s69, -1\n"
    "s_branch Lgen%=\n"
    "Loe105%=:\n"
    "s_waitcnt vmcnt(20)\n"
    "s_bitcmp1_b64 s[96:97], s79\n"
    "s_cbranch_scc0 Lgen105%=\n"
    "v_cmp_ge_u32 s[92:93], v61, v7\n"
    "v_cmp_ge_u32 vcc, v61, v13\n"
    "s_add_u32 s50, s50, 1\n"
    "v_cmp_ge_u32 s[90:91], s50, v11\n"
    "s_cmp_eq_u64 s[92:93], -1\n"
    "s_cbranch_scc1 Lall105%=\n"
    "s_cmp_eq_u64 vcc, -1\n"
    "s_cbranch_scc0 Lsp105%=\n"
    "Lok105%=:\n"
    "v_cndmask_b32 v13, 0, v12, s[90:91]\n"
    "s_bitcmp1_b64 s[62:63], s79\n"
    "s_cbranch_scc1 Lgen105%=\n"
    "s_branch Loe105r%=\n"
    "Lall105%=:\n"
    "s_mov_b64 s[96:97], 0\n"
    "s_lshr_b64 s[64:65], s[98:99], 1\n"
    "s_or_b64 s[64:65], s[64:65], s[28:29]\n"
    "s_mov_b64 s[60:61], s[62:63]\n"
    "s_branch Lok105%=\n"
    "Lsp105%=:\n"
    "s_mov_b64 s[46:47], s[90:91]\n"
    "s_getpc_b64 s[68:69]\n"
    "Lgs105%=:\n"
    "s_add_u32 s68, s68, Lok2105%=-Lgs105%=\n"
    "s_addc_u32 s69, s69, 0\n"
    "s_branch Lspin0%=\n"
    "Lok2105%=:\n"
    "s_mov_b64 s[90:91], s[46:47]\n"
    "s_branch Lok105%=\n"
    "Lgen105%=:\n"
    "s_getpc_b64 s[68:69]\n"
    "Lgg105%=:\n"
    "s_add_u32 s68, s68, Loe105r%=-Lgg105%=\n"
    "s_addc_u32 s69, s69, -1\n"
    "s_branch Lgen%=\n"
    "Loe106%=:\n"
    "s_waitcnt vmcnt(20)\n"
    "s_bitcmp1_b64 s[96:97], s79\n"
    "s_cbranch_scc0 Lgen106%=\n"
    "v_cmp_ge_u32 s[92:93], v6, v7\n"
    "v_cmp_ge_u32 vcc, v6, v13\n"
    "s_add_u32 s50, s50, 1\n"
    "v_cmp_ge_u32 s[90:91], s50, v11\n"
    "s_cmp_eq_u64 s[92:93], -1\n"
    "s_cbranch_scc1 Lall106%=\n"
    "s_cmp_eq_u64 vcc, -1\n"
    "s_cbranch_scc0 Lsp106%=\n"
    "Lok106%=:\n"
    "v_cndmask_b32 v13, 0, v12, s[90:91]\n"
    "s_bitcmp1_b64 s[62:63], s79\n"
    "s_cbranch_scc1 Lgen106%=\n"
    "s_branch Loe106r%=\n"
    "Lall106%=:\n"
    "s_mov_b64 s[96:97], 0\n"
    "s_lshr_b64 s[64:65], s[98:99], 1\n"
    "s_or_b64 s[64:65], s[64:65], s[28:29]\n"
    "s_mov_b64 s[60:61], s[62:63]\n"
    "s_branch Lok106%=\n"
    "Lsp106%=:\n"
    "s_mov_b64 s[46:47], s[90:91]\n"
    "s_getpc_b64 s[68:69]\n"
    "Lgs106%=:\n"
    "s_add_u32 s68, s68, Lok2106%=-Lgs106%=\n"
    "s_addc_u32 s69, s69, 0\n"
    "s_branch Lspin0%=\n"
    "Lok2106%=:\n"
    "s_mov_b64 s[90:91], s[46:47]\n"
    "s_branch Lok106%=\n"
    "Lgen106%=:\n"
    "s_getpc_b64 s[68:69]\n"
    "Lgg106%=:\n"
    "s_add_u32 s68, s68, Loe106r%=-Lgg106%=\n"
    "s_addc_u32 s69, s69, -1\n"
    "s_branch Lgen%=\n"
    "Loe107%=:\n"
    "s_waitcnt vmcnt(20)\n"
    "s_bitcmp1_b64 s[96:97], s79\n"
    "s_cbranch_scc0 Lgen107%=\n"
    "v_cmp_ge_u32 s[92:93], v61, v7\n"
    "v_cmp_ge_u32 vcc, v61, v13\n"
    "s_add_u32 s50, s50, 1\n"
    "v_cmp_ge_u32 s[90:91], s50, v11\n"
    "s_cmp_eq_u64 s[92:93], -1\n"
    "s_cbranch_scc1 Lall107%=\n"
    "s_cmp_eq_u64 vcc, -1\n"
    "s_cbranch_scc0 Lsp107%=\n"
    "Lok107%=:\n"
    "v_cndmask_b32 v13, 0, v12, s[90:91]\n"
    "s_bitcmp1_b64 s[62:63], s79\n"
    "s_cbranch_scc1 Lgen107%=\n"
    "s_branch Loe107r%=\n"
    "Lall107%=:\n"
    "s_mov_b64 s[96:97], 0\n"
    "s_lshr_b64 s[64:65], s[98:99], 1\n"
    "s_or_b64 s[64:65], s[64:65], s[28:29]\n"
    "s_mov_b64 s[60:61], s[62:63]\n"
    "s_branch Lok107%=\n"
    "Lsp107%=:\n"
    "s_mov_b64 s[46:47], s[90:91]\n"
    "s_getpc_b64 s[68:69]\n"
    "Lgs107%=:\n"
    "s_add_u32 s68, s68, Lok2107%=-Lgs107%=\n"
    "s_addc_u32 s69, s69, 0\n"
    "s_branch Lspin0%=\n"
    "Lok2107%=:\n"
    "s_mov_b64 s[90:91], s[46:47]\n"
    "s_branch Lok107%=\n"
    "Lgen107%=:\n"
    "s_getpc_b64 s[68:69]\n"
    "Lgg107%=:\n"
    "s_add_u32 s68, s68, Loe107r%=-Lgg107%=\n"
    "s_addc_u32 s69, s69, -1\n"
    "s_branch Lgen%=\n"
    "Lspin0%=:\n"
    "s_mov_b32 s80, 2147483647\n"
    "s_memrealtime s[90:91]\n"
    "Lspin%=:\n"
    "global_load_dword v46, v44, s[18:19] sc0 sc1\n"
    "s_waitcnt vmcnt(0) lgkmcnt(0)\n"
    "v_cmp_ge_u32 vcc, v46, v13\n"
    "s_nop 4\n"
    "s_cmp_eq_u64 vcc, -1\n"
    "s_cbranch_scc1 Lspinx%=\n"
    "v_mov_b32 v35, 66688\n"
    "global_load_dword v35, v35, s[18:19] sc1\n"
    "s_memrealtime s[92:93]\n"
    "s_waitcnt vmcnt(0) lgkmcnt(0)\n"
    "v_readfirstlane_b32 s67, v35\n"
    "s_sub_u32 s77, s92, s90\n"
    "s_nop 2\n"
    "s_cmp_eq_u32 s67, s66\n"
    "s_cbranch_scc1 Lspinx%=\n"
    "s_cmp_gt_u32 s77, 40000000\n"
    "s_cbranch_scc1 Lgiveupa%=\n"
    "s_sleep 2\n"
    "s_branch Lspin%=\n"
    "Lgiveupa%=:\n"
    "v_mov_b32 v35, 66688\n"
    "v_mov_b32 v47, s66\n"
    "s_mov_b64 exec, 1\n"
    "global_store_dword v35, v47, s[18:19] sc1\n"
    "s_mov_b64 exec, -1\n"
    "s_waitcnt vmcnt(0)\n"
    "s_branch Lspinx%=\n"
    "Lspinx%=:\n"
    "s_setpc_b64 s[68:69]\n"
    "Lgen%=:\n"
    "s_sub_u32 s67, s79, 1\n"
    "s_bitcmp1_b64 s[98:99], s67\n"
    "s_cbranch_scc0 Lnoapp%=\n"
    "s_waitcnt vmcnt(16)\n"
    "v_readfirstlane_b32 s67, v34\n"
    "s_nop 3\n"
    "s_cmp_eq_u32 s67, s54\n"
    "s_cselect_b32 s67, s53, s67\n"
    "s_and_b32 s49, s67, 0xff\n"
    "s_or_b32 s48, s48, s49\n"
    "s_bcnt1_i32_b32 s55, s49\n"
    "s_cmp_eq_u32 s55, 7\n"
    "s_cbranch_scc1 Ltabok%=\n"
    "s_lshl_b32 s67, s49, 8\n"
    "s_or_b32 s67, s67, s49\n"
    "s_add_u32 s77, s22, 1\n"
    "s_lshr_b32 s67, s67, s77\n"
    "s_and_b32 s67, s67, 0xff\n"
    "s_mov_b32 s78, 0\n"
    "s_mov_b32 s90, 0\n"
    "Llist%=:\n"
    "s_ff1_i32_b32 s91, s67\n"
    "s_bitset0_b32 s67, s91\n"
    "s_add_u32 s91, s91, s77\n"
    "s_and_b32 s91, s91, 7\n"
    "s_lshl_b32 s91, s91, s90\n"
    "s_or_b32 s78, s78, s91\n"
    "s_add_u32 s90, s90, 4\n"
    "s_cmp_lg_u32 s67, 0\n"
    "s_cbranch_scc1 Llist%=\n"
    "v_subrev_u32 v35, s51, v63\n"
    "s_lshl_b32 s90, s55, 3\n"
    "v_cmp_gt_u32 vcc, s90, v35\n"
    "v_cvt_f32_u32 v38, v35\n"
    "v_add_f32 v38, 0.5, v38\n"
    "v_cvt_f32_u32 v47, s55\n"
    "v_rcp_f32 v47, v47\n"
    "s_nop 0\n"
    "v_mul_f32 v38, v38, v47\n"
    "v_cvt_u32_f32 v38, v38\n"
    "v_mul_lo_u32 v47, v38, s55\n"
    "v_sub_u32 v35, v35, v47\n"
    "v_lshlrev_b32 v35, 2, v35\n"
    "v_lshrrev_b32 v35, v35, s78\n"
    "v_and_b32 v35, 7, v35\n"
    "s_mov_b64 exec, vcc\n"
    "v_lshlrev_b32 v47, 19, v35\n"
    "v_lshl_add_u32 v47, v38, 7, v47\n"
    "v_add_u32 v40, s34, v47\n"
    "v_lshlrev_b32 v47, 25, v35\n"
    "v_lshl_add_u32 v47, v38, 22, v47\n"
    "v_add_u32 v41, s35, v47\n"
    "s_mov_b64 exec, -1\n"
    "Ltabok%=:\n"
    "s_mov_b32 s50, 0\n"
    "s_cmp_eq_u32 s38, 0\n"
    "s_cbranch_scc1 Lnow0%=\n"
    "v_lshrrev_b32 v35, 3, v63\n"
    "v_lshrrev_b32 v35, v35, s49\n"
    "v_and_b32 v35, 1, v35\n"
    "s_lshl_b32 s67, s66, 5\n"
    "v_mul_lo_u32 v12, v35, s67\n"
    "v_cmp_gt_u32 vcc, 5, v11\n"
    "s_nop 1\n"
    "v_cndmask_b32 v7, 0, v12, vcc\n"
    "v_cmp_eq_u32 vcc, 0, v11\n"
    "s_sub_u32 s67, s51, 3\n"
    "s_bitset1_b64 s[96:97], s67\n"
    "s_add_u32 s67, s67, s55\n"
    "s_bitset1_b64 s[96:97], s67\n"
    "s_add_u32 s67, s67, s55\n"
    "s_bitset1_b64 s[96:97], s67\n"
    "s_add_u32 s67, s67, s55\n"
    "s_add_u32 s67, s67, s55\n"
    "s_bitset1_b64 s[96:97], s67\n"
    "s_add_u32 s67, s67, s55\n"
    "s_add_u32 s67, s67, s55\n"
    "s_bitset1_b64 s[96:97], s67\n"
    "v_cndmask_b32 v13, 0, v12, vcc\n"
    "Lnow0%=:\n"
    "s_lshl_b32 s90, s55, 3\n"
    "s_add_u32 s51, s51, s90\n"
    "s_add_u32 s52, s52, 1\n"
    "s_cmp_eq_u32 s48, 0xff\n"
    "s_cbranch_scc1 Lalldone%=\n"
    "s_sub_u32 s67, s51, 6\n"
    "s_bitset1_b64 s[98:99], s67\n"
    "Lalldone%=:\n"
    "s_or_b64 s[64:65], s[96:97], s[98:99]\n"
    "s_lshr_b64 s[64:65], s[64:65], 1\n"
    "s_or_b64 s[64:65], s[64:65], s[28:29]\n"
    "s_lshl_b64 s[62:63], s[98:99], 1\n"
    "s_or_b64 s[62:63], s[62:63], s[98:99]\n"
    "s_or_b64 s[60:61], s[62:63], s[96:97]\n"
    "Lnoapp%=:\n"
    "s_bitcmp1_b64 s[98:99], s79\n"
    "s_cbranch_scc0 Lnodec%=\n"
    "s_memrealtime s[90:91]\n"
    "Ldec%=:\n"
    "v_cmp_le_u32 vcc, s66, v8\n"
    "s_nop 4\n"
    "s_and_b32 s67, vcc_lo, 0xff\n"
    "s_andn2_b32 s53, s67, s48\n"
    "s_cbranch_scc1 Lcand%=\n"
    "s_mov_b32 s80, 2147483647\n"
    "global_load_dword v8, v9, s[18:19] sc0 sc1\n"
    "s_waitcnt vmcnt(0) lgkmcnt(0)\n"
    "v_cmp_le_u32 vcc, s66, v8\n"
    "s_nop 4\n"
    "s_and_b32 s67, vcc_lo, 0xff\n"
    "s_andn2_b32 s53, s67, s48\n"
    "s_cbranch_scc1 Lcand%=\n"
    "v_mov_b32 v35, 66688\n"
    "global_load_dword v35, v35, s[18:19] sc1\n"
    "s_memrealtime s[92:93]\n"
    "s_waitcnt vmcnt(0) lgkmcnt(0)\n"
    "v_readfirstlane_b32 s67, v35\n"
    "s_sub_u32 s77, s92, s90\n"
    "s_nop 2\n"
    "s_cmp_eq_u32 s67, s66\n"
    "s_cbranch_scc1 Lcandall%=\n"
    "s_cmp_gt_u32 s77, 40000000\n"
    "s_cbranch_scc1 Lgiveupb%=\n"
    "s_sleep 2\n"
    "s_branch Ldec%=\n"
    "Lgiveupb%=:\n"
    "v_mov_b32 v35, 66688\n"
    "v_mov_b32 v47, s66\n"
    "s_mov_b64 exec, 1\n"
    "global_store_dword v35, v47, s[18:19] sc1\n"
    "s_mov_b64 exec, -1\n"
    "s_waitcnt vmcnt(0)\n"
    "s_branch Lcandall%=\n"
    "Lcandall%=:\n"
    "s_andn2_b32 s53, 0xff, s48\n"
    "Lcand%=:\n"
    "v_readlane_b32 s54, v45, s52\n"
    "s_nop 3\n"
    "s_lshr_b32 s67, s54, 8\n"
    "s_and_b32 s77, s66, 0xffffff\n"
    "s_cmp_eq_u32 s67, s77\n"
    "s_cbranch_scc0 Lcas%=\n"
    "s_and_b32 s53, s54, 0xff\n"
    "v_mov_b32 v34, s54\n"
    "s_branch Lnodec%=\n"
    "Lcas%=:\n"
    "s_lshl_b32 s67, s77, 8\n"
    "s_or_b32 s67, s67, s53\n"
    "v_mov_b32 v36, s67\n"
    "v_mov_b32 v37, s54\n"
    "s_lshl_b32 s67, s52, 2\n"
    "s_add_u32 s67, s67, s36\n"
    "v_mov_b32 v35, s67\n"
    "s_mov_b64 exec, 1\n"
    "global_atomic_cmpswap v34, v35, v[36:37], s[18:19] sc0\n"
    "s_mov_b64 exec, -1\n"
    "Lnodec%=:\n"
    "s_setpc_b64 s[68:69]\n"
    "Lad2%=:\n"
    "s_getpc_b64 s[68:69]\n"
    "Lga2%=:\n"
    "s_add_u32 s68, s68, Lad2r%=-Lga2%=\n"
    "s_addc_u32 s69, s69, -1\n"
    "s_branch Lad%=\n"
    "Lad3%=:\n"
    "s_getpc_b64 s[68:69]\n"
    "Lga3%=:\n"
    "s_add_u32 s68, s68, Lad3r%=-Lga3%=\n"
    "s_addc_u32 s69, s69, -1\n"
    "s_branch Lad%=\n"
    "Lad4%=:\n"
    "s_getpc_b64 s[68:69]\n"
    "Lga4%=:\n"
    "s_add_u32 s68, s68, Lad4r%=-Lga4%=\n"
    "s_addc_u32 s69, s69, -1\n"
    "s_branch Lad%=\n"
    "Lad5%=:\n"
    "s_getpc_b64 s[68:69]\n"
    "Lga5%=:\n"
    "s_add_u32 s68, s68, Lad5r%=-Lga5%=\n"
    "s_addc_u32 s69, s69, -1\n"
    "s_branch Lad%=\n"
    "Lad6%=:\n"
    "s_getpc_b64 s[68:69]\n"
    "Lga6%=:\n"
    "s_add_u32 s68, s68, Lad6r%=-Lga6%=\n"
    "s_addc_u32 s69, s69, -1\n"
    "s_branch Lad%=\n"
    "Lad7%=:\n"
    "s_getpc_b64 s[68:69]\n"
    "Lga7%=:\n"
    "s_add_u32 s68, s68, Lad7r%=-Lga7%=\n"
    "s_addc_u32 s69, s69, -1\n"
    "s_branch Lad%=\n"
    "Lad102%=:\n"
    "s_getpc_b64 s[68:69]\n"
    "Lga102%=:\n"
    "s_add_u32 s68, s68, Lad102r%=-Lga102%=\n"
    "s_addc_u32 s69, s69, -1\n"
    "s_branch Lad%=\n"
    "Lad103%=:\n"
    "s_getpc_b64 s[68:69]\n"
    "Lga103%=:\n"
    "s_add_u32 s68, s68, Lad103r%=-Lga103%=\n"
    "s_addc_u32 s69, s69, -1\n"
    "s_branch Lad%=\n"
    "Lad104%=:\n"
    "s_getpc_b64 s[68:69]\n"
    "Lga104%=:\n"
    "s_add_u32 s68, s68, Lad104r%=-Lga104%=\n"
    "s_addc_u32 s69, s69, -1\n"
    "s_branch Lad%=\n"
    "Lad105%=:\n"
    "s_getpc_b64 s[68:69]\n"
    "Lga105%=:\n"
    "s_add_u32 s68, s68, Lad105r%=-Lga105%=\n"
    "s_addc_u32 s69, s69, -1\n"
    "s_branch Lad%=\n"
    "Lad106%=:\n"
    "s_getpc_b64 s[68:69]\n"
    "Lga106%=:\n"
    "s_add_u32 s68, s68, Lad106r%=-Lga106%=\n"
    "s_addc_u32 s69, s69, -1\n"
    "s_branch Lad%=\n"
    "Lad107%=:\n"
    "s_getpc_b64 s[68:69]\n"
    "Lga107%=:\n"
    "s_add_u32 s68, s68, Lad107r%=-Lga107%=\n"
    "s_addc_u32 s69, s69, -1\n"
    "s_branch Lad%=\n"
    "Lad%=:\n"
    "s_cmp_lg_u32 s80, 2147483647\n"
    "s_cbranch_scc1 Lax1%=\n"
    "s_mov_b32 s80, 0\n"
    "s_setpc_b64 s[68:69]\n"
    "Lax1%=:\n"
    "s_cmp_eq_u32 s30, 0\n"
    "s_cbranch_scc1 Laxt%=\n"
    "s_cmp_lt_u32 s79, 14\n"
    "s_cbranch_scc1 Laxx%=\n"
    "s_sub_u32 s67, s79, s80\n"
    "s_cmp_eq_u32 s67, 1\n"
    "s_cbranch_scc1 Lax2%=\n"
    "s_mov_b32 s80, s79\n"
    "s_mov_b32 s81, s78\n"
    "s_setpc_b64 s[68:69]\n"
    "Lax2%=:\n"
    "s_add_u32 s81, s81, s78\n"
    "s_lshr_b32 s81, s81, 1\n"
    "s_mov_b32 s41, 1\n"
    "s_mov_b32 s30, 0\n"
    "s_mov_b32 s94, 0\n"
    "s_mov_b32 s95, 0\n"
    "s_setpc_b64 s[68:69]\n"
    "Laxt%=:\n"
    "s_add_u32 s94, s94, 1\n"
    "s_cmp_le_u32 s94, 3\n"
    "s_cbranch_scc1 Laxx%=\n"
    "s_add_u32 s95, s95, s78\n"
    "s_cmp_lt_u32 s94, 6\n"
    "s_cbranch_scc1 Laxx%=\n"
    "s_mul_i32 s67, s81, 3\n"
    "s_add_u32 s95, s95, 900\n"
    "s_cmp_gt_u32 s95, s67\n"
    "s_cselect_b32 s41, 0, 1\n"
    "s_mov_b32 s30, -1\n"
    "Laxx%=:\n"
    "s_setpc_b64 s[68:69]\n"
    ".p2align 6\n"
    "Lend%=:\n"
    ".p2align 3\n"
    "v_pk_mul_f32 v[0:1], v[96:97], v[96:97]\n"
    "v_pk_mul_f32 v[2:3], v[98:99], v[98:99]\n"
    "v_pk_mul_f32 v[4:5], v[100:101], v[100:101]\n"
    "v_pk_mul_f32 v[6:7], v[102:103], v[102:103]\n"
    "v_pk_mul_f32 v[8:9], v[104:105], v[104:105]\n"
    "v_pk_mul_f32 v[10:11], v[106:107], v[106:107]\n"
    "v_pk_mul_f32 v[12:13], v[108:109], v[108:109]\n"
    "v_pk_mul_f32 v[14:15], v[110:111], v[110:111]\n"
    "v_pk_fma_f32 v[0:1], v[0:1], s[90:91], v[62:63]\n"
    "v_pk_fma_f32 v[2:3], v[2:3], s[90:91], v[62:63]\n"
    "v_pk_fma_f32 v[4:5], v[4:5], s[90:91], v[62:63]\n"
    "v_pk_fma_f32 v[6:7], v[6:7], s[90:91], v[62:63]\n"
    "v_pk_fma_f32 v[8:9], v[8:9], s[90:91], v[62:63]\n"
    "v_pk_fma_f32 v[10:11], v[10:11], s[90:91], v[62:63]\n"
    "v_pk_fma_f32 v[12:13], v[12:13], s[90:91], v[62:63]\n"
    "v_pk_fma_f32 v[14:15], v[14:15], s[90:91], v[62:63]\n"
    "v_pk_mul_f32 v[0:1], v[96:97], v[0:1]\n"
    "v_pk_mul_f32 v[2:3], v[98:99], v[2:3]\n"
    "v_pk_mul_f32 v[4:5], v[100:101], v[4:5]\n"
    "v_pk_mul_f32 v[6:7], v[102:103], v[6:7]\n"
    "v_pk_mul_f32 v[8:9], v[104:105], v[8:9]\n"
    "v_pk_mul_f32 v[10:11], v[106:107], v[10:11]\n"
    "v_pk_mul_f32 v[12:13], v[108:109], v[12:13]\n"
    "v_pk_mul_f32 v[14:15], v[110:111], v[14:15]\n"
    "v_exp_f32 v0, v0\n"
    "v_exp_f32 v1, v1\n"
    "v_exp_f32 v2, v2\n"
    "v_exp_f32 v3, v3\n"
    "v_exp_f32 v4, v4\n"
    "v_exp_f32 v5, v5\n"
    "v_exp_f32 v6, v6\n"
    "v_exp_f32 v7, v7\n"
    "v_exp_f32 v8, v8\n"
    "v_exp_f32 v9, v9\n"
    "v_exp_f32 v10, v10\n"
    "v_exp_f32 v11, v11\n"
    "v_exp_f32 v12, v12\n"
    "v_exp_f32 v13, v13\n"
    "v_exp_f32 v14, v14\n"
    "v_exp_f32 v15, v15\n"
    "v_pk_add_f32 v[0:1], v[0:1], 1.0 op_sel_hi:[1,0]\n"
    "v_pk_add_f32 v[2:3], v[2:3], 1.0 op_sel_hi:[1,0]\n"
    "v_pk_add_f32 v[4:5], v[4:5], 1.0 op_sel_hi:[1,0]\n"
    "v_pk_add_f32 v[6:7], v[6:7], 1.0 op_sel_hi:[1,0]\n"
    "v_pk_add_f32 v[8:9], v[8:9], 1.0 op_sel_hi:[1,0]\n"
    "v_pk_add_f32 v[10:11], v[10:11], 1.0 op_sel_hi:[1,0]\n"
    "v_pk_add_f32 v[12:13], v[12:13], 1.0 op_sel_hi:[1,0]\n"
    "v_pk_add_f32 v[14:15], v[14:15], 1.0 op_sel_hi:[1,0]\n"
    "v_rcp_f32 v0, v0\n"
    "v_rcp_f32 v1, v1\n"
    "v_rcp_f32 v2, v2\n"
    "v_rcp_f32 v3, v3\n"
    "v_rcp_f32 v4, v4\n"
    "v_rcp_f32 v5, v5\n"
    "v_rcp_f32 v6, v6\n"
    "v_rcp_f32 v7, v7\n"
    "v_rcp_f32 v8, v8\n"
    "v_rcp_f32 v9, v9\n"
    "v_rcp_f32 v10, v10\n"
    "v_rcp_f32 v11, v11\n"
    "v_rcp_f32 v12, v12\n"
    "v_rcp_f32 v13, v13\n"
    "v_rcp_f32 v14, v14\n"
    "v_rcp_f32 v15, v15\n"
    "v_pk_mul_f32 v[0:1], v[96:97], v[0:1]\n"
    "v_pk_mul_f32 v[2:3], v[98:99], v[2:3]\n"
    "v_pk_mul_f32 v[4:5], v[100:101], v[4:5]\n"
    "v_pk_mul_f32 v[6:7], v[102:103], v[6:7]\n"
    "v_pk_mul_f32 v[8:9], v[104:105], v[8:9]\n"
    "v_pk_mul_f32 v[10:11], v[106:107], v[10:11]\n"
    "v_pk_mul_f32 v[12:13], v[108:109], v[12:13]\n"
    "v_pk_mul_f32 v[14:15], v[110:111], v[14:15]\n"
    "v_cvt_pk_bf16_f32 v52, v0, v1\n"
    "v_cvt_pk_bf16_f32 v53, v2, v3\n"
    "v_cvt_pk_bf16_f32 v54, v4, v5\n"
    "v_cvt_pk_bf16_f32 v55, v6, v7\n"
    "v_cvt_pk_bf16_f32 v56, v8, v9\n"
    "v_cvt_pk_bf16_f32 v57, v10, v11\n"
    "v_cvt_pk_bf16_f32 v58, v12, v13\n"
    "v_cvt_pk_bf16_f32 v59, v14, v15\n"
    "v_cndmask_b32 v44, v52, v54, vcc\n"
    "v_cndmask_b32 v45, v53, v55, vcc\n"
    "v_cndmask_b32 v46, v54, v52, vcc\n"
    "v_cndmask_b32 v47, v55, v53, vcc\n"
    "v_cndmask_b32 v48, v56, v58, vcc\n"
    "v_cndmask_b32 v49, v57, v59, vcc\n"
    "v_cndmask_b32 v50, v58, v56, vcc\n"
    "v_cndmask_b32 v51, v59, v57, vcc\n"
    "buffer_store_dwordx4 v[44:47], v39, s[12:15], s72 offen offset:0 \n"
    "buffer_store_dwordx4 v[48:51], v39, s[12:15], s72 offen offset:64 \n"
    "v_pk_mul_f32 v[0:1], v[112:113], v[112:113]\n"
    "v_pk_mul_f32 v[2:3], v[114:115], v[114:115]\n"
    "v_pk_mul_f32 v[4:5], v[116:117], v[116:117]\n"
    "v_pk_mul_f32 v[6:7], v[118:119], v[118:119]\n"
    "v_pk_mul_f32 v[8:9], v[120:121], v[120:121]\n"
    "v_pk_mul_f32 v[10:11], v[122:123], v[122:123]\n"
    "v_pk_mul_f32 v[12:13], v[124:125], v[124:125]\n"
    "v_pk_mul_f32 v[14:15], v[126:127], v[126:127]\n"
    "v_pk_fma_f32 v[0:1], v[0:1], s[90:91], v[62:63]\n"
    "v_pk_fma_f32 v[2:3], v[2:3], s[90:91], v[62:63]\n"
    "v_pk_fma_f32 v[4:5], v[4:5], s[90:91], v[62:63]\n"
    "v_pk_fma_f32 v[6:7], v[6:7], s[90:91], v[62:63]\n"
    "v_pk_fma_f32 v[8:9], v[8:9], s[90:91], v[62:63]\n"
    "v_pk_fma_f32 v[10:11], v[10:11], s[90:91], v[62:63]\n"
    "v_pk_fma_f32 v[12:13], v[12:13], s[90:91], v[62:63]\n"
    "v_pk_fma_f32 v[14:15], v[14:15], s[90:91], v[62:63]\n"
    "v_pk_mul_f32 v[0:1], v[112:113], v[0:1]\n"
    "v_pk_mul_f32 v[2:3], v[114:115], v[2:3]\n"
    "v_pk_mul_f32 v[4:5], v[116:117], v[4:5]\n"
    "v_pk_mul_f32 v[6:7], v[118:119], v[6:7]\n"
    "v_pk_mul_f32 v[8:9], v[120:121], v[8:9]\n"
    "v_pk_mul_f32 v[10:11], v[122:123], v[10:11]\n"
    "v_pk_mul_f32 v[12:13], v[124:125], v[12:13]\n"
    "v_pk_mul_f32 v[14:15], v[126:127], v[14:15]\n"
    "v_exp_f32 v0, v0\n"
    "v_exp_f32 v1, v1\n"
    "v_exp_f32 v2, v2\n"
    "v_exp_f32 v3, v3\n"
    "v_exp_f32 v4, v4\n"
    "v_exp_f32 v5, v5\n"
    "v_exp_f32 v6, v6\n"
    "v_exp_f32 v7, v7\n"
    "v_exp_f32 v8, v8\n"
    "v_exp_f32 v9, v9\n"
    "v_exp_f32 v10, v10\n"
    "v_exp_f32 v11, v11\n"
    "v_exp_f32 v12, v12\n"
    "v_exp_f32 v13, v13\n"
    "v_exp_f32 v14, v14\n"
    "v_exp_f32 v15, v15\n"
    "v_pk_add_f32 v[0:1], v[0:1], 1.0 op_sel_hi:[1,0]\n"
    "v_pk_add_f32 v[2:3], v[2:3], 1.0 op_sel_hi:[1,0]\n"
    "v_pk_add_f32 v[4:5], v[4:5], 1.0 op_sel_hi:[1,0]\n"
    "v_pk_add_f32 v[6:7], v[6:7], 1.0 op_sel_hi:[1,0]\n"
    "v_pk_add_f32 v[8:9], v[8:9], 1.0 op_sel_hi:[1,0]\n"
    "v_pk_add_f32 v[10:11], v[10:11], 1.0 op_sel_hi:[1,0]\n"
    "v_pk_add_f32 v[12:13], v[12:13], 1.0 op_sel_hi:[1,0]\n"
    "v_pk_add_f32 v[14:15], v[14:15], 1.0 op_sel_hi:[1,0]\n"
    "v_rcp_f32 v0, v0\n"
    "v_rcp_f32 v1, v1\n"
    "v_rcp_f32 v2, v2\n"
    "v_rcp_f32 v3, v3\n"
    "v_rcp_f32 v4, v4\n"
    "v_rcp_f32 v5, v5\n"
    "v_rcp_f32 v6, v6\n"
    "v_rcp_f32 v7, v7\n"
    "v_rcp_f32 v8, v8\n"
    "v_rcp_f32 v9, v9\n"
    "v_rcp_f32 v10, v10\n"
    "v_rcp_f32 v11, v11\n"
    "v_rcp_f32 v12, v12\n"
    "v_rcp_f32 v13, v13\n"
    "v_rcp_f32 v14, v14\n"
    "v_rcp_f32 v15, v15\n"
    "v_pk_mul_f32 v[0:1], v[112:113], v[0:1]\n"
    "v_pk_mul_f32 v[2:3], v[114:115], v[2:3]\n"
    "v_pk_mul_f32 v[4:5], v[116:117], v[4:5]\n"
    "v_pk_mul_f32 v[6:7], v[118:119], v[6:7]\n"
    "v_pk_mul_f32 v[8:9], v[120:121], v[8:9]\n"
    "v_pk_mul_f32 v[10:11], v[122:123], v[10:11]\n"
    "v_pk_mul_f32 v[12:13], v[124:125], v[12:13]\n"
    "v_pk_mul_f32 v[14:15], v[126:127], v[14:15]\n"
    "v_cvt_pk_bf16_f32 v52, v0, v1\n"
    "v_cvt_pk_bf16_f32 v53, v2, v3\n"
    "v_cvt_pk_bf16_f32 v54, v4, v5\n"
    "v_cvt_pk_bf16_f32 v55, v6, v7\n"
    "v_cvt_pk_bf16_f32 v56, v8, v9\n"
    "v_cvt_pk_bf16_f32 v57, v10, v11\n"
    "v_cvt_pk_bf16_f32 v58, v12, v13\n"
    "v_cvt_pk_bf16_f32 v59, v14, v15\n"
    "v_cndmask_b32 v44, v52, v54, vcc\n"
    "v_cndmask_b32 v45, v53, v55, vcc\n"
    "v_cndmask_b32 v46, v54, v52, vcc\n"
    "v_cndmask_b32 v47, v55, v53, vcc\n"
    "v_cndmask_b32 v48, v56, v58, vcc\n"
    "v_cndmask_b32 v49, v57, v59, vcc\n"
    "v_cndmask_b32 v50, v58, v56, vcc\n"
    "v_cndmask_b32 v51, v59, v57, vcc\n"
    "buffer_store_dwordx4 v[44:47], v39, s[12:15], s72 offen offset:128 \n"
    "buffer_store_dwordx4 v[48:51], v39, s[12:15], s72 offen offset:192 \n"
    "s_add_u32 s72, s72, 0x100000\n"
    "v_pk_mul_f32 v[0:1], v[128:129], v[128:129]\n"
    "v_pk_mul_f32 v[2:3], v[130:131], v[130:131]\n"
    "v_pk_mul_f32 v[4:5], v[132:133], v[132:133]\n"
    "v_pk_mul_f32 v[6:7], v[134:135], v[134:135]\n"
    "v_pk_mul_f32 v[8:9], v[136:137], v[136:137]\n"
    "v_pk_mul_f32 v[10:11], v[138:139], v[138:139]\n"
    "v_pk_mul_f32 v[12:13], v[140:141], v[140:141]\n"
    "v_pk_mul_f32 v[14:15], v[142:143], v[142:143]\n"
    "v_pk_fma_f32 v[0:1], v[0:1], s[90:91], v[62:63]\n"
    "v_pk_fma_f32 v[2:3], v[2:3], s[90:91], v[62:63]\n"
    "v_pk_fma_f32 v[4:5], v[4:5], s[90:91], v[62:63]\n"
    "v_pk_fma_f32 v[6:7], v[6:7], s[90:91], v[62:63]\n"
    "v_pk_fma_f32 v[8:9], v[8:9], s[90:91], v[62:63]\n"
    "v_pk_fma_f32 v[10:11], v[10:11], s[90:91], v[62:63]\n"
    "v_pk_fma_f32 v[12:13], v[12:13], s[90:91], v[62:63]\n"
    "v_pk_fma_f32 v[14:15], v[14:15], s[90:91], v[62:63]\n"
    "v_pk_mul_f32 v[0:1], v[128:129], v[0:1]\n"
    "v_pk_mul_f32 v[2:3], v[130:131], v[2:3]\n"
    "v_pk_mul_f32 v[4:5], v[132:133], v[4:5]\n"
    "v_pk_mul_f32 v[6:7], v[134:135], v[6:7]\n"
    "v_pk_mul_f32 v[8:9], v[136:137], v[8:9]\n"
    "v_pk_mul_f32 v[10:11], v[138:139], v[10:11]\n"
    "v_pk_mul_f32 v[12:13], v[140:141], v[12:13]\n"
    "v_pk_mul_f32 v[14:15], v[142:143], v[14:15]\n"
    "v_exp_f32 v0, v0\n"
    "v_exp_f32 v1, v1\n"
    "v_exp_f32 v2, v2\n"
    "v_exp_f32 v3, v3\n"
    "v_exp_f32 v4, v4\n"
    "v_exp_f32 v5, v5\n"
    "v_exp_f32 v6, v6\n"
    "v_exp_f32 v7, v7\n"
    "v_exp_f32 v8, v8\n"
    "v_exp_f32 v9, v9\n"
    "v_exp_f32 v10, v10\n"
    "v_exp_f32 v11, v11\n"
    "v_exp_f32 v12, v12\n"
    "v_exp_f32 v13, v13\n"
    "v_exp_f32 v14, v14\n"
    "v_exp_f32 v15, v15\n"
    "v_pk_add_f32 v[0:1], v[0:1], 1.0 op_sel_hi:[1,0]\n"
    "v_pk_add_f32 v[2:3], v[2:3], 1.0 op_sel_hi:[1,0]\n"
    "v_pk_add_f32 v[4:5], v[4:5], 1.0 op_sel_hi:[1,0]\n"
    "v_pk_add_f32 v[6:7], v[6:7], 1.0 op_sel_hi:[1,0]\n"
    "v_pk_add_f32 v[8:9], v[8:9], 1.0 op_sel_hi:[1,0]\n"
    "v_pk_add_f32 v[10:11], v[10:11], 1.0 op_sel_hi:[1,0]\n"
    "v_pk_add_f32 v[12:13], v[12:13], 1.0 op_sel_hi:[1,0]\n"
    "v_pk_add_f32 v[14:15], v[14:15], 1.0 op_sel_hi:[1,0]\n"
    "v_rcp_f32 v0, v0\n"
    "v_rcp_f32 v1, v1\n"
    "v_rcp_f32 v2, v2\n"
    "v_rcp_f32 v3, v3\n"
    "v_rcp_f32 v4, v4\n"
    "v_rcp_f32 v5, v5\n"
    "v_rcp_f32 v6, v6\n"
    "v_rcp_f32 v7, v7\n"
    "v_rcp_f32 v8, v8\n"
    "v_rcp_f32 v9, v9\n"
    "v_rcp_f32 v10, v10\n"
    "v_rcp_f32 v11, v11\n"
    "v_rcp_f32 v12, v12\n"
    "v_rcp_f32 v13, v13\n"
    "v_rcp_f32 v14, v14\n"
    "v_rcp_f32 v15, v15\n"
    "v_pk_mul_f32 v[0:1], v[128:129], v[0:1]\n"
    "v_pk_mul_f32 v[2:3], v[130:131], v[2:3]\n"
    "v_pk_mul_f32 v[4:5], v[132:133], v[4:5]\n"
    "v_pk_mul_f32 v[6:7], v[134:135], v[6:7]\n"
    "v_pk_mul_f32 v[8:9], v[136:137], v[8:9]\n"
    "v_pk_mul_f32 v[10:11], v[138:139], v[10:11]\n"
    "v_pk_mul_f32 v[12:13], v[140:141], v[12:13]\n"
    "v_pk_mul_f32 v[14:15], v[142:143], v[14:15]\n"
    "v_cvt_pk_bf16_f32 v52, v0, v1\n"
    "v_cvt_pk_bf16_f32 v53, v2, v3\n"
    "v_cvt_pk_bf16_f32 v54, v4, v5\n"
    "v_cvt_pk_bf16_f32 v55, v6, v7\n"
    "v_cvt_pk_bf16_f32 v56, v8, v9\n"
    "v_cvt_pk_bf16_f32 v57, v10, v11\n"
    "v_cvt_pk_bf16_f32 v58, v12, v13\n"
    "v_cvt_pk_bf16_f32 v59, v14, v15\n"
    "v_cndmask_b32 v44, v52, v54, vcc\n"
    "v_cndmask_b32 v45, v53, v55, vcc\n"
    "v_cndmask_b32 v46, v54, v52, vcc\n"
    "v_cndmask_b32 v47, v55, v53, vcc\n"
    "v_cndmask_b32 v48, v56, v58, vcc\n"
    "v_cndmask_b32 v49, v57, v59, vcc\n"
    "v_cndmask_b32 v50, v58, v56, vcc\n"
    "v_cndmask_b32 v51, v59, v57, vcc\n"
    "buffer_store_dwordx4 v[44:47], v39, s[12:15], s72 offen offset:0 \n"
    "buffer_store_dwordx4 v[48:51], v39, s[12:15], s72 offen offset:64 \n"
    "v_pk_mul_f32 v[0:1], v[144:145], v[144:145]\n"
    "v_pk_mul_f32 v[2:3], v[146:147], v[146:147]\n"
    "v_pk_mul_f32 v[4:5], v[148:149], v[148:149]\n"
    "v_pk_mul_f32 v[6:7], v[150:151], v[150:151]\n"
    "v_pk_mul_f32 v[8:9], v[152:153], v[152:153]\n"
    "v_pk_mul_f32 v[10:11], v[154:155], v[154:155]\n"
    "v_pk_mul_f32 v[12:13], v[156:157], v[156:157]\n"
    "v_pk_mul_f32 v[14:15], v[158:159], v[158:159]\n"
    "v_pk_fma_f32 v[0:1], v[0:1], s[90:91], v[62:63]\n"
    "v_pk_fma_f32 v[2:3], v[2:3], s[90:91], v[62:63]\n"
    "v_pk_fma_f32 v[4:5], v[4:5], s[90:91], v[62:63]\n"
    "v_pk_fma_f32 v[6:7], v[6:7], s[90:91], v[62:63]\n"
    "v_pk_fma_f32 v[8:9], v[8:9], s[90:91], v[62:63]\n"
    "v_pk_fma_f32 v[10:11], v[10:11], s[90:91], v[62:63]\n"
    "v_pk_fma_f32 v[12:13], v[12:13], s[90:91], v[62:63]\n"
    "v_pk_fma_f32 v[14:15], v[14:15], s[90:91], v[62:63]\n"
    "v_pk_mul_f32 v[0:1], v[144:145], v[0:1]\n"
    "v_pk_mul_f32 v[2:3], v[146:147], v[2:3]\n"
    "v_pk_mul_f32 v[4:5], v[148:149], v[4:5]\n"
    "v_pk_mul_f32 v[6:7], v[150:151], v[6:7]\n"
    "v_pk_mul_f32 v[8:9], v[152:153], v[8:9]\n"
    "v_pk_mul_f32 v[10:11], v[154:155], v[10:11]\n"
    "v_pk_mul_f32 v[12:13], v[156:157], v[12:13]\n"
    "v_pk_mul_f32 v[14:15], v[158:159], v[14:15]\n"
    "v_exp_f32 v0, v0\n"
    "v_exp_f32 v1, v1\n"
    "v_exp_f32 v2, v2\n"
    "v_exp_f32 v3, v3\n"
    "v_exp_f32 v4, v4\n"
    "v_exp_f32 v5, v5\n"
    "v_exp_f32 v6, v6\n"
    "v_exp_f32 v7, v7\n"
    "v_exp_f32 v8, v8\n"
    "v_exp_f32 v9, v9\n"
    "v_exp_f32 v10, v10\n"
    "v_exp_f32 v11, v11\n"
    "v_exp_f32 v12, v12\n"
    "v_exp_f32 v13, v13\n"
    "v_exp_f32 v14, v14\n"
    "v_exp_f32 v15, v15\n"
    "v_pk_add_f32 v[0:1], v[0:1], 1.0 op_sel_hi:[1,0]\n"
    "v_pk_add_f32 v[2:3], v[2:3], 1.0 op_sel_hi:[1,0]\n"
    "v_pk_add_f32 v[4:5], v[4:5], 1.0 op_sel_hi:[1,0]\n"
    "v_pk_add_f32 v[6:7], v[6:7], 1.0 op_sel_hi:[1,0]\n"
    "v_pk_add_f32 v[8:9], v[8:9], 1.0 op_sel_hi:[1,0]\n"
    "v_pk_add_f32 v[10:11], v[10:11], 1.0 op_sel_hi:[1,0]\n"
    "v_pk_add_f32 v[12:13], v[12:13], 1.0 op_sel_hi:[1,0]\n"
    "v_pk_add_f32 v[14:15], v[14:15], 1.0 op_sel_hi:[1,0]\n"
    "v_rcp_f32 v0, v0\n"
    "v_rcp_f32 v1, v1\n"
    "v_rcp_f32 v2, v2\n"
    "v_rcp_f32 v3, v3\n"
    "v_rcp_f32 v4, v4\n"
    "v_rcp_f32 v5, v5\n"
    "v_rcp_f32 v6, v6\n"
    "v_rcp_f32 v7, v7\n"
    "v_rcp_f32 v8, v8\n"
    "v_rcp_f32 v9, v9\n"
    "v_rcp_f32 v10, v10\n"
    "v_rcp_f32 v11, v11\n"
    "v_rcp_f32 v12, v12\n"
    "v_rcp_f32 v13, v13\n"
    "v_rcp_f32 v14, v14\n"
    "v_rcp_f32 v15, v15\n"
    "v_pk_mul_f32 v[0:1], v[144:145], v[0:1]\n"
    "v_pk_mul_f32 v[2:3], v[146:147], v[2:3]\n"
    "v_pk_mul_f32 v[4:5], v[148:149], v[4:5]\n"
    "v_pk_mul_f32 v[6:7], v[150:151], v[6:7]\n"
    "v_pk_mul_f32 v[8:9], v[152:153], v[8:9]\n"
    "v_pk_mul_f32 v[10:11], v[154:155], v[10:11]\n"
    "v_pk_mul_f32 v[12:13], v[156:157], v[12:13]\n"
    "v_pk_mul_f32 v[14:15], v[158:159], v[14:15]\n"
    "v_cvt_pk_bf16_f32 v52, v0, v1\n"
    "v_cvt_pk_bf16_f32 v53, v2, v3\n"
    "v_cvt_pk_bf16_f32 v54, v4, v5\n"
    "v_cvt_pk_bf16_f32 v55, v6, v7\n"
    "v_cvt_pk_bf16_f32 v56, v8, v9\n"
    "v_cvt_pk_bf16_f32 v57, v10, v11\n"
    "v_cvt_pk_bf16_f32 v58, v12, v13\n"
    "v_cvt_pk_bf16_f32 v59, v14, v15\n"
    "v_cndmask_b32 v44, v52, v54, vcc\n"
    "v_cndmask_b32 v45, v53, v55, vcc\n"
    "v_cndmask_b32 v46, v54, v52, vcc\n"
    "v_cndmask_b32 v47, v55, v53, vcc\n"
    "v_cndmask_b32 v48, v56, v58, vcc\n"
    "v_cndmask_b32 v49, v57, v59, vcc\n"
    "v_cndmask_b32 v50, v58, v56, vcc\n"
    "v_cndmask_b32 v51, v59, v57, vcc\n"
    "buffer_store_dwordx4 v[44:47], v39, s[12:15], s72 offen offset:128 \n"
    "buffer_store_dwordx4 v[48:51], v39, s[12:15], s72 offen offset:192 \n"
    "s_add_u32 s72, s72, 0x100000\n"
    "v_pk_mul_f32 v[0:1], v[160:161], v[160:161]\n"
    "v_pk_mul_f32 v[2:3], v[162:163], v[162:163]\n"
    "v_pk_mul_f32 v[4:5], v[164:165], v[164:165]\n"
    "v_pk_mul_f32 v[6:7], v[166:167], v[166:167]\n"
    "v_pk_mul_f32 v[8:9], v[168:169], v[168:169]\n"
    "v_pk_mul_f32 v[10:11], v[170:171], v[170:171]\n"
    "v_pk_mul_f32 v[12:13], v[172:173], v[172:173]\n"
    "v_pk_mul_f32 v[14:15], v[174:175], v[174:175]\n"
    "v_pk_fma_f32 v[0:1], v[0:1], s[90:91], v[62:63]\n"
    "v_pk_fma_f32 v[2:3], v[2:3], s[90:91], v[62:63]\n"
    "v_pk_fma_f32 v[4:5], v[4:5], s[90:91], v[62:63]\n"
    "v_pk_fma_f32 v[6:7], v[6:7], s[90:91], v[62:63]\n"
    "v_pk_fma_f32 v[8:9], v[8:9], s[90:91], v[62:63]\n"
    "v_pk_fma_f32 v[10:11], v[10:11], s[90:91], v[62:63]\n"
    "v_pk_fma_f32 v[12:13], v[12:13], s[90:91], v[62:63]\n"
    "v_pk_fma_f32 v[14:15], v[14:15], s[90:91], v[62:63]\n"
    "v_pk_mul_f32 v[0:1], v[160:161], v[0:1]\n"
    "v_pk_mul_f32 v[2:3], v[162:163], v[2:3]\n"
    "v_pk_mul_f32 v[4:5], v[164:165], v[4:5]\n"
    "v_pk_mul_f32 v[6:7], v[166:167], v[6:7]\n"
    "v_pk_mul_f32 v[8:9], v[168:169], v[8:9]\n"
    "v_pk_mul_f32 v[10:11], v[170:171], v[10:11]\n"
    "v_pk_mul_f32 v[12:13], v[172:173], v[12:13]\n"
    "v_pk_mul_f32 v[14:15], v[174:175], v[14:15]\n"
    "v_exp_f32 v0, v0\n"
    "v_exp_f32 v1, v1\n"
    "v_exp_f32 v2, v2\n"
    "v_exp_f32 v3, v3\n"
    "v_exp_f32 v4, v4\n"
    "v_exp_f32 v5, v5\n"
    "v_exp_f32 v6, v6\n"
    "v_exp_f32 v7, v7\n"
    "v_exp_f32 v8, v8\n"
    "v_exp_f32 v9, v9\n"
    "v_exp_f32 v10, v10\n"
    "v_exp_f32 v11, v11\n"
    "v_exp_f32 v12, v12\n"
    "v_exp_f32 v13, v13\n"
    "v_exp_f32 v14, v14\n"
    "v_exp_f32 v15, v15\n"
    "v_pk_add_f32 v[0:1], v[0:1], 1.0 op_sel_hi:[1,0]\n"
    "v_pk_add_f32 v[2:3], v[2:3], 1.0 op_sel_hi:[1,0]\n"
    "v_pk_add_f32 v[4:5], v[4:5], 1.0 op_sel_hi:[1,0]\n"
    "v_pk_add_f32 v[6:7], v[6:7], 1.0 op_sel_hi:[1,0]\n"
    "v_pk_add_f32 v[8:9], v[8:9], 1.0 op_sel_hi:[1,0]\n"
    "v_pk_add_f32 v[10:11], v[10:11], 1.0 op_sel_hi:[1,0]\n"
    "v_pk_add_f32 v[12:13], v[12:13], 1.0 op_sel_hi:[1,0]\n"
    "v_pk_add_f32 v[14:15], v[14:15], 1.0 op_sel_hi:[1,0]\n"
    "v_rcp_f32 v0, v0\n"
    "v_rcp_f32 v1, v1\n"
    "v_rcp_f32 v2, v2\n"
    "v_rcp_f32 v3, v3\n"
    "v_rcp_f32 v4, v4\n"
    "v_rcp_f32 v5, v5\n"
    "v_rcp_f32 v6, v6\n"
    "v_rcp_f32 v7, v7\n"
    "v_rcp_f32 v8, v8\n"
    "v_rcp_f32 v9, v9\n"
    "v_rcp_f32 v10, v10\n"
    "v_rcp_f32 v11, v11\n"
    "v_rcp_f32 v12, v12\n"
    "v_rcp_f32 v13, v13\n"
    "v_rcp_f32 v14, v14\n"
    "v_rcp_f32 v15, v15\n"
    "v_pk_mul_f32 v[0:1], v[160:161], v[0:1]\n"
    "v_pk_mul_f32 v[2:3], v[162:163], v[2:3]\n"
    "v_pk_mul_f32 v[4:5], v[164:165], v[4:5]\n"
    "v_pk_mul_f32 v[6:7], v[166:167], v[6:7]\n"
    "v_pk_mul_f32 v[8:9], v[168:169], v[8:9]\n"
    "v_pk_mul_f32 v[10:11], v[170:171], v[10:11]\n"
    "v_pk_mul_f32 v[12:13], v[172:173], v[12:13]\n"
    "v_pk_mul_f32 v[14:15], v[174:175], v[14:15]\n"
    "v_cvt_pk_bf16_f32 v52, v0, v1\n"
    "v_cvt_pk_bf16_f32 v53, v2, v3\n"
    "v_cvt_pk_bf16_f32 v54, v4, v5\n"
    "v_cvt_pk_bf16_f32 v55, v6, v7\n"
    "v_cvt_pk_bf16_f32 v56, v8, v9\n"
    "v_cvt_pk_bf16_f32 v57, v10, v11\n"
    "v_cvt_pk_bf16_f32 v58, v12, v13\n"
    "v_cvt_pk_bf16_f32 v59, v14, v15\n"
    "v_cndmask_b32 v44, v52, v54, vcc\n"
    "v_cndmask_b32 v45, v53, v55, vcc\n"
    "v_cndmask_b32 v46, v54, v52, vcc\n"
    "v_cndmask_b32 v47, v55, v53, vcc\n"
    "v_cndmask_b32 v48, v56, v58, vcc\n"
    "v_cndmask_b32 v49, v57, v59, vcc\n"
    "v_cndmask_b32 v50, v58, v56, vcc\n"
    "v_cndmask_b32 v51, v59, v57, vcc\n"
    "buffer_store_dwordx4 v[44:47], v39, s[12:15], s72 offen offset:0 \n"
    "buffer_store_dwordx4 v[48:51], v39, s[12:15], s72 offen offset:64 \n"
    "v_pk_mul_f32 v[0:1], v[176:177], v[176:177]\n"
    "v_pk_mul_f32 v[2:3], v[178:179], v[178:179]\n"
    "v_pk_mul_f32 v[4:5], v[180:181], v[180:181]\n"
    "v_pk_mul_f32 v[6:7], v[182:183], v[182:183]\n"
    "v_pk_mul_f32 v[8:9], v[184:185], v[184:185]\n"
    "v_pk_mul_f32 v[10:11], v[186:187], v[186:187]\n"
    "v_pk_mul_f32 v[12:13], v[188:189], v[188:189]\n"
    "v_pk_mul_f32 v[14:15], v[190:191], v[190:191]\n"
    "v_pk_fma_f32 v[0:1], v[0:1], s[90:91], v[62:63]\n"
    "v_pk_fma_f32 v[2:3], v[2:3], s[90:91], v[62:63]\n"
    "v_pk_fma_f32 v[4:5], v[4:5], s[90:91], v[62:63]\n"
    "v_pk_fma_f32 v[6:7], v[6:7], s[90:91], v[62:63]\n"
    "v_pk_fma_f32 v[8:9], v[8:9], s[90:91], v[62:63]\n"
    "v_pk_fma_f32 v[10:11], v[10:11], s[90:91], v[62:63]\n"
    "v_pk_fma_f32 v[12:13], v[12:13], s[90:91], v[62:63]\n"
    "v_pk_fma_f32 v[14:15], v[14:15], s[90:91], v[62:63]\n"
    "v_pk_mul_f32 v[0:1], v[176:177], v[0:1]\n"
    "v_pk_mul_f32 v[2:3], v[178:179], v[2:3]\n"
    "v_pk_mul_f32 v[4:5], v[180:181], v[4:5]\n"
    "v_pk_mul_f32 v[6:7], v[182:183], v[6:7]\n"
    "v_pk_mul_f32 v[8:9], v[184:185], v[8:9]\n"
    "v_pk_mul_f32 v[10:11], v[186:187], v[10:11]\n"
    "v_pk_mul_f32 v[12:13], v[188:189], v[12:13]\n"
    "v_pk_mul_f32 v[14:15], v[190:191], v[14:15]\n"
    "v_exp_f32 v0, v0\n"
    "v_exp_f32 v1, v1\n"
    "v_exp_f32 v2, v2\n"
    "v_exp_f32 v3, v3\n"
    "v_exp_f32 v4, v4\n"
    "v_exp_f32 v5, v5\n"
    "v_exp_f32 v6, v6\n"
    "v_exp_f32 v7, v7\n"
    "v_exp_f32 v8, v8\n"
    "v_exp_f32 v9, v9\n"
    "v_exp_f32 v10, v10\n"
    "v_exp_f32 v11, v11\n"
    "v_exp_f32 v12, v12\n"
    "v_exp_f32 v13, v13\n"
    "v_exp_f32 v14, v14\n"
    "v_exp_f32 v15, v15\n"
    "v_pk_add_f32 v[0:1], v[0:1], 1.0 op_sel_hi:[1,0]\n"
    "v_pk_add_f32 v[2:3], v[2:3], 1.0 op_sel_hi:[1,0]\n"
    "v_pk_add_f32 v[4:5], v[4:5], 1.0 op_sel_hi:[1,0]\n"
    "v_pk_add_f32 v[6:7], v[6:7], 1.0 op_sel_hi:[1,0]\n"
    "v_pk_add_f32 v[8:9], v[8:9], 1.0 op_sel_hi:[1,0]\n"
    "v_pk_add_f32 v[10:11], v[10:11], 1.0 op_sel_hi:[1,0]\n"
    "v_pk_add_f32 v[12:13], v[12:13], 1.0 op_sel_hi:[1,0]\n"
    "v_pk_add_f32 v[14:15], v[14:15], 1.0 op_sel_hi:[1,0]\n"
    "v_rcp_f32 v0, v0\n"
    "v_rcp_f32 v1, v1\n"
    "v_rcp_f32 v2, v2\n"
    "v_rcp_f32 v3, v3\n"
    "v_rcp_f32 v4, v4\n"
    "v_rcp_f32 v5, v5\n"
    "v_rcp_f32 v6, v6\n"
    "v_rcp_f32 v7, v7\n"
    "v_rcp_f32 v8, v8\n"
    "v_rcp_f32 v9, v9\n"
    "v_rcp_f32 v10, v10\n"
    "v_rcp_f32 v11, v11\n"
    "v_rcp_f32 v12, v12\n"
    "v_rcp_f32 v13, v13\n"
    "v_rcp_f32 v14, v14\n"
    "v_rcp_f32 v15, v15\n"
    "v_pk_mul_f32 v[0:1], v[176:177], v[0:1]\n"
    "v_pk_mul_f32 v[2:3], v[178:179], v[2:3]\n"
    "v_pk_mul_f32 v[4:5], v[180:181], v[4:5]\n"
    "v_pk_mul_f32 v[6:7], v[182:183], v[6:7]\n"
    "v_pk_mul_f32 v[8:9], v[184:185], v[8:9]\n"
    "v_pk_mul_f32 v[10:11], v[186:187], v[10:11]\n"
    "v_pk_mul_f32 v[12:13], v[188:189], v[12:13]\n"
    "v_pk_mul_f32 v[14:15], v[190:191], v[14:15]\n"
    "v_cvt_pk_bf16_f32 v52, v0, v1\n"
    "v_cvt_pk_bf16_f32 v53, v2, v3\n"
    "v_cvt_pk_bf16_f32 v54, v4, v5\n"
    "v_cvt_pk_bf16_f32 v55, v6, v7\n"
    "v_cvt_pk_bf16_f32 v56, v8, v9\n"
    "v_cvt_pk_bf16_f32 v57, v10, v11\n"
    "v_cvt_pk_bf16_f32 v58, v12, v13\n"
    "v_cvt_pk_bf16_f32 v59, v14, v15\n"
    "v_cndmask_b32 v44, v52, v54, vcc\n"
    "v_cndmask_b32 v45, v53, v55, vcc\n"
    "v_cndmask_b32 v46, v54, v52, vcc\n"
    "v_cndmask_b32 v47, v55, v53, vcc\n"
    "v_cndmask_b32 v48, v56, v58, vcc\n"
    "v_cndmask_b32 v49, v57, v59, vcc\n"
    "v_cndmask_b32 v50, v58, v56, vcc\n"
    "v_cndmask_b32 v51, v59, v57, vcc\n"
    "buffer_store_dwordx4 v[44:47], v39, s[12:15], s72 offen offset:128 \n"
    "buffer_store_dwordx4 v[48:51], v39, s[12:15], s72 offen offset:192 \n"
    "s_add_u32 s72, s72, 0x100000\n"
    "v_pk_mul_f32 v[0:1], v[192:193], v[192:193]\n"
    "v_pk_mul_f32 v[2:3], v[194:195], v[194:195]\n"
    "v_pk_mul_f32 v[4:5], v[196:197], v[196:197]\n"
    "v_pk_mul_f32 v[6:7], v[198:199], v[198:199]\n"
    "v_pk_mul_f32 v[8:9], v[200:201], v[200:201]\n"
    "v_pk_mul_f32 v[10:11], v[202:203], v[202:203]\n"
    "v_pk_mul_f32 v[12:13], v[204:205], v[204:205]\n"
    "v_pk_mul_f32 v[14:15], v[206:207], v[206:207]\n"
    "v_pk_fma_f32 v[0:1], v[0:1], s[90:91], v[62:63]\n"
    "v_pk_fma_f32 v[2:3], v[2:3], s[90:91], v[62:63]\n"
    "v_pk_fma_f32 v[4:5], v[4:5], s[90:91], v[62:63]\n"
    "v_pk_fma_f32 v[6:7], v[6:7], s[90:91], v[62:63]\n"
    "v_pk_fma_f32 v[8:9], v[8:9], s[90:91], v[62:63]\n"
    "v_pk_fma_f32 v[10:11], v[10:11], s[90:91], v[62:63]\n"
    "v_pk_fma_f32 v[12:13], v[12:13], s[90:91], v[62:63]\n"
    "v_pk_fma_f32 v[14:15], v[14:15], s[90:91], v[62:63]\n"
    "v_pk_mul_f32 v[0:1], v[192:193], v[0:1]\n"
    "v_pk_mul_f32 v[2:3], v[194:195], v[2:3]\n"
    "v_pk_mul_f32 v[4:5], v[196:197], v[4:5]\n"
    "v_pk_mul_f32 v[6:7], v[198:199], v[6:7]\n"
    "v_pk_mul_f32 v[8:9], v[200:201], v[8:9]\n"
    "v_pk_mul_f32 v[10:11], v[202:203], v[10:11]\n"
    "v_pk_mul_f32 v[12:13], v[204:205], v[12:13]\n"
    "v_pk_mul_f32 v[14:15], v[206:207], v[14:15]\n"
    "v_exp_f32 v0, v0\n"
    "v_exp_f32 v1, v1\n"
    "v_exp_f32 v2, v2\n"
    "v_exp_f32 v3, v3\n"
    "v_exp_f32 v4, v4\n"
    "v_exp_f32 v5, v5\n"
    "v_exp_f32 v6, v6\n"
    "v_exp_f32 v7, v7\n"
    "v_exp_f32 v8, v8\n"
    "v_exp_f32 v9, v9\n"
    "v_exp_f32 v10, v10\n"
    "v_exp_f32 v11, v11\n"
    "v_exp_f32 v12, v12\n"
    "v_exp_f32 v13, v13\n"
    "v_exp_f32 v14, v14\n"
    "v_exp_f32 v15, v15\n"
    "v_pk_add_f32 v[0:1], v[0:1], 1.0 op_sel_hi:[1,0]\n"
    "v_pk_add_f32 v[2:3], v[2:3], 1.0 op_sel_hi:[1,0]\n"
    "v_pk_add_f32 v[4:5], v[4:5], 1.0 op_sel_hi:[1,0]\n"
    "v_pk_add_f32 v[6:7], v[6:7], 1.0 op_sel_hi:[1,0]\n"
    "v_pk_add_f32 v[8:9], v[8:9], 1.0 op_sel_hi:[1,0]\n"
    "v_pk_add_f32 v[10:11], v[10:11], 1.0 op_sel_hi:[1,0]\n"
    "v_pk_add_f32 v[12:13], v[12:13], 1.0 op_sel_hi:[1,0]\n"
    "v_pk_add_f32 v[14:15], v[14:15], 1.0 op_sel_hi:[1,0]\n"
    "v_rcp_f32 v0, v0\n"
    "v_rcp_f32 v1, v1\n"
    "v_rcp_f32 v2, v2\n"
    "v_rcp_f32 v3, v3\n"
    "v_rcp_f32 v4, v4\n"
    "v_rcp_f32 v5, v5\n"
    "v_rcp_f32 v6, v6\n"
    "v_rcp_f32 v7, v7\n"
    "v_rcp_f32 v8, v8\n"
    "v_rcp_f32 v9, v9\n"
    "v_rcp_f32 v10, v10\n"
    "v_rcp_f32 v11, v11\n"
    "v_rcp_f32 v12, v12\n"
    "v_rcp_f32 v13, v13\n"
    "v_rcp_f32 v14, v14\n"
    "v_rcp_f32 v15, v15\n"
    "v_pk_mul_f32 v[0:1], v[192:193], v[0:1]\n"
    "v_pk_mul_f32 v[2:3], v[194:195], v[2:3]\n"
    "v_pk_mul_f32 v[4:5], v[196:197], v[4:5]\n"
    "v_pk_mul_f32 v[6:7], v[198:199], v[6:7]\n"
    "v_pk_mul_f32 v[8:9], v[200:201], v[8:9]\n"
    "v_pk_mul_f32 v[10:11], v[202:203], v[10:11]\n"
    "v_pk_mul_f32 v[12:13], v[204:205], v[12:13]\n"
    "v_pk_mul_f32 v[14:15], v[206:207], v[14:15]\n"
    "v_cvt_pk_bf16_f32 v52, v0, v1\n"
    "v_cvt_pk_bf16_f32 v53, v2, v3\n"
    "v_cvt_pk_bf16_f32 v54, v4, v5\n"
    "v_cvt_pk_bf16_f32 v55, v6, v7\n"
    "v_cvt_pk_bf16_f32 v56, v8, v9\n"
    "v_cvt_pk_bf16_f32 v57, v10, v11\n"
    "v_cvt_pk_bf16_f32 v58, v12, v13\n"
    "v_cvt_pk_bf16_f32 v59, v14, v15\n"
    "v_cndmask_b32 v44, v52, v54, vcc\n"
    "v_cndmask_b32 v45, v53, v55, vcc\n"
    "v_cndmask_b32 v46, v54, v52, vcc\n"
    "v_cndmask_b32 v47, v55, v53, vcc\n"
    "v_cndmask_b32 v48, v56, v58, vcc\n"
    "v_cndmask_b32 v49, v57, v59, vcc\n"
    "v_cndmask_b32 v50, v58, v56, vcc\n"
    "v_cndmask_b32 v51, v59, v57, vcc\n"
    "buffer_store_dwordx4 v[44:47], v39, s[12:15], s72 offen offset:0 \n"
    "buffer_store_dwordx4 v[48:51], v39, s[12:15], s72 offen offset:64 \n"
    "v_pk_mul_f32 v[0:1], v[208:209], v[208:209]\n"
    "v_pk_mul_f32 v[2:3], v[210:211], v[210:211]\n"
    "v_pk_mul_f32 v[4:5], v[212:213], v[212:213]\n"
    "v_pk_mul_f32 v[6:7], v[214:215], v[214:215]\n"
    "v_pk_mul_f32 v[8:9], v[216:217], v[216:217]\n"
    "v_pk_mul_f32 v[10:11], v[218:219], v[218:219]\n"
    "v_pk_mul_f32 v[12:13], v[220:221], v[220:221]\n"
    "v_pk_mul_f32 v[14:15], v[222:223], v[222:223]\n"
    "v_pk_fma_f32 v[0:1], v[0:1], s[90:91], v[62:63]\n"
    "v_pk_fma_f32 v[2:3], v[2:3], s[90:91], v[62:63]\n"
    "v_pk_fma_f32 v[4:5], v[4:5], s[90:91], v[62:63]\n"
    "v_pk_fma_f32 v[6:7], v[6:7], s[90:91], v[62:63]\n"
    "v_pk_fma_f32 v[8:9], v[8:9], s[90:91], v[62:63]\n"
    "v_pk_fma_f32 v[10:11], v[10:11], s[90:91], v[62:63]\n"
    "v_pk_fma_f32 v[12:13], v[12:13], s[90:91], v[62:63]\n"
    "v_pk_fma_f32 v[14:15], v[14:15], s[90:91], v[62:63]\n"
    "v_pk_mul_f32 v[0:1], v[208:209], v[0:1]\n"
    "v_pk_mul_f32 v[2:3], v[210:211], v[2:3]\n"
    "v_pk_mul_f32 v[4:5], v[212:213], v[4:5]\n"
    "v_pk_mul_f32 v[6:7], v[214:215], v[6:7]\n"
    "v_pk_mul_f32 v[8:9], v[216:217], v[8:9]\n"
    "v_pk_mul_f32 v[10:11], v[218:219], v[10:11]\n"
    "v_pk_mul_f32 v[12:13], v[220:221], v[12:13]\n"
    "v_pk_mul_f32 v[14:15], v[222:223], v[14:15]\n"
    "v_exp_f32 v0, v0\n"
    "v_exp_f32 v1, v1\n"
    "v_exp_f32 v2, v2\n"
    "v_exp_f32 v3, v3\n"
    "v_exp_f32 v4, v4\n"
    "v_exp_f32 v5, v5\n"
    "v_exp_f32 v6, v6\n"
    "v_exp_f32 v7, v7\n"
    "v_exp_f32 v8, v8\n"
    "v_exp_f32 v9, v9\n"
    "v_exp_f32 v10, v10\n"
    "v_exp_f32 v11, v11\n"
    "v_exp_f32 v12, v12\n"
    "v_exp_f32 v13, v13\n"
    "v_exp_f32 v14, v14\n"
    "v_exp_f32 v15, v15\n"
    "v_pk_add_f32 v[0:1], v[0:1], 1.0 op_sel_hi:[1,0]\n"
    "v_pk_add_f32 v[2:3], v[2:3], 1.0 op_sel_hi:[1,0]\n"
    "v_pk_add_f32 v[4:5], v[4:5], 1.0 op_sel_hi:[1,0]\n"
    "v_pk_add_f32 v[6:7], v[6:7], 1.0 op_sel_hi:[1,0]\n"
    "v_pk_add_f32 v[8:9], v[8:9], 1.0 op_sel_hi:[1,0]\n"
    "v_pk_add_f32 v[10:11], v[10:11], 1.0 op_sel_hi:[1,0]\n"
    "v_pk_add_f32 v[12:13], v[12:13], 1.0 op_sel_hi:[1,0]\n"
    "v_pk_add_f32 v[14:15], v[14:15], 1.0 op_sel_hi:[1,0]\n"
    "v_rcp_f32 v0, v0\n"
    "v_rcp_f32 v1, v1\n"
    "v_rcp_f32 v2, v2\n"
    "v_rcp_f32 v3, v3\n"
    "v_rcp_f32 v4, v4\n"
    "v_rcp_f32 v5, v5\n"
    "v_rcp_f32 v6, v6\n"
    "v_rcp_f32 v7, v7\n"
    "v_rcp_f32 v8, v8\n"
    "v_rcp_f32 v9, v9\n"
    "v_rcp_f32 v10, v10\n"
    "v_rcp_f32 v11, v11\n"
    "v_rcp_f32 v12, v12\n"
    "v_rcp_f32 v13, v13\n"
    "v_rcp_f32 v14, v14\n"
    "v_rcp_f32 v15, v15\n"
    "v_pk_mul_f32 v[0:1], v[208:209], v[0:1]\n"
    "v_pk_mul_f32 v[2:3], v[210:211], v[2:3]\n"
    "v_pk_mul_f32 v[4:5], v[212:213], v[4:5]\n"
    "v_pk_mul_f32 v[6:7], v[214:215], v[6:7]\n"
    "v_pk_mul_f32 v[8:9], v[216:217], v[8:9]\n"
    "v_pk_mul_f32 v[10:11], v[218:219], v[10:11]\n"
    "v_pk_mul_f32 v[12:13], v[220:221], v[12:13]\n"
    "v_pk_mul_f32 v[14:15], v[222:223], v[14:15]\n"
    "v_cvt_pk_bf16_f32 v52, v0, v1\n"
    "v_cvt_pk_bf16_f32 v53, v2, v3\n"
    "v_cvt_pk_bf16_f32 v54, v4, v5\n"
    "v_cvt_pk_bf16_f32 v55, v6, v7\n"
    "v_cvt_pk_bf16_f32 v56, v8, v9\n"
    "v_cvt_pk_bf16_f32 v57, v10, v11\n"
    "v_cvt_pk_bf16_f32 v58, v12, v13\n"
    "v_cvt_pk_bf16_f32 v59, v14, v15\n"
    "v_cndmask_b32 v44, v52, v54, vcc\n"
    "v_cndmask_b32 v45, v53, v55, vcc\n"
    "v_cndmask_b32 v46, v54, v52, vcc\n"
    "v_cndmask_b32 v47, v55, v53, vcc\n"
    "v_cndmask_b32 v48, v56, v58, vcc\n"
    "v_cndmask_b32 v49, v57, v59, vcc\n"
    "v_cndmask_b32 v50, v58, v56, vcc\n"
    "v_cndmask_b32 v51, v59, v57, vcc\n"
    "buffer_store_dwordx4 v[44:47], v39, s[12:15], s72 offen offset:128 \n"
    "buffer_store_dwordx4 v[48:51], v39, s[12:15], s72 offen offset:192 \n"
    "s_add_u32 s72, s72, 0x100000\n"
    "v_pk_mul_f32 v[0:1], v[224:225], v[224:225]\n"
    "v_pk_mul_f32 v[2:3], v[226:227], v[226:227]\n"
    "v_pk_mul_f32 v[4:5], v[228:229], v[228:229]\n"
    "v_pk_mul_f32 v[6:7], v[230:231], v[230:231]\n"
    "v_pk_mul_f32 v[8:9], v[232:233], v[232:233]\n"
    "v_pk_mul_f32 v[10:11], v[234:235], v[234:235]\n"
    "v_pk_mul_f32 v[12:13], v[236:237], v[236:237]\n"
    "v_pk_mul_f32 v[14:15], v[238:239], v[238:239]\n"
    "v_pk_fma_f32 v[0:1], v[0:1], s[90:91], v[62:63]\n"
    "v_pk_fma_f32 v[2:3], v[2:3], s[90:91], v[62:63]\n"
    "v_pk_fma_f32 v[4:5], v[4:5], s[90:91], v[62:63]\n"
    "v_pk_fma_f32 v[6:7], v[6:7], s[90:91], v[62:63]\n"
    "v_pk_fma_f32 v[8:9], v[8:9], s[90:91], v[62:63]\n"
    "v_pk_fma_f32 v[10:11], v[10:11], s[90:91], v[62:63]\n"
    "v_pk_fma_f32 v[12:13], v[12:13], s[90:91], v[62:63]\n"
    "v_pk_fma_f32 v[14:15], v[14:15], s[90:91], v[62:63]\n"
    "v_pk_mul_f32 v[0:1], v[224:225], v[0:1]\n"
    "v_pk_mul_f32 v[2:3], v[226:227], v[2:3]\n"
    "v_pk_mul_f32 v[4:5], v[228:229], v[4:5]\n"
    "v_pk_mul_f32 v[6:7], v[230:231], v[6:7]\n"
    "v_pk_mul_f32 v[8:9], v[232:233], v[8:9]\n"
    "v_pk_mul_f32 v[10:11], v[234:235], v[10:11]\n"
    "v_pk_mul_f32 v[12:13], v[236:237], v[12:13]\n"
    "v_pk_mul_f32 v[14:15], v[238:239], v[14:15]\n"
    "v_exp_f32 v0, v0\n"
    "v_exp_f32 v1, v1\n"
    "v_exp_f32 v2, v2\n"
    "v_exp_f32 v3, v3\n"
    "v_exp_f32 v4, v4\n"
    "v_exp_f32 v5, v5\n"
    "v_exp_f32 v6, v6\n"
    "v_exp_f32 v7, v7\n"
    "v_exp_f32 v8, v8\n"
    "v_exp_f32 v9, v9\n"
    "v_exp_f32 v10, v10\n"
    "v_exp_f32 v11, v11\n"
    "v_exp_f32 v12, v12\n"
    "v_exp_f32 v13, v13\n"
    "v_exp_f32 v14, v14\n"
    "v_exp_f32 v15, v15\n"
    "v_pk_add_f32 v[0:1], v[0:1], 1.0 op_sel_hi:[1,0]\n"
    "v_pk_add_f32 v[2:3], v[2:3], 1.0 op_sel_hi:[1,0]\n"
    "v_pk_add_f32 v[4:5], v[4:5], 1.0 op_sel_hi:[1,0]\n"
    "v_pk_add_f32 v[6:7], v[6:7], 1.0 op_sel_hi:[1,0]\n"
    "v_pk_add_f32 v[8:9], v[8:9], 1.0 op_sel_hi:[1,0]\n"
    "v_pk_add_f32 v[10:11], v[10:11], 1.0 op_sel_hi:[1,0]\n"
    "v_pk_add_f32 v[12:13], v[12:13], 1.0 op_sel_hi:[1,0]\n"
    "v_pk_add_f32 v[14:15], v[14:15], 1.0 op_sel_hi:[1,0]\n"
    "v_rcp_f32 v0, v0\n"
    "v_rcp_f32 v1, v1\n"
    "v_rcp_f32 v2, v2\n"
    "v_rcp_f32 v3, v3\n"
    "v_rcp_f32 v4, v4\n"
    "v_rcp_f32 v5, v5\n"
    "v_rcp_f32 v6, v6\n"
    "v_rcp_f32 v7, v7\n"
    "v_rcp_f32 v8, v8\n"
    "v_rcp_f32 v9, v9\n"
    "v_rcp_f32 v10, v10\n"
    "v_rcp_f32 v11, v11\n"
    "v_rcp_f32 v12, v12\n"
    "v_rcp_f32 v13, v13\n"
    "v_rcp_f32 v14, v14\n"
    "v_rcp_f32 v15, v15\n"
    "v_pk_mul_f32 v[0:1], v[224:225], v[0:1]\n"
    "v_pk_mul_f32 v[2:3], v[226:227], v[2:3]\n"
    "v_pk_mul_f32 v[4:5], v[228:229], v[4:5]\n"
    "v_pk_mul_f32 v[6:7], v[230:231], v[6:7]\n"
    "v_pk_mul_f32 v[8:9], v[232:233], v[8:9]\n"
    "v_pk_mul_f32 v[10:11], v[234:235], v[10:11]\n"
    "v_pk_mul_f32 v[12:13], v[236:237], v[12:13]\n"
    "v_pk_mul_f32 v[14:15], v[238:239], v[14:15]\n"
    "v_cvt_pk_bf16_f32 v52, v0, v1\n"
    "v_cvt_pk_bf16_f32 v53, v2, v3\n"
    "v_cvt_pk_bf16_f32 v54, v4, v5\n"
    "v_cvt_pk_bf16_f32 v55, v6, v7\n"
    "v_cvt_pk_bf16_f32 v56, v8, v9\n"
    "v_cvt_pk_bf16_f32 v57, v10, v11\n"
    "v_cvt_pk_bf16_f32 v58, v12, v13\n"
    "v_cvt_pk_bf16_f32 v59, v14, v15\n"
    "v_cndmask_b32 v44, v52, v54, vcc\n"
    "v_cndmask_b32 v45, v53, v55, vcc\n"
    "v_cndmask_b32 v46, v54, v52, vcc\n"
    "v_cndmask_b32 v47, v55, v53, vcc\n"
    "v_cndmask_b32 v48, v56, v58, vcc\n"
    "v_cndmask_b32 v49, v57, v59, vcc\n"
    "v_cndmask_b32 v50, v58, v56, vcc\n"
    "v_cndmask_b32 v51, v59, v57, vcc\n"
    "buffer_store_dwordx4 v[44:47], v39, s[12:15], s72 offen offset:0 \n"
    "buffer_store_dwordx4 v[48:51], v39, s[12:15], s72 offen offset:64 \n"
    "v_pk_mul_f32 v[0:1], v[240:241], v[240:241]\n"
    "v_pk_mul_f32 v[2:3], v[242:243], v[242:243]\n"
    "v_pk_mul_f32 v[4:5], v[244:245], v[244:245]\n"
    "v_pk_mul_f32 v[6:7], v[246:247], v[246:247]\n"
    "v_pk_mul_f32 v[8:9], v[248:249], v[248:249]\n"
    "v_pk_mul_f32 v[10:11], v[250:251], v[250:251]\n"
    "v_pk_mul_f32 v[12:13], v[252:253], v[252:253]\n"
    "v_pk_mul_f32 v[14:15], v[254:255], v[254:255]\n"
    "v_pk_fma_f32 v[0:1], v[0:1], s[90:91], v[62:63]\n"
    "v_pk_fma_f32 v[2:3], v[2:3], s[90:91], v[62:63]\n"
    "v_pk_fma_f32 v[4:5], v[4:5], s[90:91], v[62:63]\n"
    "v_pk_fma_f32 v[6:7], v[6:7], s[90:91], v[62:63]\n"
    "v_pk_fma_f32 v[8:9], v[8:9], s[90:91], v[62:63]\n"
    "v_pk_fma_f32 v[10:11], v[10:11], s[90:91], v[62:63]\n"
    "v_pk_fma_f32 v[12:13], v[12:13], s[90:91], v[62:63]\n"
    "v_pk_fma_f32 v[14:15], v[14:15], s[90:91], v[62:63]\n"
    "v_pk_mul_f32 v[0:1], v[240:241], v[0:1]\n"
    "v_pk_mul_f32 v[2:3], v[242:243], v[2:3]\n"
    "v_pk_mul_f32 v[4:5], v[244:245], v[4:5]\n"
    "v_pk_mul_f32 v[6:7], v[246:247], v[6:7]\n"
    "v_pk_mul_f32 v[8:9], v[248:249], v[8:9]\n"
    "v_pk_mul_f32 v[10:11], v[250:251], v[10:11]\n"
    "v_pk_mul_f32 v[12:13], v[252:253], v[12:13]\n"
    "v_pk_mul_f32 v[14:15], v[254:255], v[14:15]\n"
    "v_exp_f32 v0, v0\n"
    "v_exp_f32 v1, v1\n"
    "v_exp_f32 v2, v2\n"
    "v_exp_f32 v3, v3\n"
    "v_exp_f32 v4, v4\n"
    "v_exp_f32 v5, v5\n"
    "v_exp_f32 v6, v6\n"
    "v_exp_f32 v7, v7\n"
    "v_exp_f32 v8, v8\n"
    "v_exp_f32 v9, v9\n"
    "v_exp_f32 v10, v10\n"
    "v_exp_f32 v11, v11\n"
    "v_exp_f32 v12, v12\n"
    "v_exp_f32 v13, v13\n"
    "v_exp_f32 v14, v14\n"
    "v_exp_f32 v15, v15\n"
    "v_pk_add_f32 v[0:1], v[0:1], 1.0 op_sel_hi:[1,0]\n"
    "v_pk_add_f32 v[2:3], v[2:3], 1.0 op_sel_hi:[1,0]\n"
    "v_pk_add_f32 v[4:5], v[4:5], 1.0 op_sel_hi:[1,0]\n"
    "v_pk_add_f32 v[6:7], v[6:7], 1.0 op_sel_hi:[1,0]\n"
    "v_pk_add_f32 v[8:9], v[8:9], 1.0 op_sel_hi:[1,0]\n"
    "v_pk_add_f32 v[10:11], v[10:11], 1.0 op_sel_hi:[1,0]\n"
    "v_pk_add_f32 v[12:13], v[12:13], 1.0 op_sel_hi:[1,0]\n"
    "v_pk_add_f32 v[14:15], v[14:15], 1.0 op_sel_hi:[1,0]\n"
    "v_rcp_f32 v0, v0\n"
    "v_rcp_f32 v1, v1\n"
    "v_rcp_f32 v2, v2\n"
    "v_rcp_f32 v3, v3\n"
    "v_rcp_f32 v4, v4\n"
    "v_rcp_f32 v5, v5\n"
    "v_rcp_f32 v6, v6\n"
    "v_rcp_f32 v7, v7\n"
    "v_rcp_f32 v8, v8\n"
    "v_rcp_f32 v9, v9\n"
    "v_rcp_f32 v10, v10\n"
    "v_rcp_f32 v11, v11\n"
    "v_rcp_f32 v12, v12\n"
    "v_rcp_f32 v13, v13\n"
    "v_rcp_f32 v14, v14\n"
    "v_rcp_f32 v15, v15\n"
    "v_pk_mul_f32 v[0:1], v[240:241], v[0:1]\n"
    "v_pk_mul_f32 v[2:3], v[242:243], v[2:3]\n"
    "v_pk_mul_f32 v[4:5], v[244:245], v[4:5]\n"
    "v_pk_mul_f32 v[6:7], v[246:247], v[6:7]\n"
    "v_pk_mul_f32 v[8:9], v[248:249], v[8:9]\n"
    "v_pk_mul_f32 v[10:11], v[250:251], v[10:11]\n"
    "v_pk_mul_f32 v[12:13], v[252:253], v[12:13]\n"
    "v_pk_mul_f32 v[14:15], v[254:255], v[14:15]\n"
    "v_cvt_pk_bf16_f32 v52, v0, v1\n"
    "v_cvt_pk_bf16_f32 v53, v2, v3\n"
    "v_cvt_pk_bf16_f32 v54, v4, v5\n"
    "v_cvt_pk_bf16_f32 v55, v6, v7\n"
    "v_cvt_pk_bf16_f32 v56, v8, v9\n"
    "v_cvt_pk_bf16_f32 v57, v10, v11\n"
    "v_cvt_pk_bf16_f32 v58, v12, v13\n"
    "v_cvt_pk_bf16_f32 v59, v14, v15\n"
    "v_cndmask_b32 v44, v52, v54, vcc\n"
    "v_cndmask_b32 v45, v53, v55, vcc\n"
    "v_cndmask_b32 v46, v54, v52, vcc\n"
    "v_cndmask_b32 v47, v55, v53, vcc\n"
    "v_cndmask_b32 v48, v56, v58, vcc\n"
    "v_cndmask_b32 v49, v57, v59, vcc\n"
    "v_cndmask_b32 v50, v58, v56, vcc\n"
    "v_cndmask_b32 v51, v59, v57, vcc\n"
    "buffer_store_dwordx4 v[44:47], v39, s[12:15], s72 offen offset:128 \n"
    "buffer_store_dwordx4 v[48:51], v39, s[12:15], s72 offen offset:192 \n"
    "s_add_u32 s72, s72, 0x100000\n"
    "v_accvgpr_read_b32 v16, a0\n"
    "v_accvgpr_read_b32 v17, a1\n"
    "v_accvgpr_read_b32 v18, a2\n"
    "v_accvgpr_read_b32 v19, a3\n"
    "v_accvgpr_read_b32 v20, a4\n"
    "v_accvgpr_read_b32 v21, a5\n"
    "v_accvgpr_read_b32 v22, a6\n"
    "v_accvgpr_read_b32 v23, a7\n"
    "v_accvgpr_read_b32 v24, a8\n"
    "v_accvgpr_read_b32 v25, a9\n"
    "v_accvgpr_read_b32 v26, a10\n"
    "v_accvgpr_read_b32 v27, a11\n"
    "v_accvgpr_read_b32 v28, a12\n"
    "v_accvgpr_read_b32 v29, a13\n"
    "v_accvgpr_read_b32 v30, a14\n"
    "v_accvgpr_read_b32 v31, a15\n"
    "v_pk_mul_f32 v[0:1], v[16:17], v[16:17]\n"
    "v_pk_mul_f32 v[2:3], v[18:19], v[18:19]\n"
    "v_pk_mul_f32 v[4:5], v[20:21], v[20:21]\n"
    "v_pk_mul_f32 v[6:7], v[22:23], v[22:23]\n"
    "v_pk_mul_f32 v[8:9], v[24:25], v[24:25]\n"
    "v_pk_mul_f32 v[10:11], v[26:27], v[26:27]\n"
    "v_pk_mul_f32 v[12:13], v[28:29], v[28:29]\n"
    "v_pk_mul_f32 v[14:15], v[30:31], v[30:31]\n"
    "v_pk_fma_f32 v[0:1], v[0:1], s[90:91], v[62:63]\n"
    "v_pk_fma_f32 v[2:3], v[2:3], s[90:91], v[62:63]\n"
    "v_pk_fma_f32 v[4:5], v[4:5], s[90:91], v[62:63]\n"
    "v_pk_fma_f32 v[6:7], v[6:7], s[90:91], v[62:63]\n"
    "v_pk_fma_f32 v[8:9], v[8:9], s[90:91], v[62:63]\n"
    "v_pk_fma_f32 v[10:11], v[10:11], s[90:91], v[62:63]\n"
    "v_pk_fma_f32 v[12:13], v[12:13], s[90:91], v[62:63]\n"
    "v_pk_fma_f32 v[14:15], v[14:15], s[90:91], v[62:63]\n"
    "v_pk_mul_f32 v[0:1], v[16:17], v[0:1]\n"
    "v_pk_mul_f32 v[2:3], v[18:19], v[2:3]\n"
    "v_pk_mul_f32 v[4:5], v[20:21], v[4:5]\n"
    "v_pk_mul_f32 v[6:7], v[22:23], v[6:7]\n"
    "v_pk_mul_f32 v[8:9], v[24:25], v[8:9]\n"
    "v_pk_mul_f32 v[10:11], v[26:27], v[10:11]\n"
    "v_pk_mul_f32 v[12:13], v[28:29], v[12:13]\n"
    "v_pk_mul_f32 v[14:15], v[30:31], v[14:15]\n"
    "v_exp_f32 v0, v0\n"
    "v_exp_f32 v1, v1\n"
    "v_exp_f32 v2, v2\n"
    "v_exp_f32 v3, v3\n"
    "v_exp_f32 v4, v4\n"
    "v_exp_f32 v5, v5\n"
    "v_exp_f32 v6, v6\n"
    "v_exp_f32 v7, v7\n"
    "v_exp_f32 v8, v8\n"
    "v_exp_f32 v9, v9\n"
    "v_exp_f32 v10, v10\n"
    "v_exp_f32 v11, v11\n"
    "v_exp_f32 v12, v12\n"
    "v_exp_f32 v13, v13\n"
    "v_exp_f32 v14, v14\n"
    "v_exp_f32 v15, v15\n"
    "v_pk_add_f32 v[0:1], v[0:1], 1.0 op_sel_hi:[1,0]\n"
    "v_pk_add_f32 v[2:3], v[2:3], 1.0 op_sel_hi:[1,0]\n"
    "v_pk_add_f32 v[4:5], v[4:5], 1.0 op_sel_hi:[1,0]\n"
    "v_pk_add_f32 v[6:7], v[6:7], 1.0 op_sel_hi:[1,0]\n"
    "v_pk_add_f32 v[8:9], v[8:9], 1.0 op_sel_hi:[1,0]\n"
    "v_pk_add_f32 v[10:11], v[10:11], 1.0 op_sel_hi:[1,0]\n"
    "v_pk_add_f32 v[12:13], v[12:13], 1.0 op_sel_hi:[1,0]\n"
    "v_pk_add_f32 v[14:15], v[14:15], 1.0 op_sel_hi:[1,0]\n"
    "v_rcp_f32 v0, v0\n"
    "v_rcp_f32 v1, v1\n"
    "v_rcp_f32 v2, v2\n"
    "v_rcp_f32 v3, v3\n"
    "v_rcp_f32 v4, v4\n"
    "v_rcp_f32 v5, v5\n"
    "v_rcp_f32 v6, v6\n"
    "v_rcp_f32 v7, v7\n"
    "v_rcp_f32 v8, v8\n"
    "v_rcp_f32 v9, v9\n"
    "v_rcp_f32 v10, v10\n"
    "v_rcp_f32 v11, v11\n"
    "v_rcp_f32 v12, v12\n"
    "v_rcp_f32 v13, v13\n"
    "v_rcp_f32 v14, v14\n"
    "v_rcp_f32 v15, v15\n"
    "v_pk_mul_f32 v[0:1], v[16:17], v[0:1]\n"
    "v_pk_mul_f32 v[2:3], v[18:19], v[2:3]\n"
    "v_pk_mul_f32 v[4:5], v[20:21], v[4:5]\n"
    "v_pk_mul_f32 v[6:7], v[22:23], v[6:7]\n"
    "v_pk_mul_f32 v[8:9], v[24:25], v[8:9]\n"
    "v_pk_mul_f32 v[10:11], v[26:27], v[10:11]\n"
    "v_pk_mul_f32 v[12:13], v[28:29], v[12:13]\n"
    "v_pk_mul_f32 v[14:15], v[30:31], v[14:15]\n"
    "v_cvt_pk_bf16_f32 v52, v0, v1\n"
    "v_cvt_pk_bf16_f32 v53, v2, v3\n"
    "v_cvt_pk_bf16_f32 v54, v4, v5\n"
    "v_cvt_pk_bf16_f32 v55, v6, v7\n"
    "v_cvt_pk_bf16_f32 v56, v8, v9\n"
    "v_cvt_pk_bf16_f32 v57, v10, v11\n"
    "v_cvt_pk_bf16_f32 v58, v12, v13\n"
    "v_cvt_pk_bf16_f32 v59, v14, v15\n"
    "v_cndmask_b32 v44, v52, v54, vcc\n"
    "v_cndmask_b32 v45, v53, v55, vcc\n"
    "v_cndmask_b32 v46, v54, v52, vcc\n"
    "v_cndmask_b32 v47, v55, v53, vcc\n"
    "v_cndmask_b32 v48, v56, v58, vcc\n"
    "v_cndmask_b32 v49, v57, v59, vcc\n"
    "v_cndmask_b32 v50, v58, v56, vcc\n"
    "v_cndmask_b32 v51, v59, v57, vcc\n"
    "buffer_store_dwordx4 v[44:47], v39, s[12:15], s72 offen offset:0 \n"
    "buffer_store_dwordx4 v[48:51], v39, s[12:15], s72 offen offset:64 \n"
    "v_accvgpr_read_b32 v16, a16\n"
    "v_accvgpr_read_b32 v17, a17\n"
    "v_accvgpr_read_b32 v18, a18\n"
    "v_accvgpr_read_b32 v19, a19\n"
    "v_accvgpr_read_b32 v20, a20\n"
    "v_accvgpr_read_b32 v21, a21\n"
    "v_accvgpr_read_b32 v22, a22\n"
    "v_accvgpr_read_b32 v23, a23\n"
    "v_accvgpr_read_b32 v24, a24\n"
    "v_accvgpr_read_b32 v25, a25\n"
    "v_accvgpr_read_b32 v26, a26\n"
    "v_accvgpr_read_b32 v27, a27\n"
    "v_accvgpr_read_b32 v28, a28\n"
    "v_accvgpr_read_b32 v29, a29\n"
    "v_accvgpr_read_b32 v30, a30\n"
    "v_accvgpr_read_b32 v31, a31\n"
    "v_pk_mul_f32 v[0:1], v[16:17], v[16:17]\n"
    "v_pk_mul_f32 v[2:3], v[18:19], v[18:19]\n"
    "v_pk_mul_f32 v[4:5], v[20:21], v[20:21]\n"
    "v_pk_mul_f32 v[6:7], v[22:23], v[22:23]\n"
    "v_pk_mul_f32 v[8:9], v[24:25], v[24:25]\n"
    "v_pk_mul_f32 v[10:11], v[26:27], v[26:27]\n"
    "v_pk_mul_f32 v[12:13], v[28:29], v[28:29]\n"
    "v_pk_mul_f32 v[14:15], v[30:31], v[30:31]\n"
    "v_pk_fma_f32 v[0:1], v[0:1], s[90:91], v[62:63]\n"
    "v_pk_fma_f32 v[2:3], v[2:3], s[90:91], v[62:63]\n"
    "v_pk_fma_f32 v[4:5], v[4:5], s[90:91], v[62:63]\n"
    "v_pk_fma_f32 v[6:7], v[6:7], s[90:91], v[62:63]\n"
    "v_pk_fma_f32 v[8:9], v[8:9], s[90:91], v[62:63]\n"
    "v_pk_fma_f32 v[10:11], v[10:11], s[90:91], v[62:63]\n"
    "v_pk_fma_f32 v[12:13], v[12:13], s[90:91], v[62:63]\n"
    "v_pk_fma_f32 v[14:15], v[14:15], s[90:91], v[62:63]\n"
    "v_pk_mul_f32 v[0:1], v[16:17], v[0:1]\n"
    "v_pk_mul_f32 v[2:3], v[18:19], v[2:3]\n"
    "v_pk_mul_f32 v[4:5], v[20:21], v[4:5]\n"
    "v_pk_mul_f32 v[6:7], v[22:23], v[6:7]\n"
    "v_pk_mul_f32 v[8:9], v[24:25], v[8:9]\n"
    "v_pk_mul_f32 v[10:11], v[26:27], v[10:11]\n"
    "v_pk_mul_f32 v[12:13], v[28:29], v[12:13]\n"
    "v_pk_mul_f32 v[14:15], v[30:31], v[14:15]\n"
    "v_exp_f32 v0, v0\n"
    "v_exp_f32 v1, v1\n"
    "v_exp_f32 v2, v2\n"
    "v_exp_f32 v3, v3\n"
    "v_exp_f32 v4, v4\n"
    "v_exp_f32 v5, v5\n"
    "v_exp_f32 v6, v6\n"
    "v_exp_f32 v7, v7\n"
    "v_exp_f32 v8, v8\n"
    "v_exp_f32 v9, v9\n"
    "v_exp_f32 v10, v10\n"
    "v_exp_f32 v11, v11\n"
    "v_exp_f32 v12, v12\n"
    "v_exp_f32 v13, v13\n"
    "v_exp_f32 v14, v14\n"
    "v_exp_f32 v15, v15\n"
    "v_pk_add_f32 v[0:1], v[0:1], 1.0 op_sel_hi:[1,0]\n"
    "v_pk_add_f32 v[2:3], v[2:3], 1.0 op_sel_hi:[1,0]\n"
    "v_pk_add_f32 v[4:5], v[4:5], 1.0 op_sel_hi:[1,0]\n"
    "v_pk_add_f32 v[6:7], v[6:7], 1.0 op_sel_hi:[1,0]\n"
    "v_pk_add_f32 v[8:9], v[8:9], 1.0 op_sel_hi:[1,0]\n"
    "v_pk_add_f32 v[10:11], v[10:11], 1.0 op_sel_hi:[1,0]\n"
    "v_pk_add_f32 v[12:13], v[12:13], 1.0 op_sel_hi:[1,0]\n"
    "v_pk_add_f32 v[14:15], v[14:15], 1.0 op_sel_hi:[1,0]\n"
    "v_rcp_f32 v0, v0\n"
    "v_rcp_f32 v1, v1\n"
    "v_rcp_f32 v2, v2\n"
    "v_rcp_f32 v3, v3\n"
    "v_rcp_f32 v4, v4\n"
    "v_rcp_f32 v5, v5\n"
    "v_rcp_f32 v6, v6\n"
    "v_rcp_f32 v7, v7\n"
    "v_rcp_f32 v8, v8\n"
    "v_rcp_f32 v9, v9\n"
    "v_rcp_f32 v10, v10\n"
    "v_rcp_f32 v11, v11\n"
    "v_rcp_f32 v12, v12\n"
    "v_rcp_f32 v13, v13\n"
    "v_rcp_f32 v14, v14\n"
    "v_rcp_f32 v15, v15\n"
    "v_pk_mul_f32 v[0:1], v[16:17], v[0:1]\n"
    "v_pk_mul_f32 v[2:3], v[18:19], v[2:3]\n"
    "v_pk_mul_f32 v[4:5], v[20:21], v[4:5]\n"
    "v_pk_mul_f32 v[6:7], v[22:23], v[6:7]\n"
    "v_pk_mul_f32 v[8:9], v[24:25], v[8:9]\n"
    "v_pk_mul_f32 v[10:11], v[26:27], v[10:11]\n"
    "v_pk_mul_f32 v[12:13], v[28:29], v[12:13]\n"
    "v_pk_mul_f32 v[14:15], v[30:31], v[14:15]\n"
    "v_cvt_pk_bf16_f32 v52, v0, v1\n"
    "v_cvt_pk_bf16_f32 v53, v2, v3\n"
    "v_cvt_pk_bf16_f32 v54, v4, v5\n"
    "v_cvt_pk_bf16_f32 v55, v6, v7\n"
    "v_cvt_pk_bf16_f32 v56, v8, v9\n"
    "v_cvt_pk_bf16_f32 v57, v10, v11\n"
    "v_cvt_pk_bf16_f32 v58, v12, v13\n"
    "v_cvt_pk_bf16_f32 v59, v14, v15\n"
    "v_cndmask_b32 v44, v52, v54, vcc\n"
    "v_cndmask_b32 v45, v53, v55, vcc\n"
    "v_cndmask_b32 v46, v54, v52, vcc\n"
    "v_cndmask_b32 v47, v55, v53, vcc\n"
    "v_cndmask_b32 v48, v56, v58, vcc\n"
    "v_cndmask_b32 v49, v57, v59, vcc\n"
    "v_cndmask_b32 v50, v58, v56, vcc\n"
    "v_cndmask_b32 v51, v59, v57, vcc\n"
    "buffer_store_dwordx4 v[44:47], v39, s[12:15], s72 offen offset:128 \n"
    "buffer_store_dwordx4 v[48:51], v39, s[12:15], s72 offen offset:192 \n"
    "s_add_u32 s72, s72, 0x100000\n"
    "v_accvgpr_read_b32 v16, a32\n"
    "v_accvgpr_read_b32 v17, a33\n"
    "v_accvgpr_read_b32 v18, a34\n"
    "v_accvgpr_read_b32 v19, a35\n"
    "v_accvgpr_read_b32 v20, a36\n"
    "v_accvgpr_read_b32 v21, a37\n"
    "v_accvgpr_read_b32 v22, a38\n"
    "v_accvgpr_read_b32 v23, a39\n"
    "v_accvgpr_read_b32 v24, a40\n"
    "v_accvgpr_read_b32 v25, a41\n"
    "v_accvgpr_read_b32 v26, a42\n"
    "v_accvgpr_read_b32 v27, a43\n"
    "v_accvgpr_read_b32 v28, a44\n"
    "v_accvgpr_read_b32 v29, a45\n"
    "v_accvgpr_read_b32 v30, a46\n"
    "v_accvgpr_read_b32 v31, a47\n"
    "v_pk_mul_f32 v[0:1], v[16:17], v[16:17]\n"
    "v_pk_mul_f32 v[2:3], v[18:19], v[18:19]\n"
    "v_pk_mul_f32 v[4:5], v[20:21], v[20:21]\n"
    "v_pk_mul_f32 v[6:7], v[22:23], v[22:23]\n"
    "v_pk_mul_f32 v[8:9], v[24:25], v[24:25]\n"
    "v_pk_mul_f32 v[10:11], v[26:27], v[26:27]\n"
    "v_pk_mul_f32 v[12:13], v[28:29], v[28:29]\n"
    "v_pk_mul_f32 v[14:15], v[30:31], v[30:31]\n"
    "v_pk_fma_f32 v[0:1], v[0:1], s[90:91], v[62:63]\n"
    "v_pk_fma_f32 v[2:3], v[2:3], s[90:91], v[62:63]\n"
    "v_pk_fma_f32 v[4:5], v[4:5], s[90:91], v[62:63]\n"
    "v_pk_fma_f32 v[6:7], v[6:7], s[90:91], v[62:63]\n"
    "v_pk_fma_f32 v[8:9], v[8:9], s[90:91], v[62:63]\n"
    "v_pk_fma_f32 v[10:11], v[10:11], s[90:91], v[62:63]\n"
    "v_pk_fma_f32 v[12:13], v[12:13], s[90:91], v[62:63]\n"
    "v_pk_fma_f32 v[14:15], v[14:15], s[90:91], v[62:63]\n"
    "v_pk_mul_f32 v[0:1], v[16:17], v[0:1]\n"
    "v_pk_mul_f32 v[2:3], v[18:19], v[2:3]\n"
    "v_pk_mul_f32 v[4:5], v[20:21], v[4:5]\n"
    "v_pk_mul_f32 v[6:7], v[22:23], v[6:7]\n"
    "v_pk_mul_f32 v[8:9], v[24:25], v[8:9]\n"
    "v_pk_mul_f32 v[10:11], v[26:27], v[10:11]\n"
    "v_pk_mul_f32 v[12:13], v[28:29], v[12:13]\n"
    "v_pk_mul_f32 v[14:15], v[30:31], v[14:15]\n"
    "v_exp_f32 v0, v0\n"
    "v_exp_f32 v1, v1\n"
    "v_exp_f32 v2, v2\n"
    "v_exp_f32 v3, v3\n"
    "v_exp_f32 v4, v4\n"
    "v_exp_f32 v5, v5\n"
    "v_exp_f32 v6, v6\n"
    "v_exp_f32 v7, v7\n"
    "v_exp_f32 v8, v8\n"
    "v_exp_f32 v9, v9\n"
    "v_exp_f32 v10, v10\n"
    "v_exp_f32 v11, v11\n"
    "v_exp_f32 v12, v12\n"
    "v_exp_f32 v13, v13\n"
    "v_exp_f32 v14, v14\n"
    "v_exp_f32 v15, v15\n"
    "v_pk_add_f32 v[0:1], v[0:1], 1.0 op_sel_hi:[1,0]\n"
    "v_pk_add_f32 v[2:3], v[2:3], 1.0 op_sel_hi:[1,0]\n"
    "v_pk_add_f32 v[4:5], v[4:5], 1.0 op_sel_hi:[1,0]\n"
    "v_pk_add_f32 v[6:7], v[6:7], 1.0 op_sel_hi:[1,0]\n"
    "v_pk_add_f32 v[8:9], v[8:9], 1.0 op_sel_hi:[1,0]\n"
    "v_pk_add_f32 v[10:11], v[10:11], 1.0 op_sel_hi:[1,0]\n"
    "v_pk_add_f32 v[12:13], v[12:13], 1.0 op_sel_hi:[1,0]\n"
    "v_pk_add_f32 v[14:15], v[14:15], 1.0 op_sel_hi:[1,0]\n"
    "v_rcp_f32 v0, v0\n"
    "v_rcp_f32 v1, v1\n"
    "v_rcp_f32 v2, v2\n"
    "v_rcp_f32 v3, v3\n"
    "v_rcp_f32 v4, v4\n"
    "v_rcp_f32 v5, v5\n"
    "v_rcp_f32 v6, v6\n"
    "v_rcp_f32 v7, v7\n"
    "v_rcp_f32 v8, v8\n"
    "v_rcp_f32 v9, v9\n"
    "v_rcp_f32 v10, v10\n"
    "v_rcp_f32 v11, v11\n"
    "v_rcp_f32 v12, v12\n"
    "v_rcp_f32 v13, v13\n"
    "v_rcp_f32 v14, v14\n"
    "v_rcp_f32 v15, v15\n"
    "v_pk_mul_f32 v[0:1], v[16:17], v[0:1]\n"
    "v_pk_mul_f32 v[2:3], v[18:19], v[2:3]\n"
    "v_pk_mul_f32 v[4:5], v[20:21], v[4:5]\n"
    "v_pk_mul_f32 v[6:7], v[22:23], v[6:7]\n"
    "v_pk_mul_f32 v[8:9], v[24:25], v[8:9]\n"
    "v_pk_mul_f32 v[10:11], v[26:27], v[10:11]\n"
    "v_pk_mul_f32 v[12:13], v[28:29], v[12:13]\n"
    "v_pk_mul_f32 v[14:15], v[30:31], v[14:15]\n"
    "v_cvt_pk_bf16_f32 v52, v0, v1\n"
    "v_cvt_pk_bf16_f32 v53, v2, v3\n"
    "v_cvt_pk_bf16_f32 v54, v4, v5\n"
    "v_cvt_pk_bf16_f32 v55, v6, v7\n"
    "v_cvt_pk_bf16_f32 v56, v8, v9\n"
    "v_cvt_pk_bf16_f32 v57, v10, v11\n"
    "v_cvt_pk_bf16_f32 v58, v12, v13\n"
    "v_cvt_pk_bf16_f32 v59, v14, v15\n"
    "v_cndmask_b32 v44, v52, v54, vcc\n"
    "v_cndmask_b32 v45, v53, v55, vcc\n"
    "v_cndmask_b32 v46, v54, v52, vcc\n"
    "v_cndmask_b32 v47, v55, v53, vcc\n"
    "v_cndmask_b32 v48, v56, v58, vcc\n"
    "v_cndmask_b32 v49, v57, v59, vcc\n"
    "v_cndmask_b32 v50, v58, v56, vcc\n"
    "v_cndmask_b32 v51, v59, v57, vcc\n"
    "buffer_store_dwordx4 v[44:47], v39, s[12:15], s72 offen offset:0 \n"
    "buffer_store_dwordx4 v[48:51], v39, s[12:15], s72 offen offset:64 \n"
    "v_accvgpr_read_b32 v16, a48\n"
    "v_accvgpr_read_b32 v17, a49\n"
    "v_accvgpr_read_b32 v18, a50\n"
    "v_accvgpr_read_b32 v19, a51\n"
    "v_accvgpr_read_b32 v20, a52\n"
    "v_accvgpr_read_b32 v21, a53\n"
    "v_accvgpr_read_b32 v22, a54\n"
    "v_accvgpr_read_b32 v23, a55\n"
    "v_accvgpr_read_b32 v24, a56\n"
    "v_accvgpr_read_b32 v25, a57\n"
    "v_accvgpr_read_b32 v26, a58\n"
    "v_accvgpr_read_b32 v27, a59\n"
    "v_accvgpr_read_b32 v28, a60\n"
    "v_accvgpr_read_b32 v29, a61\n"
    "v_accvgpr_read_b32 v30, a62\n"
    "v_accvgpr_read_b32 v31, a63\n"
    "v_pk_mul_f32 v[0:1], v[16:17], v[16:17]\n"
    "v_pk_mul_f32 v[2:3], v[18:19], v[18:19]\n"
    "v_pk_mul_f32 v[4:5], v[20:21], v[20:21]\n"
    "v_pk_mul_f32 v[6:7], v[22:23], v[22:23]\n"
    "v_pk_mul_f32 v[8:9], v[24:25], v[24:25]\n"
    "v_pk_mul_f32 v[10:11], v[26:27], v[26:27]\n"
    "v_pk_mul_f32 v[12:13], v[28:29], v[28:29]\n"
    "v_pk_mul_f32 v[14:15], v[30:31], v[30:31]\n"
    "v_pk_fma_f32 v[0:1], v[0:1], s[90:91], v[62:63]\n"
    "v_pk_fma_f32 v[2:3], v[2:3], s[90:91], v[62:63]\n"
    "v_pk_fma_f32 v[4:5], v[4:5], s[90:91], v[62:63]\n"
    "v_pk_fma_f32 v[6:7], v[6:7], s[90:91], v[62:63]\n"
    "v_pk_fma_f32 v[8:9], v[8:9], s[90:91], v[62:63]\n"
    "v_pk_fma_f32 v[10:11], v[10:11], s[90:91], v[62:63]\n"
    "v_pk_fma_f32 v[12:13], v[12:13], s[90:91], v[62:63]\n"
    "v_pk_fma_f32 v[14:15], v[14:15], s[90:91], v[62:63]\n"
    "v_pk_mul_f32 v[0:1], v[16:17], v[0:1]\n"
    "v_pk_mul_f32 v[2:3], v[18:19], v[2:3]\n"
    "v_pk_mul_f32 v[4:5], v[20:21], v[4:5]\n"
    "v_pk_mul_f32 v[6:7], v[22:23], v[6:7]\n"
    "v_pk_mul_f32 v[8:9], v[24:25], v[8:9]\n"
    "v_pk_mul_f32 v[10:11], v[26:27], v[10:11]\n"
    "v_pk_mul_f32 v[12:13], v[28:29], v[12:13]\n"
    "v_pk_mul_f32 v[14:15], v[30:31], v[14:15]\n"
    "v_exp_f32 v0, v0\n"
    "v_exp_f32 v1, v1\n"
    "v_exp_f32 v2, v2\n"
    "v_exp_f32 v3, v3\n"
    "v_exp_f32 v4, v4\n"
    "v_exp_f32 v5, v5\n"
    "v_exp_f32 v6, v6\n"
    "v_exp_f32 v7, v7\n"
    "v_exp_f32 v8, v8\n"
    "v_exp_f32 v9, v9\n"
    "v_exp_f32 v10, v10\n"
    "v_exp_f32 v11, v11\n"
    "v_exp_f32 v12, v12\n"
    "v_exp_f32 v13, v13\n"
    "v_exp_f32 v14, v14\n"
    "v_exp_f32 v15, v15\n"
    "v_pk_add_f32 v[0:1], v[0:1], 1.0 op_sel_hi:[1,0]\n"
    "v_pk_add_f32 v[2:3], v[2:3], 1.0 op_sel_hi:[1,0]\n"
    "v_pk_add_f32 v[4:5], v[4:5], 1.0 op_sel_hi:[1,0]\n"
    "v_pk_add_f32 v[6:7], v[6:7], 1.0 op_sel_hi:[1,0]\n"
    "v_pk_add_f32 v[8:9], v[8:9], 1.0 op_sel_hi:[1,0]\n"
    "v_pk_add_f32 v[10:11], v[10:11], 1.0 op_sel_hi:[1,0]\n"
    "v_pk_add_f32 v[12:13], v[12:13], 1.0 op_sel_hi:[1,0]\n"
    "v_pk_add_f32 v[14:15], v[14:15], 1.0 op_sel_hi:[1,0]\n"
    "v_rcp_f32 v0, v0\n"
    "v_rcp_f32 v1, v1\n"
    "v_rcp_f32 v2, v2\n"
    "v_rcp_f32 v3, v3\n"
    "v_rcp_f32 v4, v4\n"
    "v_rcp_f32 v5, v5\n"
    "v_rcp_f32 v6, v6\n"
    "v_rcp_f32 v7, v7\n"
    "v_rcp_f32 v8, v8\n"
    "v_rcp_f32 v9, v9\n"
    "v_rcp_f32 v10, v10\n"
    "v_rcp_f32 v11, v11\n"
    "v_rcp_f32 v12, v12\n"
    "v_rcp_f32 v13, v13\n"
    "v_rcp_f32 v14, v14\n"
    "v_rcp_f32 v15, v15\n"
    "v_pk_mul_f32 v[0:1], v[16:17], v[0:1]\n"
    "v_pk_mul_f32 v[2:3], v[18:19], v[2:3]\n"
    "v_pk_mul_f32 v[4:5], v[20:21], v[4:5]\n"
    "v_pk_mul_f32 v[6:7], v[22:23], v[6:7]\n"
    "v_pk_mul_f32 v[8:9], v[24:25], v[8:9]\n"
    "v_pk_mul_f32 v[10:11], v[26:27], v[10:11]\n"
    "v_pk_mul_f32 v[12:13], v[28:29], v[12:13]\n"
    "v_pk_mul_f32 v[14:15], v[30:31], v[14:15]\n"
    "v_cvt_pk_bf16_f32 v52, v0, v1\n"
    "v_cvt_pk_bf16_f32 v53, v2, v3\n"
    "v_cvt_pk_bf16_f32 v54, v4, v5\n"
    "v_cvt_pk_bf16_f32 v55, v6, v7\n"
    "v_cvt_pk_bf16_f32 v56, v8, v9\n"
    "v_cvt_pk_bf16_f32 v57, v10, v11\n"
    "v_cvt_pk_bf16_f32 v58, v12, v13\n"
    "v_cvt_pk_bf16_f32 v59, v14, v15\n"
    "v_cndmask_b32 v44, v52, v54, vcc\n"
    "v_cndmask_b32 v45, v53, v55, vcc\n"
    "v_cndmask_b32 v46, v54, v52, vcc\n"
    "v_cndmask_b32 v47, v55, v53, vcc\n"
    "v_cndmask_b32 v48, v56, v58, vcc\n"
    "v_cndmask_b32 v49, v57, v59, vcc\n"
    "v_cndmask_b32 v50, v58, v56, vcc\n"
    "v_cndmask_b32 v51, v59, v57, vcc\n"
    "buffer_store_dwordx4 v[44:47], v39, s[12:15], s72 offen offset:128 \n"
    "buffer_store_dwordx4 v[48:51], v39, s[12:15], s72 offen offset:192 \n"
    "s_branch Lexit%=\n"
    "Ldead%=:\n"
    "v_mov_b32 v44, 0x7fc07fc0\n"
    "v_mov_b32 v45, 0x7fc07fc0\n"
    "v_mov_b32 v46, 0x7fc07fc0\n"
    "v_mov_b32 v47, 0x7fc07fc0\n"
    "s_mov_b32 s74, 0\n"
    "Ldeadl%=:\n"
    "buffer_store_dwordx4 v[44:47], v39, s[12:15], s72 offen offset:0\n"
    "buffer_store_dwordx4 v[44:47], v39, s[12:15], s72 offen offset:64\n"
    "buffer_store_dwordx4 v[44:47], v39, s[12:15], s72 offen offset:128\n"
    "buffer_store_dwordx4 v[44:47], v39, s[12:15], s72 offen offset:192\n"
    "s_add_u32 s72, s72, 0x100000\n"
    "s_add_u32 s74, s74, 1\n"
    "s_cmp_lt_u32 s74, 8\n"
    "s_cbranch_scc1 Ldeadl%=\n"
    "Lexit%=:\n"

        :
        : [ka] "s"(__builtin_amdgcn_kernarg_segment_ptr()), [bx] "s"(blockIdx.x), [lb] "s"(ldsbase), "{v0}"(threadIdx.x)
        : "memory", "scc", "vcc", "s4", "s5", "s6", "s7", "s8", "s9", "s10", "s11", "s12", "s13", "s14", "s15", "s16", "s17", "s18", "s19", "s20", "s21", "s22", "s23", "s24", "s25", "s26", "s27", "s28", "s29", "s30", "s31", "s33", "s34", "s35", "s36", "s37", "s38", "s39", "s40", "s41", "s42", "s43", "s44", "s45", "s46", "s47", "s48", "s49", "s50", "s51", "s52", "s53", "s54", "s55", "s56", "s57", "s58", "s59", "s60", "s61", "s62", "s63", "s64", "s65", "s66", "s67", "s68", "s69", "s70", "s71", "s72", "s73", "s74", "s75", "s76", "s77", "s78", "s79", "s80", "s81", "s82", "s83", "s84", "s85", "s86", "s87", "s88", "s89", "s90", "s91", "s92", "s93", "s94", "s95", "s96", "s97", "s98", "s99",
          "v1", "v2", "v3", "v4", "v5", "v6", "v7", "v8", "v9", "v10", "v11", "v12", "v13", "v14", "v15", "v16", "v17", "v18", "v19", "v20", "v21", "v22", "v23", "v24", "v25", "v26", "v27", "v28", "v29", "v30", "v31", "v32", "v33", "v34", "v35", "v36", "v37", "v38", "v39", "v40", "v41", "v42", "v43", "v44", "v45", "v46", "v47", "v48", "v49", "v50", "v51", "v52", "v53", "v54", "v55", "v56", "v57", "v58", "v59", "v60", "v61", "v62", "v63", "v64", "v65", "v66", "v67", "v68", "v69", "v70", "v71", "v72", "v73", "v74", "v75", "v76", "v77", "v78", "v79", "v80", "v81", "v82", "v83", "v84", "v85", "v86", "v87", "v88", "v89", "v90", "v91", "v92", "v93", "v94", "v95", "v96", "v97", "v98", "v99", "v100", "v101", "v102", "v103", "v104", "v105", "v106", "v107", "v108", "v109", "v110", "v111", "v112", "v113", "v114", "v115", "v116", "v117", "v118", "v119", "v120", "v121", "v122", "v123", "v124", "v125", "v126", "v127", "v128", "v129", "v130", "v131", "v132", "v133", "v134", "v135", "v136", "v137", "v138", "v139", "v140", "v141", "v142", "v143", "v144", "v145", "v146", "v147", "v148", "v149", "v150", "v151", "v152", "v153", "v154", "v155", "v156", "v157", "v158", "v159", "v160", "v161", "v162", "v163", "v164", "v165", "v166", "v167", "v168", "v169", "v170", "v171", "v172", "v173", "v174", "v175", "v176", "v177", "v178", "v179", "v180", "v181", "v182", "v183", "v184", "v185", "v186", "v187", "v188", "v189", "v190", "v191", "v192", "v193", "v194", "v195", "v196", "v197", "v198", "v199", "v200", "v201", "v202", "v203", "v204", "v205", "v206", "v207", "v208", "v209", "v210", "v211", "v212", "v213", "v214", "v215", "v216", "v217", "v218", "v219", "v220", "v221", "v222", "v223", "v224", "v225", "v226", "v227", "v228", "v229", "v230", "v231", "v232", "v233", "v234", "v235", "v236", "v237", "v238", "v239", "v240", "v241", "v242", "v243", "v244", "v245", "v246", "v247", "v248", "v249", "v250", "v251", "v252", "v253", "v254", "v255",
          "a0", "a1", "a2", "a3", "a4", "a5", "a6", "a7", "a8", "a9", "a10", "a11", "a12", "a13", "a14", "a15", "a16", "a17", "a18", "a19", "a20", "a21", "a22", "a23", "a24", "a25", "a26", "a27", "a28", "a29", "a30", "a31", "a32", "a33", "a34", "a35", "a36", "a37", "a38", "a39", "a40", "a41", "a42", "a43", "a44", "a45", "a46", "a47", "a48", "a49", "a50", "a51", "a52", "a53", "a54", "a55", "a56", "a57", "a58", "a59", "a60", "a61", "a62", "a63", "a64", "a65", "a66", "a67", "a68", "a69", "a70", "a71", "a72", "a73", "a74", "a75", "a76", "a77", "a78", "a79", "a80", "a81", "a82", "a83", "a84", "a85", "a86", "a87", "a88", "a89", "a90", "a91", "a92", "a93", "a94", "a95", "a96", "a97", "a98", "a99", "a100", "a101", "a102", "a103", "a104", "a105", "a106", "a107", "a108", "a109", "a110", "a111", "a112", "a113", "a114", "a115", "a116", "a117", "a118", "a119", "a120", "a121", "a122", "a123", "a124", "a125", "a126", "a127", "a128", "a129", "a130", "a131", "a132", "a133", "a134", "a135", "a136", "a137", "a138", "a139", "a140", "a141", "a142", "a143", "a144", "a145", "a146", "a147", "a148", "a149", "a150", "a151", "a152", "a153", "a154", "a155", "a156", "a157", "a158", "a159", "a160", "a161", "a162", "a163", "a164", "a165", "a166", "a167", "a168", "a169", "a170", "a171", "a172", "a173", "a174", "a175", "a176", "a177", "a178", "a179", "a180", "a181", "a182", "a183", "a184", "a185", "a186", "a187", "a188", "a189", "a190", "a191", "a192", "a193", "a194", "a195", "a196", "a197", "a198", "a199", "a200", "a201", "a202", "a203", "a204", "a205", "a206", "a207", "a208", "a209", "a210", "a211", "a212", "a213", "a214", "a215", "a216", "a217", "a218", "a219", "a220", "a221", "a222", "a223", "a224", "a225", "a226", "a227", "a228", "a229", "a230", "a231", "a232", "a233", "a234", "a235", "a236", "a237", "a238", "a239", "a240", "a241", "a242", "a243", "a244", "a245", "a246", "a247", "a248", "a249", "a250", "a251", "a252", "a253", "a254", "a255");
}

extern "C" void kernel_launch(int rank, int nranks, ncclComm_t comm, hipStream_t stream,
                              void* const* d_in, const int* in_sizes, int n_in,
                              void* d_out, int out_size, const int* in_dtypes, int out_dtype,
                              void* const* d_out_all, void* const* d_ws_all, size_t ws_bytes) {
    if (nranks != NRANKS || (unsigned)rank >= (unsigned)NRANKS || n_in != 2 || ws_bytes < WS_NEED ||
        in_sizes[0] != 4096 * 512 || in_sizes[1] != 4096 * 32768 || out_size != 512 * 32768) {
        fprintf(stderr, "a2a_gemm_gelu: unexpected problem shape\n");
        return;
    }
    static bool ready[NRANKS];
    if (!ready[rank]) {
        HIP_CHECK(hipFuncSetAttribute((const void*)a2a_gemm_gelu, hipFuncAttributeMaxDynamicSharedMemorySize, LDS_BYTES));
        ready[rank] = true;
    }
    Args a;
    a.x = (const char*)d_in[0]; a.w = (const char*)d_in[1]; a.out = (char*)d_out;
    for (int p = 0; p < NRANKS; ++p) a.ws[p] = (char*)d_ws_all[p];
    a.wsme = a.ws[rank];
    a.stamps = nullptr;
    a.rank = rank; a.pad = 0;
    a2a_gemm_gelu<<<256, 256, LDS_BYTES, stream>>>(a);
    HIP_CHECK(hipGetLastError());
}
```
